```python
import numpy as np

kernel_name = "gpu_gemm_a2a_m8192_k8192_n8192_bf16_silu"


ABSMAX_RTOL = 0.008

COMPUTE_DTYPE = "bfloat16"
EPILOGUE = "silu"
IN_QUANT = None
OUT_QUANT = None
IN_DTYPES = {"x": "bfloat16", "w_mat": "bfloat16"}
WORLD_SIZE = 8

def _to_bf16(x):
    u = np.ascontiguousarray(x, dtype=np.float32).view(np.uint32)
    return ((u + 0x7FFF + ((u >> 16) & 1)) & 0xFFFF0000).view(np.float32)

def setup_inputs():

    x = _to_bf16(np.random.randn(8192, 8192).astype(np.float32))
    w_mat = _to_bf16((np.random.randn(8192, 8192) / np.sqrt(8192)).astype(np.float32))
    return {"x": x, "w_mat": w_mat}

def _epilogue(y):
    return (y / (1.0 + np.exp(-np.clip(y, -60.0, 60.0)))).astype(np.float32)

def reference(x, w_mat):
    y = (x @ w_mat).astype(np.float32)
    return _epilogue(y)
```

```cpp
#include <hip/hip_runtime.h>
#include <rccl/rccl.h>
#include <cstdio>
#include <cstdint>

#define HIP_CHECK(expr) do { hipError_t _e = (expr); if (_e != hipSuccess) { \
    fprintf(stderr, "HIP error %s at %s:%d\n", hipGetErrorString(_e), __FILE__, __LINE__); \
    return; } } while (0)

#define LAS __attribute__((address_space(3)))
#define GAS __attribute__((address_space(1)))
typedef short bf16x8 __attribute__((ext_vector_type(8)));
typedef short s16x4 __attribute__((ext_vector_type(4)));
typedef float f32x4 __attribute__((ext_vector_type(4)));
typedef unsigned u32x4 __attribute__((ext_vector_type(4)));
typedef unsigned u32x2 __attribute__((ext_vector_type(2)));
typedef __bf16 bf16x2_t __attribute__((ext_vector_type(2)));

constexpr int NR = 8;
constexpr int ML = 1024;
constexpr int NN = 8192;
constexpr int KK = 8192;
constexpr int NB = 1024;
constexpr int BM = 256, BN = 128, BK = 64;
constexpr int NSTAGE = 3;
constexpr int A_BYTES = BM * BK * 2;
constexpr int B_BYTES = BK * BN * 2;
constexpr int STAGE_BYTES = A_BYTES + B_BYTES;
constexpr int FLAG_OFF = NSTAGE * STAGE_BYTES;
constexpr int LDS_BYTES = FLAG_OFF + 256;
constexpr int NWG = 256;
#define NLOAD 4
constexpr int NT = KK / BK;
constexpr unsigned long long DEADLINE = 50000000ull;

constexpr int WS_TICKET = 0;
constexpr int WS_TMO = 128;
constexpr int WS_ENTER = 1024;
constexpr int WS_DONE = 4096;

struct Params {
    const void* x; const void* w;
    void* out[NR];
    void* ws[NR];
    int rank;
    int solo;
};

__device__ __forceinline__ float silu_f(float y) {
    float c = __builtin_amdgcn_fmed3f(y, -60.0f, 60.0f);
    float e = __builtin_amdgcn_exp2f(c * -1.44269504088896341f);
    return y * __builtin_amdgcn_rcpf(1.0f + e);
}
__device__ __forceinline__ unsigned pack_bf16(float lo, float hi) {
    bf16x2_t v; v.x = (__bf16)lo; v.y = (__bf16)hi;
    return __builtin_bit_cast(unsigned, v);
}

#define SCHED() __builtin_amdgcn_sched_barrier(0)

__global__ __launch_bounds__(512, 2) void gemm_a2a(Params p) {
    __shared__ __attribute__((aligned(1024))) unsigned char lds_raw[LDS_BYTES];
    LAS unsigned char* lds = (LAS unsigned char*)lds_raw;
    const int tid = threadIdx.x;
    const int wave = __builtin_amdgcn_readfirstlane(tid >> 6);
    const int lane = tid & 63;
    volatile LAS unsigned* FULL = (volatile LAS unsigned*)(lds + FLAG_OFF);
    volatile LAS unsigned* FREE = (volatile LAS unsigned*)(lds + FLAG_OFF + 16);
    volatile LAS unsigned* MISC = (volatile LAS unsigned*)(lds + FLAG_OFF + 32);
    if (tid < 16) ((LAS unsigned*)(lds + FLAG_OFF))[tid] = (tid < 4 && tid >= NLOAD) ? 0x7fffffffu : 0u;
    __syncthreads();

    const int b = blockIdx.x;
    const int xcd = b & 7, idx = b >> 3;
    const int dest = xcd;
    const int pm = idx & 3, pn = idx >> 2;
    const int row0 = pm * BM;
    const int col0 = dest * NB + pn * BN;

    if (wave >= 4) {
        const int lw = wave - 4;
        if (lw >= NLOAD) return;
        constexpr int APW = 32 / NLOAD, BPW = 16 / NLOAD, PW = APW + BPW;
        const __amdgpu_buffer_rsrc_t rsA = __builtin_amdgcn_make_buffer_rsrc((void*)p.x, 0, ML * KK * 2, 0x00020000);
        const __amdgpu_buffer_rsrc_t rsB = __builtin_amdgcn_make_buffer_rsrc((void*)p.w, 0, KK * NN * 2, 0x00020000);
        unsigned voffA[2], voffB[2];
#pragma unroll
        for (int h = 0; h < 2; ++h) {
            voffA[h] = (unsigned)((lane >> 3) * (KK * 2) + (((lane & 7) ^ ((h << 2) | (lane >> 4))) << 4));
            const int kq = lane >> 4;
            const int sw = (kq & 1) | ((kq & 2) << 1) | (h << 3);
            voffB[h] = (unsigned)(kq * (NN * 2) + (((lane & 15) ^ sw) << 4));
        }
        unsigned sA = (unsigned)((row0 + lw * APW * 8) * (KK * 2));
        unsigned sB = (unsigned)((lw * BPW * 4) * (NN * 2) + col0 * 2);
        int stage = 0;
        for (int t = 0; t < NT; ++t) {
            if (t >= NSTAGE) {
                const unsigned need = (unsigned)(t - NSTAGE + 1);
                while (true) {
                    u32x4 f = *(volatile LAS u32x4*)FREE;
                    unsigned mn = min(min(f.x, f.y), min(f.z, f.w));
                    if (__builtin_amdgcn_readfirstlane(mn) >= need) break;
                    __builtin_amdgcn_s_sleep(1);
                }
                asm volatile("" ::: "memory");
            }
            LAS unsigned char* sa = lds + stage * STAGE_BYTES + lw * APW * 1024;
            LAS unsigned char* sb = lds + stage * STAGE_BYTES + A_BYTES + lw * BPW * 1024;
#pragma unroll
            for (int j = 0; j < BPW; ++j) {
                const int jj = lw * BPW + j;
                (void)jj;
                __builtin_amdgcn_raw_ptr_buffer_load_lds(rsB, (LAS void*)(sb + j * 1024), 16, voffB[(j >> 1) & 1], sB + j * 4 * (NN * 2), 0, 0);
            }
#pragma unroll
            for (int i = 0; i < APW; ++i)
                __builtin_amdgcn_raw_ptr_buffer_load_lds(rsA, (LAS void*)(sa + i * 1024), 16, voffA[i & 1], sA + i * 8 * (KK * 2), 0, 0);
            sA += BK * 2; sB += BK * (NN * 2);
            stage = (stage == NSTAGE - 1) ? 0 : stage + 1;
            if (t >= 1) {
                asm volatile("s_waitcnt vmcnt(%0)" :: "n"(PW) : "memory");
                if (lane == 0) FULL[lw] = (unsigned)t;
            }
        }
        asm volatile("s_waitcnt vmcnt(0)" ::: "memory");
        if (lane == 0) FULL[lw] = (unsigned)NT;
        return;
    }

    const int wm = wave >> 1, wn = wave & 1;
    const int g = lane >> 4, r16 = lane & 15;
    unsigned call = 0;
    if (wave == 0) {
        unsigned tk = 0;
        if (lane == 0) tk = __hip_atomic_fetch_add((GAS unsigned*)((char*)p.ws[p.rank] + WS_TICKET), 1u, __ATOMIC_RELAXED, __HIP_MEMORY_SCOPE_AGENT);
        tk = __builtin_amdgcn_readfirstlane(tk);
        call = tk / NWG + 1;
        if ((tk % NWG) == 0 && lane < NR && lane != p.rank && !p.solo)
            __hip_atomic_store((GAS unsigned*)((char*)p.ws[lane] + WS_ENTER + 128 * p.rank), call, __ATOMIC_RELAXED, __HIP_MEMORY_SCOPE_SYSTEM);
        if (lane == 0) MISC[0] = call;
    }
    unsigned aoff[2], boff[4];
#pragma unroll
    for (int ks = 0; ks < 2; ++ks)
        aoff[ks] = (unsigned)((wm * 128 + r16) * 128 + ((((ks << 2) | g) ^ ((r16 >> 1) & 7)) << 4));
    {
        const int q = r16 >> 2, pp = r16 & 3;
        const int k = 8 * g + q;
        const int sw = (k & 1) | ((k & 2) << 1) | (k & 8);
#pragma unroll
        for (int n = 0; n < 4; ++n) {
            const int u = n >> 1, np = n & 1;
            const int chunk = wn * 8 + 4 * u + pp;
            boff[n] = (unsigned)(A_BYTES + k * 256 + ((chunk ^ sw) << 4) + ((np ^ (pp & 1)) << 3));
        }
    }
    f32x4 acc[8][4];
#pragma unroll
    for (int m = 0; m < 8; ++m)
#pragma unroll
        for (int n = 0; n < 4; ++n) acc[m][n] = (f32x4){0.f, 0.f, 0.f, 0.f};
    bf16x8 a[8], b0[4], b1[4];

#define LDA(m, base, ks) a[m] = *(const LAS bf16x8*)((base) + aoff[ks] + (m) * 2048)
#define LDB(dst, n, base, ks) do { \
        s16x4 lo_ = __builtin_amdgcn_ds_read_tr16_b64_v4i16((LAS s16x4*)((base) + boff[n] + (ks) * 8192)); \
        s16x4 hi_ = __builtin_amdgcn_ds_read_tr16_b64_v4i16((LAS s16x4*)((base) + boff[n] + (ks) * 8192 + 1024)); \
        dst[n] = __builtin_shufflevector(lo_, hi_, 0, 1, 2, 3, 4, 5, 6, 7); } while (0)
#define WAITFULL(need) do { \
        while (true) { u32x4 f_ = *(volatile LAS u32x4*)FULL; \
            unsigned mn_ = min(min(f_.x, f_.y), min(f_.z, f_.w)); \
            if (__builtin_amdgcn_readfirstlane(mn_) >= (unsigned)(need)) break; } \
        asm volatile("" ::: "memory"); } while (0)

    WAITFULL(1);
    {
        LAS unsigned char* base = lds;
#pragma unroll
        for (int n = 0; n < 4; ++n) LDB(b0, n, base, 0);
#pragma unroll
        for (int m = 0; m < 8; ++m) LDA(m, base, 0);
    }
    int stage = 0;
    for (int t = 0; t < NT; ++t) {
        LAS unsigned char* cur = lds + stage * STAGE_BYTES;
        const int nstage = (stage == NSTAGE - 1) ? 0 : stage + 1;
        LAS unsigned char* nxt = lds + nstage * STAGE_BYTES;
        u32x4 fl = *(volatile LAS u32x4*)FULL;
        SCHED();
#pragma unroll
        for (int m = 0; m < 8; ++m) {
#pragma unroll
            for (int n = 0; n < 4; ++n) acc[m][n] = __builtin_amdgcn_mfma_f32_16x16x32_bf16(b0[n], a[m], acc[m][n], 0, 0, 0);
            if (m < 4) LDB(b1, m, cur, 1);
            LDA(m, cur, 1);
            SCHED();
        }
        if (lane == 0) FREE[wave] = (unsigned)(t + 1);
        {
            unsigned mn = min(min(fl.x, fl.y), min(fl.z, fl.w));
            const unsigned need = (unsigned)min(t + 2, NT);
            if (__builtin_amdgcn_readfirstlane(mn) < need) WAITFULL(need);
            asm volatile("" ::: "memory");
        }
        SCHED();
#pragma unroll
        for (int m = 0; m < 8; ++m) {
#pragma unroll
            for (int n = 0; n < 4; ++n) acc[m][n] = __builtin_amdgcn_mfma_f32_16x16x32_bf16(b1[n], a[m], acc[m][n], 0, 0, 0);
            if (m < 4) LDB(b0, m, nxt, 0);
            LDA(m, nxt, 0);
            SCHED();
        }
        stage = nstage;
    }

    call = MISC[0];
    GAS unsigned* tmo = (GAS unsigned*)((char*)p.ws[p.rank] + WS_TMO);
    bool bad = false;
    if (dest != p.rank && !p.solo) {
        GAS unsigned* fl = (GAS unsigned*)((char*)p.ws[p.rank] + WS_ENTER + 128 * dest);
        unsigned long long t0 = __builtin_amdgcn_s_memrealtime();
        while (__hip_atomic_load(fl, __ATOMIC_RELAXED, __HIP_MEMORY_SCOPE_SYSTEM) < call) {
            if (__hip_atomic_load(tmo, __ATOMIC_RELAXED, __HIP_MEMORY_SCOPE_AGENT) == call || __builtin_amdgcn_s_memrealtime() - t0 > DEADLINE) {
                __hip_atomic_store(tmo, call, __ATOMIC_RELAXED, __HIP_MEMORY_SCOPE_AGENT); bad = true; break; }
            __builtin_amdgcn_s_sleep(2);
        }
    }
    if (!bad) {
        char* obase = (char*)p.out[dest] + ((size_t)(p.rank * ML + row0 + wm * 128 + r16) * NB + pn * BN + wn * 64 + 8 * g) * 2;
        const bool odd = g & 1;
#pragma unroll
        for (int m = 0; m < 8; ++m)
#pragma unroll
            for (int u = 0; u < 2; ++u) {
                f32x4 v0 = acc[m][2 * u], v1 = acc[m][2 * u + 1];
                unsigned w0 = pack_bf16(silu_f(v0[0]), silu_f(v0[1])), w1 = pack_bf16(silu_f(v0[2]), silu_f(v0[3]));
                unsigned w2 = pack_bf16(silu_f(v1[0]), silu_f(v1[1])), w3 = pack_bf16(silu_f(v1[2]), silu_f(v1[3]));
                u32x4 o; o.x = odd ? w2 : w0; o.y = odd ? w3 : w1; o.z = odd ? w0 : w2; o.w = odd ? w1 : w3;
                *(GAS u32x4*)(obase + (size_t)m * 16 * NB * 2 + u * 64) = o;
            }
    }
    asm volatile("s_waitcnt vmcnt(0)" ::: "memory");
    unsigned arr = 0;
    if (lane == 0) arr = __hip_atomic_fetch_add((LAS unsigned*)&MISC[1], 1u, __ATOMIC_RELAXED, __HIP_MEMORY_SCOPE_WORKGROUP);
    arr = __builtin_amdgcn_readfirstlane(arr);
    if (arr == 3 && !p.solo) {
        if (dest != p.rank && lane == 0) {
            __builtin_amdgcn_fence(__ATOMIC_RELEASE, "");
            asm volatile("s_waitcnt vmcnt(0)" ::: "memory");
            __hip_atomic_fetch_add((GAS unsigned*)((char*)p.ws[dest] + WS_DONE + 128 * p.rank), 1u, __ATOMIC_RELAXED, __HIP_MEMORY_SCOPE_SYSTEM);
        }
        if (dest == p.rank && idx == 0) {
            if (lane < NR && lane != p.rank) {
                GAS unsigned* fl = (GAS unsigned*)((char*)p.ws[p.rank] + WS_DONE + 128 * lane);
                unsigned long long t0 = __builtin_amdgcn_s_memrealtime();
                while (__hip_atomic_load(fl, __ATOMIC_RELAXED, __HIP_MEMORY_SCOPE_SYSTEM) < call * 32u) {
                    if (__hip_atomic_load(tmo, __ATOMIC_RELAXED, __HIP_MEMORY_SCOPE_AGENT) == call || __builtin_amdgcn_s_memrealtime() - t0 > DEADLINE) {
                        __hip_atomic_store(tmo, call, __ATOMIC_RELAXED, __HIP_MEMORY_SCOPE_AGENT); break; }
                    __builtin_amdgcn_s_sleep(2);
                }
            }
            __builtin_amdgcn_fence(__ATOMIC_ACQUIRE, "");
            if (lane == 0 && __hip_atomic_load(tmo, __ATOMIC_RELAXED, __HIP_MEMORY_SCOPE_AGENT) == call)
                *(GAS unsigned short*)((char*)p.out[p.rank] + (size_t)(p.rank * ML) * NB * 2) = 0x7fc0;
        }
    }
}

#define BENCH_SOLO 0

extern "C" void kernel_launch(int rank, int nranks, ncclComm_t comm, hipStream_t stream,
                              void* const* d_in, const int* in_sizes, int n_in,
                              void* d_out, int out_size, const int* in_dtypes, int out_dtype,
                              void* const* d_out_all, void* const* d_ws_all, size_t ws_bytes) {
    if (nranks != NR) { fprintf(stderr, "need %d ranks\n", NR); return; }
    Params p;
    p.x = d_in[0]; p.w = d_in[1];
    for (int i = 0; i < NR; ++i) { p.out[i] = d_out_all[i]; p.ws[i] = d_ws_all[i]; }
    p.rank = rank; p.solo = BENCH_SOLO;
    hipLaunchKernelGGL(gemm_a2a, dim3(NWG), dim3(512), 0, stream, p);
    HIP_CHECK(hipGetLastError());
}
```

```cpp
#include <hip/hip_runtime.h>
#include <rccl/rccl.h>
#include <cstdio>
#include <cstdint>

#define HIP_CHECK(expr) do { hipError_t _e = (expr); if (_e != hipSuccess) { \
    fprintf(stderr, "HIP error %s at %s:%d\n", hipGetErrorString(_e), __FILE__, __LINE__); \
    return; } } while (0)

#define GAS __attribute__((address_space(1)))
typedef unsigned u32x4 __attribute__((ext_vector_type(4)));
typedef __amdgpu_buffer_rsrc_t rsrc_t;

#define DAUXS "sc1"
#define LAUXS "sc1 nt"

#define ASM_PLAIN_A \
    ".p2align 6\n\t" \
    "1:\n\t" \
    "v_mfma_f32_16x16x32_bf16 a[0:3], a[192:195], a[128:131], a[0:3]\n\t" \
    "ds_read_b128 a[176:179], v20 offset:12288\n\t" \
    "v_mfma_f32_16x16x32_bf16 a[0:3], a[196:199], a[132:135], a[0:3]\n\t" \
    "buffer_load_dwordx4 v26, %[rs], %[so] offen offset:0 lds\n\t" \
    "v_mfma_f32_16x16x32_bf16 a[4:7], a[200:203], a[128:131], a[4:7]\n\t" \
    "ds_read_b128 a[180:183], v21 offset:12288\n\t" \
    "v_mfma_f32_16x16x32_bf16 a[4:7], a[204:207], a[132:135], a[4:7]\n\t" \
    "buffer_load_dwordx4 v27, %[rs], %[so] offen offset:1024 lds\n\t" \
    "v_mfma_f32_16x16x32_bf16 a[8:11], a[208:211], a[128:131], a[8:11]\n\t" \
    "ds_read_b128 a[184:187], v20 offset:14336\n\t" \
    "v_mfma_f32_16x16x32_bf16 a[8:11], a[212:215], a[132:135], a[8:11]\n\t" \
    "buffer_load_dwordx4 v28, %[rs], %[so] offen offset:2048 lds\n\t" \
    "v_mfma_f32_16x16x32_bf16 a[12:15], a[216:219], a[128:131], a[12:15]\n\t" \
    "ds_read_b128 a[188:191], v21 offset:14336\n\t" \
    "v_mfma_f32_16x16x32_bf16 a[12:15], a[220:223], a[132:135], a[12:15]\n\t" \
    "buffer_load_dwordx4 v29, %[rs], %[so] offen offset:3072 lds\n\t" \
    "v_mfma_f32_16x16x32_bf16 a[32:35], a[192:195], a[136:139], a[32:35]\n\t" \
    "ds_read_b64_tr_b16 a[224:225], v22 offset:0\n\t" \
    "s_add_u32 m0, m0, 0x1000\n\t" \
    "v_mfma_f32_16x16x32_bf16 a[32:35], a[196:199], a[140:143], a[32:35]\n\t" \
    "s_add_u32 %[so], %[so], 0x80000\n\t" \
    "v_xor_b32 v20, 0x38000, v20\n\t" \
    "v_mfma_f32_16x16x32_bf16 a[36:39], a[200:203], a[136:139], a[36:39]\n\t" \
    "ds_read_b64_tr_b16 a[226:227], v22 offset:1024\n\t" \
    "v_xor_b32 v21, 0x38000, v21\n\t" \
    "v_mfma_f32_16x16x32_bf16 a[36:39], a[204:207], a[140:143], a[36:39]\n\t" \
    "buffer_load_dwordx4 v26, %[rs], %[so] offen offset:0 lds\n\t" \
    "v_mfma_f32_16x16x32_bf16 a[40:43], a[208:211], a[136:139], a[40:43]\n\t" \
    "ds_read_b64_tr_b16 a[228:229], v22 offset:8192\n\t" \
    "v_mfma_f32_16x16x32_bf16 a[40:43], a[212:215], a[140:143], a[40:43]\n\t" \
    "buffer_load_dwordx4 v27, %[rs], %[so] offen offset:1024 lds\n\t" \
    "v_mfma_f32_16x16x32_bf16 a[44:47], a[216:219], a[136:139], a[44:47]\n\t" \
    "ds_read_b64_tr_b16 a[230:231], v22 offset:9216\n\t" \
    "v_mfma_f32_16x16x32_bf16 a[44:47], a[220:223], a[140:143], a[44:47]\n\t" \
    "buffer_load_dwordx4 v28, %[rs], %[so] offen offset:2048 lds\n\t" \
    "v_mfma_f32_16x16x32_bf16 v[48:51], a[192:195], a[144:147], v[48:51]\n\t" \
    "ds_read_b64_tr_b16 a[232:233], v23 offset:0\n\t" \
    "v_mfma_f32_16x16x32_bf16 v[48:51], a[196:199], a[148:151], v[48:51]\n\t" \
    "buffer_load_dwordx4 v29, %[rs], %[so] offen offset:3072 lds\n\t" \
    "v_mfma_f32_16x16x32_bf16 v[52:55], a[200:203], a[144:147], v[52:55]\n\t" \
    "ds_read_b64_tr_b16 a[234:235], v23 offset:1024\n\t" \
    "s_add_u32 m0, m0, 0x1000\n\t" \
    "v_mfma_f32_16x16x32_bf16 v[52:55], a[204:207], a[148:151], v[52:55]\n\t" \
    "s_add_u32 %[so], %[so], 0x80000\n\t" \
    "v_mfma_f32_16x16x32_bf16 v[56:59], a[208:211], a[144:147], v[56:59]\n\t" \
    "ds_read_b64_tr_b16 a[236:237], v23 offset:8192\n\t" \
    "v_mfma_f32_16x16x32_bf16 v[56:59], a[212:215], a[148:151], v[56:59]\n\t" \
    "buffer_load_dwordx4 v26, %[rs], %[so] offen offset:0 lds\n\t" \
    "v_mfma_f32_16x16x32_bf16 v[60:63], a[216:219], a[144:147], v[60:63]\n\t" \
    "ds_read_b64_tr_b16 a[238:239], v23 offset:9216\n\t" \
    "v_mfma_f32_16x16x32_bf16 v[60:63], a[220:223], a[148:151], v[60:63]\n\t" \
    "buffer_load_dwordx4 v27, %[rs], %[so] offen offset:1024 lds\n\t" \
    "v_mfma_f32_16x16x32_bf16 v[80:83], a[192:195], a[152:155], v[80:83]\n\t" \
    "ds_read_b64_tr_b16 a[240:241], v24 offset:0\n\t" \
    "v_mfma_f32_16x16x32_bf16 v[80:83], a[196:199], a[156:159], v[80:83]\n\t" \
    "buffer_load_dwordx4 v28, %[rs], %[so] offen offset:2048 lds\n\t" \
    "v_mfma_f32_16x16x32_bf16 v[84:87], a[200:203], a[152:155], v[84:87]\n\t" \
    "ds_read_b64_tr_b16 a[242:243], v24 offset:1024\n\t" \
    "v_mfma_f32_16x16x32_bf16 v[84:87], a[204:207], a[156:159], v[84:87]\n\t" \
    "buffer_load_dwordx4 v29, %[rs], %[so] offen offset:3072 lds\n\t" \
    "v_mfma_f32_16x16x32_bf16 v[88:91], a[208:211], a[152:155], v[88:91]\n\t" \
    "ds_read_b64_tr_b16 a[244:245], v24 offset:8192\n\t" \
    "s_add_u32 m0, m0, 0x2000\n\t" \
    "v_mfma_f32_16x16x32_bf16 v[88:91], a[212:215], a[156:159], v[88:91]\n\t" \
    "s_add_u32 %[so], %[so], 0x100000\n\t" \
    "v_mfma_f32_16x16x32_bf16 v[92:95], a[216:219], a[152:155], v[92:95]\n\t" \
    "ds_read_b64_tr_b16 a[246:247], v24 offset:9216\n\t" \
    "v_mfma_f32_16x16x32_bf16 v[92:95], a[220:223], a[156:159], v[92:95]\n\t" \
    "buffer_load_dwordx4 v26, %[rs], %[so] offen offset:0 lds\n\t" \
    "v_mfma_f32_16x16x32_bf16 v[112:115], a[192:195], a[160:163], v[112:115]\n\t" \
    "ds_read_b64_tr_b16 a[248:249], v25 offset:0\n\t" \
    "v_mfma_f32_16x16x32_bf16 v[112:115], a[196:199], a[164:167], v[112:115]\n\t" \
    "buffer_load_dwordx4 v27, %[rs], %[so] offen offset:1024 lds\n\t" \
    "v_mfma_f32_16x16x32_bf16 v[116:119], a[200:203], a[160:163], v[116:119]\n\t" \
    "ds_read_b64_tr_b16 a[250:251], v25 offset:1024\n\t" \
    "v_mfma_f32_16x16x32_bf16 v[116:119], a[204:207], a[164:167], v[116:119]\n\t" \
    "buffer_load_dwordx4 v28, %[rs], %[so] offen offset:2048 lds\n\t" \
    "v_mfma_f32_16x16x32_bf16 v[120:123], a[208:211], a[160:163], v[120:123]\n\t" \
    "ds_read_b64_tr_b16 a[252:253], v25 offset:8192\n\t" \
    "v_mfma_f32_16x16x32_bf16 v[120:123], a[212:215], a[164:167], v[120:123]\n\t" \
    "buffer_load_dwordx4 v29, %[rs], %[so] offen offset:3072 lds\n\t" \
    "v_mfma_f32_16x16x32_bf16 v[124:127], a[216:219], a[160:163], v[124:127]\n\t" \
    "ds_read_b64_tr_b16 a[254:255], v25 offset:9216\n\t" \
    "s_add_u32 m0, m0, 0x1000\n\t" \
    "v_mfma_f32_16x16x32_bf16 v[124:127], a[220:223], a[164:167], v[124:127]\n\t" \
    "s_add_u32 %[so], %[so], 0x80000\n\t" \
    "v_mfma_f32_16x16x32_bf16 v[144:147], a[192:195], a[168:171], v[144:147]\n\t" \
    "buffer_load_dwordx4 v26, %[rs], %[so] offen offset:0 lds\n\t" \
    "v_mfma_f32_16x16x32_bf16 v[144:147], a[196:199], a[172:175], v[144:147]\n\t" \
    "buffer_load_dwordx4 v27, %[rs], %[so] offen offset:1024 lds\n\t" \
    "v_mfma_f32_16x16x32_bf16 v[148:151], a[200:203], a[168:171], v[148:151]\n\t" \
    "buffer_load_dwordx4 v28, %[rs], %[so] offen offset:2048 lds\n\t" \
    "v_mfma_f32_16x16x32_bf16 v[148:151], a[204:207], a[172:175], v[148:151]\n\t" \
    "buffer_load_dwordx4 v29, %[rs], %[so] offen offset:3072 lds\n\t" \
    "v_mfma_f32_16x16x32_bf16 v[152:155], a[208:211], a[168:171], v[152:155]\n\t" \
    "s_add_u32 m0, m0, 0x1000\n\t" \
    "s_add_u32 %[so], %[so], 0x80000\n\t" \
    "v_mfma_f32_16x16x32_bf16 v[152:155], a[212:215], a[172:175], v[152:155]\n\t" \
    "buffer_load_dwordx4 v26, %[rs], %[so] offen offset:0 lds\n\t" \
    "v_mfma_f32_16x16x32_bf16 v[156:159], a[216:219], a[168:171], v[156:159]\n\t" \
    "buffer_load_dwordx4 v27, %[rs], %[so] offen offset:1024 lds\n\t" \
    "v_mfma_f32_16x16x32_bf16 v[156:159], a[220:223], a[172:175], v[156:159]\n\t" \
    "s_waitcnt lgkmcnt(8)\n\ts_nop 0\n\t" \
    "v_mfma_f32_16x16x32_bf16 v[176:179], a[192:195], a[176:179], v[176:179]\n\t" \
    "buffer_load_dwordx4 v28, %[rs], %[so] offen offset:2048 lds\n\t" \
    "v_mfma_f32_16x16x32_bf16 v[176:179], a[196:199], a[180:183], v[176:179]\n\t" \
    "buffer_load_dwordx4 v29, %[rs], %[so] offen offset:3072 lds\n\t" \
    "v_mfma_f32_16x16x32_bf16 v[180:183], a[200:203], a[176:179], v[180:183]\n\t" \
    "s_add_u32 m0, m0, 0xffffd000\n\t" \
    "v_mfma_f32_16x16x32_bf16 v[180:183], a[204:207], a[180:183], v[180:183]\n\t" \
    "s_add_u32 %[so], %[so], 0xffe80000\n\t" \
    "v_mfma_f32_16x16x32_bf16 v[184:187], a[208:211], a[176:179], v[184:187]\n\t" \
    "v_mfma_f32_16x16x32_bf16 v[184:187], a[212:215], a[180:183], v[184:187]\n\t" \
    "v_mfma_f32_16x16x32_bf16 v[188:191], a[216:219], a[176:179], v[188:191]\n\t" \
    "v_mfma_f32_16x16x32_bf16 v[188:191], a[220:223], a[180:183], v[188:191]\n\t" \
    "v_mfma_f32_16x16x32_bf16 v[208:211], a[192:195], a[184:187], v[208:211]\n\t" \
    "v_mfma_f32_16x16x32_bf16 v[208:211], a[196:199], a[188:191], v[208:211]\n\t" \
    "v_mfma_f32_16x16x32_bf16 v[212:215], a[200:203], a[184:187], v[212:215]\n\t" \
    "v_mfma_f32_16x16x32_bf16 v[212:215], a[204:207], a[188:191], v[212:215]\n\t" \
    "v_mfma_f32_16x16x32_bf16 v[216:219], a[208:211], a[184:187], v[216:219]\n\t" \
    "v_mfma_f32_16x16x32_bf16 v[216:219], a[212:215], a[188:191], v[216:219]\n\t" \
    "v_mfma_f32_16x16x32_bf16 v[220:223], a[216:219], a[184:187], v[220:223]\n\t" \
    "v_mfma_f32_16x16x32_bf16 v[220:223], a[220:223], a[188:191], v[220:223]\n\t" \
    "s_waitcnt vmcnt(32) lgkmcnt(0)\n\ts_barrier\n\t" \
    "v_mfma_f32_16x16x32_bf16 a[16:19], a[224:227], a[128:131], a[16:19]\n\t" \
    "ds_read_b64_tr_b16 a[192:193], v22 offset:16384\n\t" \
    "v_mfma_f32_16x16x32_bf16 a[16:19], a[228:231], a[132:135], a[16:19]\n\t" \
    "buffer_load_dwordx4 v26, %[rs], %[so] offen offset:0 lds\n\t" \
    "v_mfma_f32_16x16x32_bf16 a[20:23], a[232:235], a[128:131], a[20:23]\n\t" \
    "ds_read_b64_tr_b16 a[194:195], v22 offset:17408\n\t" \
    "v_mfma_f32_16x16x32_bf16 a[20:23], a[236:239], a[132:135], a[20:23]\n\t" \
    "buffer_load_dwordx4 v27, %[rs], %[so] offen offset:1024 lds\n\t" \
    "v_mfma_f32_16x16x32_bf16 a[24:27], a[240:243], a[128:131], a[24:27]\n\t" \
    "ds_read_b64_tr_b16 a[196:197], v22 offset:24576\n\t" \
    "v_mfma_f32_16x16x32_bf16 a[24:27], a[244:247], a[132:135], a[24:27]\n\t" \
    "buffer_load_dwordx4 v28, %[rs], %[so] offen offset:2048 lds\n\t" \
    "v_mfma_f32_16x16x32_bf16 a[28:31], a[248:251], a[128:131], a[28:31]\n\t" \
    "ds_read_b64_tr_b16 a[198:199], v22 offset:25600\n\t" \
    "v_mfma_f32_16x16x32_bf16 a[28:31], a[252:255], a[132:135], a[28:31]\n\t" \
    "buffer_load_dwordx4 v29, %[rs], %[so] offen offset:3072 lds\n\t" \
    "v_mfma_f32_16x16x32_bf16 a[48:51], a[224:227], a[136:139], a[48:51]\n\t" \
    "ds_read_b128 a[128:131], v20 offset:0\n\t" \
    "s_add_u32 m0, m0, 0x4000\n\t" \
    "v_mfma_f32_16x16x32_bf16 a[48:51], a[228:231], a[140:143], a[48:51]\n\t" \
    "s_add_u32 %[so], %[so], 0x200000\n\t" \
    "v_mfma_f32_16x16x32_bf16 a[52:55], a[232:235], a[136:139], a[52:55]\n\t" \
    "ds_read_b128 a[132:135], v21 offset:0\n\t" \
    "v_mfma_f32_16x16x32_bf16 a[52:55], a[236:239], a[140:143], a[52:55]\n\t" \
    "buffer_load_dwordx4 v26, %[rs], %[so] offen offset:0 lds\n\t" \
    "v_mfma_f32_16x16x32_bf16 a[56:59], a[240:243], a[136:139], a[56:59]\n\t" \
    "ds_read_b64_tr_b16 a[200:201], v23 offset:16384\n\t" \
    "v_mfma_f32_16x16x32_bf16 a[56:59], a[244:247], a[140:143], a[56:59]\n\t" \
    "buffer_load_dwordx4 v27, %[rs], %[so] offen offset:1024 lds\n\t" \
    "v_mfma_f32_16x16x32_bf16 a[60:63], a[248:251], a[136:139], a[60:63]\n\t" \
    "ds_read_b64_tr_b16 a[202:203], v23 offset:17408\n\t" \
    "v_mfma_f32_16x16x32_bf16 a[60:63], a[252:255], a[140:143], a[60:63]\n\t" \
    "buffer_load_dwordx4 v28, %[rs], %[so] offen offset:2048 lds\n\t" \
    "v_mfma_f32_16x16x32_bf16 v[64:67], a[224:227], a[144:147], v[64:67]\n\t" \
    "ds_read_b128 a[136:139], v20 offset:2048\n\t" \
    "v_mfma_f32_16x16x32_bf16 v[64:67], a[228:231], a[148:151], v[64:67]\n\t" \
    "buffer_load_dwordx4 v29, %[rs], %[so] offen offset:3072 lds\n\t" \
    "v_mfma_f32_16x16x32_bf16 v[68:71], a[232:235], a[144:147], v[68:71]\n\t" \
    "ds_read_b128 a[140:143], v21 offset:2048\n\t" \
    "s_xor_b32 m0, m0, 0x3f000\n\t" \
    "v_mfma_f32_16x16x32_bf16 v[68:71], a[236:239], a[148:151], v[68:71]\n\t" \
    "s_add_u32 %[so], %[so], 0xffc80080\n\t" \
    "v_mfma_f32_16x16x32_bf16 v[72:75], a[240:243], a[144:147], v[72:75]\n\t" \
    "ds_read_b64_tr_b16 a[204:205], v23 offset:24576\n\t" \
    "v_mfma_f32_16x16x32_bf16 v[72:75], a[244:247], a[148:151], v[72:75]\n\t" \
    "s_cmp_eq_u32 %[rem], 3\n\ts_cselect_b32 %[so], %[nx], %[so]\n\t" \
    "v_mfma_f32_16x16x32_bf16 v[76:79], a[248:251], a[144:147], v[76:79]\n\t" \
    "ds_read_b64_tr_b16 a[206:207], v23 offset:25600\n\t" \
    "v_mfma_f32_16x16x32_bf16 v[76:79], a[252:255], a[148:151], v[76:79]\n\t" \
    "v_mfma_f32_16x16x32_bf16 v[96:99], a[224:227], a[152:155], v[96:99]\n\t" \
    "ds_read_b128 a[144:147], v20 offset:4096\n\t" \
    "v_mfma_f32_16x16x32_bf16 v[96:99], a[228:231], a[156:159], v[96:99]\n\t" \
    "v_mfma_f32_16x16x32_bf16 v[100:103], a[232:235], a[152:155], v[100:103]\n\t" \
    "ds_read_b128 a[148:151], v21 offset:4096\n\t" \
    "v_mfma_f32_16x16x32_bf16 v[100:103], a[236:239], a[156:159], v[100:103]\n\t" \
    "v_mfma_f32_16x16x32_bf16 v[104:107], a[240:243], a[152:155], v[104:107]\n\t" \
    "ds_read_b64_tr_b16 a[208:209], v24 offset:16384\n\t" \
    "v_mfma_f32_16x16x32_bf16 v[104:107], a[244:247], a[156:159], v[104:107]\n\t" \
    "v_mfma_f32_16x16x32_bf16 v[108:111], a[248:251], a[152:155], v[108:111]\n\t" \
    "ds_read_b64_tr_b16 a[210:211], v24 offset:17408\n\t" \
    "v_mfma_f32_16x16x32_bf16 v[108:111], a[252:255], a[156:159], v[108:111]\n\t" \
    "v_mfma_f32_16x16x32_bf16 v[128:131], a[224:227], a[160:163], v[128:131]\n\t" \
    "ds_read_b128 a[152:155], v20 offset:6144\n\t" \
    "v_mfma_f32_16x16x32_bf16 v[128:131], a[228:231], a[164:167], v[128:131]\n\t" \
    "v_mfma_f32_16x16x32_bf16 v[132:135], a[232:235], a[160:163], v[132:135]\n\t" \
    "ds_read_b128 a[156:159], v21 offset:6144\n\t" \
    "v_mfma_f32_16x16x32_bf16 v[132:135], a[236:239], a[164:167], v[132:135]\n\t" \
    "v_mfma_f32_16x16x32_bf16 v[136:139], a[240:243], a[160:163], v[136:139]\n\t" \
    "ds_read_b64_tr_b16 a[212:213], v24 offset:24576\n\t" \
    "v_mfma_f32_16x16x32_bf16 v[136:139], a[244:247], a[164:167], v[136:139]\n\t" \
    "v_mfma_f32_16x16x32_bf16 v[140:143], a[248:251], a[160:163], v[140:143]\n\t" \
    "ds_read_b64_tr_b16 a[214:215], v24 offset:25600\n\t" \
    "v_mfma_f32_16x16x32_bf16 v[140:143], a[252:255], a[164:167], v[140:143]\n\t" \
    "v_mfma_f32_16x16x32_bf16 v[160:163], a[224:227], a[168:171], v[160:163]\n\t" \
    "ds_read_b128 a[160:163], v20 offset:8192\n\t" \
    "v_mfma_f32_16x16x32_bf16 v[160:163], a[228:231], a[172:175], v[160:163]\n\t" \
    "v_mfma_f32_16x16x32_bf16 v[164:167], a[232:235], a[168:171], v[164:167]\n\t" \
    "ds_read_b128 a[164:167], v21 offset:8192\n\t" \
    "v_mfma_f32_16x16x32_bf16 v[164:167], a[236:239], a[172:175], v[164:167]\n\t" \
    "v_mfma_f32_16x16x32_bf16 v[168:171], a[240:243], a[168:171], v[168:171]\n\t" \
    "ds_read_b64_tr_b16 a[216:217], v25 offset:16384\n\t" \
    "v_mfma_f32_16x16x32_bf16 v[168:171], a[244:247], a[172:175], v[168:171]\n\t" \
    "v_mfma_f32_16x16x32_bf16 v[172:175], a[248:251], a[168:171], v[172:175]\n\t" \
    "ds_read_b64_tr_b16 a[218:219], v25 offset:17408\n\t" \
    "v_mfma_f32_16x16x32_bf16 v[172:175], a[252:255], a[172:175], v[172:175]\n\t" \
    "v_mfma_f32_16x16x32_bf16 v[192:195], a[224:227], a[176:179], v[192:195]\n\t" \
    "ds_read_b128 a[168:171], v20 offset:10240\n\t" \
    "v_mfma_f32_16x16x32_bf16 v[192:195], a[228:231], a[180:183], v[192:195]\n\t" \
    "v_mfma_f32_16x16x32_bf16 v[196:199], a[232:235], a[176:179], v[196:199]\n\t" \
    "ds_read_b128 a[172:175], v21 offset:10240\n\t" \
    "v_mfma_f32_16x16x32_bf16 v[196:199], a[236:239], a[180:183], v[196:199]\n\t" \
    "v_mfma_f32_16x16x32_bf16 v[200:203], a[240:243], a[176:179], v[200:203]\n\t" \
    "ds_read_b64_tr_b16 a[220:221], v25 offset:24576\n\t" \
    "v_mfma_f32_16x16x32_bf16 v[200:203], a[244:247], a[180:183], v[200:203]\n\t" \
    "v_mfma_f32_16x16x32_bf16 v[204:207], a[248:251], a[176:179], v[204:207]\n\t" \
    "ds_read_b64_tr_b16 a[222:223], v25 offset:25600\n\t" \
    "v_mfma_f32_16x16x32_bf16 v[204:207], a[252:255], a[180:183], v[204:207]\n\t" \
    "v_mfma_f32_16x16x32_bf16 v[224:227], a[224:227], a[184:187], v[224:227]\n\t" \
    "v_add_u32 v23, 0x8000, v22\n\t" \
    "v_add_u32 v22, 0xffff0000, v22\n\t" \
    "v_mfma_f32_16x16x32_bf16 v[224:227], a[228:231], a[188:191], v[224:227]\n\t" \
    "v_min_u32_e64 v22, v22, v23\n\t" \
    "v_mfma_f32_16x16x32_bf16 v[228:231], a[232:235], a[184:187], v[228:231]\n\t" \
    "v_xor_b32_e64 v23, 32, v22\n\t" \
    "v_xor_b32_e64 v24, 64, v22\n\t" \
    "v_mfma_f32_16x16x32_bf16 v[228:231], a[236:239], a[188:191], v[228:231]\n\t" \
    "v_xor_b32 v25, 0x60, v22\n\t" \
    "v_mfma_f32_16x16x32_bf16 v[232:235], a[240:243], a[184:187], v[232:235]\n\t" \
    "v_mfma_f32_16x16x32_bf16 v[232:235], a[244:247], a[188:191], v[232:235]\n\t" \
    "v_mfma_f32_16x16x32_bf16 v[236:239], a[248:251], a[184:187], v[236:239]\n\t" \
    "v_mfma_f32_16x16x32_bf16 v[236:239], a[252:255], a[188:191], v[236:239]\n\t" \
    "s_sub_u32 %[rem], %[rem], 1\n\ts_cmp_lg_u32 %[rem], %[stop]\n\t" \
    "s_waitcnt vmcnt(32) lgkmcnt(0)\n\ts_barrier\n\t" \
    "s_cbranch_scc1 1b\n\t" \
    ""
#define ASM_END_A \
    "v_mfma_f32_16x16x32_bf16 a[0:3], a[192:195], a[128:131], a[0:3]\n\t" \
    "ds_read_b128 a[176:179], v20 offset:12288\n\t" \
    "v_mfma_f32_16x16x32_bf16 a[0:3], a[196:199], a[132:135], a[0:3]\n\t" \
    "buffer_load_dwordx4 v26, %[rs], %[so] offen offset:0 lds\n\t" \
    "v_mfma_f32_16x16x32_bf16 a[4:7], a[200:203], a[128:131], a[4:7]\n\t" \
    "ds_read_b128 a[180:183], v21 offset:12288\n\t" \
    "v_mfma_f32_16x16x32_bf16 a[4:7], a[204:207], a[132:135], a[4:7]\n\t" \
    "buffer_load_dwordx4 v27, %[rs], %[so] offen offset:1024 lds\n\t" \
    "v_mfma_f32_16x16x32_bf16 a[8:11], a[208:211], a[128:131], a[8:11]\n\t" \
    "ds_read_b128 a[184:187], v20 offset:14336\n\t" \
    "v_mfma_f32_16x16x32_bf16 a[8:11], a[212:215], a[132:135], a[8:11]\n\t" \
    "buffer_load_dwordx4 v28, %[rs], %[so] offen offset:2048 lds\n\t" \
    "v_mfma_f32_16x16x32_bf16 a[12:15], a[216:219], a[128:131], a[12:15]\n\t" \
    "ds_read_b128 a[188:191], v21 offset:14336\n\t" \
    "v_mfma_f32_16x16x32_bf16 a[12:15], a[220:223], a[132:135], a[12:15]\n\t" \
    "buffer_load_dwordx4 v29, %[rs], %[so] offen offset:3072 lds\n\t" \
    "v_mfma_f32_16x16x32_bf16 a[32:35], a[192:195], a[136:139], a[32:35]\n\t" \
    "ds_read_b64_tr_b16 a[224:225], v22 offset:0\n\t" \
    "s_add_u32 m0, m0, 0x1000\n\t" \
    "v_mfma_f32_16x16x32_bf16 a[32:35], a[196:199], a[140:143], a[32:35]\n\t" \
    "s_add_u32 %[so], %[so], 0x80000\n\t" \
    "v_xor_b32 v20, 0x38000, v20\n\t" \
    "v_mfma_f32_16x16x32_bf16 a[36:39], a[200:203], a[136:139], a[36:39]\n\t" \
    "ds_read_b64_tr_b16 a[226:227], v22 offset:1024\n\t" \
    "v_xor_b32 v21, 0x38000, v21\n\t" \
    "v_mfma_f32_16x16x32_bf16 a[36:39], a[204:207], a[140:143], a[36:39]\n\t" \
    "buffer_load_dwordx4 v26, %[rs], %[so] offen offset:0 lds\n\t" \
    "v_mfma_f32_16x16x32_bf16 a[40:43], a[208:211], a[136:139], a[40:43]\n\t" \
    "ds_read_b64_tr_b16 a[228:229], v22 offset:8192\n\t" \
    "v_mfma_f32_16x16x32_bf16 a[40:43], a[212:215], a[140:143], a[40:43]\n\t" \
    "buffer_load_dwordx4 v27, %[rs], %[so] offen offset:1024 lds\n\t" \
    "v_mfma_f32_16x16x32_bf16 a[44:47], a[216:219], a[136:139], a[44:47]\n\t" \
    "ds_read_b64_tr_b16 a[230:231], v22 offset:9216\n\t" \
    "v_mfma_f32_16x16x32_bf16 a[44:47], a[220:223], a[140:143], a[44:47]\n\t" \
    "buffer_load_dwordx4 v28, %[rs], %[so] offen offset:2048 lds\n\t" \
    "v_mfma_f32_16x16x32_bf16 v[48:51], a[192:195], a[144:147], v[48:51]\n\t" \
    "ds_read_b64_tr_b16 a[232:233], v23 offset:0\n\t" \
    "v_mfma_f32_16x16x32_bf16 v[48:51], a[196:199], a[148:151], v[48:51]\n\t" \
    "buffer_load_dwordx4 v29, %[rs], %[so] offen offset:3072 lds\n\t" \
    "v_mfma_f32_16x16x32_bf16 v[52:55], a[200:203], a[144:147], v[52:55]\n\t" \
    "ds_read_b64_tr_b16 a[234:235], v23 offset:1024\n\t" \
    "s_add_u32 m0, m0, 0x1000\n\t" \
    "v_mfma_f32_16x16x32_bf16 v[52:55], a[204:207], a[148:151], v[52:55]\n\t" \
    "s_add_u32 %[so], %[so], 0x80000\n\t" \
    "v_mfma_f32_16x16x32_bf16 v[56:59], a[208:211], a[144:147], v[56:59]\n\t" \
    "ds_read_b64_tr_b16 a[236:237], v23 offset:8192\n\t" \
    "v_mfma_f32_16x16x32_bf16 v[56:59], a[212:215], a[148:151], v[56:59]\n\t" \
    "buffer_load_dwordx4 v26, %[rs], %[so] offen offset:0 lds\n\t" \
    "v_mfma_f32_16x16x32_bf16 v[60:63], a[216:219], a[144:147], v[60:63]\n\t" \
    "ds_read_b64_tr_b16 a[238:239], v23 offset:9216\n\t" \
    "v_mfma_f32_16x16x32_bf16 v[60:63], a[220:223], a[148:151], v[60:63]\n\t" \
    "buffer_load_dwordx4 v27, %[rs], %[so] offen offset:1024 lds\n\t" \
    "v_mfma_f32_16x16x32_bf16 v[80:83], a[192:195], a[152:155], v[80:83]\n\t" \
    "ds_read_b64_tr_b16 a[240:241], v24 offset:0\n\t" \
    "v_mfma_f32_16x16x32_bf16 v[80:83], a[196:199], a[156:159], v[80:83]\n\t" \
    "buffer_load_dwordx4 v28, %[rs], %[so] offen offset:2048 lds\n\t" \
    "v_mfma_f32_16x16x32_bf16 v[84:87], a[200:203], a[152:155], v[84:87]\n\t" \
    "ds_read_b64_tr_b16 a[242:243], v24 offset:1024\n\t" \
    "v_mfma_f32_16x16x32_bf16 v[84:87], a[204:207], a[156:159], v[84:87]\n\t" \
    "buffer_load_dwordx4 v29, %[rs], %[so] offen offset:3072 lds\n\t" \
    "v_mfma_f32_16x16x32_bf16 v[88:91], a[208:211], a[152:155], v[88:91]\n\t" \
    "ds_read_b64_tr_b16 a[244:245], v24 offset:8192\n\t" \
    "s_add_u32 m0, m0, 0x2000\n\t" \
    "v_mfma_f32_16x16x32_bf16 v[88:91], a[212:215], a[156:159], v[88:91]\n\t" \
    "s_add_u32 %[so], %[so], 0x100000\n\t" \
    "v_mfma_f32_16x16x32_bf16 v[92:95], a[216:219], a[152:155], v[92:95]\n\t" \
    "ds_read_b64_tr_b16 a[246:247], v24 offset:9216\n\t" \
    "v_mfma_f32_16x16x32_bf16 v[92:95], a[220:223], a[156:159], v[92:95]\n\t" \
    "buffer_load_dwordx4 v26, %[rs], %[so] offen offset:0 lds\n\t" \
    "v_mfma_f32_16x16x32_bf16 v[112:115], a[192:195], a[160:163], v[112:115]\n\t" \
    "ds_read_b64_tr_b16 a[248:249], v25 offset:0\n\t" \
    "v_mfma_f32_16x16x32_bf16 v[112:115], a[196:199], a[164:167], v[112:115]\n\t" \
    "buffer_load_dwordx4 v27, %[rs], %[so] offen offset:1024 lds\n\t" \
    "v_mfma_f32_16x16x32_bf16 v[116:119], a[200:203], a[160:163], v[116:119]\n\t" \
    "ds_read_b64_tr_b16 a[250:251], v25 offset:1024\n\t" \
    "v_mfma_f32_16x16x32_bf16 v[116:119], a[204:207], a[164:167], v[116:119]\n\t" \
    "buffer_load_dwordx4 v28, %[rs], %[so] offen offset:2048 lds\n\t" \
    "v_mfma_f32_16x16x32_bf16 v[120:123], a[208:211], a[160:163], v[120:123]\n\t" \
    "ds_read_b64_tr_b16 a[252:253], v25 offset:8192\n\t" \
    "v_mfma_f32_16x16x32_bf16 v[120:123], a[212:215], a[164:167], v[120:123]\n\t" \
    "buffer_load_dwordx4 v29, %[rs], %[so] offen offset:3072 lds\n\t" \
    "v_mfma_f32_16x16x32_bf16 v[124:127], a[216:219], a[160:163], v[124:127]\n\t" \
    "ds_read_b64_tr_b16 a[254:255], v25 offset:9216\n\t" \
    "s_add_u32 m0, m0, 0x1000\n\t" \
    "v_mfma_f32_16x16x32_bf16 v[124:127], a[220:223], a[164:167], v[124:127]\n\t" \
    "s_add_u32 %[so], %[so], 0x80000\n\t" \
    "v_mfma_f32_16x16x32_bf16 v[144:147], a[192:195], a[168:171], v[144:147]\n\t" \
    "buffer_load_dwordx4 v26, %[rs], %[so] offen offset:0 lds\n\t" \
    "v_mfma_f32_16x16x32_bf16 v[144:147], a[196:199], a[172:175], v[144:147]\n\t" \
    "buffer_load_dwordx4 v27, %[rs], %[so] offen offset:1024 lds\n\t" \
    "v_mfma_f32_16x16x32_bf16 v[148:151], a[200:203], a[168:171], v[148:151]\n\t" \
    "buffer_load_dwordx4 v28, %[rs], %[so] offen offset:2048 lds\n\t" \
    "v_mfma_f32_16x16x32_bf16 v[148:151], a[204:207], a[172:175], v[148:151]\n\t" \
    "buffer_load_dwordx4 v29, %[rs], %[so] offen offset:3072 lds\n\t" \
    "v_mfma_f32_16x16x32_bf16 v[152:155], a[208:211], a[168:171], v[152:155]\n\t" \
    "s_add_u32 m0, m0, 0x1000\n\t" \
    "s_add_u32 %[so], %[so], 0x80000\n\t" \
    "v_mfma_f32_16x16x32_bf16 v[152:155], a[212:215], a[172:175], v[152:155]\n\t" \
    "buffer_load_dwordx4 v26, %[rs], %[so] offen offset:0 lds\n\t" \
    "v_mfma_f32_16x16x32_bf16 v[156:159], a[216:219], a[168:171], v[156:159]\n\t" \
    "buffer_load_dwordx4 v27, %[rs], %[so] offen offset:1024 lds\n\t" \
    "v_mfma_f32_16x16x32_bf16 v[156:159], a[220:223], a[172:175], v[156:159]\n\t" \
    "s_waitcnt lgkmcnt(8)\n\ts_nop 0\n\t" \
    "v_mfma_f32_16x16x32_bf16 v[176:179], a[192:195], a[176:179], v[176:179]\n\t" \
    "buffer_load_dwordx4 v28, %[rs], %[so] offen offset:2048 lds\n\t" \
    "v_mfma_f32_16x16x32_bf16 v[176:179], a[196:199], a[180:183], v[176:179]\n\t" \
    "buffer_load_dwordx4 v29, %[rs], %[so] offen offset:3072 lds\n\t" \
    "v_mfma_f32_16x16x32_bf16 v[180:183], a[200:203], a[176:179], v[180:183]\n\t" \
    "s_add_u32 m0, m0, 0xffffd000\n\t" \
    "v_mfma_f32_16x16x32_bf16 v[180:183], a[204:207], a[180:183], v[180:183]\n\t" \
    "s_add_u32 %[so], %[so], 0xffe80000\n\t" \
    "v_mfma_f32_16x16x32_bf16 v[184:187], a[208:211], a[176:179], v[184:187]\n\t" \
    "v_mfma_f32_16x16x32_bf16 v[184:187], a[212:215], a[180:183], v[184:187]\n\t" \
    "v_mfma_f32_16x16x32_bf16 v[188:191], a[216:219], a[176:179], v[188:191]\n\t" \
    "v_mfma_f32_16x16x32_bf16 v[188:191], a[220:223], a[180:183], v[188:191]\n\t" \
    "v_mfma_f32_16x16x32_bf16 v[208:211], a[192:195], a[184:187], v[208:211]\n\t" \
    "v_mfma_f32_16x16x32_bf16 v[208:211], a[196:199], a[188:191], v[208:211]\n\t" \
    "v_mfma_f32_16x16x32_bf16 v[212:215], a[200:203], a[184:187], v[212:215]\n\t" \
    "v_mfma_f32_16x16x32_bf16 v[212:215], a[204:207], a[188:191], v[212:215]\n\t" \
    "v_mfma_f32_16x16x32_bf16 v[216:219], a[208:211], a[184:187], v[216:219]\n\t" \
    "v_mfma_f32_16x16x32_bf16 v[216:219], a[212:215], a[188:191], v[216:219]\n\t" \
    "v_mfma_f32_16x16x32_bf16 v[220:223], a[216:219], a[184:187], v[220:223]\n\t" \
    "v_mfma_f32_16x16x32_bf16 v[220:223], a[220:223], a[188:191], v[220:223]\n\t" \
    "s_waitcnt vmcnt(32) lgkmcnt(0)\n\ts_barrier\n\t" \
    "v_mfma_f32_16x16x32_bf16 a[16:19], a[224:227], a[128:131], a[16:19]\n\t" \
    "ds_read_b64_tr_b16 a[192:193], v22 offset:16384\n\t" \
    "v_mfma_f32_16x16x32_bf16 a[16:19], a[228:231], a[132:135], a[16:19]\n\t" \
    "buffer_load_dwordx4 v26, %[rs], %[so] offen offset:0 lds\n\t" \
    "v_accvgpr_read_b32 v32, a0\n\t" \
    "v_mfma_f32_16x16x32_bf16 a[20:23], a[232:235], a[128:131], a[20:23]\n\t" \
    "ds_read_b64_tr_b16 a[194:195], v22 offset:17408\n\t" \
    "buffer_load_dwordx4 v27, %[rs], %[so] offen offset:1024 lds\n\t" \
    "v_mfma_f32_16x16x32_bf16 a[20:23], a[236:239], a[132:135], a[20:23]\n\t" \
    "v_accvgpr_read_b32 v33, a1\n\t" \
    "v_accvgpr_read_b32 v34, a2\n\t" \
    "v_accvgpr_read_b32 v35, a3\n\t" \
    "v_mfma_f32_16x16x32_bf16 a[24:27], a[240:243], a[128:131], a[24:27]\n\t" \
    "ds_read_b64_tr_b16 a[196:197], v22 offset:24576\n\t" \
    "buffer_load_dwordx4 v28, %[rs], %[so] offen offset:2048 lds\n\t" \
    "v_mfma_f32_16x16x32_bf16 a[24:27], a[244:247], a[132:135], a[24:27]\n\t" \
    "buffer_load_dwordx4 v29, %[rs], %[so] offen offset:3072 lds\n\t" \
    "v_cvt_pk_bf16_f32 v248, v32, v33\n\t" \
    "v_mfma_f32_16x16x32_bf16 a[28:31], a[248:251], a[128:131], a[28:31]\n\t" \
    "ds_read_b64_tr_b16 a[198:199], v22 offset:25600\n\t" \
    "v_cvt_pk_bf16_f32 v249, v34, v35\n\t" \
    "v_accvgpr_read_b32 v32, a4\n\t" \
    "v_mfma_f32_16x16x32_bf16 a[28:31], a[252:255], a[132:135], a[28:31]\n\t" \
    "s_add_u32 m0, m0, 0x4000\n\t" \
    "v_accvgpr_read_b32 v33, a5\n\t" \
    "v_accvgpr_read_b32 v34, a6\n\t" \
    "v_mfma_f32_16x16x32_bf16 a[48:51], a[224:227], a[136:139], a[48:51]\n\t" \
    "ds_read_b128 a[128:131], v20 offset:0\n\t" \
    "s_add_u32 %[so], %[so], 0x200000\n\t" \
    "v_accvgpr_read_b32 v35, a7\n\t" \
    "v_mfma_f32_16x16x32_bf16 a[48:51], a[228:231], a[140:143], a[48:51]\n\t" \
    "v_cvt_pk_bf16_f32 v250, v32, v33\n\t" \
    "v_cvt_pk_bf16_f32 v251, v34, v35\n\t" \
    "v_mfma_f32_16x16x32_bf16 a[52:55], a[232:235], a[136:139], a[52:55]\n\t" \
    "ds_read_b128 a[132:135], v21 offset:0\n\t" \
    "buffer_load_dwordx4 v26, %[rs], %[so] offen offset:0 lds\n\t" \
    "v_mfma_f32_16x16x32_bf16 a[52:55], a[236:239], a[140:143], a[52:55]\n\t" \
    "buffer_load_dwordx4 v27, %[rs], %[so] offen offset:1024 lds\n\t" \
    "v_mfma_f32_16x16x32_bf16 a[56:59], a[240:243], a[136:139], a[56:59]\n\t" \
    "ds_read_b64_tr_b16 a[200:201], v23 offset:16384\n\t" \
    "s_xor_b32 %[t0], %[sd], 0x0\n\tbuffer_store_dwordx4 v[248:251], v30, %[rss], %[t0] offen " DAUXS "\n\t" \
    "v_mfma_f32_16x16x32_bf16 a[56:59], a[244:247], a[140:143], a[56:59]\n\t" \
    "buffer_load_dwordx4 v28, %[rs], %[so] offen offset:2048 lds\n\t" \
    "v_accvgpr_read_b32 v32, a8\n\t" \
    "v_mfma_f32_16x16x32_bf16 a[60:63], a[248:251], a[136:139], a[60:63]\n\t" \
    "ds_read_b64_tr_b16 a[202:203], v23 offset:17408\n\t" \
    "buffer_load_dwordx4 v29, %[rs], %[so] offen offset:3072 lds\n\t" \
    "v_mfma_f32_16x16x32_bf16 a[60:63], a[252:255], a[140:143], a[60:63]\n\t" \
    "v_accvgpr_read_b32 v33, a9\n\t" \
    "v_accvgpr_read_b32 v34, a10\n\t" \
    "v_accvgpr_read_b32 v35, a11\n\t" \
    "v_mfma_f32_16x16x32_bf16 v[64:67], a[224:227], a[144:147], v[64:67]\n\t" \
    "ds_read_b128 a[136:139], v20 offset:2048\n\t" \
    "s_xor_b32 m0, m0, 0x3f000\n\t" \
    "v_cvt_pk_bf16_f32 v252, v32, v33\n\t" \
    "v_mfma_f32_16x16x32_bf16 v[64:67], a[228:231], a[148:151], v[64:67]\n\t" \
    "s_add_u32 %[so], %[so], 0xffc80080\n\t" \
    "v_cvt_pk_bf16_f32 v253, v34, v35\n\t" \
    "v_accvgpr_read_b32 v32, a12\n\t" \
    "v_mfma_f32_16x16x32_bf16 v[68:71], a[232:235], a[144:147], v[68:71]\n\t" \
    "ds_read_b128 a[140:143], v21 offset:2048\n\t" \
    "v_accvgpr_read_b32 v33, a13\n\t" \
    "v_accvgpr_read_b32 v34, a14\n\t" \
    "v_mfma_f32_16x16x32_bf16 v[68:71], a[236:239], a[148:151], v[68:71]\n\t" \
    "s_cmp_eq_u32 %[rem], 3\n\ts_cselect_b32 %[so], %[nx], %[so]\n\t" \
    "v_accvgpr_read_b32 v35, a15\n\t" \
    "v_mfma_f32_16x16x32_bf16 v[72:75], a[240:243], a[144:147], v[72:75]\n\t" \
    "ds_read_b64_tr_b16 a[204:205], v23 offset:24576\n\t" \
    "v_cvt_pk_bf16_f32 v254, v32, v33\n\t" \
    "v_cvt_pk_bf16_f32 v255, v34, v35\n\t" \
    "v_mfma_f32_16x16x32_bf16 v[72:75], a[244:247], a[148:151], v[72:75]\n\t" \
    "s_xor_b32 %[t0], %[sd], 0x400\n\tbuffer_store_dwordx4 v[252:255], v30, %[rss], %[t0] offen " DAUXS "\n\t" \
    "v_accvgpr_read_b32 v32, a32\n\t" \
    "v_mfma_f32_16x16x32_bf16 v[76:79], a[248:251], a[144:147], v[76:79]\n\t" \
    "ds_read_b64_tr_b16 a[206:207], v23 offset:25600\n\t" \
    "v_accvgpr_read_b32 v33, a33\n\t" \
    "v_accvgpr_read_b32 v34, a34\n\t" \
    "v_mfma_f32_16x16x32_bf16 v[76:79], a[252:255], a[148:151], v[76:79]\n\t" \
    "v_accvgpr_read_b32 v35, a35\n\t" \
    "v_cvt_pk_bf16_f32 v248, v32, v33\n\t" \
    "v_cvt_pk_bf16_f32 v249, v34, v35\n\t" \
    "v_mfma_f32_16x16x32_bf16 v[96:99], a[224:227], a[152:155], v[96:99]\n\t" \
    "ds_read_b128 a[144:147], v20 offset:4096\n\t" \
    "v_accvgpr_read_b32 v32, a36\n\t" \
    "v_accvgpr_read_b32 v33, a37\n\t" \
    "v_mfma_f32_16x16x32_bf16 v[96:99], a[228:231], a[156:159], v[96:99]\n\t" \
    "v_accvgpr_read_b32 v34, a38\n\t" \
    "v_accvgpr_read_b32 v35, a39\n\t" \
    "v_cvt_pk_bf16_f32 v250, v32, v33\n\t" \
    "v_mfma_f32_16x16x32_bf16 v[100:103], a[232:235], a[152:155], v[100:103]\n\t" \
    "ds_read_b128 a[148:151], v21 offset:4096\n\t" \
    "v_cvt_pk_bf16_f32 v251, v34, v35\n\t" \
    "v_mfma_f32_16x16x32_bf16 v[100:103], a[236:239], a[156:159], v[100:103]\n\t" \
    "s_xor_b32 %[t0], %[sd], 0x800\n\tbuffer_store_dwordx4 v[248:251], v30, %[rss], %[t0] offen " DAUXS "\n\t" \
    "v_accvgpr_read_b32 v32, a40\n\t" \
    "v_mfma_f32_16x16x32_bf16 v[104:107], a[240:243], a[152:155], v[104:107]\n\t" \
    "ds_read_b64_tr_b16 a[208:209], v24 offset:16384\n\t" \
    "v_accvgpr_read_b32 v33, a41\n\t" \
    "v_accvgpr_read_b32 v34, a42\n\t" \
    "v_mfma_f32_16x16x32_bf16 v[104:107], a[244:247], a[156:159], v[104:107]\n\t" \
    "v_accvgpr_read_b32 v35, a43\n\t" \
    "v_cvt_pk_bf16_f32 v252, v32, v33\n\t" \
    "v_cvt_pk_bf16_f32 v253, v34, v35\n\t" \
    "v_mfma_f32_16x16x32_bf16 v[108:111], a[248:251], a[152:155], v[108:111]\n\t" \
    "ds_read_b64_tr_b16 a[210:211], v24 offset:17408\n\t" \
    "v_accvgpr_read_b32 v32, a44\n\t" \
    "v_accvgpr_read_b32 v33, a45\n\t" \
    "v_mfma_f32_16x16x32_bf16 v[108:111], a[252:255], a[156:159], v[108:111]\n\t" \
    "v_accvgpr_read_b32 v34, a46\n\t" \
    "v_accvgpr_read_b32 v35, a47\n\t" \
    "v_cvt_pk_bf16_f32 v254, v32, v33\n\t" \
    "v_mfma_f32_16x16x32_bf16 v[128:131], a[224:227], a[160:163], v[128:131]\n\t" \
    "ds_read_b128 a[152:155], v20 offset:6144\n\t" \
    "v_cvt_pk_bf16_f32 v255, v34, v35\n\t" \
    "v_mfma_f32_16x16x32_bf16 v[128:131], a[228:231], a[164:167], v[128:131]\n\t" \
    "s_xor_b32 %[t0], %[sd], 0xc00\n\tbuffer_store_dwordx4 v[252:255], v30, %[rss], %[t0] offen " DAUXS "\n\t" \
    "v_cvt_pk_bf16_f32 v248, v48, v49\n\t" \
    "v_mfma_f32_16x16x32_bf16 v[132:135], a[232:235], a[160:163], v[132:135]\n\t" \
    "ds_read_b128 a[156:159], v21 offset:6144\n\t" \
    "v_cvt_pk_bf16_f32 v249, v50, v51\n\t" \
    "v_cvt_pk_bf16_f32 v250, v52, v53\n\t" \
    "v_mfma_f32_16x16x32_bf16 v[132:135], a[236:239], a[164:167], v[132:135]\n\t" \
    "v_cvt_pk_bf16_f32 v251, v54, v55\n\t" \
    "s_xor_b32 %[t0], %[sd], 0x1000\n\tbuffer_store_dwordx4 v[248:251], v30, %[rss], %[t0] offen " DAUXS "\n\t" \
    "v_mfma_f32_16x16x32_bf16 v[136:139], a[240:243], a[160:163], v[136:139]\n\t" \
    "ds_read_b64_tr_b16 a[212:213], v24 offset:24576\n\t" \
    "v_cvt_pk_bf16_f32 v252, v56, v57\n\t" \
    "v_cvt_pk_bf16_f32 v253, v58, v59\n\t" \
    "v_mfma_f32_16x16x32_bf16 v[136:139], a[244:247], a[164:167], v[136:139]\n\t" \
    "v_cvt_pk_bf16_f32 v254, v60, v61\n\t" \
    "v_cvt_pk_bf16_f32 v255, v62, v63\n\t" \
    "v_mfma_f32_16x16x32_bf16 v[140:143], a[248:251], a[160:163], v[140:143]\n\t" \
    "ds_read_b64_tr_b16 a[214:215], v24 offset:25600\n\t" \
    "s_xor_b32 %[t0], %[sd], 0x1400\n\tbuffer_store_dwordx4 v[252:255], v30, %[rss], %[t0] offen " DAUXS "\n\t" \
    "v_mfma_f32_16x16x32_bf16 v[140:143], a[252:255], a[164:167], v[140:143]\n\t" \
    "v_cvt_pk_bf16_f32 v248, v80, v81\n\t" \
    "v_cvt_pk_bf16_f32 v249, v82, v83\n\t" \
    "v_cvt_pk_bf16_f32 v250, v84, v85\n\t" \
    "v_mfma_f32_16x16x32_bf16 v[160:163], a[224:227], a[168:171], v[160:163]\n\t" \
    "ds_read_b128 a[160:163], v20 offset:8192\n\t" \
    "v_cvt_pk_bf16_f32 v251, v86, v87\n\t" \
    "v_mfma_f32_16x16x32_bf16 v[160:163], a[228:231], a[172:175], v[160:163]\n\t" \
    "s_xor_b32 %[t0], %[sd], 0x1800\n\tbuffer_store_dwordx4 v[248:251], v30, %[rss], %[t0] offen " DAUXS "\n\t" \
    "v_cvt_pk_bf16_f32 v252, v88, v89\n\t" \
    "v_mfma_f32_16x16x32_bf16 v[164:167], a[232:235], a[168:171], v[164:167]\n\t" \
    "ds_read_b128 a[164:167], v21 offset:8192\n\t" \
    "v_cvt_pk_bf16_f32 v253, v90, v91\n\t" \
    "v_cvt_pk_bf16_f32 v254, v92, v93\n\t" \
    "v_mfma_f32_16x16x32_bf16 v[164:167], a[236:239], a[172:175], v[164:167]\n\t" \
    "v_cvt_pk_bf16_f32 v255, v94, v95\n\t" \
    "s_xor_b32 %[t0], %[sd], 0x1c00\n\tbuffer_store_dwordx4 v[252:255], v30, %[rss], %[t0] offen " DAUXS "\n\t" \
    "v_mfma_f32_16x16x32_bf16 v[168:171], a[240:243], a[168:171], v[168:171]\n\t" \
    "ds_read_b64_tr_b16 a[216:217], v25 offset:16384\n\t" \
    "v_cvt_pk_bf16_f32 v248, v112, v113\n\t" \
    "v_cvt_pk_bf16_f32 v249, v114, v115\n\t" \
    "v_mfma_f32_16x16x32_bf16 v[168:171], a[244:247], a[172:175], v[168:171]\n\t" \
    "v_cvt_pk_bf16_f32 v250, v116, v117\n\t" \
    "v_cvt_pk_bf16_f32 v251, v118, v119\n\t" \
    "v_mfma_f32_16x16x32_bf16 v[172:175], a[248:251], a[168:171], v[172:175]\n\t" \
    "ds_read_b64_tr_b16 a[218:219], v25 offset:17408\n\t" \
    "s_xor_b32 %[t0], %[sd], 0x2000\n\tbuffer_store_dwordx4 v[248:251], v30, %[rss], %[t0] offen " DAUXS "\n\t" \
    "v_mfma_f32_16x16x32_bf16 v[172:175], a[252:255], a[172:175], v[172:175]\n\t" \
    "v_cvt_pk_bf16_f32 v252, v120, v121\n\t" \
    "v_cvt_pk_bf16_f32 v253, v122, v123\n\t" \
    "v_cvt_pk_bf16_f32 v254, v124, v125\n\t" \
    "v_mfma_f32_16x16x32_bf16 v[192:195], a[224:227], a[176:179], v[192:195]\n\t" \
    "ds_read_b128 a[168:171], v20 offset:10240\n\t" \
    "v_cvt_pk_bf16_f32 v255, v126, v127\n\t" \
    "v_mfma_f32_16x16x32_bf16 v[192:195], a[228:231], a[180:183], v[192:195]\n\t" \
    "s_xor_b32 %[t0], %[sd], 0x2400\n\tbuffer_store_dwordx4 v[252:255], v30, %[rss], %[t0] offen " DAUXS "\n\t" \
    "v_cvt_pk_bf16_f32 v248, v144, v145\n\t" \
    "v_mfma_f32_16x16x32_bf16 v[196:199], a[232:235], a[176:179], v[196:199]\n\t" \
    "ds_read_b128 a[172:175], v21 offset:10240\n\t" \
    "v_cvt_pk_bf16_f32 v249, v146, v147\n\t" \
    "v_cvt_pk_bf16_f32 v250, v148, v149\n\t" \
    "v_mfma_f32_16x16x32_bf16 v[196:199], a[236:239], a[180:183], v[196:199]\n\t" \
    "v_cvt_pk_bf16_f32 v251, v150, v151\n\t" \
    "s_xor_b32 %[t0], %[sd], 0x2800\n\tbuffer_store_dwordx4 v[248:251], v30, %[rss], %[t0] offen " DAUXS "\n\t" \
    "v_mfma_f32_16x16x32_bf16 v[200:203], a[240:243], a[176:179], v[200:203]\n\t" \
    "ds_read_b64_tr_b16 a[220:221], v25 offset:24576\n\t" \
    "v_cvt_pk_bf16_f32 v252, v152, v153\n\t" \
    "v_cvt_pk_bf16_f32 v253, v154, v155\n\t" \
    "v_mfma_f32_16x16x32_bf16 v[200:203], a[244:247], a[180:183], v[200:203]\n\t" \
    "v_cvt_pk_bf16_f32 v254, v156, v157\n\t" \
    "v_cvt_pk_bf16_f32 v255, v158, v159\n\t" \
    "v_mfma_f32_16x16x32_bf16 v[204:207], a[248:251], a[176:179], v[204:207]\n\t" \
    "ds_read_b64_tr_b16 a[222:223], v25 offset:25600\n\t" \
    "s_xor_b32 %[t0], %[sd], 0x2c00\n\tbuffer_store_dwordx4 v[252:255], v30, %[rss], %[t0] offen " DAUXS "\n\t" \
    "v_mfma_f32_16x16x32_bf16 v[204:207], a[252:255], a[180:183], v[204:207]\n\t" \
    "v_cvt_pk_bf16_f32 v248, v176, v177\n\t" \
    "v_cvt_pk_bf16_f32 v249, v178, v179\n\t" \
    "v_cvt_pk_bf16_f32 v250, v180, v181\n\t" \
    "v_mfma_f32_16x16x32_bf16 v[224:227], a[224:227], a[184:187], v[224:227]\n\t" \
    "v_add_u32 v23, 0x8000, v22\n\t" \
    "v_add_u32 v22, 0xffff0000, v22\n\t" \
    "v_cvt_pk_bf16_f32 v251, v182, v183\n\t" \
    "v_mfma_f32_16x16x32_bf16 v[224:227], a[228:231], a[188:191], v[224:227]\n\t" \
    "v_min_u32_e64 v22, v22, v23\n\t" \
    "s_xor_b32 %[t0], %[sd], 0x3000\n\tbuffer_store_dwordx4 v[248:251], v30, %[rss], %[t0] offen " DAUXS "\n\t" \
    "v_mfma_f32_16x16x32_bf16 v[228:231], a[232:235], a[184:187], v[228:231]\n\t" \
    "v_xor_b32_e64 v23, 32, v22\n\t" \
    "v_xor_b32_e64 v24, 64, v22\n\t" \
    "v_cvt_pk_bf16_f32 v252, v184, v185\n\t" \
    "v_mfma_f32_16x16x32_bf16 v[228:231], a[236:239], a[188:191], v[228:231]\n\t" \
    "v_xor_b32 v25, 0x60, v22\n\t" \
    "v_cvt_pk_bf16_f32 v253, v186, v187\n\t" \
    "v_cvt_pk_bf16_f32 v254, v188, v189\n\t" \
    "v_mfma_f32_16x16x32_bf16 v[232:235], a[240:243], a[184:187], v[232:235]\n\t" \
    "v_cvt_pk_bf16_f32 v255, v190, v191\n\t" \
    "s_xor_b32 %[t0], %[sd], 0x3400\n\tbuffer_store_dwordx4 v[252:255], v30, %[rss], %[t0] offen " DAUXS "\n\t" \
    "v_mfma_f32_16x16x32_bf16 v[232:235], a[244:247], a[188:191], v[232:235]\n\t" \
    "v_cvt_pk_bf16_f32 v248, v208, v209\n\t" \
    "v_cvt_pk_bf16_f32 v249, v210, v211\n\t" \
    "v_cvt_pk_bf16_f32 v250, v212, v213\n\t" \
    "v_mfma_f32_16x16x32_bf16 v[236:239], a[248:251], a[184:187], v[236:239]\n\t" \
    "v_cvt_pk_bf16_f32 v251, v214, v215\n\t" \
    "s_xor_b32 %[t0], %[sd], 0x3800\n\tbuffer_store_dwordx4 v[248:251], v30, %[rss], %[t0] offen " DAUXS "\n\t" \
    "v_mfma_f32_16x16x32_bf16 v[236:239], a[252:255], a[188:191], v[236:239]\n\t" \
    "v_cvt_pk_bf16_f32 v252, v216, v217\n\t" \
    "v_cvt_pk_bf16_f32 v253, v218, v219\n\t" \
    "v_cvt_pk_bf16_f32 v254, v220, v221\n\t" \
    "v_cvt_pk_bf16_f32 v255, v222, v223\n\t" \
    "s_xor_b32 %[t0], %[sd], 0x3c00\n\tbuffer_store_dwordx4 v[252:255], v30, %[rss], %[t0] offen " DAUXS "\n\t" \
    "s_sub_u32 %[rem], %[rem], 1\n\t" \
    "s_waitcnt vmcnt(48) lgkmcnt(0)\n\ts_barrier\n\t" \
    ""
#define ASM_START_A \
    "v_mfma_f32_16x16x32_bf16 a[0:3], a[192:195], a[128:131], 0\n\t" \
    "ds_read_b128 a[176:179], v20 offset:12288\n\t" \
    "v_mfma_f32_16x16x32_bf16 a[0:3], a[196:199], a[132:135], a[0:3]\n\t" \
    "buffer_load_dwordx4 v26, %[rs], %[so] offen offset:0 lds\n\t" \
    "v_accvgpr_read_b32 v32, a16\n\t" \
    "v_mfma_f32_16x16x32_bf16 a[4:7], a[200:203], a[128:131], 0\n\t" \
    "ds_read_b128 a[180:183], v21 offset:12288\n\t" \
    "buffer_load_dwordx4 v27, %[rs], %[so] offen offset:1024 lds\n\t" \
    "v_mfma_f32_16x16x32_bf16 a[4:7], a[204:207], a[132:135], a[4:7]\n\t" \
    "buffer_load_dwordx4 v28, %[rs], %[so] offen offset:2048 lds\n\t" \
    "v_accvgpr_read_b32 v33, a17\n\t" \
    "v_mfma_f32_16x16x32_bf16 a[8:11], a[208:211], a[128:131], 0\n\t" \
    "ds_read_b128 a[184:187], v20 offset:14336\n\t" \
    "v_accvgpr_read_b32 v34, a18\n\t" \
    "v_accvgpr_read_b32 v35, a19\n\t" \
    "v_mfma_f32_16x16x32_bf16 a[8:11], a[212:215], a[132:135], a[8:11]\n\t" \
    "buffer_load_dwordx4 v29, %[rs], %[so] offen offset:3072 lds\n\t" \
    "v_cvt_pk_bf16_f32 v248, v32, v33\n\t" \
    "v_mfma_f32_16x16x32_bf16 a[12:15], a[216:219], a[128:131], 0\n\t" \
    "ds_read_b128 a[188:191], v21 offset:14336\n\t" \
    "s_add_u32 m0, m0, 0x1000\n\t" \
    "v_cvt_pk_bf16_f32 v249, v34, v35\n\t" \
    "v_mfma_f32_16x16x32_bf16 a[12:15], a[220:223], a[132:135], a[12:15]\n\t" \
    "v_accvgpr_read_b32 v32, a20\n\t" \
    "v_accvgpr_read_b32 v33, a21\n\t" \
    "v_accvgpr_read_b32 v34, a22\n\t" \
    "v_mfma_f32_16x16x32_bf16 a[32:35], a[192:195], a[136:139], 0\n\t" \
    "ds_read_b64_tr_b16 a[224:225], v22 offset:0\n\t" \
    "s_add_u32 %[so], %[so], 0x80000\n\t" \
    "v_accvgpr_read_b32 v35, a23\n\t" \
    "v_mfma_f32_16x16x32_bf16 a[32:35], a[196:199], a[140:143], a[32:35]\n\t" \
    "buffer_load_dwordx4 v26, %[rs], %[so] offen offset:0 lds\n\t" \
    "v_xor_b32 v20, 0x38000, v20\n\t" \
    "v_mfma_f32_16x16x32_bf16 a[36:39], a[200:203], a[136:139], 0\n\t" \
    "ds_read_b64_tr_b16 a[226:227], v22 offset:1024\n\t" \
    "v_xor_b32 v21, 0x38000, v21\n\t" \
    "v_cvt_pk_bf16_f32 v250, v32, v33\n\t" \
    "v_mfma_f32_16x16x32_bf16 a[36:39], a[204:207], a[140:143], a[36:39]\n\t" \
    "buffer_load_dwordx4 v27, %[rs], %[so] offen offset:1024 lds\n\t" \
    "v_cvt_pk_bf16_f32 v251, v34, v35\n\t" \
    "v_mfma_f32_16x16x32_bf16 a[40:43], a[208:211], a[136:139], 0\n\t" \
    "ds_read_b64_tr_b16 a[228:229], v22 offset:8192\n\t" \
    "buffer_load_dwordx4 v28, %[rs], %[so] offen offset:2048 lds\n\t" \
    "v_mfma_f32_16x16x32_bf16 a[40:43], a[212:215], a[140:143], a[40:43]\n\t" \
    "s_xor_b32 %[t0], %[sd], 0x4000\n\tbuffer_store_dwordx4 v[248:251], v30, %[rss], %[t0] offen " DAUXS "\n\t" \
    "v_accvgpr_read_b32 v32, a24\n\t" \
    "v_mfma_f32_16x16x32_bf16 a[44:47], a[216:219], a[136:139], 0\n\t" \
    "ds_read_b64_tr_b16 a[230:231], v22 offset:9216\n\t" \
    "buffer_load_dwordx4 v29, %[rs], %[so] offen offset:3072 lds\n\t" \
    "v_mfma_f32_16x16x32_bf16 a[44:47], a[220:223], a[140:143], a[44:47]\n\t" \
    "s_add_u32 m0, m0, 0x1000\n\t" \
    "v_accvgpr_read_b32 v33, a25\n\t" \
    "v_accvgpr_read_b32 v34, a26\n\t" \
    "v_mfma_f32_16x16x32_bf16 v[48:51], a[192:195], a[144:147], 0\n\t" \
    "ds_read_b64_tr_b16 a[232:233], v23 offset:0\n\t" \
    "s_add_u32 %[so], %[so], 0x80000\n\t" \
    "v_accvgpr_read_b32 v35, a27\n\t" \
    "v_mfma_f32_16x16x32_bf16 v[48:51], a[196:199], a[148:151], v[48:51]\n\t" \
    "v_cvt_pk_bf16_f32 v252, v32, v33\n\t" \
    "v_cvt_pk_bf16_f32 v253, v34, v35\n\t" \
    "v_accvgpr_read_b32 v32, a28\n\t" \
    "v_mfma_f32_16x16x32_bf16 v[52:55], a[200:203], a[144:147], 0\n\t" \
    "ds_read_b64_tr_b16 a[234:235], v23 offset:1024\n\t" \
    "buffer_load_dwordx4 v26, %[rs], %[so] offen offset:0 lds\n\t" \
    "v_mfma_f32_16x16x32_bf16 v[52:55], a[204:207], a[148:151], v[52:55]\n\t" \
    "buffer_load_dwordx4 v27, %[rs], %[so] offen offset:1024 lds\n\t" \
    "v_accvgpr_read_b32 v33, a29\n\t" \
    "v_mfma_f32_16x16x32_bf16 v[56:59], a[208:211], a[144:147], 0\n\t" \
    "ds_read_b64_tr_b16 a[236:237], v23 offset:8192\n\t" \
    "v_accvgpr_read_b32 v34, a30\n\t" \
    "v_accvgpr_read_b32 v35, a31\n\t" \
    "v_mfma_f32_16x16x32_bf16 v[56:59], a[212:215], a[148:151], v[56:59]\n\t" \
    "buffer_load_dwordx4 v28, %[rs], %[so] offen offset:2048 lds\n\t" \
    "v_cvt_pk_bf16_f32 v254, v32, v33\n\t" \
    "v_mfma_f32_16x16x32_bf16 v[60:63], a[216:219], a[144:147], 0\n\t" \
    "ds_read_b64_tr_b16 a[238:239], v23 offset:9216\n\t" \
    "buffer_load_dwordx4 v29, %[rs], %[so] offen offset:3072 lds\n\t" \
    "v_mfma_f32_16x16x32_bf16 v[60:63], a[220:223], a[148:151], v[60:63]\n\t" \
    "v_cvt_pk_bf16_f32 v255, v34, v35\n\t" \
    "s_xor_b32 %[t0], %[sd], 0x4400\n\tbuffer_store_dwordx4 v[252:255], v30, %[rss], %[t0] offen " DAUXS "\n\t" \
    "v_mfma_f32_16x16x32_bf16 v[80:83], a[192:195], a[152:155], 0\n\t" \
    "ds_read_b64_tr_b16 a[240:241], v24 offset:0\n\t" \
    "s_add_u32 m0, m0, 0x2000\n\t" \
    "v_accvgpr_read_b32 v32, a48\n\t" \
    "v_mfma_f32_16x16x32_bf16 v[80:83], a[196:199], a[156:159], v[80:83]\n\t" \
    "s_add_u32 %[so], %[so], 0x100000\n\t" \
    "v_accvgpr_read_b32 v33, a49\n\t" \
    "v_accvgpr_read_b32 v34, a50\n\t" \
    "v_mfma_f32_16x16x32_bf16 v[84:87], a[200:203], a[152:155], 0\n\t" \
    "ds_read_b64_tr_b16 a[242:243], v24 offset:1024\n\t" \
    "v_accvgpr_read_b32 v35, a51\n\t" \
    "v_cvt_pk_bf16_f32 v248, v32, v33\n\t" \
    "v_mfma_f32_16x16x32_bf16 v[84:87], a[204:207], a[156:159], v[84:87]\n\t" \
    "buffer_load_dwordx4 v26, %[rs], %[so] offen offset:0 lds\n\t" \
    "v_cvt_pk_bf16_f32 v249, v34, v35\n\t" \
    "v_mfma_f32_16x16x32_bf16 v[88:91], a[208:211], a[152:155], 0\n\t" \
    "ds_read_b64_tr_b16 a[244:245], v24 offset:8192\n\t" \
    "buffer_load_dwordx4 v27, %[rs], %[so] offen offset:1024 lds\n\t" \
    "v_mfma_f32_16x16x32_bf16 v[88:91], a[212:215], a[156:159], v[88:91]\n\t" \
    "buffer_load_dwordx4 v28, %[rs], %[so] offen offset:2048 lds\n\t" \
    "v_accvgpr_read_b32 v32, a52\n\t" \
    "v_mfma_f32_16x16x32_bf16 v[92:95], a[216:219], a[152:155], 0\n\t" \
    "ds_read_b64_tr_b16 a[246:247], v24 offset:9216\n\t" \
    "v_accvgpr_read_b32 v33, a53\n\t" \
    "v_accvgpr_read_b32 v34, a54\n\t" \
    "v_mfma_f32_16x16x32_bf16 v[92:95], a[220:223], a[156:159], v[92:95]\n\t" \
    "buffer_load_dwordx4 v29, %[rs], %[so] offen offset:3072 lds\n\t" \
    "v_accvgpr_read_b32 v35, a55\n\t" \
    "v_mfma_f32_16x16x32_bf16 v[112:115], a[192:195], a[160:163], 0\n\t" \
    "ds_read_b64_tr_b16 a[248:249], v25 offset:0\n\t" \
    "s_add_u32 m0, m0, 0x1000\n\t" \
    "v_cvt_pk_bf16_f32 v250, v32, v33\n\t" \
    "v_mfma_f32_16x16x32_bf16 v[112:115], a[196:199], a[164:167], v[112:115]\n\t" \
    "v_cvt_pk_bf16_f32 v251, v34, v35\n\t" \
    "s_xor_b32 %[t0], %[sd], 0x4800\n\tbuffer_store_dwordx4 v[248:251], v30, %[rss], %[t0] offen " DAUXS "\n\t" \
    "v_mfma_f32_16x16x32_bf16 v[116:119], a[200:203], a[160:163], 0\n\t" \
    "ds_read_b64_tr_b16 a[250:251], v25 offset:1024\n\t" \
    "s_add_u32 %[so], %[so], 0x80000\n\t" \
    "v_accvgpr_read_b32 v32, a56\n\t" \
    "v_mfma_f32_16x16x32_bf16 v[116:119], a[204:207], a[164:167], v[116:119]\n\t" \
    "buffer_load_dwordx4 v26, %[rs], %[so] offen offset:0 lds\n\t" \
    "v_accvgpr_read_b32 v33, a57\n\t" \
    "v_mfma_f32_16x16x32_bf16 v[120:123], a[208:211], a[160:163], 0\n\t" \
    "ds_read_b64_tr_b16 a[252:253], v25 offset:8192\n\t" \
    "v_accvgpr_read_b32 v34, a58\n\t" \
    "v_accvgpr_read_b32 v35, a59\n\t" \
    "v_mfma_f32_16x16x32_bf16 v[120:123], a[212:215], a[164:167], v[120:123]\n\t" \
    "buffer_load_dwordx4 v27, %[rs], %[so] offen offset:1024 lds\n\t" \
    "v_cvt_pk_bf16_f32 v252, v32, v33\n\t" \
    "v_mfma_f32_16x16x32_bf16 v[124:127], a[216:219], a[160:163], 0\n\t" \
    "ds_read_b64_tr_b16 a[254:255], v25 offset:9216\n\t" \
    "buffer_load_dwordx4 v28, %[rs], %[so] offen offset:2048 lds\n\t" \
    "v_mfma_f32_16x16x32_bf16 v[124:127], a[220:223], a[164:167], v[124:127]\n\t" \
    "v_cvt_pk_bf16_f32 v253, v34, v35\n\t" \
    "v_accvgpr_read_b32 v32, a60\n\t" \
    "v_accvgpr_read_b32 v33, a61\n\t" \
    "v_mfma_f32_16x16x32_bf16 v[144:147], a[192:195], a[168:171], 0\n\t" \
    "buffer_load_dwordx4 v29, %[rs], %[so] offen offset:3072 lds\n\t" \
    "v_accvgpr_read_b32 v34, a62\n\t" \
    "v_mfma_f32_16x16x32_bf16 v[144:147], a[196:199], a[172:175], v[144:147]\n\t" \
    "s_add_u32 m0, m0, 0x1000\n\t" \
    "v_accvgpr_read_b32 v35, a63\n\t" \
    "v_cvt_pk_bf16_f32 v254, v32, v33\n\t" \
    "v_mfma_f32_16x16x32_bf16 v[148:151], a[200:203], a[168:171], 0\n\t" \
    "s_add_u32 %[so], %[so], 0x80000\n\t" \
    "v_cvt_pk_bf16_f32 v255, v34, v35\n\t" \
    "v_mfma_f32_16x16x32_bf16 v[148:151], a[204:207], a[172:175], v[148:151]\n\t" \
    "s_xor_b32 %[t0], %[sd], 0x4c00\n\tbuffer_store_dwordx4 v[252:255], v30, %[rss], %[t0] offen " DAUXS "\n\t" \
    "v_cvt_pk_bf16_f32 v248, v64, v65\n\t" \
    "v_mfma_f32_16x16x32_bf16 v[152:155], a[208:211], a[168:171], 0\n\t" \
    "buffer_load_dwordx4 v26, %[rs], %[so] offen offset:0 lds\n\t" \
    "v_cvt_pk_bf16_f32 v249, v66, v67\n\t" \
    "v_mfma_f32_16x16x32_bf16 v[152:155], a[212:215], a[172:175], v[152:155]\n\t" \
    "buffer_load_dwordx4 v27, %[rs], %[so] offen offset:1024 lds\n\t" \
    "v_cvt_pk_bf16_f32 v250, v68, v69\n\t" \
    "v_mfma_f32_16x16x32_bf16 v[156:159], a[216:219], a[168:171], 0\n\t" \
    "v_cvt_pk_bf16_f32 v251, v70, v71\n\t" \
    "s_xor_b32 %[t0], %[sd], 0x5000\n\tbuffer_store_dwordx4 v[248:251], v30, %[rss], %[t0] offen " DAUXS "\n\t" \
    "v_mfma_f32_16x16x32_bf16 v[156:159], a[220:223], a[172:175], v[156:159]\n\t" \
    "s_waitcnt lgkmcnt(8)\n\ts_nop 0\n\t" \
    "buffer_load_dwordx4 v28, %[rs], %[so] offen offset:2048 lds\n\t" \
    "v_mfma_f32_16x16x32_bf16 v[176:179], a[192:195], a[176:179], 0\n\t" \
    "buffer_load_dwordx4 v29, %[rs], %[so] offen offset:3072 lds\n\t" \
    "v_cvt_pk_bf16_f32 v252, v72, v73\n\t" \
    "v_mfma_f32_16x16x32_bf16 v[176:179], a[196:199], a[180:183], v[176:179]\n\t" \
    "v_cvt_pk_bf16_f32 v253, v74, v75\n\t" \
    "v_cvt_pk_bf16_f32 v254, v76, v77\n\t" \
    "v_cvt_pk_bf16_f32 v255, v78, v79\n\t" \
    "v_mfma_f32_16x16x32_bf16 v[180:183], a[200:203], a[176:179], 0\n\t" \
    "s_add_u32 m0, m0, 0xffffd000\n\t" \
    "s_xor_b32 %[t0], %[sd], 0x5400\n\tbuffer_store_dwordx4 v[252:255], v30, %[rss], %[t0] offen " DAUXS "\n\t" \
    "v_mfma_f32_16x16x32_bf16 v[180:183], a[204:207], a[180:183], v[180:183]\n\t" \
    "s_add_u32 %[so], %[so], 0xffe80000\n\t" \
    "v_cvt_pk_bf16_f32 v248, v96, v97\n\t" \
    "v_cvt_pk_bf16_f32 v249, v98, v99\n\t" \
    "v_mfma_f32_16x16x32_bf16 v[184:187], a[208:211], a[176:179], 0\n\t" \
    "v_cvt_pk_bf16_f32 v250, v100, v101\n\t" \
    "v_cvt_pk_bf16_f32 v251, v102, v103\n\t" \
    "v_mfma_f32_16x16x32_bf16 v[184:187], a[212:215], a[180:183], v[184:187]\n\t" \
    "s_xor_b32 %[t0], %[sd], 0x5800\n\tbuffer_store_dwordx4 v[248:251], v30, %[rss], %[t0] offen " DAUXS "\n\t" \
    "v_cvt_pk_bf16_f32 v252, v104, v105\n\t" \
    "v_mfma_f32_16x16x32_bf16 v[188:191], a[216:219], a[176:179], 0\n\t" \
    "v_cvt_pk_bf16_f32 v253, v106, v107\n\t" \
    "v_cvt_pk_bf16_f32 v254, v108, v109\n\t" \
    "v_cvt_pk_bf16_f32 v255, v110, v111\n\t" \
    "v_mfma_f32_16x16x32_bf16 v[188:191], a[220:223], a[180:183], v[188:191]\n\t" \
    "s_xor_b32 %[t0], %[sd], 0x5c00\n\tbuffer_store_dwordx4 v[252:255], v30, %[rss], %[t0] offen " DAUXS "\n\t" \
    "v_cvt_pk_bf16_f32 v248, v128, v129\n\t" \
    "v_mfma_f32_16x16x32_bf16 v[208:211], a[192:195], a[184:187], 0\n\t" \
    "v_cvt_pk_bf16_f32 v249, v130, v131\n\t" \
    "v_cvt_pk_bf16_f32 v250, v132, v133\n\t" \
    "v_cvt_pk_bf16_f32 v251, v134, v135\n\t" \
    "v_mfma_f32_16x16x32_bf16 v[208:211], a[196:199], a[188:191], v[208:211]\n\t" \
    "s_xor_b32 %[t0], %[sd], 0x6000\n\tbuffer_store_dwordx4 v[248:251], v30, %[rss], %[t0] offen " DAUXS "\n\t" \
    "v_cvt_pk_bf16_f32 v252, v136, v137\n\t" \
    "v_mfma_f32_16x16x32_bf16 v[212:215], a[200:203], a[184:187], 0\n\t" \
    "v_cvt_pk_bf16_f32 v253, v138, v139\n\t" \
    "v_cvt_pk_bf16_f32 v254, v140, v141\n\t" \
    "v_cvt_pk_bf16_f32 v255, v142, v143\n\t" \
    "v_mfma_f32_16x16x32_bf16 v[212:215], a[204:207], a[188:191], v[212:215]\n\t" \
    "s_xor_b32 %[t0], %[sd], 0x6400\n\tbuffer_store_dwordx4 v[252:255], v30, %[rss], %[t0] offen " DAUXS "\n\t" \
    "v_cvt_pk_bf16_f32 v248, v160, v161\n\t" \
    "v_mfma_f32_16x16x32_bf16 v[216:219], a[208:211], a[184:187], 0\n\t" \
    "v_cvt_pk_bf16_f32 v249, v162, v163\n\t" \
    "v_cvt_pk_bf16_f32 v250, v164, v165\n\t" \
    "v_cvt_pk_bf16_f32 v251, v166, v167\n\t" \
    "v_mfma_f32_16x16x32_bf16 v[216:219], a[212:215], a[188:191], v[216:219]\n\t" \
    "s_xor_b32 %[t0], %[sd], 0x6800\n\tbuffer_store_dwordx4 v[248:251], v30, %[rss], %[t0] offen " DAUXS "\n\t" \
    "v_cvt_pk_bf16_f32 v252, v168, v169\n\t" \
    "v_mfma_f32_16x16x32_bf16 v[220:223], a[216:219], a[184:187], 0\n\t" \
    "v_cvt_pk_bf16_f32 v253, v170, v171\n\t" \
    "v_cvt_pk_bf16_f32 v254, v172, v173\n\t" \
    "v_cvt_pk_bf16_f32 v255, v174, v175\n\t" \
    "v_mfma_f32_16x16x32_bf16 v[220:223], a[220:223], a[188:191], v[220:223]\n\t" \
    "s_xor_b32 %[t0], %[sd], 0x6c00\n\tbuffer_store_dwordx4 v[252:255], v30, %[rss], %[t0] offen " DAUXS "\n\t" \
    "v_cvt_pk_bf16_f32 v248, v192, v193\n\t" \
    "v_cvt_pk_bf16_f32 v249, v194, v195\n\t" \
    "v_cvt_pk_bf16_f32 v250, v196, v197\n\t" \
    "v_cvt_pk_bf16_f32 v251, v198, v199\n\t" \
    "s_xor_b32 %[t0], %[sd], 0x7000\n\tbuffer_store_dwordx4 v[248:251], v30, %[rss], %[t0] offen " DAUXS "\n\t" \
    "v_cvt_pk_bf16_f32 v252, v200, v201\n\t" \
    "v_cvt_pk_bf16_f32 v253, v202, v203\n\t" \
    "v_cvt_pk_bf16_f32 v254, v204, v205\n\t" \
    "v_cvt_pk_bf16_f32 v255, v206, v207\n\t" \
    "s_xor_b32 %[t0], %[sd], 0x7400\n\tbuffer_store_dwordx4 v[252:255], v30, %[rss], %[t0] offen " DAUXS "\n\t" \
    "v_cvt_pk_bf16_f32 v248, v224, v225\n\t" \
    "v_cvt_pk_bf16_f32 v249, v226, v227\n\t" \
    "v_cvt_pk_bf16_f32 v250, v228, v229\n\t" \
    "v_cvt_pk_bf16_f32 v251, v230, v231\n\t" \
    "s_xor_b32 %[t0], %[sd], 0x7800\n\tbuffer_store_dwordx4 v[248:251], v30, %[rss], %[t0] offen " DAUXS "\n\t" \
    "v_cvt_pk_bf16_f32 v252, v232, v233\n\t" \
    "v_cvt_pk_bf16_f32 v253, v234, v235\n\t" \
    "v_cvt_pk_bf16_f32 v254, v236, v237\n\t" \
    "v_cvt_pk_bf16_f32 v255, v238, v239\n\t" \
    "s_xor_b32 %[t0], %[sd], 0x7c00\n\tbuffer_store_dwordx4 v[252:255], v30, %[rss], %[t0] offen " DAUXS "\n\t" \
    "s_waitcnt vmcnt(63) lgkmcnt(0)\n\ts_barrier\n\t" \
    "v_mfma_f32_16x16x32_bf16 a[16:19], a[224:227], a[128:131], 0\n\t" \
    "ds_read_b64_tr_b16 a[192:193], v22 offset:16384\n\t" \
    "v_mfma_f32_16x16x32_bf16 a[16:19], a[228:231], a[132:135], a[16:19]\n\t" \
    "buffer_load_dwordx4 v26, %[rs], %[so] offen offset:0 lds\n\t" \
    "v_mfma_f32_16x16x32_bf16 a[20:23], a[232:235], a[128:131], 0\n\t" \
    "ds_read_b64_tr_b16 a[194:195], v22 offset:17408\n\t" \
    "v_mfma_f32_16x16x32_bf16 a[20:23], a[236:239], a[132:135], a[20:23]\n\t" \
    "buffer_load_dwordx4 v27, %[rs], %[so] offen offset:1024 lds\n\t" \
    "v_mfma_f32_16x16x32_bf16 a[24:27], a[240:243], a[128:131], 0\n\t" \
    "ds_read_b64_tr_b16 a[196:197], v22 offset:24576\n\t" \
    "v_mfma_f32_16x16x32_bf16 a[24:27], a[244:247], a[132:135], a[24:27]\n\t" \
    "buffer_load_dwordx4 v28, %[rs], %[so] offen offset:2048 lds\n\t" \
    "v_mfma_f32_16x16x32_bf16 a[28:31], a[248:251], a[128:131], 0\n\t" \
    "ds_read_b64_tr_b16 a[198:199], v22 offset:25600\n\t" \
    "v_mfma_f32_16x16x32_bf16 a[28:31], a[252:255], a[132:135], a[28:31]\n\t" \
    "buffer_load_dwordx4 v29, %[rs], %[so] offen offset:3072 lds\n\t" \
    "v_mfma_f32_16x16x32_bf16 a[48:51], a[224:227], a[136:139], 0\n\t" \
    "ds_read_b128 a[128:131], v20 offset:0\n\t" \
    "s_add_u32 m0, m0, 0x4000\n\t" \
    "v_mfma_f32_16x16x32_bf16 a[48:51], a[228:231], a[140:143], a[48:51]\n\t" \
    "s_add_u32 %[so], %[so], 0x200000\n\t" \
    "v_mfma_f32_16x16x32_bf16 a[52:55], a[232:235], a[136:139], 0\n\t" \
    "ds_read_b128 a[132:135], v21 offset:0\n\t" \
    "v_mfma_f32_16x16x32_bf16 a[52:55], a[236:239], a[140:143], a[52:55]\n\t" \
    "buffer_load_dwordx4 v26, %[rs], %[so] offen offset:0 lds\n\t" \
    "v_mfma_f32_16x16x32_bf16 a[56:59], a[240:243], a[136:139], 0\n\t" \
    "ds_read_b64_tr_b16 a[200:201], v23 offset:16384\n\t" \
    "v_mfma_f32_16x16x32_bf16 a[56:59], a[244:247], a[140:143], a[56:59]\n\t" \
    "buffer_load_dwordx4 v27, %[rs], %[so] offen offset:1024 lds\n\t" \
    "v_mfma_f32_16x16x32_bf16 a[60:63], a[248:251], a[136:139], 0\n\t" \
    "ds_read_b64_tr_b16 a[202:203], v23 offset:17408\n\t" \
    "v_mfma_f32_16x16x32_bf16 a[60:63], a[252:255], a[140:143], a[60:63]\n\t" \
    "buffer_load_dwordx4 v28, %[rs], %[so] offen offset:2048 lds\n\t" \
    "v_mfma_f32_16x16x32_bf16 v[64:67], a[224:227], a[144:147], 0\n\t" \
    "ds_read_b128 a[136:139], v20 offset:2048\n\t" \
    "v_mfma_f32_16x16x32_bf16 v[64:67], a[228:231], a[148:151], v[64:67]\n\t" \
    "buffer_load_dwordx4 v29, %[rs], %[so] offen offset:3072 lds\n\t" \
    "v_mfma_f32_16x16x32_bf16 v[68:71], a[232:235], a[144:147], 0\n\t" \
    "ds_read_b128 a[140:143], v21 offset:2048\n\t" \
    "s_xor_b32 m0, m0, 0x3f000\n\t" \
    "v_mfma_f32_16x16x32_bf16 v[68:71], a[236:239], a[148:151], v[68:71]\n\t" \
    "s_add_u32 %[so], %[so], 0xffc80080\n\t" \
    "v_mfma_f32_16x16x32_bf16 v[72:75], a[240:243], a[144:147], 0\n\t" \
    "ds_read_b64_tr_b16 a[204:205], v23 offset:24576\n\t" \
    "v_mfma_f32_16x16x32_bf16 v[72:75], a[244:247], a[148:151], v[72:75]\n\t" \
    "s_cmp_eq_u32 %[rem], 3\n\ts_cselect_b32 %[so], %[nx], %[so]\n\t" \
    "v_mfma_f32_16x16x32_bf16 v[76:79], a[248:251], a[144:147], 0\n\t" \
    "ds_read_b64_tr_b16 a[206:207], v23 offset:25600\n\t" \
    "v_mfma_f32_16x16x32_bf16 v[76:79], a[252:255], a[148:151], v[76:79]\n\t" \
    "v_mfma_f32_16x16x32_bf16 v[96:99], a[224:227], a[152:155], 0\n\t" \
    "ds_read_b128 a[144:147], v20 offset:4096\n\t" \
    "v_mfma_f32_16x16x32_bf16 v[96:99], a[228:231], a[156:159], v[96:99]\n\t" \
    "v_mfma_f32_16x16x32_bf16 v[100:103], a[232:235], a[152:155], 0\n\t" \
    "ds_read_b128 a[148:151], v21 offset:4096\n\t" \
    "v_mfma_f32_16x16x32_bf16 v[100:103], a[236:239], a[156:159], v[100:103]\n\t" \
    "v_mfma_f32_16x16x32_bf16 v[104:107], a[240:243], a[152:155], 0\n\t" \
    "ds_read_b64_tr_b16 a[208:209], v24 offset:16384\n\t" \
    "v_mfma_f32_16x16x32_bf16 v[104:107], a[244:247], a[156:159], v[104:107]\n\t" \
    "v_mfma_f32_16x16x32_bf16 v[108:111], a[248:251], a[152:155], 0\n\t" \
    "ds_read_b64_tr_b16 a[210:211], v24 offset:17408\n\t" \
    "v_mfma_f32_16x16x32_bf16 v[108:111], a[252:255], a[156:159], v[108:111]\n\t" \
    "v_mfma_f32_16x16x32_bf16 v[128:131], a[224:227], a[160:163], 0\n\t" \
    "ds_read_b128 a[152:155], v20 offset:6144\n\t" \
    "v_mfma_f32_16x16x32_bf16 v[128:131], a[228:231], a[164:167], v[128:131]\n\t" \
    "v_mfma_f32_16x16x32_bf16 v[132:135], a[232:235], a[160:163], 0\n\t" \
    "ds_read_b128 a[156:159], v21 offset:6144\n\t" \
    "v_mfma_f32_16x16x32_bf16 v[132:135], a[236:239], a[164:167], v[132:135]\n\t" \
    "v_mfma_f32_16x16x32_bf16 v[136:139], a[240:243], a[160:163], 0\n\t" \
    "ds_read_b64_tr_b16 a[212:213], v24 offset:24576\n\t" \
    "v_mfma_f32_16x16x32_bf16 v[136:139], a[244:247], a[164:167], v[136:139]\n\t" \
    "v_mfma_f32_16x16x32_bf16 v[140:143], a[248:251], a[160:163], 0\n\t" \
    "ds_read_b64_tr_b16 a[214:215], v24 offset:25600\n\t" \
    "v_mfma_f32_16x16x32_bf16 v[140:143], a[252:255], a[164:167], v[140:143]\n\t" \
    "v_mfma_f32_16x16x32_bf16 v[160:163], a[224:227], a[168:171], 0\n\t" \
    "ds_read_b128 a[160:163], v20 offset:8192\n\t" \
    "v_mfma_f32_16x16x32_bf16 v[160:163], a[228:231], a[172:175], v[160:163]\n\t" \
    "v_mfma_f32_16x16x32_bf16 v[164:167], a[232:235], a[168:171], 0\n\t" \
    "ds_read_b128 a[164:167], v21 offset:8192\n\t" \
    "v_mfma_f32_16x16x32_bf16 v[164:167], a[236:239], a[172:175], v[164:167]\n\t" \
    "v_mfma_f32_16x16x32_bf16 v[168:171], a[240:243], a[168:171], 0\n\t" \
    "ds_read_b64_tr_b16 a[216:217], v25 offset:16384\n\t" \
    "v_mfma_f32_16x16x32_bf16 v[168:171], a[244:247], a[172:175], v[168:171]\n\t" \
    "v_mfma_f32_16x16x32_bf16 v[172:175], a[248:251], a[168:171], 0\n\t" \
    "ds_read_b64_tr_b16 a[218:219], v25 offset:17408\n\t" \
    "v_mfma_f32_16x16x32_bf16 v[172:175], a[252:255], a[172:175], v[172:175]\n\t" \
    "v_mfma_f32_16x16x32_bf16 v[192:195], a[224:227], a[176:179], 0\n\t" \
    "ds_read_b128 a[168:171], v20 offset:10240\n\t" \
    "v_mfma_f32_16x16x32_bf16 v[192:195], a[228:231], a[180:183], v[192:195]\n\t" \
    "v_mfma_f32_16x16x32_bf16 v[196:199], a[232:235], a[176:179], 0\n\t" \
    "ds_read_b128 a[172:175], v21 offset:10240\n\t" \
    "v_mfma_f32_16x16x32_bf16 v[196:199], a[236:239], a[180:183], v[196:199]\n\t" \
    "v_mfma_f32_16x16x32_bf16 v[200:203], a[240:243], a[176:179], 0\n\t" \
    "ds_read_b64_tr_b16 a[220:221], v25 offset:24576\n\t" \
    "v_mfma_f32_16x16x32_bf16 v[200:203], a[244:247], a[180:183], v[200:203]\n\t" \
    "v_mfma_f32_16x16x32_bf16 v[204:207], a[248:251], a[176:179], 0\n\t" \
    "ds_read_b64_tr_b16 a[222:223], v25 offset:25600\n\t" \
    "v_mfma_f32_16x16x32_bf16 v[204:207], a[252:255], a[180:183], v[204:207]\n\t" \
    "v_mfma_f32_16x16x32_bf16 v[224:227], a[224:227], a[184:187], 0\n\t" \
    "v_add_u32 v23, 0x8000, v22\n\t" \
    "v_add_u32 v22, 0xffff0000, v22\n\t" \
    "v_mfma_f32_16x16x32_bf16 v[224:227], a[228:231], a[188:191], v[224:227]\n\t" \
    "v_min_u32_e64 v22, v22, v23\n\t" \
    "v_mfma_f32_16x16x32_bf16 v[228:231], a[232:235], a[184:187], 0\n\t" \
    "v_xor_b32_e64 v23, 32, v22\n\t" \
    "v_xor_b32_e64 v24, 64, v22\n\t" \
    "v_mfma_f32_16x16x32_bf16 v[228:231], a[236:239], a[188:191], v[228:231]\n\t" \
    "v_xor_b32 v25, 0x60, v22\n\t" \
    "v_mfma_f32_16x16x32_bf16 v[232:235], a[240:243], a[184:187], 0\n\t" \
    "v_mfma_f32_16x16x32_bf16 v[232:235], a[244:247], a[188:191], v[232:235]\n\t" \
    "v_mfma_f32_16x16x32_bf16 v[236:239], a[248:251], a[184:187], 0\n\t" \
    "v_mfma_f32_16x16x32_bf16 v[236:239], a[252:255], a[188:191], v[236:239]\n\t" \
    "s_sub_u32 %[rem], %[rem], 1\n\t" \
    "s_waitcnt vmcnt(48) lgkmcnt(0)\n\ts_barrier\n\t" \
    ""
#define ASM_PLAIN_B \
    ".p2align 6\n\t" \
    "1:\n\t" \
    "v_mfma_f32_16x16x32_bf16 a[0:3], a[192:195], a[128:131], a[0:3]\n\t" \
    "ds_read_b128 a[176:179], v20 offset:12288\n\t" \
    "v_mfma_f32_16x16x32_bf16 a[0:3], a[196:199], a[132:135], a[0:3]\n\t" \
    "buffer_load_dwordx4 v26, %[rs], %[so] offen offset:0 lds\n\t" \
    "v_mfma_f32_16x16x32_bf16 a[4:7], a[200:203], a[128:131], a[4:7]\n\t" \
    "ds_read_b128 a[180:183], v21 offset:12288\n\t" \
    "v_mfma_f32_16x16x32_bf16 a[4:7], a[204:207], a[132:135], a[4:7]\n\t" \
    "buffer_load_dwordx4 v27, %[rs], %[so] offen offset:1024 lds\n\t" \
    "v_mfma_f32_16x16x32_bf16 a[8:11], a[208:211], a[128:131], a[8:11]\n\t" \
    "ds_read_b128 a[184:187], v20 offset:14336\n\t" \
    "v_mfma_f32_16x16x32_bf16 a[8:11], a[212:215], a[132:135], a[8:11]\n\t" \
    "buffer_load_dwordx4 v28, %[rs], %[so] offen offset:2048 lds\n\t" \
    "v_mfma_f32_16x16x32_bf16 a[12:15], a[216:219], a[128:131], a[12:15]\n\t" \
    "ds_read_b128 a[188:191], v21 offset:14336\n\t" \
    "v_mfma_f32_16x16x32_bf16 a[12:15], a[220:223], a[132:135], a[12:15]\n\t" \
    "v_mfma_f32_16x16x32_bf16 a[32:35], a[192:195], a[136:139], a[32:35]\n\t" \
    "ds_read_b64_tr_b16 a[224:225], v22 offset:0\n\t" \
    "v_mfma_f32_16x16x32_bf16 a[32:35], a[196:199], a[140:143], a[32:35]\n\t" \
    "buffer_load_dwordx4 v29, %[rs], %[so] offen offset:3072 lds\n\t" \
    "v_mfma_f32_16x16x32_bf16 a[36:39], a[200:203], a[136:139], a[36:39]\n\t" \
    "ds_read_b64_tr_b16 a[226:227], v22 offset:1024\n\t" \
    "s_add_u32 m0, m0, 0x1000\n\t" \
    "v_mfma_f32_16x16x32_bf16 a[36:39], a[204:207], a[140:143], a[36:39]\n\t" \
    "v_xor_b32 v20, 0x38000, v20\n\t" \
    "v_xor_b32 v21, 0x38000, v21\n\t" \
    "v_mfma_f32_16x16x32_bf16 a[40:43], a[208:211], a[136:139], a[40:43]\n\t" \
    "ds_read_b64_tr_b16 a[228:229], v22 offset:8192\n\t" \
    "v_mfma_f32_16x16x32_bf16 a[40:43], a[212:215], a[140:143], a[40:43]\n\t" \
    "s_add_u32 %[so], %[so], 0x40000\n\t" \
    "v_mfma_f32_16x16x32_bf16 a[44:47], a[216:219], a[136:139], a[44:47]\n\t" \
    "ds_read_b64_tr_b16 a[230:231], v22 offset:9216\n\t" \
    "v_mfma_f32_16x16x32_bf16 a[44:47], a[220:223], a[140:143], a[44:47]\n\t" \
    "buffer_load_dwordx4 v26, %[rs], %[so] offen offset:0 lds\n\t" \
    "v_mfma_f32_16x16x32_bf16 v[48:51], a[192:195], a[144:147], v[48:51]\n\t" \
    "ds_read_b64_tr_b16 a[232:233], v23 offset:0\n\t" \
    "v_mfma_f32_16x16x32_bf16 v[48:51], a[196:199], a[148:151], v[48:51]\n\t" \
    "v_mfma_f32_16x16x32_bf16 v[52:55], a[200:203], a[144:147], v[52:55]\n\t" \
    "ds_read_b64_tr_b16 a[234:235], v23 offset:1024\n\t" \
    "v_mfma_f32_16x16x32_bf16 v[52:55], a[204:207], a[148:151], v[52:55]\n\t" \
    "buffer_load_dwordx4 v27, %[rs], %[so] offen offset:1024 lds\n\t" \
    "v_mfma_f32_16x16x32_bf16 v[56:59], a[208:211], a[144:147], v[56:59]\n\t" \
    "ds_read_b64_tr_b16 a[236:237], v23 offset:8192\n\t" \
    "v_mfma_f32_16x16x32_bf16 v[56:59], a[212:215], a[148:151], v[56:59]\n\t" \
    "buffer_load_dwordx4 v28, %[rs], %[so] offen offset:2048 lds\n\t" \
    "v_mfma_f32_16x16x32_bf16 v[60:63], a[216:219], a[144:147], v[60:63]\n\t" \
    "ds_read_b64_tr_b16 a[238:239], v23 offset:9216\n\t" \
    "v_mfma_f32_16x16x32_bf16 v[60:63], a[220:223], a[148:151], v[60:63]\n\t" \
    "buffer_load_dwordx4 v29, %[rs], %[so] offen offset:3072 lds\n\t" \
    "v_mfma_f32_16x16x32_bf16 v[80:83], a[192:195], a[152:155], v[80:83]\n\t" \
    "ds_read_b64_tr_b16 a[240:241], v24 offset:0\n\t" \
    "v_mfma_f32_16x16x32_bf16 v[80:83], a[196:199], a[156:159], v[80:83]\n\t" \
    "s_add_u32 m0, m0, 0x1000\n\t" \
    "v_mfma_f32_16x16x32_bf16 v[84:87], a[200:203], a[152:155], v[84:87]\n\t" \
    "ds_read_b64_tr_b16 a[242:243], v24 offset:1024\n\t" \
    "v_mfma_f32_16x16x32_bf16 v[84:87], a[204:207], a[156:159], v[84:87]\n\t" \
    "s_add_u32 %[so], %[so], 0x40000\n\t" \
    "v_mfma_f32_16x16x32_bf16 v[88:91], a[208:211], a[152:155], v[88:91]\n\t" \
    "ds_read_b64_tr_b16 a[244:245], v24 offset:8192\n\t" \
    "v_mfma_f32_16x16x32_bf16 v[88:91], a[212:215], a[156:159], v[88:91]\n\t" \
    "v_mfma_f32_16x16x32_bf16 v[92:95], a[216:219], a[152:155], v[92:95]\n\t" \
    "ds_read_b64_tr_b16 a[246:247], v24 offset:9216\n\t" \
    "v_mfma_f32_16x16x32_bf16 v[92:95], a[220:223], a[156:159], v[92:95]\n\t" \
    "buffer_load_dwordx4 v26, %[rs], %[so] offen offset:0 lds\n\t" \
    "v_mfma_f32_16x16x32_bf16 v[112:115], a[192:195], a[160:163], v[112:115]\n\t" \
    "ds_read_b64_tr_b16 a[248:249], v25 offset:0\n\t" \
    "v_mfma_f32_16x16x32_bf16 v[112:115], a[196:199], a[164:167], v[112:115]\n\t" \
    "buffer_load_dwordx4 v27, %[rs], %[so] offen offset:1024 lds\n\t" \
    "v_mfma_f32_16x16x32_bf16 v[116:119], a[200:203], a[160:163], v[116:119]\n\t" \
    "ds_read_b64_tr_b16 a[250:251], v25 offset:1024\n\t" \
    "v_mfma_f32_16x16x32_bf16 v[116:119], a[204:207], a[164:167], v[116:119]\n\t" \
    "buffer_load_dwordx4 v28, %[rs], %[so] offen offset:2048 lds\n\t" \
    "v_mfma_f32_16x16x32_bf16 v[120:123], a[208:211], a[160:163], v[120:123]\n\t" \
    "ds_read_b64_tr_b16 a[252:253], v25 offset:8192\n\t" \
    "v_mfma_f32_16x16x32_bf16 v[120:123], a[212:215], a[164:167], v[120:123]\n\t" \
    "buffer_load_dwordx4 v29, %[rs], %[so] offen offset:3072 lds\n\t" \
    "v_mfma_f32_16x16x32_bf16 v[124:127], a[216:219], a[160:163], v[124:127]\n\t" \
    "ds_read_b64_tr_b16 a[254:255], v25 offset:9216\n\t" \
    "v_mfma_f32_16x16x32_bf16 v[124:127], a[220:223], a[164:167], v[124:127]\n\t" \
    "v_mfma_f32_16x16x32_bf16 v[144:147], a[192:195], a[168:171], v[144:147]\n\t" \
    "s_add_u32 m0, m0, 0x1000\n\t" \
    "v_mfma_f32_16x16x32_bf16 v[144:147], a[196:199], a[172:175], v[144:147]\n\t" \
    "v_mfma_f32_16x16x32_bf16 v[148:151], a[200:203], a[168:171], v[148:151]\n\t" \
    "s_add_u32 %[so], %[so], 0x40000\n\t" \
    "v_mfma_f32_16x16x32_bf16 v[148:151], a[204:207], a[172:175], v[148:151]\n\t" \
    "v_mfma_f32_16x16x32_bf16 v[152:155], a[208:211], a[168:171], v[152:155]\n\t" \
    "buffer_load_dwordx4 v26, %[rs], %[so] offen offset:0 lds\n\t" \
    "v_mfma_f32_16x16x32_bf16 v[152:155], a[212:215], a[172:175], v[152:155]\n\t" \
    "v_mfma_f32_16x16x32_bf16 v[156:159], a[216:219], a[168:171], v[156:159]\n\t" \
    "v_mfma_f32_16x16x32_bf16 v[156:159], a[220:223], a[172:175], v[156:159]\n\t" \
    "s_waitcnt lgkmcnt(8)\n\ts_nop 0\n\t" \
    "v_mfma_f32_16x16x32_bf16 v[176:179], a[192:195], a[176:179], v[176:179]\n\t" \
    "buffer_load_dwordx4 v27, %[rs], %[so] offen offset:1024 lds\n\t" \
    "v_mfma_f32_16x16x32_bf16 v[176:179], a[196:199], a[180:183], v[176:179]\n\t" \
    "buffer_load_dwordx4 v28, %[rs], %[so] offen offset:2048 lds\n\t" \
    "v_mfma_f32_16x16x32_bf16 v[180:183], a[200:203], a[176:179], v[180:183]\n\t" \
    "v_mfma_f32_16x16x32_bf16 v[180:183], a[204:207], a[180:183], v[180:183]\n\t" \
    "v_mfma_f32_16x16x32_bf16 v[184:187], a[208:211], a[176:179], v[184:187]\n\t" \
    "buffer_load_dwordx4 v29, %[rs], %[so] offen offset:3072 lds\n\t" \
    "v_mfma_f32_16x16x32_bf16 v[184:187], a[212:215], a[180:183], v[184:187]\n\t" \
    "v_mfma_f32_16x16x32_bf16 v[188:191], a[216:219], a[176:179], v[188:191]\n\t" \
    "s_add_u32 m0, m0, 0x1000\n\t" \
    "v_mfma_f32_16x16x32_bf16 v[188:191], a[220:223], a[180:183], v[188:191]\n\t" \
    "v_mfma_f32_16x16x32_bf16 v[208:211], a[192:195], a[184:187], v[208:211]\n\t" \
    "v_mfma_f32_16x16x32_bf16 v[208:211], a[196:199], a[188:191], v[208:211]\n\t" \
    "s_cmp_eq_u32 m0, 0x18000\n\ts_cselect_b32 m0, 0, m0\n\ts_nop 0\n\t" \
    "v_mfma_f32_16x16x32_bf16 v[212:215], a[200:203], a[184:187], v[212:215]\n\t" \
    "v_mfma_f32_16x16x32_bf16 v[212:215], a[204:207], a[188:191], v[212:215]\n\t" \
    "s_add_u32 %[so], %[so], 0xfff40080\n\t" \
    "v_mfma_f32_16x16x32_bf16 v[216:219], a[208:211], a[184:187], v[216:219]\n\t" \
    "v_mfma_f32_16x16x32_bf16 v[216:219], a[212:215], a[188:191], v[216:219]\n\t" \
    "v_mfma_f32_16x16x32_bf16 v[220:223], a[216:219], a[184:187], v[220:223]\n\t" \
    "v_mfma_f32_16x16x32_bf16 v[220:223], a[220:223], a[188:191], v[220:223]\n\t" \
    "s_waitcnt vmcnt(48) lgkmcnt(0)\n\ts_barrier\n\t" \
    "v_mfma_f32_16x16x32_bf16 a[16:19], a[224:227], a[128:131], a[16:19]\n\t" \
    "ds_read_b64_tr_b16 a[192:193], v22 offset:16384\n\t" \
    "v_mfma_f32_16x16x32_bf16 a[16:19], a[228:231], a[132:135], a[16:19]\n\t" \
    "buffer_load_dwordx4 v26, %[rs], %[so] offen offset:0 lds\n\t" \
    "v_mfma_f32_16x16x32_bf16 a[20:23], a[232:235], a[128:131], a[20:23]\n\t" \
    "ds_read_b64_tr_b16 a[194:195], v22 offset:17408\n\t" \
    "v_mfma_f32_16x16x32_bf16 a[20:23], a[236:239], a[132:135], a[20:23]\n\t" \
    "buffer_load_dwordx4 v27, %[rs], %[so] offen offset:1024 lds\n\t" \
    "v_mfma_f32_16x16x32_bf16 a[24:27], a[240:243], a[128:131], a[24:27]\n\t" \
    "ds_read_b64_tr_b16 a[196:197], v22 offset:24576\n\t" \
    "v_mfma_f32_16x16x32_bf16 a[24:27], a[244:247], a[132:135], a[24:27]\n\t" \
    "buffer_load_dwordx4 v28, %[rs], %[so] offen offset:2048 lds\n\t" \
    "v_mfma_f32_16x16x32_bf16 a[28:31], a[248:251], a[128:131], a[28:31]\n\t" \
    "ds_read_b64_tr_b16 a[198:199], v22 offset:25600\n\t" \
    "v_mfma_f32_16x16x32_bf16 a[28:31], a[252:255], a[132:135], a[28:31]\n\t" \
    "v_mfma_f32_16x16x32_bf16 a[48:51], a[224:227], a[136:139], a[48:51]\n\t" \
    "ds_read_b128 a[128:131], v20 offset:0\n\t" \
    "v_mfma_f32_16x16x32_bf16 a[48:51], a[228:231], a[140:143], a[48:51]\n\t" \
    "buffer_load_dwordx4 v29, %[rs], %[so] offen offset:3072 lds\n\t" \
    "v_mfma_f32_16x16x32_bf16 a[52:55], a[232:235], a[136:139], a[52:55]\n\t" \
    "ds_read_b128 a[132:135], v21 offset:0\n\t" \
    "s_add_u32 m0, m0, 0x1000\n\t" \
    "v_mfma_f32_16x16x32_bf16 a[52:55], a[236:239], a[140:143], a[52:55]\n\t" \
    "v_mfma_f32_16x16x32_bf16 a[56:59], a[240:243], a[136:139], a[56:59]\n\t" \
    "ds_read_b64_tr_b16 a[200:201], v23 offset:16384\n\t" \
    "v_mfma_f32_16x16x32_bf16 a[56:59], a[244:247], a[140:143], a[56:59]\n\t" \
    "s_add_u32 %[so], %[so], 0x40000\n\t" \
    "v_mfma_f32_16x16x32_bf16 a[60:63], a[248:251], a[136:139], a[60:63]\n\t" \
    "ds_read_b64_tr_b16 a[202:203], v23 offset:17408\n\t" \
    "v_mfma_f32_16x16x32_bf16 a[60:63], a[252:255], a[140:143], a[60:63]\n\t" \
    "buffer_load_dwordx4 v26, %[rs], %[so] offen offset:0 lds\n\t" \
    "v_mfma_f32_16x16x32_bf16 v[64:67], a[224:227], a[144:147], v[64:67]\n\t" \
    "ds_read_b128 a[136:139], v20 offset:2048\n\t" \
    "v_mfma_f32_16x16x32_bf16 v[64:67], a[228:231], a[148:151], v[64:67]\n\t" \
    "v_mfma_f32_16x16x32_bf16 v[68:71], a[232:235], a[144:147], v[68:71]\n\t" \
    "ds_read_b128 a[140:143], v21 offset:2048\n\t" \
    "v_mfma_f32_16x16x32_bf16 v[68:71], a[236:239], a[148:151], v[68:71]\n\t" \
    "buffer_load_dwordx4 v27, %[rs], %[so] offen offset:1024 lds\n\t" \
    "v_mfma_f32_16x16x32_bf16 v[72:75], a[240:243], a[144:147], v[72:75]\n\t" \
    "ds_read_b64_tr_b16 a[204:205], v23 offset:24576\n\t" \
    "v_mfma_f32_16x16x32_bf16 v[72:75], a[244:247], a[148:151], v[72:75]\n\t" \
    "buffer_load_dwordx4 v28, %[rs], %[so] offen offset:2048 lds\n\t" \
    "v_mfma_f32_16x16x32_bf16 v[76:79], a[248:251], a[144:147], v[76:79]\n\t" \
    "ds_read_b64_tr_b16 a[206:207], v23 offset:25600\n\t" \
    "v_mfma_f32_16x16x32_bf16 v[76:79], a[252:255], a[148:151], v[76:79]\n\t" \
    "buffer_load_dwordx4 v29, %[rs], %[so] offen offset:3072 lds\n\t" \
    "v_mfma_f32_16x16x32_bf16 v[96:99], a[224:227], a[152:155], v[96:99]\n\t" \
    "ds_read_b128 a[144:147], v20 offset:4096\n\t" \
    "v_mfma_f32_16x16x32_bf16 v[96:99], a[228:231], a[156:159], v[96:99]\n\t" \
    "s_add_u32 m0, m0, 0x1000\n\t" \
    "v_mfma_f32_16x16x32_bf16 v[100:103], a[232:235], a[152:155], v[100:103]\n\t" \
    "ds_read_b128 a[148:151], v21 offset:4096\n\t" \
    "v_mfma_f32_16x16x32_bf16 v[100:103], a[236:239], a[156:159], v[100:103]\n\t" \
    "s_add_u32 %[so], %[so], 0x40000\n\t" \
    "v_mfma_f32_16x16x32_bf16 v[104:107], a[240:243], a[152:155], v[104:107]\n\t" \
    "ds_read_b64_tr_b16 a[208:209], v24 offset:16384\n\t" \
    "v_mfma_f32_16x16x32_bf16 v[104:107], a[244:247], a[156:159], v[104:107]\n\t" \
    "v_mfma_f32_16x16x32_bf16 v[108:111], a[248:251], a[152:155], v[108:111]\n\t" \
    "ds_read_b64_tr_b16 a[210:211], v24 offset:17408\n\t" \
    "v_mfma_f32_16x16x32_bf16 v[108:111], a[252:255], a[156:159], v[108:111]\n\t" \
    "buffer_load_dwordx4 v26, %[rs], %[so] offen offset:0 lds\n\t" \
    "v_mfma_f32_16x16x32_bf16 v[128:131], a[224:227], a[160:163], v[128:131]\n\t" \
    "ds_read_b128 a[152:155], v20 offset:6144\n\t" \
    "v_mfma_f32_16x16x32_bf16 v[128:131], a[228:231], a[164:167], v[128:131]\n\t" \
    "buffer_load_dwordx4 v27, %[rs], %[so] offen offset:1024 lds\n\t" \
    "v_mfma_f32_16x16x32_bf16 v[132:135], a[232:235], a[160:163], v[132:135]\n\t" \
    "ds_read_b128 a[156:159], v21 offset:6144\n\t" \
    "v_mfma_f32_16x16x32_bf16 v[132:135], a[236:239], a[164:167], v[132:135]\n\t" \
    "buffer_load_dwordx4 v28, %[rs], %[so] offen offset:2048 lds\n\t" \
    "v_mfma_f32_16x16x32_bf16 v[136:139], a[240:243], a[160:163], v[136:139]\n\t" \
    "ds_read_b64_tr_b16 a[212:213], v24 offset:24576\n\t" \
    "v_mfma_f32_16x16x32_bf16 v[136:139], a[244:247], a[164:167], v[136:139]\n\t" \
    "buffer_load_dwordx4 v29, %[rs], %[so] offen offset:3072 lds\n\t" \
    "v_mfma_f32_16x16x32_bf16 v[140:143], a[248:251], a[160:163], v[140:143]\n\t" \
    "ds_read_b64_tr_b16 a[214:215], v24 offset:25600\n\t" \
    "v_mfma_f32_16x16x32_bf16 v[140:143], a[252:255], a[164:167], v[140:143]\n\t" \
    "v_mfma_f32_16x16x32_bf16 v[160:163], a[224:227], a[168:171], v[160:163]\n\t" \
    "ds_read_b128 a[160:163], v20 offset:8192\n\t" \
    "s_add_u32 m0, m0, 0x1000\n\t" \
    "v_mfma_f32_16x16x32_bf16 v[160:163], a[228:231], a[172:175], v[160:163]\n\t" \
    "v_mfma_f32_16x16x32_bf16 v[164:167], a[232:235], a[168:171], v[164:167]\n\t" \
    "ds_read_b128 a[164:167], v21 offset:8192\n\t" \
    "s_add_u32 %[so], %[so], 0x40000\n\t" \
    "v_mfma_f32_16x16x32_bf16 v[164:167], a[236:239], a[172:175], v[164:167]\n\t" \
    "v_mfma_f32_16x16x32_bf16 v[168:171], a[240:243], a[168:171], v[168:171]\n\t" \
    "ds_read_b64_tr_b16 a[216:217], v25 offset:16384\n\t" \
    "v_mfma_f32_16x16x32_bf16 v[168:171], a[244:247], a[172:175], v[168:171]\n\t" \
    "buffer_load_dwordx4 v26, %[rs], %[so] offen offset:0 lds\n\t" \
    "v_mfma_f32_16x16x32_bf16 v[172:175], a[248:251], a[168:171], v[172:175]\n\t" \
    "ds_read_b64_tr_b16 a[218:219], v25 offset:17408\n\t" \
    "v_mfma_f32_16x16x32_bf16 v[172:175], a[252:255], a[172:175], v[172:175]\n\t" \
    "buffer_load_dwordx4 v27, %[rs], %[so] offen offset:1024 lds\n\t" \
    "v_mfma_f32_16x16x32_bf16 v[192:195], a[224:227], a[176:179], v[192:195]\n\t" \
    "ds_read_b128 a[168:171], v20 offset:10240\n\t" \
    "v_mfma_f32_16x16x32_bf16 v[192:195], a[228:231], a[180:183], v[192:195]\n\t" \
    "buffer_load_dwordx4 v28, %[rs], %[so] offen offset:2048 lds\n\t" \
    "v_mfma_f32_16x16x32_bf16 v[196:199], a[232:235], a[176:179], v[196:199]\n\t" \
    "ds_read_b128 a[172:175], v21 offset:10240\n\t" \
    "v_mfma_f32_16x16x32_bf16 v[196:199], a[236:239], a[180:183], v[196:199]\n\t" \
    "v_mfma_f32_16x16x32_bf16 v[200:203], a[240:243], a[176:179], v[200:203]\n\t" \
    "ds_read_b64_tr_b16 a[220:221], v25 offset:24576\n\t" \
    "v_mfma_f32_16x16x32_bf16 v[200:203], a[244:247], a[180:183], v[200:203]\n\t" \
    "buffer_load_dwordx4 v29, %[rs], %[so] offen offset:3072 lds\n\t" \
    "v_mfma_f32_16x16x32_bf16 v[204:207], a[248:251], a[176:179], v[204:207]\n\t" \
    "ds_read_b64_tr_b16 a[222:223], v25 offset:25600\n\t" \
    "s_add_u32 m0, m0, 0x1000\n\t" \
    "v_mfma_f32_16x16x32_bf16 v[204:207], a[252:255], a[180:183], v[204:207]\n\t" \
    "v_mfma_f32_16x16x32_bf16 v[224:227], a[224:227], a[184:187], v[224:227]\n\t" \
    "v_add_u32 v23, 0x8000, v22\n\t" \
    "v_add_u32 v22, 0xffff0000, v22\n\t" \
    "v_mfma_f32_16x16x32_bf16 v[224:227], a[228:231], a[188:191], v[224:227]\n\t" \
    "s_add_u32 %[so], %[so], 0x3ff80\n\t" \
    "v_min_u32_e64 v22, v22, v23\n\t" \
    "v_mfma_f32_16x16x32_bf16 v[228:231], a[232:235], a[184:187], v[228:231]\n\t" \
    "v_xor_b32_e64 v23, 32, v22\n\t" \
    "v_xor_b32_e64 v24, 64, v22\n\t" \
    "v_mfma_f32_16x16x32_bf16 v[228:231], a[236:239], a[188:191], v[228:231]\n\t" \
    "s_cmp_eq_u32 %[rem], 4\n\ts_cselect_b32 %[so], %[nx], %[so]\n\t" \
    "v_mfma_f32_16x16x32_bf16 v[232:235], a[240:243], a[184:187], v[232:235]\n\t" \
    "v_xor_b32 v25, 0x60, v22\n\t" \
    "v_mfma_f32_16x16x32_bf16 v[232:235], a[244:247], a[188:191], v[232:235]\n\t" \
    "v_mfma_f32_16x16x32_bf16 v[236:239], a[248:251], a[184:187], v[236:239]\n\t" \
    "v_mfma_f32_16x16x32_bf16 v[236:239], a[252:255], a[188:191], v[236:239]\n\t" \
    "s_sub_u32 %[rem], %[rem], 1\n\ts_cmp_lg_u32 %[rem], %[stop]\n\t" \
    "s_waitcnt vmcnt(48) lgkmcnt(0)\n\ts_barrier\n\t" \
    "s_cbranch_scc1 1b\n\t" \
    ""
#define ASM_END_B \
    "v_mfma_f32_16x16x32_bf16 a[0:3], a[192:195], a[128:131], a[0:3]\n\t" \
    "ds_read_b128 a[176:179], v20 offset:12288\n\t" \
    "v_mfma_f32_16x16x32_bf16 a[0:3], a[196:199], a[132:135], a[0:3]\n\t" \
    "buffer_load_dwordx4 v26, %[rs], %[so] offen offset:0 lds\n\t" \
    "v_mfma_f32_16x16x32_bf16 a[4:7], a[200:203], a[128:131], a[4:7]\n\t" \
    "ds_read_b128 a[180:183], v21 offset:12288\n\t" \
    "v_mfma_f32_16x16x32_bf16 a[4:7], a[204:207], a[132:135], a[4:7]\n\t" \
    "buffer_load_dwordx4 v27, %[rs], %[so] offen offset:1024 lds\n\t" \
    "v_mfma_f32_16x16x32_bf16 a[8:11], a[208:211], a[128:131], a[8:11]\n\t" \
    "ds_read_b128 a[184:187], v20 offset:14336\n\t" \
    "v_mfma_f32_16x16x32_bf16 a[8:11], a[212:215], a[132:135], a[8:11]\n\t" \
    "buffer_load_dwordx4 v28, %[rs], %[so] offen offset:2048 lds\n\t" \
    "v_mfma_f32_16x16x32_bf16 a[12:15], a[216:219], a[128:131], a[12:15]\n\t" \
    "ds_read_b128 a[188:191], v21 offset:14336\n\t" \
    "v_mfma_f32_16x16x32_bf16 a[12:15], a[220:223], a[132:135], a[12:15]\n\t" \
    "v_mfma_f32_16x16x32_bf16 a[32:35], a[192:195], a[136:139], a[32:35]\n\t" \
    "ds_read_b64_tr_b16 a[224:225], v22 offset:0\n\t" \
    "v_mfma_f32_16x16x32_bf16 a[32:35], a[196:199], a[140:143], a[32:35]\n\t" \
    "buffer_load_dwordx4 v29, %[rs], %[so] offen offset:3072 lds\n\t" \
    "v_mfma_f32_16x16x32_bf16 a[36:39], a[200:203], a[136:139], a[36:39]\n\t" \
    "ds_read_b64_tr_b16 a[226:227], v22 offset:1024\n\t" \
    "s_add_u32 m0, m0, 0x1000\n\t" \
    "v_mfma_f32_16x16x32_bf16 a[36:39], a[204:207], a[140:143], a[36:39]\n\t" \
    "v_xor_b32 v20, 0x38000, v20\n\t" \
    "v_xor_b32 v21, 0x38000, v21\n\t" \
    "v_mfma_f32_16x16x32_bf16 a[40:43], a[208:211], a[136:139], a[40:43]\n\t" \
    "ds_read_b64_tr_b16 a[228:229], v22 offset:8192\n\t" \
    "v_mfma_f32_16x16x32_bf16 a[40:43], a[212:215], a[140:143], a[40:43]\n\t" \
    "s_add_u32 %[so], %[so], 0x40000\n\t" \
    "v_mfma_f32_16x16x32_bf16 a[44:47], a[216:219], a[136:139], a[44:47]\n\t" \
    "ds_read_b64_tr_b16 a[230:231], v22 offset:9216\n\t" \
    "v_mfma_f32_16x16x32_bf16 a[44:47], a[220:223], a[140:143], a[44:47]\n\t" \
    "buffer_load_dwordx4 v26, %[rs], %[so] offen offset:0 lds\n\t" \
    "v_mfma_f32_16x16x32_bf16 v[48:51], a[192:195], a[144:147], v[48:51]\n\t" \
    "ds_read_b64_tr_b16 a[232:233], v23 offset:0\n\t" \
    "v_mfma_f32_16x16x32_bf16 v[48:51], a[196:199], a[148:151], v[48:51]\n\t" \
    "v_mfma_f32_16x16x32_bf16 v[52:55], a[200:203], a[144:147], v[52:55]\n\t" \
    "ds_read_b64_tr_b16 a[234:235], v23 offset:1024\n\t" \
    "v_mfma_f32_16x16x32_bf16 v[52:55], a[204:207], a[148:151], v[52:55]\n\t" \
    "buffer_load_dwordx4 v27, %[rs], %[so] offen offset:1024 lds\n\t" \
    "v_mfma_f32_16x16x32_bf16 v[56:59], a[208:211], a[144:147], v[56:59]\n\t" \
    "ds_read_b64_tr_b16 a[236:237], v23 offset:8192\n\t" \
    "v_mfma_f32_16x16x32_bf16 v[56:59], a[212:215], a[148:151], v[56:59]\n\t" \
    "buffer_load_dwordx4 v28, %[rs], %[so] offen offset:2048 lds\n\t" \
    "v_mfma_f32_16x16x32_bf16 v[60:63], a[216:219], a[144:147], v[60:63]\n\t" \
    "ds_read_b64_tr_b16 a[238:239], v23 offset:9216\n\t" \
    "v_mfma_f32_16x16x32_bf16 v[60:63], a[220:223], a[148:151], v[60:63]\n\t" \
    "buffer_load_dwordx4 v29, %[rs], %[so] offen offset:3072 lds\n\t" \
    "v_mfma_f32_16x16x32_bf16 v[80:83], a[192:195], a[152:155], v[80:83]\n\t" \
    "ds_read_b64_tr_b16 a[240:241], v24 offset:0\n\t" \
    "v_mfma_f32_16x16x32_bf16 v[80:83], a[196:199], a[156:159], v[80:83]\n\t" \
    "s_add_u32 m0, m0, 0x1000\n\t" \
    "v_mfma_f32_16x16x32_bf16 v[84:87], a[200:203], a[152:155], v[84:87]\n\t" \
    "ds_read_b64_tr_b16 a[242:243], v24 offset:1024\n\t" \
    "v_mfma_f32_16x16x32_bf16 v[84:87], a[204:207], a[156:159], v[84:87]\n\t" \
    "s_add_u32 %[so], %[so], 0x40000\n\t" \
    "v_mfma_f32_16x16x32_bf16 v[88:91], a[208:211], a[152:155], v[88:91]\n\t" \
    "ds_read_b64_tr_b16 a[244:245], v24 offset:8192\n\t" \
    "v_mfma_f32_16x16x32_bf16 v[88:91], a[212:215], a[156:159], v[88:91]\n\t" \
    "v_mfma_f32_16x16x32_bf16 v[92:95], a[216:219], a[152:155], v[92:95]\n\t" \
    "ds_read_b64_tr_b16 a[246:247], v24 offset:9216\n\t" \
    "v_mfma_f32_16x16x32_bf16 v[92:95], a[220:223], a[156:159], v[92:95]\n\t" \
    "buffer_load_dwordx4 v26, %[rs], %[so] offen offset:0 lds\n\t" \
    "v_mfma_f32_16x16x32_bf16 v[112:115], a[192:195], a[160:163], v[112:115]\n\t" \
    "ds_read_b64_tr_b16 a[248:249], v25 offset:0\n\t" \
    "v_mfma_f32_16x16x32_bf16 v[112:115], a[196:199], a[164:167], v[112:115]\n\t" \
    "buffer_load_dwordx4 v27, %[rs], %[so] offen offset:1024 lds\n\t" \
    "v_mfma_f32_16x16x32_bf16 v[116:119], a[200:203], a[160:163], v[116:119]\n\t" \
    "ds_read_b64_tr_b16 a[250:251], v25 offset:1024\n\t" \
    "v_mfma_f32_16x16x32_bf16 v[116:119], a[204:207], a[164:167], v[116:119]\n\t" \
    "buffer_load_dwordx4 v28, %[rs], %[so] offen offset:2048 lds\n\t" \
    "v_mfma_f32_16x16x32_bf16 v[120:123], a[208:211], a[160:163], v[120:123]\n\t" \
    "ds_read_b64_tr_b16 a[252:253], v25 offset:8192\n\t" \
    "v_mfma_f32_16x16x32_bf16 v[120:123], a[212:215], a[164:167], v[120:123]\n\t" \
    "buffer_load_dwordx4 v29, %[rs], %[so] offen offset:3072 lds\n\t" \
    "v_mfma_f32_16x16x32_bf16 v[124:127], a[216:219], a[160:163], v[124:127]\n\t" \
    "ds_read_b64_tr_b16 a[254:255], v25 offset:9216\n\t" \
    "v_mfma_f32_16x16x32_bf16 v[124:127], a[220:223], a[164:167], v[124:127]\n\t" \
    "v_mfma_f32_16x16x32_bf16 v[144:147], a[192:195], a[168:171], v[144:147]\n\t" \
    "s_add_u32 m0, m0, 0x1000\n\t" \
    "v_mfma_f32_16x16x32_bf16 v[144:147], a[196:199], a[172:175], v[144:147]\n\t" \
    "v_mfma_f32_16x16x32_bf16 v[148:151], a[200:203], a[168:171], v[148:151]\n\t" \
    "s_add_u32 %[so], %[so], 0x40000\n\t" \
    "v_mfma_f32_16x16x32_bf16 v[148:151], a[204:207], a[172:175], v[148:151]\n\t" \
    "v_mfma_f32_16x16x32_bf16 v[152:155], a[208:211], a[168:171], v[152:155]\n\t" \
    "buffer_load_dwordx4 v26, %[rs], %[so] offen offset:0 lds\n\t" \
    "v_mfma_f32_16x16x32_bf16 v[152:155], a[212:215], a[172:175], v[152:155]\n\t" \
    "v_mfma_f32_16x16x32_bf16 v[156:159], a[216:219], a[168:171], v[156:159]\n\t" \
    "v_mfma_f32_16x16x32_bf16 v[156:159], a[220:223], a[172:175], v[156:159]\n\t" \
    "s_waitcnt lgkmcnt(8)\n\ts_nop 0\n\t" \
    "v_mfma_f32_16x16x32_bf16 v[176:179], a[192:195], a[176:179], v[176:179]\n\t" \
    "buffer_load_dwordx4 v27, %[rs], %[so] offen offset:1024 lds\n\t" \
    "v_mfma_f32_16x16x32_bf16 v[176:179], a[196:199], a[180:183], v[176:179]\n\t" \
    "buffer_load_dwordx4 v28, %[rs], %[so] offen offset:2048 lds\n\t" \
    "v_mfma_f32_16x16x32_bf16 v[180:183], a[200:203], a[176:179], v[180:183]\n\t" \
    "v_mfma_f32_16x16x32_bf16 v[180:183], a[204:207], a[180:183], v[180:183]\n\t" \
    "v_mfma_f32_16x16x32_bf16 v[184:187], a[208:211], a[176:179], v[184:187]\n\t" \
    "buffer_load_dwordx4 v29, %[rs], %[so] offen offset:3072 lds\n\t" \
    "v_mfma_f32_16x16x32_bf16 v[184:187], a[212:215], a[180:183], v[184:187]\n\t" \
    "v_mfma_f32_16x16x32_bf16 v[188:191], a[216:219], a[176:179], v[188:191]\n\t" \
    "s_add_u32 m0, m0, 0x1000\n\t" \
    "v_mfma_f32_16x16x32_bf16 v[188:191], a[220:223], a[180:183], v[188:191]\n\t" \
    "v_mfma_f32_16x16x32_bf16 v[208:211], a[192:195], a[184:187], v[208:211]\n\t" \
    "v_mfma_f32_16x16x32_bf16 v[208:211], a[196:199], a[188:191], v[208:211]\n\t" \
    "s_cmp_eq_u32 m0, 0x18000\n\ts_cselect_b32 m0, 0, m0\n\ts_nop 0\n\t" \
    "v_mfma_f32_16x16x32_bf16 v[212:215], a[200:203], a[184:187], v[212:215]\n\t" \
    "v_mfma_f32_16x16x32_bf16 v[212:215], a[204:207], a[188:191], v[212:215]\n\t" \
    "s_add_u32 %[so], %[so], 0xfff40080\n\t" \
    "v_mfma_f32_16x16x32_bf16 v[216:219], a[208:211], a[184:187], v[216:219]\n\t" \
    "v_mfma_f32_16x16x32_bf16 v[216:219], a[212:215], a[188:191], v[216:219]\n\t" \
    "v_mfma_f32_16x16x32_bf16 v[220:223], a[216:219], a[184:187], v[220:223]\n\t" \
    "v_mfma_f32_16x16x32_bf16 v[220:223], a[220:223], a[188:191], v[220:223]\n\t" \
    "s_waitcnt vmcnt(48) lgkmcnt(0)\n\ts_barrier\n\t" \
    "v_mfma_f32_16x16x32_bf16 a[16:19], a[224:227], a[128:131], a[16:19]\n\t" \
    "ds_read_b64_tr_b16 a[192:193], v22 offset:16384\n\t" \
    "v_mfma_f32_16x16x32_bf16 a[16:19], a[228:231], a[132:135], a[16:19]\n\t" \
    "buffer_load_dwordx4 v26, %[rs], %[so] offen offset:0 lds\n\t" \
    "v_accvgpr_read_b32 v32, a0\n\t" \
    "v_mfma_f32_16x16x32_bf16 a[20:23], a[232:235], a[128:131], a[20:23]\n\t" \
    "ds_read_b64_tr_b16 a[194:195], v22 offset:17408\n\t" \
    "v_accvgpr_read_b32 v33, a1\n\t" \
    "v_accvgpr_read_b32 v34, a2\n\t" \
    "v_mfma_f32_16x16x32_bf16 a[20:23], a[236:239], a[132:135], a[20:23]\n\t" \
    "buffer_load_dwordx4 v27, %[rs], %[so] offen offset:1024 lds\n\t" \
    "v_accvgpr_read_b32 v35, a3\n\t" \
    "v_mfma_f32_16x16x32_bf16 a[24:27], a[240:243], a[128:131], a[24:27]\n\t" \
    "ds_read_b64_tr_b16 a[196:197], v22 offset:24576\n\t" \
    "v_cvt_pk_bf16_f32 v248, v32, v33\n\t" \
    "v_cvt_pk_bf16_f32 v249, v34, v35\n\t" \
    "v_mfma_f32_16x16x32_bf16 a[24:27], a[244:247], a[132:135], a[24:27]\n\t" \
    "buffer_load_dwordx4 v28, %[rs], %[so] offen offset:2048 lds\n\t" \
    "v_accvgpr_read_b32 v32, a4\n\t" \
    "v_mfma_f32_16x16x32_bf16 a[28:31], a[248:251], a[128:131], a[28:31]\n\t" \
    "ds_read_b64_tr_b16 a[198:199], v22 offset:25600\n\t" \
    "v_accvgpr_read_b32 v33, a5\n\t" \
    "v_accvgpr_read_b32 v34, a6\n\t" \
    "v_mfma_f32_16x16x32_bf16 a[28:31], a[252:255], a[132:135], a[28:31]\n\t" \
    "v_accvgpr_read_b32 v35, a7\n\t" \
    "v_cvt_pk_bf16_f32 v250, v32, v33\n\t" \
    "v_cvt_pk_bf16_f32 v251, v34, v35\n\t" \
    "v_mfma_f32_16x16x32_bf16 a[48:51], a[224:227], a[136:139], a[48:51]\n\t" \
    "ds_read_b128 a[128:131], v20 offset:0\n\t" \
    "buffer_load_dwordx4 v29, %[rs], %[so] offen offset:3072 lds\n\t" \
    "v_mfma_f32_16x16x32_bf16 a[48:51], a[228:231], a[140:143], a[48:51]\n\t" \
    "s_xor_b32 %[t0], %[sd], 0x0\n\tbuffer_store_dwordx4 v[248:251], v30, %[rss], %[t0] offen " DAUXS "\n\t" \
    "v_accvgpr_read_b32 v32, a8\n\t" \
    "v_mfma_f32_16x16x32_bf16 a[52:55], a[232:235], a[136:139], a[52:55]\n\t" \
    "ds_read_b128 a[132:135], v21 offset:0\n\t" \
    "s_add_u32 m0, m0, 0x1000\n\t" \
    "v_accvgpr_read_b32 v33, a9\n\t" \
    "v_mfma_f32_16x16x32_bf16 a[52:55], a[236:239], a[140:143], a[52:55]\n\t" \
    "v_accvgpr_read_b32 v34, a10\n\t" \
    "v_accvgpr_read_b32 v35, a11\n\t" \
    "v_cvt_pk_bf16_f32 v252, v32, v33\n\t" \
    "v_mfma_f32_16x16x32_bf16 a[56:59], a[240:243], a[136:139], a[56:59]\n\t" \
    "ds_read_b64_tr_b16 a[200:201], v23 offset:16384\n\t" \
    "v_cvt_pk_bf16_f32 v253, v34, v35\n\t" \
    "v_accvgpr_read_b32 v32, a12\n\t" \
    "v_mfma_f32_16x16x32_bf16 a[56:59], a[244:247], a[140:143], a[56:59]\n\t" \
    "s_add_u32 %[so], %[so], 0x40000\n\t" \
    "v_accvgpr_read_b32 v33, a13\n\t" \
    "v_accvgpr_read_b32 v34, a14\n\t" \
    "v_mfma_f32_16x16x32_bf16 a[60:63], a[248:251], a[136:139], a[60:63]\n\t" \
    "ds_read_b64_tr_b16 a[202:203], v23 offset:17408\n\t" \
    "v_accvgpr_read_b32 v35, a15\n\t" \
    "v_cvt_pk_bf16_f32 v254, v32, v33\n\t" \
    "v_mfma_f32_16x16x32_bf16 a[60:63], a[252:255], a[140:143], a[60:63]\n\t" \
    "buffer_load_dwordx4 v26, %[rs], %[so] offen offset:0 lds\n\t" \
    "v_cvt_pk_bf16_f32 v255, v34, v35\n\t" \
    "v_mfma_f32_16x16x32_bf16 v[64:67], a[224:227], a[144:147], v[64:67]\n\t" \
    "ds_read_b128 a[136:139], v20 offset:2048\n\t" \
    "s_xor_b32 %[t0], %[sd], 0x400\n\tbuffer_store_dwordx4 v[252:255], v30, %[rss], %[t0] offen " DAUXS "\n\t" \
    "v_mfma_f32_16x16x32_bf16 v[64:67], a[228:231], a[148:151], v[64:67]\n\t" \
    "v_accvgpr_read_b32 v32, a32\n\t" \
    "v_accvgpr_read_b32 v33, a33\n\t" \
    "v_accvgpr_read_b32 v34, a34\n\t" \
    "v_mfma_f32_16x16x32_bf16 v[68:71], a[232:235], a[144:147], v[68:71]\n\t" \
    "ds_read_b128 a[140:143], v21 offset:2048\n\t" \
    "buffer_load_dwordx4 v27, %[rs], %[so] offen offset:1024 lds\n\t" \
    "v_mfma_f32_16x16x32_bf16 v[68:71], a[236:239], a[148:151], v[68:71]\n\t" \
    "v_accvgpr_read_b32 v35, a35\n\t" \
    "v_cvt_pk_bf16_f32 v248, v32, v33\n\t" \
    "v_cvt_pk_bf16_f32 v249, v34, v35\n\t" \
    "v_mfma_f32_16x16x32_bf16 v[72:75], a[240:243], a[144:147], v[72:75]\n\t" \
    "ds_read_b64_tr_b16 a[204:205], v23 offset:24576\n\t" \
    "buffer_load_dwordx4 v28, %[rs], %[so] offen offset:2048 lds\n\t" \
    "v_mfma_f32_16x16x32_bf16 v[72:75], a[244:247], a[148:151], v[72:75]\n\t" \
    "v_accvgpr_read_b32 v32, a36\n\t" \
    "v_accvgpr_read_b32 v33, a37\n\t" \
    "v_accvgpr_read_b32 v34, a38\n\t" \
    "v_mfma_f32_16x16x32_bf16 v[76:79], a[248:251], a[144:147], v[76:79]\n\t" \
    "ds_read_b64_tr_b16 a[206:207], v23 offset:25600\n\t" \
    "buffer_load_dwordx4 v29, %[rs], %[so] offen offset:3072 lds\n\t" \
    "v_mfma_f32_16x16x32_bf16 v[76:79], a[252:255], a[148:151], v[76:79]\n\t" \
    "v_accvgpr_read_b32 v35, a39\n\t" \
    "v_cvt_pk_bf16_f32 v250, v32, v33\n\t" \
    "v_cvt_pk_bf16_f32 v251, v34, v35\n\t" \
    "v_mfma_f32_16x16x32_bf16 v[96:99], a[224:227], a[152:155], v[96:99]\n\t" \
    "ds_read_b128 a[144:147], v20 offset:4096\n\t" \
    "s_xor_b32 %[t0], %[sd], 0x800\n\tbuffer_store_dwordx4 v[248:251], v30, %[rss], %[t0] offen " DAUXS "\n\t" \
    "v_mfma_f32_16x16x32_bf16 v[96:99], a[228:231], a[156:159], v[96:99]\n\t" \
    "s_add_u32 m0, m0, 0x1000\n\t" \
    "v_accvgpr_read_b32 v32, a40\n\t" \
    "v_accvgpr_read_b32 v33, a41\n\t" \
    "v_mfma_f32_16x16x32_bf16 v[100:103], a[232:235], a[152:155], v[100:103]\n\t" \
    "ds_read_b128 a[148:151], v21 offset:4096\n\t" \
    "v_accvgpr_read_b32 v34, a42\n\t" \
    "v_accvgpr_read_b32 v35, a43\n\t" \
    "v_mfma_f32_16x16x32_bf16 v[100:103], a[236:239], a[156:159], v[100:103]\n\t" \
    "s_add_u32 %[so], %[so], 0x40000\n\t" \
    "v_cvt_pk_bf16_f32 v252, v32, v33\n\t" \
    "v_cvt_pk_bf16_f32 v253, v34, v35\n\t" \
    "v_mfma_f32_16x16x32_bf16 v[104:107], a[240:243], a[152:155], v[104:107]\n\t" \
    "ds_read_b64_tr_b16 a[208:209], v24 offset:16384\n\t" \
    "v_accvgpr_read_b32 v32, a44\n\t" \
    "v_accvgpr_read_b32 v33, a45\n\t" \
    "v_mfma_f32_16x16x32_bf16 v[104:107], a[244:247], a[156:159], v[104:107]\n\t" \
    "v_accvgpr_read_b32 v34, a46\n\t" \
    "v_accvgpr_read_b32 v35, a47\n\t" \
    "v_cvt_pk_bf16_f32 v254, v32, v33\n\t" \
    "v_mfma_f32_16x16x32_bf16 v[108:111], a[248:251], a[152:155], v[108:111]\n\t" \
    "ds_read_b64_tr_b16 a[210:211], v24 offset:17408\n\t" \
    "buffer_load_dwordx4 v26, %[rs], %[so] offen offset:0 lds\n\t" \
    "v_mfma_f32_16x16x32_bf16 v[108:111], a[252:255], a[156:159], v[108:111]\n\t" \
    "v_cvt_pk_bf16_f32 v255, v34, v35\n\t" \
    "s_xor_b32 %[t0], %[sd], 0xc00\n\tbuffer_store_dwordx4 v[252:255], v30, %[rss], %[t0] offen " DAUXS "\n\t" \
    "v_mfma_f32_16x16x32_bf16 v[128:131], a[224:227], a[160:163], v[128:131]\n\t" \
    "ds_read_b128 a[152:155], v20 offset:6144\n\t" \
    "buffer_load_dwordx4 v27, %[rs], %[so] offen offset:1024 lds\n\t" \
    "v_mfma_f32_16x16x32_bf16 v[128:131], a[228:231], a[164:167], v[128:131]\n\t" \
    "v_cvt_pk_bf16_f32 v248, v48, v49\n\t" \
    "v_cvt_pk_bf16_f32 v249, v50, v51\n\t" \
    "v_cvt_pk_bf16_f32 v250, v52, v53\n\t" \
    "v_mfma_f32_16x16x32_bf16 v[132:135], a[232:235], a[160:163], v[132:135]\n\t" \
    "ds_read_b128 a[156:159], v21 offset:6144\n\t" \
    "v_cvt_pk_bf16_f32 v251, v54, v55\n\t" \
    "v_mfma_f32_16x16x32_bf16 v[132:135], a[236:239], a[164:167], v[132:135]\n\t" \
    "buffer_load_dwordx4 v28, %[rs], %[so] offen offset:2048 lds\n\t" \
    "v_mfma_f32_16x16x32_bf16 v[136:139], a[240:243], a[160:163], v[136:139]\n\t" \
    "ds_read_b64_tr_b16 a[212:213], v24 offset:24576\n\t" \
    "s_xor_b32 %[t0], %[sd], 0x1000\n\tbuffer_store_dwordx4 v[248:251], v30, %[rss], %[t0] offen " DAUXS "\n\t" \
    "v_mfma_f32_16x16x32_bf16 v[136:139], a[244:247], a[164:167], v[136:139]\n\t" \
    "buffer_load_dwordx4 v29, %[rs], %[so] offen offset:3072 lds\n\t" \
    "v_cvt_pk_bf16_f32 v252, v56, v57\n\t" \
    "v_mfma_f32_16x16x32_bf16 v[140:143], a[248:251], a[160:163], v[140:143]\n\t" \
    "ds_read_b64_tr_b16 a[214:215], v24 offset:25600\n\t" \
    "v_cvt_pk_bf16_f32 v253, v58, v59\n\t" \
    "v_cvt_pk_bf16_f32 v254, v60, v61\n\t" \
    "v_mfma_f32_16x16x32_bf16 v[140:143], a[252:255], a[164:167], v[140:143]\n\t" \
    "v_cvt_pk_bf16_f32 v255, v62, v63\n\t" \
    "s_xor_b32 %[t0], %[sd], 0x1400\n\tbuffer_store_dwordx4 v[252:255], v30, %[rss], %[t0] offen " DAUXS "\n\t" \
    "v_mfma_f32_16x16x32_bf16 v[160:163], a[224:227], a[168:171], v[160:163]\n\t" \
    "ds_read_b128 a[160:163], v20 offset:8192\n\t" \
    "s_add_u32 m0, m0, 0x1000\n\t" \
    "v_cvt_pk_bf16_f32 v248, v80, v81\n\t" \
    "v_mfma_f32_16x16x32_bf16 v[160:163], a[228:231], a[172:175], v[160:163]\n\t" \
    "v_cvt_pk_bf16_f32 v249, v82, v83\n\t" \
    "v_cvt_pk_bf16_f32 v250, v84, v85\n\t" \
    "v_cvt_pk_bf16_f32 v251, v86, v87\n\t" \
    "v_mfma_f32_16x16x32_bf16 v[164:167], a[232:235], a[168:171], v[164:167]\n\t" \
    "ds_read_b128 a[164:167], v21 offset:8192\n\t" \
    "s_add_u32 %[so], %[so], 0x40000\n\t" \
    "v_mfma_f32_16x16x32_bf16 v[164:167], a[236:239], a[172:175], v[164:167]\n\t" \
    "s_xor_b32 %[t0], %[sd], 0x1800\n\tbuffer_store_dwordx4 v[248:251], v30, %[rss], %[t0] offen " DAUXS "\n\t" \
    "v_cvt_pk_bf16_f32 v252, v88, v89\n\t" \
    "v_mfma_f32_16x16x32_bf16 v[168:171], a[240:243], a[168:171], v[168:171]\n\t" \
    "ds_read_b64_tr_b16 a[216:217], v25 offset:16384\n\t" \
    "buffer_load_dwordx4 v26, %[rs], %[so] offen offset:0 lds\n\t" \
    "v_mfma_f32_16x16x32_bf16 v[168:171], a[244:247], a[172:175], v[168:171]\n\t" \
    "v_cvt_pk_bf16_f32 v253, v90, v91\n\t" \
    "v_cvt_pk_bf16_f32 v254, v92, v93\n\t" \
    "v_cvt_pk_bf16_f32 v255, v94, v95\n\t" \
    "v_mfma_f32_16x16x32_bf16 v[172:175], a[248:251], a[168:171], v[172:175]\n\t" \
    "ds_read_b64_tr_b16 a[218:219], v25 offset:17408\n\t" \
    "s_xor_b32 %[t0], %[sd], 0x1c00\n\tbuffer_store_dwordx4 v[252:255], v30, %[rss], %[t0] offen " DAUXS "\n\t" \
    "v_mfma_f32_16x16x32_bf16 v[172:175], a[252:255], a[172:175], v[172:175]\n\t" \
    "buffer_load_dwordx4 v27, %[rs], %[so] offen offset:1024 lds\n\t" \
    "v_cvt_pk_bf16_f32 v248, v112, v113\n\t" \
    "v_mfma_f32_16x16x32_bf16 v[192:195], a[224:227], a[176:179], v[192:195]\n\t" \
    "ds_read_b128 a[168:171], v20 offset:10240\n\t" \
    "v_cvt_pk_bf16_f32 v249, v114, v115\n\t" \
    "v_cvt_pk_bf16_f32 v250, v116, v117\n\t" \
    "v_mfma_f32_16x16x32_bf16 v[192:195], a[228:231], a[180:183], v[192:195]\n\t" \
    "buffer_load_dwordx4 v28, %[rs], %[so] offen offset:2048 lds\n\t" \
    "v_cvt_pk_bf16_f32 v251, v118, v119\n\t" \
    "v_mfma_f32_16x16x32_bf16 v[196:199], a[232:235], a[176:179], v[196:199]\n\t" \
    "ds_read_b128 a[172:175], v21 offset:10240\n\t" \
    "s_xor_b32 %[t0], %[sd], 0x2000\n\tbuffer_store_dwordx4 v[248:251], v30, %[rss], %[t0] offen " DAUXS "\n\t" \
    "v_mfma_f32_16x16x32_bf16 v[196:199], a[236:239], a[180:183], v[196:199]\n\t" \
    "v_cvt_pk_bf16_f32 v252, v120, v121\n\t" \
    "v_cvt_pk_bf16_f32 v253, v122, v123\n\t" \
    "v_cvt_pk_bf16_f32 v254, v124, v125\n\t" \
    "v_mfma_f32_16x16x32_bf16 v[200:203], a[240:243], a[176:179], v[200:203]\n\t" \
    "ds_read_b64_tr_b16 a[220:221], v25 offset:24576\n\t" \
    "buffer_load_dwordx4 v29, %[rs], %[so] offen offset:3072 lds\n\t" \
    "v_mfma_f32_16x16x32_bf16 v[200:203], a[244:247], a[180:183], v[200:203]\n\t" \
    "v_cvt_pk_bf16_f32 v255, v126, v127\n\t" \
    "s_xor_b32 %[t0], %[sd], 0x2400\n\tbuffer_store_dwordx4 v[252:255], v30, %[rss], %[t0] offen " DAUXS "\n\t" \
    "v_mfma_f32_16x16x32_bf16 v[204:207], a[248:251], a[176:179], v[204:207]\n\t" \
    "ds_read_b64_tr_b16 a[222:223], v25 offset:25600\n\t" \
    "s_add_u32 m0, m0, 0x1000\n\t" \
    "v_cvt_pk_bf16_f32 v248, v144, v145\n\t" \
    "v_mfma_f32_16x16x32_bf16 v[204:207], a[252:255], a[180:183], v[204:207]\n\t" \
    "v_cvt_pk_bf16_f32 v249, v146, v147\n\t" \
    "v_cvt_pk_bf16_f32 v250, v148, v149\n\t" \
    "v_cvt_pk_bf16_f32 v251, v150, v151\n\t" \
    "v_mfma_f32_16x16x32_bf16 v[224:227], a[224:227], a[184:187], v[224:227]\n\t" \
    "v_add_u32 v23, 0x8000, v22\n\t" \
    "v_add_u32 v22, 0xffff0000, v22\n\t" \
    "v_mfma_f32_16x16x32_bf16 v[224:227], a[228:231], a[188:191], v[224:227]\n\t" \
    "s_add_u32 %[so], %[so], 0x3ff80\n\t" \
    "v_min_u32_e64 v22, v22, v23\n\t" \
    "v_mfma_f32_16x16x32_bf16 v[228:231], a[232:235], a[184:187], v[228:231]\n\t" \
    "v_xor_b32_e64 v23, 32, v22\n\t" \
    "v_xor_b32_e64 v24, 64, v22\n\t" \
    "v_mfma_f32_16x16x32_bf16 v[228:231], a[236:239], a[188:191], v[228:231]\n\t" \
    "s_cmp_eq_u32 %[rem], 4\n\ts_cselect_b32 %[so], %[nx], %[so]\n\t" \
    "v_xor_b32 v25, 0x60, v22\n\t" \
    "v_mfma_f32_16x16x32_bf16 v[232:235], a[240:243], a[184:187], v[232:235]\n\t" \
    "s_xor_b32 %[t0], %[sd], 0x2800\n\tbuffer_store_dwordx4 v[248:251], v30, %[rss], %[t0] offen " DAUXS "\n\t" \
    "v_cvt_pk_bf16_f32 v252, v152, v153\n\t" \
    "v_mfma_f32_16x16x32_bf16 v[232:235], a[244:247], a[188:191], v[232:235]\n\t" \
    "v_cvt_pk_bf16_f32 v253, v154, v155\n\t" \
    "v_cvt_pk_bf16_f32 v254, v156, v157\n\t" \
    "v_cvt_pk_bf16_f32 v255, v158, v159\n\t" \
    "v_mfma_f32_16x16x32_bf16 v[236:239], a[248:251], a[184:187], v[236:239]\n\t" \
    "s_xor_b32 %[t0], %[sd], 0x2c00\n\tbuffer_store_dwordx4 v[252:255], v30, %[rss], %[t0] offen " DAUXS "\n\t" \
    "v_cvt_pk_bf16_f32 v248, v176, v177\n\t" \
    "v_mfma_f32_16x16x32_bf16 v[236:239], a[252:255], a[188:191], v[236:239]\n\t" \
    "v_cvt_pk_bf16_f32 v249, v178, v179\n\t" \
    "v_cvt_pk_bf16_f32 v250, v180, v181\n\t" \
    "v_cvt_pk_bf16_f32 v251, v182, v183\n\t" \
    "s_xor_b32 %[t0], %[sd], 0x3000\n\tbuffer_store_dwordx4 v[248:251], v30, %[rss], %[t0] offen " DAUXS "\n\t" \
    "v_cvt_pk_bf16_f32 v252, v184, v185\n\t" \
    "v_cvt_pk_bf16_f32 v253, v186, v187\n\t" \
    "v_cvt_pk_bf16_f32 v254, v188, v189\n\t" \
    "v_cvt_pk_bf16_f32 v255, v190, v191\n\t" \
    "s_xor_b32 %[t0], %[sd], 0x3400\n\tbuffer_store_dwordx4 v[252:255], v30, %[rss], %[t0] offen " DAUXS "\n\t" \
    "v_cvt_pk_bf16_f32 v248, v208, v209\n\t" \
    "v_cvt_pk_bf16_f32 v249, v210, v211\n\t" \
    "v_cvt_pk_bf16_f32 v250, v212, v213\n\t" \
    "v_cvt_pk_bf16_f32 v251, v214, v215\n\t" \
    "s_xor_b32 %[t0], %[sd], 0x3800\n\tbuffer_store_dwordx4 v[248:251], v30, %[rss], %[t0] offen " DAUXS "\n\t" \
    "v_cvt_pk_bf16_f32 v252, v216, v217\n\t" \
    "v_cvt_pk_bf16_f32 v253, v218, v219\n\t" \
    "v_cvt_pk_bf16_f32 v254, v220, v221\n\t" \
    "v_cvt_pk_bf16_f32 v255, v222, v223\n\t" \
    "s_xor_b32 %[t0], %[sd], 0x3c00\n\tbuffer_store_dwordx4 v[252:255], v30, %[rss], %[t0] offen " DAUXS "\n\t" \
    "s_sub_u32 %[rem], %[rem], 1\n\t" \
    "s_waitcnt vmcnt(63) lgkmcnt(0)\n\ts_barrier\n\t" \
    ""
#define ASM_START_B \
    "v_mfma_f32_16x16x32_bf16 a[0:3], a[192:195], a[128:131], 0\n\t" \
    "ds_read_b128 a[176:179], v20 offset:12288\n\t" \
    "v_mfma_f32_16x16x32_bf16 a[0:3], a[196:199], a[132:135], a[0:3]\n\t" \
    "buffer_load_dwordx4 v26, %[rs], %[so] offen offset:0 lds\n\t" \
    "v_accvgpr_read_b32 v32, a16\n\t" \
    "v_mfma_f32_16x16x32_bf16 a[4:7], a[200:203], a[128:131], 0\n\t" \
    "ds_read_b128 a[180:183], v21 offset:12288\n\t" \
    "v_accvgpr_read_b32 v33, a17\n\t" \
    "v_accvgpr_read_b32 v34, a18\n\t" \
    "v_mfma_f32_16x16x32_bf16 a[4:7], a[204:207], a[132:135], a[4:7]\n\t" \
    "buffer_load_dwordx4 v27, %[rs], %[so] offen offset:1024 lds\n\t" \
    "v_accvgpr_read_b32 v35, a19\n\t" \
    "v_mfma_f32_16x16x32_bf16 a[8:11], a[208:211], a[128:131], 0\n\t" \
    "ds_read_b128 a[184:187], v20 offset:14336\n\t" \
    "v_cvt_pk_bf16_f32 v248, v32, v33\n\t" \
    "v_cvt_pk_bf16_f32 v249, v34, v35\n\t" \
    "v_mfma_f32_16x16x32_bf16 a[8:11], a[212:215], a[132:135], a[8:11]\n\t" \
    "buffer_load_dwordx4 v28, %[rs], %[so] offen offset:2048 lds\n\t" \
    "v_accvgpr_read_b32 v32, a20\n\t" \
    "v_mfma_f32_16x16x32_bf16 a[12:15], a[216:219], a[128:131], 0\n\t" \
    "ds_read_b128 a[188:191], v21 offset:14336\n\t" \
    "v_accvgpr_read_b32 v33, a21\n\t" \
    "v_accvgpr_read_b32 v34, a22\n\t" \
    "v_mfma_f32_16x16x32_bf16 a[12:15], a[220:223], a[132:135], a[12:15]\n\t" \
    "v_accvgpr_read_b32 v35, a23\n\t" \
    "v_cvt_pk_bf16_f32 v250, v32, v33\n\t" \
    "v_cvt_pk_bf16_f32 v251, v34, v35\n\t" \
    "v_mfma_f32_16x16x32_bf16 a[32:35], a[192:195], a[136:139], 0\n\t" \
    "ds_read_b64_tr_b16 a[224:225], v22 offset:0\n\t" \
    "buffer_load_dwordx4 v29, %[rs], %[so] offen offset:3072 lds\n\t" \
    "v_mfma_f32_16x16x32_bf16 a[32:35], a[196:199], a[140:143], a[32:35]\n\t" \
    "v_xor_b32 v20, 0x38000, v20\n\t" \
    "v_xor_b32 v21, 0x38000, v21\n\t" \
    "v_mfma_f32_16x16x32_bf16 a[36:39], a[200:203], a[136:139], 0\n\t" \
    "ds_read_b64_tr_b16 a[226:227], v22 offset:1024\n\t" \
    "s_add_u32 m0, m0, 0x1000\n\t" \
    "v_mfma_f32_16x16x32_bf16 a[36:39], a[204:207], a[140:143], a[36:39]\n\t" \
    "s_xor_b32 %[t0], %[sd], 0x4000\n\tbuffer_store_dwordx4 v[248:251], v30, %[rss], %[t0] offen " DAUXS "\n\t" \
    "v_accvgpr_read_b32 v32, a24\n\t" \
    "v_mfma_f32_16x16x32_bf16 a[40:43], a[208:211], a[136:139], 0\n\t" \
    "ds_read_b64_tr_b16 a[228:229], v22 offset:8192\n\t" \
    "v_accvgpr_read_b32 v33, a25\n\t" \
    "v_accvgpr_read_b32 v34, a26\n\t" \
    "v_mfma_f32_16x16x32_bf16 a[40:43], a[212:215], a[140:143], a[40:43]\n\t" \
    "s_add_u32 %[so], %[so], 0x40000\n\t" \
    "v_accvgpr_read_b32 v35, a27\n\t" \
    "v_cvt_pk_bf16_f32 v252, v32, v33\n\t" \
    "v_mfma_f32_16x16x32_bf16 a[44:47], a[216:219], a[136:139], 0\n\t" \
    "ds_read_b64_tr_b16 a[230:231], v22 offset:9216\n\t" \
    "v_cvt_pk_bf16_f32 v253, v34, v35\n\t" \
    "v_accvgpr_read_b32 v32, a28\n\t" \
    "v_mfma_f32_16x16x32_bf16 a[44:47], a[220:223], a[140:143], a[44:47]\n\t" \
    "buffer_load_dwordx4 v26, %[rs], %[so] offen offset:0 lds\n\t" \
    "v_accvgpr_read_b32 v33, a29\n\t" \
    "v_mfma_f32_16x16x32_bf16 v[48:51], a[192:195], a[144:147], 0\n\t" \
    "ds_read_b64_tr_b16 a[232:233], v23 offset:0\n\t" \
    "v_accvgpr_read_b32 v34, a30\n\t" \
    "v_accvgpr_read_b32 v35, a31\n\t" \
    "v_mfma_f32_16x16x32_bf16 v[48:51], a[196:199], a[148:151], v[48:51]\n\t" \
    "v_cvt_pk_bf16_f32 v254, v32, v33\n\t" \
    "v_cvt_pk_bf16_f32 v255, v34, v35\n\t" \
    "v_mfma_f32_16x16x32_bf16 v[52:55], a[200:203], a[144:147], 0\n\t" \
    "ds_read_b64_tr_b16 a[234:235], v23 offset:1024\n\t" \
    "buffer_load_dwordx4 v27, %[rs], %[so] offen offset:1024 lds\n\t" \
    "v_mfma_f32_16x16x32_bf16 v[52:55], a[204:207], a[148:151], v[52:55]\n\t" \
    "s_xor_b32 %[t0], %[sd], 0x4400\n\tbuffer_store_dwordx4 v[252:255], v30, %[rss], %[t0] offen " DAUXS "\n\t" \
    "v_accvgpr_read_b32 v32, a48\n\t" \
    "v_mfma_f32_16x16x32_bf16 v[56:59], a[208:211], a[144:147], 0\n\t" \
    "ds_read_b64_tr_b16 a[236:237], v23 offset:8192\n\t" \
    "buffer_load_dwordx4 v28, %[rs], %[so] offen offset:2048 lds\n\t" \
    "v_mfma_f32_16x16x32_bf16 v[56:59], a[212:215], a[148:151], v[56:59]\n\t" \
    "v_accvgpr_read_b32 v33, a49\n\t" \
    "v_accvgpr_read_b32 v34, a50\n\t" \
    "v_accvgpr_read_b32 v35, a51\n\t" \
    "v_mfma_f32_16x16x32_bf16 v[60:63], a[216:219], a[144:147], 0\n\t" \
    "ds_read_b64_tr_b16 a[238:239], v23 offset:9216\n\t" \
    "buffer_load_dwordx4 v29, %[rs], %[so] offen offset:3072 lds\n\t" \
    "v_mfma_f32_16x16x32_bf16 v[60:63], a[220:223], a[148:151], v[60:63]\n\t" \
    "v_cvt_pk_bf16_f32 v248, v32, v33\n\t" \
    "v_cvt_pk_bf16_f32 v249, v34, v35\n\t" \
    "v_accvgpr_read_b32 v32, a52\n\t" \
    "v_mfma_f32_16x16x32_bf16 v[80:83], a[192:195], a[152:155], 0\n\t" \
    "ds_read_b64_tr_b16 a[240:241], v24 offset:0\n\t" \
    "v_accvgpr_read_b32 v33, a53\n\t" \
    "v_accvgpr_read_b32 v34, a54\n\t" \
    "v_mfma_f32_16x16x32_bf16 v[80:83], a[196:199], a[156:159], v[80:83]\n\t" \
    "s_add_u32 m0, m0, 0x1000\n\t" \
    "v_accvgpr_read_b32 v35, a55\n\t" \
    "v_cvt_pk_bf16_f32 v250, v32, v33\n\t" \
    "v_mfma_f32_16x16x32_bf16 v[84:87], a[200:203], a[152:155], 0\n\t" \
    "ds_read_b64_tr_b16 a[242:243], v24 offset:1024\n\t" \
    "v_cvt_pk_bf16_f32 v251, v34, v35\n\t" \
    "v_mfma_f32_16x16x32_bf16 v[84:87], a[204:207], a[156:159], v[84:87]\n\t" \
    "s_add_u32 %[so], %[so], 0x40000\n\t" \
    "s_xor_b32 %[t0], %[sd], 0x4800\n\tbuffer_store_dwordx4 v[248:251], v30, %[rss], %[t0] offen " DAUXS "\n\t" \
    "v_mfma_f32_16x16x32_bf16 v[88:91], a[208:211], a[152:155], 0\n\t" \
    "ds_read_b64_tr_b16 a[244:245], v24 offset:8192\n\t" \
    "v_accvgpr_read_b32 v32, a56\n\t" \
    "v_accvgpr_read_b32 v33, a57\n\t" \
    "v_mfma_f32_16x16x32_bf16 v[88:91], a[212:215], a[156:159], v[88:91]\n\t" \
    "v_accvgpr_read_b32 v34, a58\n\t" \
    "v_accvgpr_read_b32 v35, a59\n\t" \
    "v_cvt_pk_bf16_f32 v252, v32, v33\n\t" \
    "v_mfma_f32_16x16x32_bf16 v[92:95], a[216:219], a[152:155], 0\n\t" \
    "ds_read_b64_tr_b16 a[246:247], v24 offset:9216\n\t" \
    "buffer_load_dwordx4 v26, %[rs], %[so] offen offset:0 lds\n\t" \
    "v_mfma_f32_16x16x32_bf16 v[92:95], a[220:223], a[156:159], v[92:95]\n\t" \
    "v_cvt_pk_bf16_f32 v253, v34, v35\n\t" \
    "v_accvgpr_read_b32 v32, a60\n\t" \
    "v_accvgpr_read_b32 v33, a61\n\t" \
    "v_mfma_f32_16x16x32_bf16 v[112:115], a[192:195], a[160:163], 0\n\t" \
    "ds_read_b64_tr_b16 a[248:249], v25 offset:0\n\t" \
    "buffer_load_dwordx4 v27, %[rs], %[so] offen offset:1024 lds\n\t" \
    "v_mfma_f32_16x16x32_bf16 v[112:115], a[196:199], a[164:167], v[112:115]\n\t" \
    "v_accvgpr_read_b32 v34, a62\n\t" \
    "v_accvgpr_read_b32 v35, a63\n\t" \
    "v_cvt_pk_bf16_f32 v254, v32, v33\n\t" \
    "v_mfma_f32_16x16x32_bf16 v[116:119], a[200:203], a[160:163], 0\n\t" \
    "ds_read_b64_tr_b16 a[250:251], v25 offset:1024\n\t" \
    "v_cvt_pk_bf16_f32 v255, v34, v35\n\t" \
    "v_mfma_f32_16x16x32_bf16 v[116:119], a[204:207], a[164:167], v[116:119]\n\t" \
    "buffer_load_dwordx4 v28, %[rs], %[so] offen offset:2048 lds\n\t" \
    "v_mfma_f32_16x16x32_bf16 v[120:123], a[208:211], a[160:163], 0\n\t" \
    "ds_read_b64_tr_b16 a[252:253], v25 offset:8192\n\t" \
    "s_xor_b32 %[t0], %[sd], 0x4c00\n\tbuffer_store_dwordx4 v[252:255], v30, %[rss], %[t0] offen " DAUXS "\n\t" \
    "v_mfma_f32_16x16x32_bf16 v[120:123], a[212:215], a[164:167], v[120:123]\n\t" \
    "buffer_load_dwordx4 v29, %[rs], %[so] offen offset:3072 lds\n\t" \
    "v_cvt_pk_bf16_f32 v248, v64, v65\n\t" \
    "v_mfma_f32_16x16x32_bf16 v[124:127], a[216:219], a[160:163], 0\n\t" \
    "ds_read_b64_tr_b16 a[254:255], v25 offset:9216\n\t" \
    "v_cvt_pk_bf16_f32 v249, v66, v67\n\t" \
    "v_cvt_pk_bf16_f32 v250, v68, v69\n\t" \
    "v_mfma_f32_16x16x32_bf16 v[124:127], a[220:223], a[164:167], v[124:127]\n\t" \
    "v_cvt_pk_bf16_f32 v251, v70, v71\n\t" \
    "s_xor_b32 %[t0], %[sd], 0x5000\n\tbuffer_store_dwordx4 v[248:251], v30, %[rss], %[t0] offen " DAUXS "\n\t" \
    "v_mfma_f32_16x16x32_bf16 v[144:147], a[192:195], a[168:171], 0\n\t" \
    "s_add_u32 m0, m0, 0x1000\n\t" \
    "v_cvt_pk_bf16_f32 v252, v72, v73\n\t" \
    "v_cvt_pk_bf16_f32 v253, v74, v75\n\t" \
    "v_mfma_f32_16x16x32_bf16 v[144:147], a[196:199], a[172:175], v[144:147]\n\t" \
    "v_cvt_pk_bf16_f32 v254, v76, v77\n\t" \
    "v_cvt_pk_bf16_f32 v255, v78, v79\n\t" \
    "v_mfma_f32_16x16x32_bf16 v[148:151], a[200:203], a[168:171], 0\n\t" \
    "s_add_u32 %[so], %[so], 0x40000\n\t" \
    "s_xor_b32 %[t0], %[sd], 0x5400\n\tbuffer_store_dwordx4 v[252:255], v30, %[rss], %[t0] offen " DAUXS "\n\t" \
    "v_mfma_f32_16x16x32_bf16 v[148:151], a[204:207], a[172:175], v[148:151]\n\t" \
    "v_cvt_pk_bf16_f32 v248, v96, v97\n\t" \
    "v_cvt_pk_bf16_f32 v249, v98, v99\n\t" \
    "v_cvt_pk_bf16_f32 v250, v100, v101\n\t" \
    "v_mfma_f32_16x16x32_bf16 v[152:155], a[208:211], a[168:171], 0\n\t" \
    "buffer_load_dwordx4 v26, %[rs], %[so] offen offset:0 lds\n\t" \
    "v_cvt_pk_bf16_f32 v251, v102, v103\n\t" \
    "v_mfma_f32_16x16x32_bf16 v[152:155], a[212:215], a[172:175], v[152:155]\n\t" \
    "s_xor_b32 %[t0], %[sd], 0x5800\n\tbuffer_store_dwordx4 v[248:251], v30, %[rss], %[t0] offen " DAUXS "\n\t" \
    "v_cvt_pk_bf16_f32 v252, v104, v105\n\t" \
    "v_mfma_f32_16x16x32_bf16 v[156:159], a[216:219], a[168:171], 0\n\t" \
    "v_cvt_pk_bf16_f32 v253, v106, v107\n\t" \
    "v_cvt_pk_bf16_f32 v254, v108, v109\n\t" \
    "v_cvt_pk_bf16_f32 v255, v110, v111\n\t" \
    "v_mfma_f32_16x16x32_bf16 v[156:159], a[220:223], a[172:175], v[156:159]\n\t" \
    "s_waitcnt lgkmcnt(8)\n\ts_nop 0\n\t" \
    "buffer_load_dwordx4 v27, %[rs], %[so] offen offset:1024 lds\n\t" \
    "v_mfma_f32_16x16x32_bf16 v[176:179], a[192:195], a[176:179], 0\n\t" \
    "s_xor_b32 %[t0], %[sd], 0x5c00\n\tbuffer_store_dwordx4 v[252:255], v30, %[rss], %[t0] offen " DAUXS "\n\t" \
    "v_cvt_pk_bf16_f32 v248, v128, v129\n\t" \
    "v_mfma_f32_16x16x32_bf16 v[176:179], a[196:199], a[180:183], v[176:179]\n\t" \
    "buffer_load_dwordx4 v28, %[rs], %[so] offen offset:2048 lds\n\t" \
    "v_cvt_pk_bf16_f32 v249, v130, v131\n\t" \
    "v_mfma_f32_16x16x32_bf16 v[180:183], a[200:203], a[176:179], 0\n\t" \
    "v_cvt_pk_bf16_f32 v250, v132, v133\n\t" \
    "v_cvt_pk_bf16_f32 v251, v134, v135\n\t" \
    "v_mfma_f32_16x16x32_bf16 v[180:183], a[204:207], a[180:183], v[180:183]\n\t" \
    "s_xor_b32 %[t0], %[sd], 0x6000\n\tbuffer_store_dwordx4 v[248:251], v30, %[rss], %[t0] offen " DAUXS "\n\t" \
    "v_cvt_pk_bf16_f32 v252, v136, v137\n\t" \
    "v_mfma_f32_16x16x32_bf16 v[184:187], a[208:211], a[176:179], 0\n\t" \
    "buffer_load_dwordx4 v29, %[rs], %[so] offen offset:3072 lds\n\t" \
    "v_cvt_pk_bf16_f32 v253, v138, v139\n\t" \
    "v_mfma_f32_16x16x32_bf16 v[184:187], a[212:215], a[180:183], v[184:187]\n\t" \
    "v_cvt_pk_bf16_f32 v254, v140, v141\n\t" \
    "v_cvt_pk_bf16_f32 v255, v142, v143\n\t" \
    "v_mfma_f32_16x16x32_bf16 v[188:191], a[216:219], a[176:179], 0\n\t" \
    "s_add_u32 m0, m0, 0x1000\n\t" \
    "s_xor_b32 %[t0], %[sd], 0x6400\n\tbuffer_store_dwordx4 v[252:255], v30, %[rss], %[t0] offen " DAUXS "\n\t" \
    "v_mfma_f32_16x16x32_bf16 v[188:191], a[220:223], a[180:183], v[188:191]\n\t" \
    "v_cvt_pk_bf16_f32 v248, v160, v161\n\t" \
    "v_cvt_pk_bf16_f32 v249, v162, v163\n\t" \
    "v_cvt_pk_bf16_f32 v250, v164, v165\n\t" \
    "v_mfma_f32_16x16x32_bf16 v[208:211], a[192:195], a[184:187], 0\n\t" \
    "v_cvt_pk_bf16_f32 v251, v166, v167\n\t" \
    "s_xor_b32 %[t0], %[sd], 0x6800\n\tbuffer_store_dwordx4 v[248:251], v30, %[rss], %[t0] offen " DAUXS "\n\t" \
    "v_mfma_f32_16x16x32_bf16 v[208:211], a[196:199], a[188:191], v[208:211]\n\t" \
    "s_cmp_eq_u32 m0, 0x18000\n\ts_cselect_b32 m0, 0, m0\n\ts_nop 0\n\t" \
    "v_cvt_pk_bf16_f32 v252, v168, v169\n\t" \
    "v_mfma_f32_16x16x32_bf16 v[212:215], a[200:203], a[184:187], 0\n\t" \
    "v_cvt_pk_bf16_f32 v253, v170, v171\n\t" \
    "v_cvt_pk_bf16_f32 v254, v172, v173\n\t" \
    "v_cvt_pk_bf16_f32 v255, v174, v175\n\t" \
    "v_mfma_f32_16x16x32_bf16 v[212:215], a[204:207], a[188:191], v[212:215]\n\t" \
    "s_add_u32 %[so], %[so], 0xfff40080\n\t" \
    "s_xor_b32 %[t0], %[sd], 0x6c00\n\tbuffer_store_dwordx4 v[252:255], v30, %[rss], %[t0] offen " DAUXS "\n\t" \
    "v_mfma_f32_16x16x32_bf16 v[216:219], a[208:211], a[184:187], 0\n\t" \
    "v_cvt_pk_bf16_f32 v248, v192, v193\n\t" \
    "v_cvt_pk_bf16_f32 v249, v194, v195\n\t" \
    "v_cvt_pk_bf16_f32 v250, v196, v197\n\t" \
    "v_mfma_f32_16x16x32_bf16 v[216:219], a[212:215], a[188:191], v[216:219]\n\t" \
    "v_cvt_pk_bf16_f32 v251, v198, v199\n\t" \
    "s_xor_b32 %[t0], %[sd], 0x7000\n\tbuffer_store_dwordx4 v[248:251], v30, %[rss], %[t0] offen " DAUXS "\n\t" \
    "v_mfma_f32_16x16x32_bf16 v[220:223], a[216:219], a[184:187], 0\n\t" \
    "v_cvt_pk_bf16_f32 v252, v200, v201\n\t" \
    "v_cvt_pk_bf16_f32 v253, v202, v203\n\t" \
    "v_cvt_pk_bf16_f32 v254, v204, v205\n\t" \
    "v_mfma_f32_16x16x32_bf16 v[220:223], a[220:223], a[188:191], v[220:223]\n\t" \
    "v_cvt_pk_bf16_f32 v255, v206, v207\n\t" \
    "s_xor_b32 %[t0], %[sd], 0x7400\n\tbuffer_store_dwordx4 v[252:255], v30, %[rss], %[t0] offen " DAUXS "\n\t" \
    "v_cvt_pk_bf16_f32 v248, v224, v225\n\t" \
    "v_cvt_pk_bf16_f32 v249, v226, v227\n\t" \
    "v_cvt_pk_bf16_f32 v250, v228, v229\n\t" \
    "v_cvt_pk_bf16_f32 v251, v230, v231\n\t" \
    "s_xor_b32 %[t0], %[sd], 0x7800\n\tbuffer_store_dwordx4 v[248:251], v30, %[rss], %[t0] offen " DAUXS "\n\t" \
    "v_cvt_pk_bf16_f32 v252, v232, v233\n\t" \
    "v_cvt_pk_bf16_f32 v253, v234, v235\n\t" \
    "v_cvt_pk_bf16_f32 v254, v236, v237\n\t" \
    "v_cvt_pk_bf16_f32 v255, v238, v239\n\t" \
    "s_xor_b32 %[t0], %[sd], 0x7c00\n\tbuffer_store_dwordx4 v[252:255], v30, %[rss], %[t0] offen " DAUXS "\n\t" \
    "s_waitcnt vmcnt(63) lgkmcnt(0)\n\ts_barrier\n\t" \
    "v_mfma_f32_16x16x32_bf16 a[16:19], a[224:227], a[128:131], 0\n\t" \
    "ds_read_b64_tr_b16 a[192:193], v22 offset:16384\n\t" \
    "v_mfma_f32_16x16x32_bf16 a[16:19], a[228:231], a[132:135], a[16:19]\n\t" \
    "buffer_load_dwordx4 v26, %[rs], %[so] offen offset:0 lds\n\t" \
    "v_mfma_f32_16x16x32_bf16 a[20:23], a[232:235], a[128:131], 0\n\t" \
    "ds_read_b64_tr_b16 a[194:195], v22 offset:17408\n\t" \
    "v_mfma_f32_16x16x32_bf16 a[20:23], a[236:239], a[132:135], a[20:23]\n\t" \
    "buffer_load_dwordx4 v27, %[rs], %[so] offen offset:1024 lds\n\t" \
    "v_mfma_f32_16x16x32_bf16 a[24:27], a[240:243], a[128:131], 0\n\t" \
    "ds_read_b64_tr_b16 a[196:197], v22 offset:24576\n\t" \
    "v_mfma_f32_16x16x32_bf16 a[24:27], a[244:247], a[132:135], a[24:27]\n\t" \
    "buffer_load_dwordx4 v28, %[rs], %[so] offen offset:2048 lds\n\t" \
    "v_mfma_f32_16x16x32_bf16 a[28:31], a[248:251], a[128:131], 0\n\t" \
    "ds_read_b64_tr_b16 a[198:199], v22 offset:25600\n\t" \
    "v_mfma_f32_16x16x32_bf16 a[28:31], a[252:255], a[132:135], a[28:31]\n\t" \
    "v_mfma_f32_16x16x32_bf16 a[48:51], a[224:227], a[136:139], 0\n\t" \
    "ds_read_b128 a[128:131], v20 offset:0\n\t" \
    "v_mfma_f32_16x16x32_bf16 a[48:51], a[228:231], a[140:143], a[48:51]\n\t" \
    "buffer_load_dwordx4 v29, %[rs], %[so] offen offset:3072 lds\n\t" \
    "v_mfma_f32_16x16x32_bf16 a[52:55], a[232:235], a[136:139], 0\n\t" \
    "ds_read_b128 a[132:135], v21 offset:0\n\t" \
    "s_add_u32 m0, m0, 0x1000\n\t" \
    "v_mfma_f32_16x16x32_bf16 a[52:55], a[236:239], a[140:143], a[52:55]\n\t" \
    "v_mfma_f32_16x16x32_bf16 a[56:59], a[240:243], a[136:139], 0\n\t" \
    "ds_read_b64_tr_b16 a[200:201], v23 offset:16384\n\t" \
    "v_mfma_f32_16x16x32_bf16 a[56:59], a[244:247], a[140:143], a[56:59]\n\t" \
    "s_add_u32 %[so], %[so], 0x40000\n\t" \
    "v_mfma_f32_16x16x32_bf16 a[60:63], a[248:251], a[136:139], 0\n\t" \
    "ds_read_b64_tr_b16 a[202:203], v23 offset:17408\n\t" \
    "v_mfma_f32_16x16x32_bf16 a[60:63], a[252:255], a[140:143], a[60:63]\n\t" \
    "buffer_load_dwordx4 v26, %[rs], %[so] offen offset:0 lds\n\t" \
    "v_mfma_f32_16x16x32_bf16 v[64:67], a[224:227], a[144:147], 0\n\t" \
    "ds_read_b128 a[136:139], v20 offset:2048\n\t" \
    "v_mfma_f32_16x16x32_bf16 v[64:67], a[228:231], a[148:151], v[64:67]\n\t" \
    "v_mfma_f32_16x16x32_bf16 v[68:71], a[232:235], a[144:147], 0\n\t" \
    "ds_read_b128 a[140:143], v21 offset:2048\n\t" \
    "v_mfma_f32_16x16x32_bf16 v[68:71], a[236:239], a[148:151], v[68:71]\n\t" \
    "buffer_load_dwordx4 v27, %[rs], %[so] offen offset:1024 lds\n\t" \
    "v_mfma_f32_16x16x32_bf16 v[72:75], a[240:243], a[144:147], 0\n\t" \
    "ds_read_b64_tr_b16 a[204:205], v23 offset:24576\n\t" \
    "v_mfma_f32_16x16x32_bf16 v[72:75], a[244:247], a[148:151], v[72:75]\n\t" \
    "buffer_load_dwordx4 v28, %[rs], %[so] offen offset:2048 lds\n\t" \
    "v_mfma_f32_16x16x32_bf16 v[76:79], a[248:251], a[144:147], 0\n\t" \
    "ds_read_b64_tr_b16 a[206:207], v23 offset:25600\n\t" \
    "v_mfma_f32_16x16x32_bf16 v[76:79], a[252:255], a[148:151], v[76:79]\n\t" \
    "buffer_load_dwordx4 v29, %[rs], %[so] offen offset:3072 lds\n\t" \
    "v_mfma_f32_16x16x32_bf16 v[96:99], a[224:227], a[152:155], 0\n\t" \
    "ds_read_b128 a[144:147], v20 offset:4096\n\t" \
    "v_mfma_f32_16x16x32_bf16 v[96:99], a[228:231], a[156:159], v[96:99]\n\t" \
    "s_add_u32 m0, m0, 0x1000\n\t" \
    "v_mfma_f32_16x16x32_bf16 v[100:103], a[232:235], a[152:155], 0\n\t" \
    "ds_read_b128 a[148:151], v21 offset:4096\n\t" \
    "v_mfma_f32_16x16x32_bf16 v[100:103], a[236:239], a[156:159], v[100:103]\n\t" \
    "s_add_u32 %[so], %[so], 0x40000\n\t" \
    "v_mfma_f32_16x16x32_bf16 v[104:107], a[240:243], a[152:155], 0\n\t" \
    "ds_read_b64_tr_b16 a[208:209], v24 offset:16384\n\t" \
    "v_mfma_f32_16x16x32_bf16 v[104:107], a[244:247], a[156:159], v[104:107]\n\t" \
    "v_mfma_f32_16x16x32_bf16 v[108:111], a[248:251], a[152:155], 0\n\t" \
    "ds_read_b64_tr_b16 a[210:211], v24 offset:17408\n\t" \
    "v_mfma_f32_16x16x32_bf16 v[108:111], a[252:255], a[156:159], v[108:111]\n\t" \
    "buffer_load_dwordx4 v26, %[rs], %[so] offen offset:0 lds\n\t" \
    "v_mfma_f32_16x16x32_bf16 v[128:131], a[224:227], a[160:163], 0\n\t" \
    "ds_read_b128 a[152:155], v20 offset:6144\n\t" \
    "v_mfma_f32_16x16x32_bf16 v[128:131], a[228:231], a[164:167], v[128:131]\n\t" \
    "buffer_load_dwordx4 v27, %[rs], %[so] offen offset:1024 lds\n\t" \
    "v_mfma_f32_16x16x32_bf16 v[132:135], a[232:235], a[160:163], 0\n\t" \
    "ds_read_b128 a[156:159], v21 offset:6144\n\t" \
    "v_mfma_f32_16x16x32_bf16 v[132:135], a[236:239], a[164:167], v[132:135]\n\t" \
    "buffer_load_dwordx4 v28, %[rs], %[so] offen offset:2048 lds\n\t" \
    "v_mfma_f32_16x16x32_bf16 v[136:139], a[240:243], a[160:163], 0\n\t" \
    "ds_read_b64_tr_b16 a[212:213], v24 offset:24576\n\t" \
    "v_mfma_f32_16x16x32_bf16 v[136:139], a[244:247], a[164:167], v[136:139]\n\t" \
    "buffer_load_dwordx4 v29, %[rs], %[so] offen offset:3072 lds\n\t" \
    "v_mfma_f32_16x16x32_bf16 v[140:143], a[248:251], a[160:163], 0\n\t" \
    "ds_read_b64_tr_b16 a[214:215], v24 offset:25600\n\t" \
    "v_mfma_f32_16x16x32_bf16 v[140:143], a[252:255], a[164:167], v[140:143]\n\t" \
    "v_mfma_f32_16x16x32_bf16 v[160:163], a[224:227], a[168:171], 0\n\t" \
    "ds_read_b128 a[160:163], v20 offset:8192\n\t" \
    "s_add_u32 m0, m0, 0x1000\n\t" \
    "v_mfma_f32_16x16x32_bf16 v[160:163], a[228:231], a[172:175], v[160:163]\n\t" \
    "v_mfma_f32_16x16x32_bf16 v[164:167], a[232:235], a[168:171], 0\n\t" \
    "ds_read_b128 a[164:167], v21 offset:8192\n\t" \
    "s_add_u32 %[so], %[so], 0x40000\n\t" \
    "v_mfma_f32_16x16x32_bf16 v[164:167], a[236:239], a[172:175], v[164:167]\n\t" \
    "v_mfma_f32_16x16x32_bf16 v[168:171], a[240:243], a[168:171], 0\n\t" \
    "ds_read_b64_tr_b16 a[216:217], v25 offset:16384\n\t" \
    "v_mfma_f32_16x16x32_bf16 v[168:171], a[244:247], a[172:175], v[168:171]\n\t" \
    "buffer_load_dwordx4 v26, %[rs], %[so] offen offset:0 lds\n\t" \
    "v_mfma_f32_16x16x32_bf16 v[172:175], a[248:251], a[168:171], 0\n\t" \
    "ds_read_b64_tr_b16 a[218:219], v25 offset:17408\n\t" \
    "v_mfma_f32_16x16x32_bf16 v[172:175], a[252:255], a[172:175], v[172:175]\n\t" \
    "buffer_load_dwordx4 v27, %[rs], %[so] offen offset:1024 lds\n\t" \
    "v_mfma_f32_16x16x32_bf16 v[192:195], a[224:227], a[176:179], 0\n\t" \
    "ds_read_b128 a[168:171], v20 offset:10240\n\t" \
    "v_mfma_f32_16x16x32_bf16 v[192:195], a[228:231], a[180:183], v[192:195]\n\t" \
    "buffer_load_dwordx4 v28, %[rs], %[so] offen offset:2048 lds\n\t" \
    "v_mfma_f32_16x16x32_bf16 v[196:199], a[232:235], a[176:179], 0\n\t" \
    "ds_read_b128 a[172:175], v21 offset:10240\n\t" \
    "v_mfma_f32_16x16x32_bf16 v[196:199], a[236:239], a[180:183], v[196:199]\n\t" \
    "v_mfma_f32_16x16x32_bf16 v[200:203], a[240:243], a[176:179], 0\n\t" \
    "ds_read_b64_tr_b16 a[220:221], v25 offset:24576\n\t" \
    "v_mfma_f32_16x16x32_bf16 v[200:203], a[244:247], a[180:183], v[200:203]\n\t" \
    "buffer_load_dwordx4 v29, %[rs], %[so] offen offset:3072 lds\n\t" \
    "v_mfma_f32_16x16x32_bf16 v[204:207], a[248:251], a[176:179], 0\n\t" \
    "ds_read_b64_tr_b16 a[222:223], v25 offset:25600\n\t" \
    "s_add_u32 m0, m0, 0x1000\n\t" \
    "v_mfma_f32_16x16x32_bf16 v[204:207], a[252:255], a[180:183], v[204:207]\n\t" \
    "v_mfma_f32_16x16x32_bf16 v[224:227], a[224:227], a[184:187], 0\n\t" \
    "v_add_u32 v23, 0x8000, v22\n\t" \
    "v_add_u32 v22, 0xffff0000, v22\n\t" \
    "v_mfma_f32_16x16x32_bf16 v[224:227], a[228:231], a[188:191], v[224:227]\n\t" \
    "s_add_u32 %[so], %[so], 0x3ff80\n\t" \
    "v_min_u32_e64 v22, v22, v23\n\t" \
    "v_mfma_f32_16x16x32_bf16 v[228:231], a[232:235], a[184:187], 0\n\t" \
    "v_xor_b32_e64 v23, 32, v22\n\t" \
    "v_xor_b32_e64 v24, 64, v22\n\t" \
    "v_mfma_f32_16x16x32_bf16 v[228:231], a[236:239], a[188:191], v[228:231]\n\t" \
    "s_cmp_eq_u32 %[rem], 4\n\ts_cselect_b32 %[so], %[nx], %[so]\n\t" \
    "v_mfma_f32_16x16x32_bf16 v[232:235], a[240:243], a[184:187], 0\n\t" \
    "v_xor_b32 v25, 0x60, v22\n\t" \
    "v_mfma_f32_16x16x32_bf16 v[232:235], a[244:247], a[188:191], v[232:235]\n\t" \
    "v_mfma_f32_16x16x32_bf16 v[236:239], a[248:251], a[184:187], 0\n\t" \
    "v_mfma_f32_16x16x32_bf16 v[236:239], a[252:255], a[188:191], v[236:239]\n\t" \
    "s_sub_u32 %[rem], %[rem], 1\n\t" \
    "s_waitcnt vmcnt(63) lgkmcnt(0)\n\ts_barrier\n\t" \
    ""
#define ASM_END_C \
    "v_mfma_f32_16x16x32_bf16 a[0:3], a[192:195], a[128:131], a[0:3]\n\t" \
    "ds_read_b128 a[176:179], v20 offset:12288\n\t" \
    "v_mfma_f32_16x16x32_bf16 a[0:3], a[196:199], a[132:135], a[0:3]\n\t" \
    "v_mfma_f32_16x16x32_bf16 a[4:7], a[200:203], a[128:131], a[4:7]\n\t" \
    "ds_read_b128 a[180:183], v21 offset:12288\n\t" \
    "v_mfma_f32_16x16x32_bf16 a[4:7], a[204:207], a[132:135], a[4:7]\n\t" \
    "v_mfma_f32_16x16x32_bf16 a[8:11], a[208:211], a[128:131], a[8:11]\n\t" \
    "ds_read_b128 a[184:187], v20 offset:14336\n\t" \
    "v_mfma_f32_16x16x32_bf16 a[8:11], a[212:215], a[132:135], a[8:11]\n\t" \
    "v_mfma_f32_16x16x32_bf16 a[12:15], a[216:219], a[128:131], a[12:15]\n\t" \
    "ds_read_b128 a[188:191], v21 offset:14336\n\t" \
    "v_mfma_f32_16x16x32_bf16 a[12:15], a[220:223], a[132:135], a[12:15]\n\t" \
    "v_mfma_f32_16x16x32_bf16 a[32:35], a[192:195], a[136:139], a[32:35]\n\t" \
    "ds_read_b64_tr_b16 a[224:225], v22 offset:0\n\t" \
    "v_mfma_f32_16x16x32_bf16 a[32:35], a[196:199], a[140:143], a[32:35]\n\t" \
    "v_xor_b32 v20, 0x38000, v20\n\t" \
    "v_xor_b32 v21, 0x38000, v21\n\t" \
    "v_mfma_f32_16x16x32_bf16 a[36:39], a[200:203], a[136:139], a[36:39]\n\t" \
    "ds_read_b64_tr_b16 a[226:227], v22 offset:1024\n\t" \
    "v_mfma_f32_16x16x32_bf16 a[36:39], a[204:207], a[140:143], a[36:39]\n\t" \
    "v_mfma_f32_16x16x32_bf16 a[40:43], a[208:211], a[136:139], a[40:43]\n\t" \
    "ds_read_b64_tr_b16 a[228:229], v22 offset:8192\n\t" \
    "v_mfma_f32_16x16x32_bf16 a[40:43], a[212:215], a[140:143], a[40:43]\n\t" \
    "v_mfma_f32_16x16x32_bf16 a[44:47], a[216:219], a[136:139], a[44:47]\n\t" \
    "ds_read_b64_tr_b16 a[230:231], v22 offset:9216\n\t" \
    "v_mfma_f32_16x16x32_bf16 a[44:47], a[220:223], a[140:143], a[44:47]\n\t" \
    "v_mfma_f32_16x16x32_bf16 v[48:51], a[192:195], a[144:147], v[48:51]\n\t" \
    "ds_read_b64_tr_b16 a[232:233], v23 offset:0\n\t" \
    "v_mfma_f32_16x16x32_bf16 v[48:51], a[196:199], a[148:151], v[48:51]\n\t" \
    "v_mfma_f32_16x16x32_bf16 v[52:55], a[200:203], a[144:147], v[52:55]\n\t" \
    "ds_read_b64_tr_b16 a[234:235], v23 offset:1024\n\t" \
    "v_mfma_f32_16x16x32_bf16 v[52:55], a[204:207], a[148:151], v[52:55]\n\t" \
    "v_mfma_f32_16x16x32_bf16 v[56:59], a[208:211], a[144:147], v[56:59]\n\t" \
    "ds_read_b64_tr_b16 a[236:237], v23 offset:8192\n\t" \
    "v_mfma_f32_16x16x32_bf16 v[56:59], a[212:215], a[148:151], v[56:59]\n\t" \
    "v_mfma_f32_16x16x32_bf16 v[60:63], a[216:219], a[144:147], v[60:63]\n\t" \
    "ds_read_b64_tr_b16 a[238:239], v23 offset:9216\n\t" \
    "v_mfma_f32_16x16x32_bf16 v[60:63], a[220:223], a[148:151], v[60:63]\n\t" \
    "v_mfma_f32_16x16x32_bf16 v[80:83], a[192:195], a[152:155], v[80:83]\n\t" \
    "ds_read_b64_tr_b16 a[240:241], v24 offset:0\n\t" \
    "v_mfma_f32_16x16x32_bf16 v[80:83], a[196:199], a[156:159], v[80:83]\n\t" \
    "v_mfma_f32_16x16x32_bf16 v[84:87], a[200:203], a[152:155], v[84:87]\n\t" \
    "ds_read_b64_tr_b16 a[242:243], v24 offset:1024\n\t" \
    "v_mfma_f32_16x16x32_bf16 v[84:87], a[204:207], a[156:159], v[84:87]\n\t" \
    "v_mfma_f32_16x16x32_bf16 v[88:91], a[208:211], a[152:155], v[88:91]\n\t" \
    "ds_read_b64_tr_b16 a[244:245], v24 offset:8192\n\t" \
    "v_mfma_f32_16x16x32_bf16 v[88:91], a[212:215], a[156:159], v[88:91]\n\t" \
    "v_mfma_f32_16x16x32_bf16 v[92:95], a[216:219], a[152:155], v[92:95]\n\t" \
    "ds_read_b64_tr_b16 a[246:247], v24 offset:9216\n\t" \
    "v_mfma_f32_16x16x32_bf16 v[92:95], a[220:223], a[156:159], v[92:95]\n\t" \
    "v_mfma_f32_16x16x32_bf16 v[112:115], a[192:195], a[160:163], v[112:115]\n\t" \
    "ds_read_b64_tr_b16 a[248:249], v25 offset:0\n\t" \
    "v_mfma_f32_16x16x32_bf16 v[112:115], a[196:199], a[164:167], v[112:115]\n\t" \
    "v_mfma_f32_16x16x32_bf16 v[116:119], a[200:203], a[160:163], v[116:119]\n\t" \
    "ds_read_b64_tr_b16 a[250:251], v25 offset:1024\n\t" \
    "v_mfma_f32_16x16x32_bf16 v[116:119], a[204:207], a[164:167], v[116:119]\n\t" \
    "v_mfma_f32_16x16x32_bf16 v[120:123], a[208:211], a[160:163], v[120:123]\n\t" \
    "ds_read_b64_tr_b16 a[252:253], v25 offset:8192\n\t" \
    "v_mfma_f32_16x16x32_bf16 v[120:123], a[212:215], a[164:167], v[120:123]\n\t" \
    "v_mfma_f32_16x16x32_bf16 v[124:127], a[216:219], a[160:163], v[124:127]\n\t" \
    "ds_read_b64_tr_b16 a[254:255], v25 offset:9216\n\t" \
    "v_mfma_f32_16x16x32_bf16 v[124:127], a[220:223], a[164:167], v[124:127]\n\t" \
    "v_mfma_f32_16x16x32_bf16 v[144:147], a[192:195], a[168:171], v[144:147]\n\t" \
    "v_mfma_f32_16x16x32_bf16 v[144:147], a[196:199], a[172:175], v[144:147]\n\t" \
    "v_mfma_f32_16x16x32_bf16 v[148:151], a[200:203], a[168:171], v[148:151]\n\t" \
    "v_mfma_f32_16x16x32_bf16 v[148:151], a[204:207], a[172:175], v[148:151]\n\t" \
    "v_mfma_f32_16x16x32_bf16 v[152:155], a[208:211], a[168:171], v[152:155]\n\t" \
    "v_mfma_f32_16x16x32_bf16 v[152:155], a[212:215], a[172:175], v[152:155]\n\t" \
    "v_mfma_f32_16x16x32_bf16 v[156:159], a[216:219], a[168:171], v[156:159]\n\t" \
    "v_mfma_f32_16x16x32_bf16 v[156:159], a[220:223], a[172:175], v[156:159]\n\t" \
    "s_waitcnt lgkmcnt(8)\n\ts_nop 0\n\t" \
    "v_mfma_f32_16x16x32_bf16 v[176:179], a[192:195], a[176:179], v[176:179]\n\t" \
    "v_mfma_f32_16x16x32_bf16 v[176:179], a[196:199], a[180:183], v[176:179]\n\t" \
    "v_mfma_f32_16x16x32_bf16 v[180:183], a[200:203], a[176:179], v[180:183]\n\t" \
    "v_mfma_f32_16x16x32_bf16 v[180:183], a[204:207], a[180:183], v[180:183]\n\t" \
    "v_mfma_f32_16x16x32_bf16 v[184:187], a[208:211], a[176:179], v[184:187]\n\t" \
    "v_mfma_f32_16x16x32_bf16 v[184:187], a[212:215], a[180:183], v[184:187]\n\t" \
    "v_mfma_f32_16x16x32_bf16 v[188:191], a[216:219], a[176:179], v[188:191]\n\t" \
    "v_mfma_f32_16x16x32_bf16 v[188:191], a[220:223], a[180:183], v[188:191]\n\t" \
    "v_mfma_f32_16x16x32_bf16 v[208:211], a[192:195], a[184:187], v[208:211]\n\t" \
    "v_mfma_f32_16x16x32_bf16 v[208:211], a[196:199], a[188:191], v[208:211]\n\t" \
    "v_mfma_f32_16x16x32_bf16 v[212:215], a[200:203], a[184:187], v[212:215]\n\t" \
    "v_mfma_f32_16x16x32_bf16 v[212:215], a[204:207], a[188:191], v[212:215]\n\t" \
    "v_mfma_f32_16x16x32_bf16 v[216:219], a[208:211], a[184:187], v[216:219]\n\t" \
    "v_mfma_f32_16x16x32_bf16 v[216:219], a[212:215], a[188:191], v[216:219]\n\t" \
    "v_mfma_f32_16x16x32_bf16 v[220:223], a[216:219], a[184:187], v[220:223]\n\t" \
    "v_mfma_f32_16x16x32_bf16 v[220:223], a[220:223], a[188:191], v[220:223]\n\t" \
    "s_waitcnt lgkmcnt(0)\n\ts_barrier\n\t" \
    "v_mfma_f32_16x16x32_bf16 a[16:19], a[224:227], a[128:131], a[16:19]\n\t" \
    "ds_read_b64_tr_b16 a[192:193], v22 offset:16384\n\t" \
    "v_mfma_f32_16x16x32_bf16 a[16:19], a[228:231], a[132:135], a[16:19]\n\t" \
    "v_accvgpr_read_b32 v32, a0\n\t" \
    "v_accvgpr_read_b32 v33, a1\n\t" \
    "v_mfma_f32_16x16x32_bf16 a[20:23], a[232:235], a[128:131], a[20:23]\n\t" \
    "ds_read_b64_tr_b16 a[194:195], v22 offset:17408\n\t" \
    "v_accvgpr_read_b32 v34, a2\n\t" \
    "v_accvgpr_read_b32 v35, a3\n\t" \
    "v_mfma_f32_16x16x32_bf16 a[20:23], a[236:239], a[132:135], a[20:23]\n\t" \
    "v_cvt_pk_bf16_f32 v248, v32, v33\n\t" \
    "v_cvt_pk_bf16_f32 v249, v34, v35\n\t" \
    "v_mfma_f32_16x16x32_bf16 a[24:27], a[240:243], a[128:131], a[24:27]\n\t" \
    "ds_read_b64_tr_b16 a[196:197], v22 offset:24576\n\t" \
    "v_accvgpr_read_b32 v32, a4\n\t" \
    "v_accvgpr_read_b32 v33, a5\n\t" \
    "v_mfma_f32_16x16x32_bf16 a[24:27], a[244:247], a[132:135], a[24:27]\n\t" \
    "v_accvgpr_read_b32 v34, a6\n\t" \
    "v_accvgpr_read_b32 v35, a7\n\t" \
    "v_mfma_f32_16x16x32_bf16 a[28:31], a[248:251], a[128:131], a[28:31]\n\t" \
    "ds_read_b64_tr_b16 a[198:199], v22 offset:25600\n\t" \
    "v_cvt_pk_bf16_f32 v250, v32, v33\n\t" \
    "v_cvt_pk_bf16_f32 v251, v34, v35\n\t" \
    "v_mfma_f32_16x16x32_bf16 a[28:31], a[252:255], a[132:135], a[28:31]\n\t" \
    "s_xor_b32 %[t0], %[sd], 0x0\n\tbuffer_store_dwordx4 v[248:251], v30, %[rss], %[t0] offen " DAUXS "\n\t" \
    "v_accvgpr_read_b32 v32, a8\n\t" \
    "v_mfma_f32_16x16x32_bf16 a[48:51], a[224:227], a[136:139], a[48:51]\n\t" \
    "ds_read_b128 a[128:131], v20 offset:0\n\t" \
    "v_accvgpr_read_b32 v33, a9\n\t" \
    "v_mfma_f32_16x16x32_bf16 a[48:51], a[228:231], a[140:143], a[48:51]\n\t" \
    "v_accvgpr_read_b32 v34, a10\n\t" \
    "v_accvgpr_read_b32 v35, a11\n\t" \
    "v_mfma_f32_16x16x32_bf16 a[52:55], a[232:235], a[136:139], a[52:55]\n\t" \
    "ds_read_b128 a[132:135], v21 offset:0\n\t" \
    "v_cvt_pk_bf16_f32 v252, v32, v33\n\t" \
    "v_cvt_pk_bf16_f32 v253, v34, v35\n\t" \
    "v_mfma_f32_16x16x32_bf16 a[52:55], a[236:239], a[140:143], a[52:55]\n\t" \
    "v_accvgpr_read_b32 v32, a12\n\t" \
    "v_accvgpr_read_b32 v33, a13\n\t" \
    "v_mfma_f32_16x16x32_bf16 a[56:59], a[240:243], a[136:139], a[56:59]\n\t" \
    "ds_read_b64_tr_b16 a[200:201], v23 offset:16384\n\t" \
    "v_accvgpr_read_b32 v34, a14\n\t" \
    "v_accvgpr_read_b32 v35, a15\n\t" \
    "v_mfma_f32_16x16x32_bf16 a[56:59], a[244:247], a[140:143], a[56:59]\n\t" \
    "v_cvt_pk_bf16_f32 v254, v32, v33\n\t" \
    "v_cvt_pk_bf16_f32 v255, v34, v35\n\t" \
    "v_mfma_f32_16x16x32_bf16 a[60:63], a[248:251], a[136:139], a[60:63]\n\t" \
    "ds_read_b64_tr_b16 a[202:203], v23 offset:17408\n\t" \
    "s_xor_b32 %[t0], %[sd], 0x400\n\tbuffer_store_dwordx4 v[252:255], v30, %[rss], %[t0] offen " DAUXS "\n\t" \
    "v_mfma_f32_16x16x32_bf16 a[60:63], a[252:255], a[140:143], a[60:63]\n\t" \
    "v_accvgpr_read_b32 v32, a32\n\t" \
    "v_accvgpr_read_b32 v33, a33\n\t" \
    "v_mfma_f32_16x16x32_bf16 v[64:67], a[224:227], a[144:147], v[64:67]\n\t" \
    "ds_read_b128 a[136:139], v20 offset:2048\n\t" \
    "v_accvgpr_read_b32 v34, a34\n\t" \
    "v_accvgpr_read_b32 v35, a35\n\t" \
    "v_mfma_f32_16x16x32_bf16 v[64:67], a[228:231], a[148:151], v[64:67]\n\t" \
    "v_cvt_pk_bf16_f32 v248, v32, v33\n\t" \
    "v_cvt_pk_bf16_f32 v249, v34, v35\n\t" \
    "v_mfma_f32_16x16x32_bf16 v[68:71], a[232:235], a[144:147], v[68:71]\n\t" \
    "ds_read_b128 a[140:143], v21 offset:2048\n\t" \
    "v_accvgpr_read_b32 v32, a36\n\t" \
    "v_accvgpr_read_b32 v33, a37\n\t" \
    "v_mfma_f32_16x16x32_bf16 v[68:71], a[236:239], a[148:151], v[68:71]\n\t" \
    "v_accvgpr_read_b32 v34, a38\n\t" \
    "v_accvgpr_read_b32 v35, a39\n\t" \
    "v_mfma_f32_16x16x32_bf16 v[72:75], a[240:243], a[144:147], v[72:75]\n\t" \
    "ds_read_b64_tr_b16 a[204:205], v23 offset:24576\n\t" \
    "v_cvt_pk_bf16_f32 v250, v32, v33\n\t" \
    "v_cvt_pk_bf16_f32 v251, v34, v35\n\t" \
    "v_mfma_f32_16x16x32_bf16 v[72:75], a[244:247], a[148:151], v[72:75]\n\t" \
    "s_xor_b32 %[t0], %[sd], 0x800\n\tbuffer_store_dwordx4 v[248:251], v30, %[rss], %[t0] offen " DAUXS "\n\t" \
    "v_accvgpr_read_b32 v32, a40\n\t" \
    "v_mfma_f32_16x16x32_bf16 v[76:79], a[248:251], a[144:147], v[76:79]\n\t" \
    "ds_read_b64_tr_b16 a[206:207], v23 offset:25600\n\t" \
    "v_accvgpr_read_b32 v33, a41\n\t" \
    "v_accvgpr_read_b32 v34, a42\n\t" \
    "v_mfma_f32_16x16x32_bf16 v[76:79], a[252:255], a[148:151], v[76:79]\n\t" \
    "buffer_load_dword v16, v17, %[rs], %[so] offen \n\t" \
    "v_accvgpr_read_b32 v35, a43\n\t" \
    "v_mfma_f32_16x16x32_bf16 v[96:99], a[224:227], a[152:155], v[96:99]\n\t" \
    "ds_read_b128 a[144:147], v20 offset:4096\n\t" \
    "v_cvt_pk_bf16_f32 v252, v32, v33\n\t" \
    "v_cvt_pk_bf16_f32 v253, v34, v35\n\t" \
    "v_mfma_f32_16x16x32_bf16 v[96:99], a[228:231], a[156:159], v[96:99]\n\t" \
    "s_add_u32 %[so], %[so], %[pfi]\n\ts_nop 0\n\t" \
    "v_accvgpr_read_b32 v32, a44\n\t" \
    "v_accvgpr_read_b32 v33, a45\n\t" \
    "v_mfma_f32_16x16x32_bf16 v[100:103], a[232:235], a[152:155], v[100:103]\n\t" \
    "ds_read_b128 a[148:151], v21 offset:4096\n\t" \
    "v_accvgpr_read_b32 v34, a46\n\t" \
    "v_accvgpr_read_b32 v35, a47\n\t" \
    "v_mfma_f32_16x16x32_bf16 v[100:103], a[236:239], a[156:159], v[100:103]\n\t" \
    "s_cmp_eq_u32 %[rem], %[psw]\n\ts_cselect_b32 %[so], %[nx], %[so]\n\t" \
    "v_cvt_pk_bf16_f32 v254, v32, v33\n\t" \
    "v_mfma_f32_16x16x32_bf16 v[104:107], a[240:243], a[152:155], v[104:107]\n\t" \
    "ds_read_b64_tr_b16 a[208:209], v24 offset:16384\n\t" \
    "v_cvt_pk_bf16_f32 v255, v34, v35\n\t" \
    "v_mfma_f32_16x16x32_bf16 v[104:107], a[244:247], a[156:159], v[104:107]\n\t" \
    "s_xor_b32 %[t0], %[sd], 0xc00\n\tbuffer_store_dwordx4 v[252:255], v30, %[rss], %[t0] offen " DAUXS "\n\t" \
    "v_cvt_pk_bf16_f32 v248, v48, v49\n\t" \
    "v_mfma_f32_16x16x32_bf16 v[108:111], a[248:251], a[152:155], v[108:111]\n\t" \
    "ds_read_b64_tr_b16 a[210:211], v24 offset:17408\n\t" \
    "v_cvt_pk_bf16_f32 v249, v50, v51\n\t" \
    "v_cvt_pk_bf16_f32 v250, v52, v53\n\t" \
    "v_mfma_f32_16x16x32_bf16 v[108:111], a[252:255], a[156:159], v[108:111]\n\t" \
    "v_cvt_pk_bf16_f32 v251, v54, v55\n\t" \
    "s_xor_b32 %[t0], %[sd], 0x1000\n\tbuffer_store_dwordx4 v[248:251], v30, %[rss], %[t0] offen " DAUXS "\n\t" \
    "v_mfma_f32_16x16x32_bf16 v[128:131], a[224:227], a[160:163], v[128:131]\n\t" \
    "ds_read_b128 a[152:155], v20 offset:6144\n\t" \
    "v_cvt_pk_bf16_f32 v252, v56, v57\n\t" \
    "v_cvt_pk_bf16_f32 v253, v58, v59\n\t" \
    "v_mfma_f32_16x16x32_bf16 v[128:131], a[228:231], a[164:167], v[128:131]\n\t" \
    "v_cvt_pk_bf16_f32 v254, v60, v61\n\t" \
    "v_cvt_pk_bf16_f32 v255, v62, v63\n\t" \
    "v_mfma_f32_16x16x32_bf16 v[132:135], a[232:235], a[160:163], v[132:135]\n\t" \
    "ds_read_b128 a[156:159], v21 offset:6144\n\t" \
    "s_xor_b32 %[t0], %[sd], 0x1400\n\tbuffer_store_dwordx4 v[252:255], v30, %[rss], %[t0] offen " DAUXS "\n\t" \
    "v_mfma_f32_16x16x32_bf16 v[132:135], a[236:239], a[164:167], v[132:135]\n\t" \
    "v_cvt_pk_bf16_f32 v248, v80, v81\n\t" \
    "v_cvt_pk_bf16_f32 v249, v82, v83\n\t" \
    "v_cvt_pk_bf16_f32 v250, v84, v85\n\t" \
    "v_mfma_f32_16x16x32_bf16 v[136:139], a[240:243], a[160:163], v[136:139]\n\t" \
    "ds_read_b64_tr_b16 a[212:213], v24 offset:24576\n\t" \
    "v_cvt_pk_bf16_f32 v251, v86, v87\n\t" \
    "v_mfma_f32_16x16x32_bf16 v[136:139], a[244:247], a[164:167], v[136:139]\n\t" \
    "s_xor_b32 %[t0], %[sd], 0x1800\n\tbuffer_store_dwordx4 v[248:251], v30, %[rss], %[t0] offen " DAUXS "\n\t" \
    "v_cvt_pk_bf16_f32 v252, v88, v89\n\t" \
    "v_mfma_f32_16x16x32_bf16 v[140:143], a[248:251], a[160:163], v[140:143]\n\t" \
    "ds_read_b64_tr_b16 a[214:215], v24 offset:25600\n\t" \
    "v_cvt_pk_bf16_f32 v253, v90, v91\n\t" \
    "v_cvt_pk_bf16_f32 v254, v92, v93\n\t" \
    "v_mfma_f32_16x16x32_bf16 v[140:143], a[252:255], a[164:167], v[140:143]\n\t" \
    "v_cvt_pk_bf16_f32 v255, v94, v95\n\t" \
    "s_xor_b32 %[t0], %[sd], 0x1c00\n\tbuffer_store_dwordx4 v[252:255], v30, %[rss], %[t0] offen " DAUXS "\n\t" \
    "v_mfma_f32_16x16x32_bf16 v[160:163], a[224:227], a[168:171], v[160:163]\n\t" \
    "ds_read_b128 a[160:163], v20 offset:8192\n\t" \
    "v_cvt_pk_bf16_f32 v248, v112, v113\n\t" \
    "v_cvt_pk_bf16_f32 v249, v114, v115\n\t" \
    "v_mfma_f32_16x16x32_bf16 v[160:163], a[228:231], a[172:175], v[160:163]\n\t" \
    "v_cvt_pk_bf16_f32 v250, v116, v117\n\t" \
    "v_cvt_pk_bf16_f32 v251, v118, v119\n\t" \
    "v_mfma_f32_16x16x32_bf16 v[164:167], a[232:235], a[168:171], v[164:167]\n\t" \
    "ds_read_b128 a[164:167], v21 offset:8192\n\t" \
    "s_xor_b32 %[t0], %[sd], 0x2000\n\tbuffer_store_dwordx4 v[248:251], v30, %[rss], %[t0] offen " DAUXS "\n\t" \
    "v_mfma_f32_16x16x32_bf16 v[164:167], a[236:239], a[172:175], v[164:167]\n\t" \
    "v_cvt_pk_bf16_f32 v252, v120, v121\n\t" \
    "v_cvt_pk_bf16_f32 v253, v122, v123\n\t" \
    "v_cvt_pk_bf16_f32 v254, v124, v125\n\t" \
    "v_mfma_f32_16x16x32_bf16 v[168:171], a[240:243], a[168:171], v[168:171]\n\t" \
    "ds_read_b64_tr_b16 a[216:217], v25 offset:16384\n\t" \
    "v_cvt_pk_bf16_f32 v255, v126, v127\n\t" \
    "v_mfma_f32_16x16x32_bf16 v[168:171], a[244:247], a[172:175], v[168:171]\n\t" \
    "s_xor_b32 %[t0], %[sd], 0x2400\n\tbuffer_store_dwordx4 v[252:255], v30, %[rss], %[t0] offen " DAUXS "\n\t" \
    "v_cvt_pk_bf16_f32 v248, v144, v145\n\t" \
    "v_mfma_f32_16x16x32_bf16 v[172:175], a[248:251], a[168:171], v[172:175]\n\t" \
    "ds_read_b64_tr_b16 a[218:219], v25 offset:17408\n\t" \
    "v_cvt_pk_bf16_f32 v249, v146, v147\n\t" \
    "v_cvt_pk_bf16_f32 v250, v148, v149\n\t" \
    "v_mfma_f32_16x16x32_bf16 v[172:175], a[252:255], a[172:175], v[172:175]\n\t" \
    "v_cvt_pk_bf16_f32 v251, v150, v151\n\t" \
    "s_xor_b32 %[t0], %[sd], 0x2800\n\tbuffer_store_dwordx4 v[248:251], v30, %[rss], %[t0] offen " DAUXS "\n\t" \
    "v_mfma_f32_16x16x32_bf16 v[192:195], a[224:227], a[176:179], v[192:195]\n\t" \
    "ds_read_b128 a[168:171], v20 offset:10240\n\t" \
    "v_cvt_pk_bf16_f32 v252, v152, v153\n\t" \
    "v_cvt_pk_bf16_f32 v253, v154, v155\n\t" \
    "v_mfma_f32_16x16x32_bf16 v[192:195], a[228:231], a[180:183], v[192:195]\n\t" \
    "v_cvt_pk_bf16_f32 v254, v156, v157\n\t" \
    "v_cvt_pk_bf16_f32 v255, v158, v159\n\t" \
    "v_mfma_f32_16x16x32_bf16 v[196:199], a[232:235], a[176:179], v[196:199]\n\t" \
    "ds_read_b128 a[172:175], v21 offset:10240\n\t" \
    "s_xor_b32 %[t0], %[sd], 0x2c00\n\tbuffer_store_dwordx4 v[252:255], v30, %[rss], %[t0] offen " DAUXS "\n\t" \
    "v_mfma_f32_16x16x32_bf16 v[196:199], a[236:239], a[180:183], v[196:199]\n\t" \
    "v_cvt_pk_bf16_f32 v248, v176, v177\n\t" \
    "v_cvt_pk_bf16_f32 v249, v178, v179\n\t" \
    "v_cvt_pk_bf16_f32 v250, v180, v181\n\t" \
    "v_mfma_f32_16x16x32_bf16 v[200:203], a[240:243], a[176:179], v[200:203]\n\t" \
    "ds_read_b64_tr_b16 a[220:221], v25 offset:24576\n\t" \
    "v_cvt_pk_bf16_f32 v251, v182, v183\n\t" \
    "v_mfma_f32_16x16x32_bf16 v[200:203], a[244:247], a[180:183], v[200:203]\n\t" \
    "s_xor_b32 %[t0], %[sd], 0x3000\n\tbuffer_store_dwordx4 v[248:251], v30, %[rss], %[t0] offen " DAUXS "\n\t" \
    "v_cvt_pk_bf16_f32 v252, v184, v185\n\t" \
    "v_mfma_f32_16x16x32_bf16 v[204:207], a[248:251], a[176:179], v[204:207]\n\t" \
    "ds_read_b64_tr_b16 a[222:223], v25 offset:25600\n\t" \
    "v_cvt_pk_bf16_f32 v253, v186, v187\n\t" \
    "v_cvt_pk_bf16_f32 v254, v188, v189\n\t" \
    "v_mfma_f32_16x16x32_bf16 v[204:207], a[252:255], a[180:183], v[204:207]\n\t" \
    "v_cvt_pk_bf16_f32 v255, v190, v191\n\t" \
    "s_xor_b32 %[t0], %[sd], 0x3400\n\tbuffer_store_dwordx4 v[252:255], v30, %[rss], %[t0] offen " DAUXS "\n\t" \
    "v_mfma_f32_16x16x32_bf16 v[224:227], a[224:227], a[184:187], v[224:227]\n\t" \
    "v_add_u32 v23, 0x8000, v22\n\t" \
    "v_add_u32 v22, 0xffff0000, v22\n\t" \
    "v_cvt_pk_bf16_f32 v248, v208, v209\n\t" \
    "v_mfma_f32_16x16x32_bf16 v[224:227], a[228:231], a[188:191], v[224:227]\n\t" \
    "v_min_u32_e64 v22, v22, v23\n\t" \
    "v_cvt_pk_bf16_f32 v249, v210, v211\n\t" \
    "v_cvt_pk_bf16_f32 v250, v212, v213\n\t" \
    "v_mfma_f32_16x16x32_bf16 v[228:231], a[232:235], a[184:187], v[228:231]\n\t" \
    "v_xor_b32_e64 v23, 32, v22\n\t" \
    "v_xor_b32_e64 v24, 64, v22\n\t" \
    "v_cvt_pk_bf16_f32 v251, v214, v215\n\t" \
    "v_mfma_f32_16x16x32_bf16 v[228:231], a[236:239], a[188:191], v[228:231]\n\t" \
    "v_xor_b32 v25, 0x60, v22\n\t" \
    "s_xor_b32 %[t0], %[sd], 0x3800\n\tbuffer_store_dwordx4 v[248:251], v30, %[rss], %[t0] offen " DAUXS "\n\t" \
    "v_mfma_f32_16x16x32_bf16 v[232:235], a[240:243], a[184:187], v[232:235]\n\t" \
    "v_cvt_pk_bf16_f32 v252, v216, v217\n\t" \
    "v_cvt_pk_bf16_f32 v253, v218, v219\n\t" \
    "v_cvt_pk_bf16_f32 v254, v220, v221\n\t" \
    "v_mfma_f32_16x16x32_bf16 v[232:235], a[244:247], a[188:191], v[232:235]\n\t" \
    "v_cvt_pk_bf16_f32 v255, v222, v223\n\t" \
    "s_xor_b32 %[t0], %[sd], 0x3c00\n\tbuffer_store_dwordx4 v[252:255], v30, %[rss], %[t0] offen " DAUXS "\n\t" \
    "v_mfma_f32_16x16x32_bf16 v[236:239], a[248:251], a[184:187], v[236:239]\n\t" \
    "v_mfma_f32_16x16x32_bf16 v[236:239], a[252:255], a[188:191], v[236:239]\n\t" \
    "s_sub_u32 %[rem], %[rem], 1\n\t" \
    "s_waitcnt lgkmcnt(0)\n\ts_barrier\n\t" \
    ""
#define ASM_START_C \
    "v_mfma_f32_16x16x32_bf16 a[0:3], a[192:195], a[128:131], 0\n\t" \
    "ds_read_b128 a[176:179], v20 offset:12288\n\t" \
    "v_mfma_f32_16x16x32_bf16 a[0:3], a[196:199], a[132:135], a[0:3]\n\t" \
    "v_accvgpr_read_b32 v32, a16\n\t" \
    "v_accvgpr_read_b32 v33, a17\n\t" \
    "v_mfma_f32_16x16x32_bf16 a[4:7], a[200:203], a[128:131], 0\n\t" \
    "ds_read_b128 a[180:183], v21 offset:12288\n\t" \
    "v_accvgpr_read_b32 v34, a18\n\t" \
    "v_accvgpr_read_b32 v35, a19\n\t" \
    "v_mfma_f32_16x16x32_bf16 a[4:7], a[204:207], a[132:135], a[4:7]\n\t" \
    "v_cvt_pk_bf16_f32 v248, v32, v33\n\t" \
    "v_cvt_pk_bf16_f32 v249, v34, v35\n\t" \
    "v_mfma_f32_16x16x32_bf16 a[8:11], a[208:211], a[128:131], 0\n\t" \
    "ds_read_b128 a[184:187], v20 offset:14336\n\t" \
    "v_accvgpr_read_b32 v32, a20\n\t" \
    "v_accvgpr_read_b32 v33, a21\n\t" \
    "v_mfma_f32_16x16x32_bf16 a[8:11], a[212:215], a[132:135], a[8:11]\n\t" \
    "v_accvgpr_read_b32 v34, a22\n\t" \
    "v_accvgpr_read_b32 v35, a23\n\t" \
    "v_mfma_f32_16x16x32_bf16 a[12:15], a[216:219], a[128:131], 0\n\t" \
    "ds_read_b128 a[188:191], v21 offset:14336\n\t" \
    "v_cvt_pk_bf16_f32 v250, v32, v33\n\t" \
    "v_cvt_pk_bf16_f32 v251, v34, v35\n\t" \
    "v_mfma_f32_16x16x32_bf16 a[12:15], a[220:223], a[132:135], a[12:15]\n\t" \
    "s_xor_b32 %[t0], %[sd], 0x4000\n\tbuffer_store_dwordx4 v[248:251], v30, %[rss], %[t0] offen " DAUXS "\n\t" \
    "v_accvgpr_read_b32 v32, a24\n\t" \
    "v_mfma_f32_16x16x32_bf16 a[32:35], a[192:195], a[136:139], 0\n\t" \
    "ds_read_b64_tr_b16 a[224:225], v22 offset:0\n\t" \
    "v_accvgpr_read_b32 v33, a25\n\t" \
    "v_mfma_f32_16x16x32_bf16 a[32:35], a[196:199], a[140:143], a[32:35]\n\t" \
    "v_xor_b32 v20, 0x38000, v20\n\t" \
    "v_xor_b32 v21, 0x38000, v21\n\t" \
    "v_accvgpr_read_b32 v34, a26\n\t" \
    "v_mfma_f32_16x16x32_bf16 a[36:39], a[200:203], a[136:139], 0\n\t" \
    "ds_read_b64_tr_b16 a[226:227], v22 offset:1024\n\t" \
    "v_accvgpr_read_b32 v35, a27\n\t" \
    "v_cvt_pk_bf16_f32 v252, v32, v33\n\t" \
    "v_mfma_f32_16x16x32_bf16 a[36:39], a[204:207], a[140:143], a[36:39]\n\t" \
    "v_cvt_pk_bf16_f32 v253, v34, v35\n\t" \
    "v_accvgpr_read_b32 v32, a28\n\t" \
    "v_accvgpr_read_b32 v33, a29\n\t" \
    "v_mfma_f32_16x16x32_bf16 a[40:43], a[208:211], a[136:139], 0\n\t" \
    "ds_read_b64_tr_b16 a[228:229], v22 offset:8192\n\t" \
    "v_accvgpr_read_b32 v34, a30\n\t" \
    "v_accvgpr_read_b32 v35, a31\n\t" \
    "v_mfma_f32_16x16x32_bf16 a[40:43], a[212:215], a[140:143], a[40:43]\n\t" \
    "v_cvt_pk_bf16_f32 v254, v32, v33\n\t" \
    "v_cvt_pk_bf16_f32 v255, v34, v35\n\t" \
    "v_mfma_f32_16x16x32_bf16 a[44:47], a[216:219], a[136:139], 0\n\t" \
    "ds_read_b64_tr_b16 a[230:231], v22 offset:9216\n\t" \
    "s_xor_b32 %[t0], %[sd], 0x4400\n\tbuffer_store_dwordx4 v[252:255], v30, %[rss], %[t0] offen " DAUXS "\n\t" \
    "v_mfma_f32_16x16x32_bf16 a[44:47], a[220:223], a[140:143], a[44:47]\n\t" \
    "v_accvgpr_read_b32 v32, a48\n\t" \
    "v_accvgpr_read_b32 v33, a49\n\t" \
    "v_mfma_f32_16x16x32_bf16 v[48:51], a[192:195], a[144:147], 0\n\t" \
    "ds_read_b64_tr_b16 a[232:233], v23 offset:0\n\t" \
    "v_accvgpr_read_b32 v34, a50\n\t" \
    "v_accvgpr_read_b32 v35, a51\n\t" \
    "v_mfma_f32_16x16x32_bf16 v[48:51], a[196:199], a[148:151], v[48:51]\n\t" \
    "v_cvt_pk_bf16_f32 v248, v32, v33\n\t" \
    "v_cvt_pk_bf16_f32 v249, v34, v35\n\t" \
    "v_mfma_f32_16x16x32_bf16 v[52:55], a[200:203], a[144:147], 0\n\t" \
    "ds_read_b64_tr_b16 a[234:235], v23 offset:1024\n\t" \
    "v_accvgpr_read_b32 v32, a52\n\t" \
    "v_accvgpr_read_b32 v33, a53\n\t" \
    "v_mfma_f32_16x16x32_bf16 v[52:55], a[204:207], a[148:151], v[52:55]\n\t" \
    "v_accvgpr_read_b32 v34, a54\n\t" \
    "v_accvgpr_read_b32 v35, a55\n\t" \
    "v_mfma_f32_16x16x32_bf16 v[56:59], a[208:211], a[144:147], 0\n\t" \
    "ds_read_b64_tr_b16 a[236:237], v23 offset:8192\n\t" \
    "v_cvt_pk_bf16_f32 v250, v32, v33\n\t" \
    "v_cvt_pk_bf16_f32 v251, v34, v35\n\t" \
    "v_mfma_f32_16x16x32_bf16 v[56:59], a[212:215], a[148:151], v[56:59]\n\t" \
    "s_xor_b32 %[t0], %[sd], 0x4800\n\tbuffer_store_dwordx4 v[248:251], v30, %[rss], %[t0] offen " DAUXS "\n\t" \
    "v_accvgpr_read_b32 v32, a56\n\t" \
    "v_mfma_f32_16x16x32_bf16 v[60:63], a[216:219], a[144:147], 0\n\t" \
    "ds_read_b64_tr_b16 a[238:239], v23 offset:9216\n\t" \
    "v_accvgpr_read_b32 v33, a57\n\t" \
    "v_accvgpr_read_b32 v34, a58\n\t" \
    "v_mfma_f32_16x16x32_bf16 v[60:63], a[220:223], a[148:151], v[60:63]\n\t" \
    "v_accvgpr_read_b32 v35, a59\n\t" \
    "v_mfma_f32_16x16x32_bf16 v[80:83], a[192:195], a[152:155], 0\n\t" \
    "ds_read_b64_tr_b16 a[240:241], v24 offset:0\n\t" \
    "v_cvt_pk_bf16_f32 v252, v32, v33\n\t" \
    "v_cvt_pk_bf16_f32 v253, v34, v35\n\t" \
    "v_mfma_f32_16x16x32_bf16 v[80:83], a[196:199], a[156:159], v[80:83]\n\t" \
    "v_accvgpr_read_b32 v32, a60\n\t" \
    "v_accvgpr_read_b32 v33, a61\n\t" \
    "v_mfma_f32_16x16x32_bf16 v[84:87], a[200:203], a[152:155], 0\n\t" \
    "ds_read_b64_tr_b16 a[242:243], v24 offset:1024\n\t" \
    "v_accvgpr_read_b32 v34, a62\n\t" \
    "v_accvgpr_read_b32 v35, a63\n\t" \
    "v_mfma_f32_16x16x32_bf16 v[84:87], a[204:207], a[156:159], v[84:87]\n\t" \
    "v_cvt_pk_bf16_f32 v254, v32, v33\n\t" \
    "v_cvt_pk_bf16_f32 v255, v34, v35\n\t" \
    "v_mfma_f32_16x16x32_bf16 v[88:91], a[208:211], a[152:155], 0\n\t" \
    "ds_read_b64_tr_b16 a[244:245], v24 offset:8192\n\t" \
    "s_xor_b32 %[t0], %[sd], 0x4c00\n\tbuffer_store_dwordx4 v[252:255], v30, %[rss], %[t0] offen " DAUXS "\n\t" \
    "v_mfma_f32_16x16x32_bf16 v[88:91], a[212:215], a[156:159], v[88:91]\n\t" \
    "v_cvt_pk_bf16_f32 v248, v64, v65\n\t" \
    "v_cvt_pk_bf16_f32 v249, v66, v67\n\t" \
    "v_cvt_pk_bf16_f32 v250, v68, v69\n\t" \
    "v_mfma_f32_16x16x32_bf16 v[92:95], a[216:219], a[152:155], 0\n\t" \
    "ds_read_b64_tr_b16 a[246:247], v24 offset:9216\n\t" \
    "v_cvt_pk_bf16_f32 v251, v70, v71\n\t" \
    "v_mfma_f32_16x16x32_bf16 v[92:95], a[220:223], a[156:159], v[92:95]\n\t" \
    "s_xor_b32 %[t0], %[sd], 0x5000\n\tbuffer_store_dwordx4 v[248:251], v30, %[rss], %[t0] offen " DAUXS "\n\t" \
    "v_cvt_pk_bf16_f32 v252, v72, v73\n\t" \
    "v_mfma_f32_16x16x32_bf16 v[112:115], a[192:195], a[160:163], 0\n\t" \
    "ds_read_b64_tr_b16 a[248:249], v25 offset:0\n\t" \
    "v_cvt_pk_bf16_f32 v253, v74, v75\n\t" \
    "v_cvt_pk_bf16_f32 v254, v76, v77\n\t" \
    "v_mfma_f32_16x16x32_bf16 v[112:115], a[196:199], a[164:167], v[112:115]\n\t" \
    "v_cvt_pk_bf16_f32 v255, v78, v79\n\t" \
    "s_xor_b32 %[t0], %[sd], 0x5400\n\tbuffer_store_dwordx4 v[252:255], v30, %[rss], %[t0] offen " DAUXS "\n\t" \
    "v_mfma_f32_16x16x32_bf16 v[116:119], a[200:203], a[160:163], 0\n\t" \
    "ds_read_b64_tr_b16 a[250:251], v25 offset:1024\n\t" \
    "v_cvt_pk_bf16_f32 v248, v96, v97\n\t" \
    "v_cvt_pk_bf16_f32 v249, v98, v99\n\t" \
    "v_mfma_f32_16x16x32_bf16 v[116:119], a[204:207], a[164:167], v[116:119]\n\t" \
    "v_cvt_pk_bf16_f32 v250, v100, v101\n\t" \
    "v_cvt_pk_bf16_f32 v251, v102, v103\n\t" \
    "v_mfma_f32_16x16x32_bf16 v[120:123], a[208:211], a[160:163], 0\n\t" \
    "ds_read_b64_tr_b16 a[252:253], v25 offset:8192\n\t" \
    "s_xor_b32 %[t0], %[sd], 0x5800\n\tbuffer_store_dwordx4 v[248:251], v30, %[rss], %[t0] offen " DAUXS "\n\t" \
    "v_mfma_f32_16x16x32_bf16 v[120:123], a[212:215], a[164:167], v[120:123]\n\t" \
    "v_cvt_pk_bf16_f32 v252, v104, v105\n\t" \
    "v_cvt_pk_bf16_f32 v253, v106, v107\n\t" \
    "v_cvt_pk_bf16_f32 v254, v108, v109\n\t" \
    "v_mfma_f32_16x16x32_bf16 v[124:127], a[216:219], a[160:163], 0\n\t" \
    "ds_read_b64_tr_b16 a[254:255], v25 offset:9216\n\t" \
    "v_cvt_pk_bf16_f32 v255, v110, v111\n\t" \
    "v_mfma_f32_16x16x32_bf16 v[124:127], a[220:223], a[164:167], v[124:127]\n\t" \
    "s_xor_b32 %[t0], %[sd], 0x5c00\n\tbuffer_store_dwordx4 v[252:255], v30, %[rss], %[t0] offen " DAUXS "\n\t" \
    "v_cvt_pk_bf16_f32 v248, v128, v129\n\t" \
    "v_mfma_f32_16x16x32_bf16 v[144:147], a[192:195], a[168:171], 0\n\t" \
    "v_cvt_pk_bf16_f32 v249, v130, v131\n\t" \
    "v_cvt_pk_bf16_f32 v250, v132, v133\n\t" \
    "v_mfma_f32_16x16x32_bf16 v[144:147], a[196:199], a[172:175], v[144:147]\n\t" \
    "v_cvt_pk_bf16_f32 v251, v134, v135\n\t" \
    "s_xor_b32 %[t0], %[sd], 0x6000\n\tbuffer_store_dwordx4 v[248:251], v30, %[rss], %[t0] offen " DAUXS "\n\t" \
    "v_mfma_f32_16x16x32_bf16 v[148:151], a[200:203], a[168:171], 0\n\t" \
    "v_cvt_pk_bf16_f32 v252, v136, v137\n\t" \
    "v_cvt_pk_bf16_f32 v253, v138, v139\n\t" \
    "v_mfma_f32_16x16x32_bf16 v[148:151], a[204:207], a[172:175], v[148:151]\n\t" \
    "v_cvt_pk_bf16_f32 v254, v140, v141\n\t" \
    "v_cvt_pk_bf16_f32 v255, v142, v143\n\t" \
    "v_mfma_f32_16x16x32_bf16 v[152:155], a[208:211], a[168:171], 0\n\t" \
    "s_xor_b32 %[t0], %[sd], 0x6400\n\tbuffer_store_dwordx4 v[252:255], v30, %[rss], %[t0] offen " DAUXS "\n\t" \
    "v_cvt_pk_bf16_f32 v248, v160, v161\n\t" \
    "v_mfma_f32_16x16x32_bf16 v[152:155], a[212:215], a[172:175], v[152:155]\n\t" \
    "v_cvt_pk_bf16_f32 v249, v162, v163\n\t" \
    "v_mfma_f32_16x16x32_bf16 v[156:159], a[216:219], a[168:171], 0\n\t" \
    "v_cvt_pk_bf16_f32 v250, v164, v165\n\t" \
    "v_cvt_pk_bf16_f32 v251, v166, v167\n\t" \
    "v_mfma_f32_16x16x32_bf16 v[156:159], a[220:223], a[172:175], v[156:159]\n\t" \
    "s_waitcnt lgkmcnt(8)\n\ts_nop 0\n\t" \
    "s_xor_b32 %[t0], %[sd], 0x6800\n\tbuffer_store_dwordx4 v[248:251], v30, %[rss], %[t0] offen " DAUXS "\n\t" \
    "v_mfma_f32_16x16x32_bf16 v[176:179], a[192:195], a[176:179], 0\n\t" \
    "v_cvt_pk_bf16_f32 v252, v168, v169\n\t" \
    "v_cvt_pk_bf16_f32 v253, v170, v171\n\t" \
    "v_cvt_pk_bf16_f32 v254, v172, v173\n\t" \
    "v_mfma_f32_16x16x32_bf16 v[176:179], a[196:199], a[180:183], v[176:179]\n\t" \
    "v_cvt_pk_bf16_f32 v255, v174, v175\n\t" \
    "s_xor_b32 %[t0], %[sd], 0x6c00\n\tbuffer_store_dwordx4 v[252:255], v30, %[rss], %[t0] offen " DAUXS "\n\t" \
    "v_mfma_f32_16x16x32_bf16 v[180:183], a[200:203], a[176:179], 0\n\t" \
    "v_cvt_pk_bf16_f32 v248, v192, v193\n\t" \
    "v_cvt_pk_bf16_f32 v249, v194, v195\n\t" \
    "v_mfma_f32_16x16x32_bf16 v[180:183], a[204:207], a[180:183], v[180:183]\n\t" \
    "v_cvt_pk_bf16_f32 v250, v196, v197\n\t" \
    "v_cvt_pk_bf16_f32 v251, v198, v199\n\t" \
    "v_mfma_f32_16x16x32_bf16 v[184:187], a[208:211], a[176:179], 0\n\t" \
    "s_xor_b32 %[t0], %[sd], 0x7000\n\tbuffer_store_dwordx4 v[248:251], v30, %[rss], %[t0] offen " DAUXS "\n\t" \
    "v_cvt_pk_bf16_f32 v252, v200, v201\n\t" \
    "v_mfma_f32_16x16x32_bf16 v[184:187], a[212:215], a[180:183], v[184:187]\n\t" \
    "v_cvt_pk_bf16_f32 v253, v202, v203\n\t" \
    "v_mfma_f32_16x16x32_bf16 v[188:191], a[216:219], a[176:179], 0\n\t" \
    "v_cvt_pk_bf16_f32 v254, v204, v205\n\t" \
    "v_cvt_pk_bf16_f32 v255, v206, v207\n\t" \
    "v_mfma_f32_16x16x32_bf16 v[188:191], a[220:223], a[180:183], v[188:191]\n\t" \
    "s_xor_b32 %[t0], %[sd], 0x7400\n\tbuffer_store_dwordx4 v[252:255], v30, %[rss], %[t0] offen " DAUXS "\n\t" \
    "v_cvt_pk_bf16_f32 v248, v224, v225\n\t" \
    "v_mfma_f32_16x16x32_bf16 v[208:211], a[192:195], a[184:187], 0\n\t" \
    "v_cvt_pk_bf16_f32 v249, v226, v227\n\t" \
    "v_cvt_pk_bf16_f32 v250, v228, v229\n\t" \
    "v_mfma_f32_16x16x32_bf16 v[208:211], a[196:199], a[188:191], v[208:211]\n\t" \
    "v_cvt_pk_bf16_f32 v251, v230, v231\n\t" \
    "s_xor_b32 %[t0], %[sd], 0x7800\n\tbuffer_store_dwordx4 v[248:251], v30, %[rss], %[t0] offen " DAUXS "\n\t" \
    "v_mfma_f32_16x16x32_bf16 v[212:215], a[200:203], a[184:187], 0\n\t" \
    "v_cvt_pk_bf16_f32 v252, v232, v233\n\t" \
    "v_cvt_pk_bf16_f32 v253, v234, v235\n\t" \
    "v_mfma_f32_16x16x32_bf16 v[212:215], a[204:207], a[188:191], v[212:215]\n\t" \
    "v_cvt_pk_bf16_f32 v254, v236, v237\n\t" \
    "v_cvt_pk_bf16_f32 v255, v238, v239\n\t" \
    "v_mfma_f32_16x16x32_bf16 v[216:219], a[208:211], a[184:187], 0\n\t" \
    "s_xor_b32 %[t0], %[sd], 0x7c00\n\tbuffer_store_dwordx4 v[252:255], v30, %[rss], %[t0] offen " DAUXS "\n\t" \
    "v_mfma_f32_16x16x32_bf16 v[216:219], a[212:215], a[188:191], v[216:219]\n\t" \
    "v_mfma_f32_16x16x32_bf16 v[220:223], a[216:219], a[184:187], 0\n\t" \
    "v_mfma_f32_16x16x32_bf16 v[220:223], a[220:223], a[188:191], v[220:223]\n\t" \
    "s_waitcnt lgkmcnt(0)\n\ts_barrier\n\t" \
    "v_mfma_f32_16x16x32_bf16 a[16:19], a[224:227], a[128:131], 0\n\t" \
    "ds_read_b64_tr_b16 a[192:193], v22 offset:16384\n\t" \
    "v_mfma_f32_16x16x32_bf16 a[16:19], a[228:231], a[132:135], a[16:19]\n\t" \
    "v_mfma_f32_16x16x32_bf16 a[20:23], a[232:235], a[128:131], 0\n\t" \
    "ds_read_b64_tr_b16 a[194:195], v22 offset:17408\n\t" \
    "v_mfma_f32_16x16x32_bf16 a[20:23], a[236:239], a[132:135], a[20:23]\n\t" \
    "v_mfma_f32_16x16x32_bf16 a[24:27], a[240:243], a[128:131], 0\n\t" \
    "ds_read_b64_tr_b16 a[196:197], v22 offset:24576\n\t" \
    "v_mfma_f32_16x16x32_bf16 a[24:27], a[244:247], a[132:135], a[24:27]\n\t" \
    "v_mfma_f32_16x16x32_bf16 a[28:31], a[248:251], a[128:131], 0\n\t" \
    "ds_read_b64_tr_b16 a[198:199], v22 offset:25600\n\t" \
    "v_mfma_f32_16x16x32_bf16 a[28:31], a[252:255], a[132:135], a[28:31]\n\t" \
    "v_mfma_f32_16x16x32_bf16 a[48:51], a[224:227], a[136:139], 0\n\t" \
    "ds_read_b128 a[128:131], v20 offset:0\n\t" \
    "v_mfma_f32_16x16x32_bf16 a[48:51], a[228:231], a[140:143], a[48:51]\n\t" \
    "v_mfma_f32_16x16x32_bf16 a[52:55], a[232:235], a[136:139], 0\n\t" \
    "ds_read_b128 a[132:135], v21 offset:0\n\t" \
    "v_mfma_f32_16x16x32_bf16 a[52:55], a[236:239], a[140:143], a[52:55]\n\t" \
    "v_mfma_f32_16x16x32_bf16 a[56:59], a[240:243], a[136:139], 0\n\t" \
    "ds_read_b64_tr_b16 a[200:201], v23 offset:16384\n\t" \
    "v_mfma_f32_16x16x32_bf16 a[56:59], a[244:247], a[140:143], a[56:59]\n\t" \
    "v_mfma_f32_16x16x32_bf16 a[60:63], a[248:251], a[136:139], 0\n\t" \
    "ds_read_b64_tr_b16 a[202:203], v23 offset:17408\n\t" \
    "v_mfma_f32_16x16x32_bf16 a[60:63], a[252:255], a[140:143], a[60:63]\n\t" \
    "v_mfma_f32_16x16x32_bf16 v[64:67], a[224:227], a[144:147], 0\n\t" \
    "ds_read_b128 a[136:139], v20 offset:2048\n\t" \
    "v_mfma_f32_16x16x32_bf16 v[64:67], a[228:231], a[148:151], v[64:67]\n\t" \
    "v_mfma_f32_16x16x32_bf16 v[68:71], a[232:235], a[144:147], 0\n\t" \
    "ds_read_b128 a[140:143], v21 offset:2048\n\t" \
    "v_mfma_f32_16x16x32_bf16 v[68:71], a[236:239], a[148:151], v[68:71]\n\t" \
    "v_mfma_f32_16x16x32_bf16 v[72:75], a[240:243], a[144:147], 0\n\t" \
    "ds_read_b64_tr_b16 a[204:205], v23 offset:24576\n\t" \
    "v_mfma_f32_16x16x32_bf16 v[72:75], a[244:247], a[148:151], v[72:75]\n\t" \
    "v_mfma_f32_16x16x32_bf16 v[76:79], a[248:251], a[144:147], 0\n\t" \
    "ds_read_b64_tr_b16 a[206:207], v23 offset:25600\n\t" \
    "v_mfma_f32_16x16x32_bf16 v[76:79], a[252:255], a[148:151], v[76:79]\n\t" \
    "buffer_load_dword v16, v17, %[rs], %[so] offen \n\t" \
    "v_mfma_f32_16x16x32_bf16 v[96:99], a[224:227], a[152:155], 0\n\t" \
    "ds_read_b128 a[144:147], v20 offset:4096\n\t" \
    "v_mfma_f32_16x16x32_bf16 v[96:99], a[228:231], a[156:159], v[96:99]\n\t" \
    "s_add_u32 %[so], %[so], %[pfi]\n\ts_nop 0\n\t" \
    "v_mfma_f32_16x16x32_bf16 v[100:103], a[232:235], a[152:155], 0\n\t" \
    "ds_read_b128 a[148:151], v21 offset:4096\n\t" \
    "v_mfma_f32_16x16x32_bf16 v[100:103], a[236:239], a[156:159], v[100:103]\n\t" \
    "s_cmp_eq_u32 %[rem], %[psw]\n\ts_cselect_b32 %[so], %[nx], %[so]\n\t" \
    "v_mfma_f32_16x16x32_bf16 v[104:107], a[240:243], a[152:155], 0\n\t" \
    "ds_read_b64_tr_b16 a[208:209], v24 offset:16384\n\t" \
    "v_mfma_f32_16x16x32_bf16 v[104:107], a[244:247], a[156:159], v[104:107]\n\t" \
    "v_mfma_f32_16x16x32_bf16 v[108:111], a[248:251], a[152:155], 0\n\t" \
    "ds_read_b64_tr_b16 a[210:211], v24 offset:17408\n\t" \
    "v_mfma_f32_16x16x32_bf16 v[108:111], a[252:255], a[156:159], v[108:111]\n\t" \
    "v_mfma_f32_16x16x32_bf16 v[128:131], a[224:227], a[160:163], 0\n\t" \
    "ds_read_b128 a[152:155], v20 offset:6144\n\t" \
    "v_mfma_f32_16x16x32_bf16 v[128:131], a[228:231], a[164:167], v[128:131]\n\t" \
    "v_mfma_f32_16x16x32_bf16 v[132:135], a[232:235], a[160:163], 0\n\t" \
    "ds_read_b128 a[156:159], v21 offset:6144\n\t" \
    "v_mfma_f32_16x16x32_bf16 v[132:135], a[236:239], a[164:167], v[132:135]\n\t" \
    "v_mfma_f32_16x16x32_bf16 v[136:139], a[240:243], a[160:163], 0\n\t" \
    "ds_read_b64_tr_b16 a[212:213], v24 offset:24576\n\t" \
    "v_mfma_f32_16x16x32_bf16 v[136:139], a[244:247], a[164:167], v[136:139]\n\t" \
    "v_mfma_f32_16x16x32_bf16 v[140:143], a[248:251], a[160:163], 0\n\t" \
    "ds_read_b64_tr_b16 a[214:215], v24 offset:25600\n\t" \
    "v_mfma_f32_16x16x32_bf16 v[140:143], a[252:255], a[164:167], v[140:143]\n\t" \
    "v_mfma_f32_16x16x32_bf16 v[160:163], a[224:227], a[168:171], 0\n\t" \
    "ds_read_b128 a[160:163], v20 offset:8192\n\t" \
    "v_mfma_f32_16x16x32_bf16 v[160:163], a[228:231], a[172:175], v[160:163]\n\t" \
    "v_mfma_f32_16x16x32_bf16 v[164:167], a[232:235], a[168:171], 0\n\t" \
    "ds_read_b128 a[164:167], v21 offset:8192\n\t" \
    "v_mfma_f32_16x16x32_bf16 v[164:167], a[236:239], a[172:175], v[164:167]\n\t" \
    "v_mfma_f32_16x16x32_bf16 v[168:171], a[240:243], a[168:171], 0\n\t" \
    "ds_read_b64_tr_b16 a[216:217], v25 offset:16384\n\t" \
    "v_mfma_f32_16x16x32_bf16 v[168:171], a[244:247], a[172:175], v[168:171]\n\t" \
    "v_mfma_f32_16x16x32_bf16 v[172:175], a[248:251], a[168:171], 0\n\t" \
    "ds_read_b64_tr_b16 a[218:219], v25 offset:17408\n\t" \
    "v_mfma_f32_16x16x32_bf16 v[172:175], a[252:255], a[172:175], v[172:175]\n\t" \
    "v_mfma_f32_16x16x32_bf16 v[192:195], a[224:227], a[176:179], 0\n\t" \
    "ds_read_b128 a[168:171], v20 offset:10240\n\t" \
    "v_mfma_f32_16x16x32_bf16 v[192:195], a[228:231], a[180:183], v[192:195]\n\t" \
    "v_mfma_f32_16x16x32_bf16 v[196:199], a[232:235], a[176:179], 0\n\t" \
    "ds_read_b128 a[172:175], v21 offset:10240\n\t" \
    "v_mfma_f32_16x16x32_bf16 v[196:199], a[236:239], a[180:183], v[196:199]\n\t" \
    "v_mfma_f32_16x16x32_bf16 v[200:203], a[240:243], a[176:179], 0\n\t" \
    "ds_read_b64_tr_b16 a[220:221], v25 offset:24576\n\t" \
    "v_mfma_f32_16x16x32_bf16 v[200:203], a[244:247], a[180:183], v[200:203]\n\t" \
    "v_mfma_f32_16x16x32_bf16 v[204:207], a[248:251], a[176:179], 0\n\t" \
    "ds_read_b64_tr_b16 a[222:223], v25 offset:25600\n\t" \
    "v_mfma_f32_16x16x32_bf16 v[204:207], a[252:255], a[180:183], v[204:207]\n\t" \
    "v_mfma_f32_16x16x32_bf16 v[224:227], a[224:227], a[184:187], 0\n\t" \
    "v_add_u32 v23, 0x8000, v22\n\t" \
    "v_add_u32 v22, 0xffff0000, v22\n\t" \
    "v_mfma_f32_16x16x32_bf16 v[224:227], a[228:231], a[188:191], v[224:227]\n\t" \
    "v_min_u32_e64 v22, v22, v23\n\t" \
    "v_mfma_f32_16x16x32_bf16 v[228:231], a[232:235], a[184:187], 0\n\t" \
    "v_xor_b32_e64 v23, 32, v22\n\t" \
    "v_xor_b32_e64 v24, 64, v22\n\t" \
    "v_mfma_f32_16x16x32_bf16 v[228:231], a[236:239], a[188:191], v[228:231]\n\t" \
    "v_xor_b32 v25, 0x60, v22\n\t" \
    "v_mfma_f32_16x16x32_bf16 v[232:235], a[240:243], a[184:187], 0\n\t" \
    "v_mfma_f32_16x16x32_bf16 v[232:235], a[244:247], a[188:191], v[232:235]\n\t" \
    "v_mfma_f32_16x16x32_bf16 v[236:239], a[248:251], a[184:187], 0\n\t" \
    "v_mfma_f32_16x16x32_bf16 v[236:239], a[252:255], a[188:191], v[236:239]\n\t" \
    "s_sub_u32 %[rem], %[rem], 1\n\t" \
    "s_waitcnt lgkmcnt(0)\n\ts_barrier\n\t" \
    ""
#define ASM_BG \
    "v_readlane_b32 %[ld], v14, %[c]\n\t" \
    "v_readlane_b32 %[oo], v15, %[c]\n\t" \
    "v_readfirstlane_b32 %[go], v18\n\t" \
    "v_readfirstlane_b32 %[t0], v44\n\t" \
    "s_bitcmp1_b32 %[c], 0\n\ts_cselect_b32 %[t0], %[t0], %[go]\n\t" \
    "v_readfirstlane_b32 %[go], v19\n\t" \
    "s_cmp_ge_u32 %[ph], 2\n\ts_cselect_b32 %[go], %[t0], %[go]\n\t" \
    "s_cmp_ge_u32 %[go], %[thr]\n\ts_cselect_b32 %[go], %[zls], 0\n\t" \
    "s_cmp_le_i32 %[gap], 0\n\ts_cselect_b32 %[go], %[go], 0\n\t" \
    "s_cmp_lg_u32 %[go], 0\n\t" \
    "s_cbranch_scc1 20f\n\t" \
    "10:\n\t" \
    "v_mfma_f32_16x16x32_bf16 a[0:3], a[192:195], a[128:131], a[0:3]\n\t" \
    "ds_read_b128 a[176:179], v20 offset:12288\n\t" \
    "v_mfma_f32_16x16x32_bf16 a[0:3], a[196:199], a[132:135], a[0:3]\n\t" \
    "buffer_load_dword v19, off, %[rss], %[pw] sc1\n\t" \
    "v_mfma_f32_16x16x32_bf16 a[4:7], a[200:203], a[128:131], a[4:7]\n\t" \
    "ds_read_b128 a[180:183], v21 offset:12288\n\t" \
    "v_mfma_f32_16x16x32_bf16 a[4:7], a[204:207], a[132:135], a[4:7]\n\t" \
    "v_mfma_f32_16x16x32_bf16 a[8:11], a[208:211], a[128:131], a[8:11]\n\t" \
    "ds_read_b128 a[184:187], v20 offset:14336\n\t" \
    "v_mfma_f32_16x16x32_bf16 a[8:11], a[212:215], a[132:135], a[8:11]\n\t" \
    "v_mfma_f32_16x16x32_bf16 a[12:15], a[216:219], a[128:131], a[12:15]\n\t" \
    "ds_read_b128 a[188:191], v21 offset:14336\n\t" \
    "v_mfma_f32_16x16x32_bf16 a[12:15], a[220:223], a[132:135], a[12:15]\n\t" \
    "v_mfma_f32_16x16x32_bf16 a[32:35], a[192:195], a[136:139], a[32:35]\n\t" \
    "ds_read_b64_tr_b16 a[224:225], v22 offset:0\n\t" \
    "v_mfma_f32_16x16x32_bf16 a[32:35], a[196:199], a[140:143], a[32:35]\n\t" \
    "v_xor_b32 v20, 0x38000, v20\n\t" \
    "v_xor_b32 v21, 0x38000, v21\n\t" \
    "v_mfma_f32_16x16x32_bf16 a[36:39], a[200:203], a[136:139], a[36:39]\n\t" \
    "ds_read_b64_tr_b16 a[226:227], v22 offset:1024\n\t" \
    "v_mfma_f32_16x16x32_bf16 a[36:39], a[204:207], a[140:143], a[36:39]\n\t" \
    "v_mfma_f32_16x16x32_bf16 a[40:43], a[208:211], a[136:139], a[40:43]\n\t" \
    "ds_read_b64_tr_b16 a[228:229], v22 offset:8192\n\t" \
    "v_mfma_f32_16x16x32_bf16 a[40:43], a[212:215], a[140:143], a[40:43]\n\t" \
    "v_mfma_f32_16x16x32_bf16 a[44:47], a[216:219], a[136:139], a[44:47]\n\t" \
    "ds_read_b64_tr_b16 a[230:231], v22 offset:9216\n\t" \
    "v_mfma_f32_16x16x32_bf16 a[44:47], a[220:223], a[140:143], a[44:47]\n\t" \
    "v_mfma_f32_16x16x32_bf16 v[48:51], a[192:195], a[144:147], v[48:51]\n\t" \
    "ds_read_b64_tr_b16 a[232:233], v23 offset:0\n\t" \
    "v_mfma_f32_16x16x32_bf16 v[48:51], a[196:199], a[148:151], v[48:51]\n\t" \
    "v_mfma_f32_16x16x32_bf16 v[52:55], a[200:203], a[144:147], v[52:55]\n\t" \
    "ds_read_b64_tr_b16 a[234:235], v23 offset:1024\n\t" \
    "v_mfma_f32_16x16x32_bf16 v[52:55], a[204:207], a[148:151], v[52:55]\n\t" \
    "v_mfma_f32_16x16x32_bf16 v[56:59], a[208:211], a[144:147], v[56:59]\n\t" \
    "ds_read_b64_tr_b16 a[236:237], v23 offset:8192\n\t" \
    "v_mfma_f32_16x16x32_bf16 v[56:59], a[212:215], a[148:151], v[56:59]\n\t" \
    "v_mfma_f32_16x16x32_bf16 v[60:63], a[216:219], a[144:147], v[60:63]\n\t" \
    "ds_read_b64_tr_b16 a[238:239], v23 offset:9216\n\t" \
    "v_mfma_f32_16x16x32_bf16 v[60:63], a[220:223], a[148:151], v[60:63]\n\t" \
    "v_mfma_f32_16x16x32_bf16 v[80:83], a[192:195], a[152:155], v[80:83]\n\t" \
    "ds_read_b64_tr_b16 a[240:241], v24 offset:0\n\t" \
    "v_mfma_f32_16x16x32_bf16 v[80:83], a[196:199], a[156:159], v[80:83]\n\t" \
    "v_mfma_f32_16x16x32_bf16 v[84:87], a[200:203], a[152:155], v[84:87]\n\t" \
    "ds_read_b64_tr_b16 a[242:243], v24 offset:1024\n\t" \
    "v_mfma_f32_16x16x32_bf16 v[84:87], a[204:207], a[156:159], v[84:87]\n\t" \
    "v_mfma_f32_16x16x32_bf16 v[88:91], a[208:211], a[152:155], v[88:91]\n\t" \
    "ds_read_b64_tr_b16 a[244:245], v24 offset:8192\n\t" \
    "v_mfma_f32_16x16x32_bf16 v[88:91], a[212:215], a[156:159], v[88:91]\n\t" \
    "v_mfma_f32_16x16x32_bf16 v[92:95], a[216:219], a[152:155], v[92:95]\n\t" \
    "ds_read_b64_tr_b16 a[246:247], v24 offset:9216\n\t" \
    "v_mfma_f32_16x16x32_bf16 v[92:95], a[220:223], a[156:159], v[92:95]\n\t" \
    "v_mfma_f32_16x16x32_bf16 v[112:115], a[192:195], a[160:163], v[112:115]\n\t" \
    "ds_read_b64_tr_b16 a[248:249], v25 offset:0\n\t" \
    "v_mfma_f32_16x16x32_bf16 v[112:115], a[196:199], a[164:167], v[112:115]\n\t" \
    "v_mfma_f32_16x16x32_bf16 v[116:119], a[200:203], a[160:163], v[116:119]\n\t" \
    "ds_read_b64_tr_b16 a[250:251], v25 offset:1024\n\t" \
    "v_mfma_f32_16x16x32_bf16 v[116:119], a[204:207], a[164:167], v[116:119]\n\t" \
    "v_mfma_f32_16x16x32_bf16 v[120:123], a[208:211], a[160:163], v[120:123]\n\t" \
    "ds_read_b64_tr_b16 a[252:253], v25 offset:8192\n\t" \
    "v_mfma_f32_16x16x32_bf16 v[120:123], a[212:215], a[164:167], v[120:123]\n\t" \
    "v_mfma_f32_16x16x32_bf16 v[124:127], a[216:219], a[160:163], v[124:127]\n\t" \
    "ds_read_b64_tr_b16 a[254:255], v25 offset:9216\n\t" \
    "v_mfma_f32_16x16x32_bf16 v[124:127], a[220:223], a[164:167], v[124:127]\n\t" \
    "v_mfma_f32_16x16x32_bf16 v[144:147], a[192:195], a[168:171], v[144:147]\n\t" \
    "v_mfma_f32_16x16x32_bf16 v[144:147], a[196:199], a[172:175], v[144:147]\n\t" \
    "v_mfma_f32_16x16x32_bf16 v[148:151], a[200:203], a[168:171], v[148:151]\n\t" \
    "v_mfma_f32_16x16x32_bf16 v[148:151], a[204:207], a[172:175], v[148:151]\n\t" \
    "v_mfma_f32_16x16x32_bf16 v[152:155], a[208:211], a[168:171], v[152:155]\n\t" \
    "v_mfma_f32_16x16x32_bf16 v[152:155], a[212:215], a[172:175], v[152:155]\n\t" \
    "v_mfma_f32_16x16x32_bf16 v[156:159], a[216:219], a[168:171], v[156:159]\n\t" \
    "v_mfma_f32_16x16x32_bf16 v[156:159], a[220:223], a[172:175], v[156:159]\n\t" \
    "s_waitcnt lgkmcnt(8)\n\ts_nop 0\n\t" \
    "v_mfma_f32_16x16x32_bf16 v[176:179], a[192:195], a[176:179], v[176:179]\n\t" \
    "v_mfma_f32_16x16x32_bf16 v[176:179], a[196:199], a[180:183], v[176:179]\n\t" \
    "v_mfma_f32_16x16x32_bf16 v[180:183], a[200:203], a[176:179], v[180:183]\n\t" \
    "v_mfma_f32_16x16x32_bf16 v[180:183], a[204:207], a[180:183], v[180:183]\n\t" \
    "v_mfma_f32_16x16x32_bf16 v[184:187], a[208:211], a[176:179], v[184:187]\n\t" \
    "v_mfma_f32_16x16x32_bf16 v[184:187], a[212:215], a[180:183], v[184:187]\n\t" \
    "v_mfma_f32_16x16x32_bf16 v[188:191], a[216:219], a[176:179], v[188:191]\n\t" \
    "v_mfma_f32_16x16x32_bf16 v[188:191], a[220:223], a[180:183], v[188:191]\n\t" \
    "v_mfma_f32_16x16x32_bf16 v[208:211], a[192:195], a[184:187], v[208:211]\n\t" \
    "v_mfma_f32_16x16x32_bf16 v[208:211], a[196:199], a[188:191], v[208:211]\n\t" \
    "v_mfma_f32_16x16x32_bf16 v[212:215], a[200:203], a[184:187], v[212:215]\n\t" \
    "v_mfma_f32_16x16x32_bf16 v[212:215], a[204:207], a[188:191], v[212:215]\n\t" \
    "v_mfma_f32_16x16x32_bf16 v[216:219], a[208:211], a[184:187], v[216:219]\n\t" \
    "v_mfma_f32_16x16x32_bf16 v[216:219], a[212:215], a[188:191], v[216:219]\n\t" \
    "v_mfma_f32_16x16x32_bf16 v[220:223], a[216:219], a[184:187], v[220:223]\n\t" \
    "v_mfma_f32_16x16x32_bf16 v[220:223], a[220:223], a[188:191], v[220:223]\n\t" \
    "s_waitcnt lgkmcnt(0)\n\ts_barrier\n\t" \
    "v_mfma_f32_16x16x32_bf16 a[16:19], a[224:227], a[128:131], a[16:19]\n\t" \
    "ds_read_b64_tr_b16 a[192:193], v22 offset:16384\n\t" \
    "v_mfma_f32_16x16x32_bf16 a[16:19], a[228:231], a[132:135], a[16:19]\n\t" \
    "v_mfma_f32_16x16x32_bf16 a[20:23], a[232:235], a[128:131], a[20:23]\n\t" \
    "ds_read_b64_tr_b16 a[194:195], v22 offset:17408\n\t" \
    "v_mfma_f32_16x16x32_bf16 a[20:23], a[236:239], a[132:135], a[20:23]\n\t" \
    "v_mfma_f32_16x16x32_bf16 a[24:27], a[240:243], a[128:131], a[24:27]\n\t" \
    "ds_read_b64_tr_b16 a[196:197], v22 offset:24576\n\t" \
    "v_mfma_f32_16x16x32_bf16 a[24:27], a[244:247], a[132:135], a[24:27]\n\t" \
    "v_mfma_f32_16x16x32_bf16 a[28:31], a[248:251], a[128:131], a[28:31]\n\t" \
    "ds_read_b64_tr_b16 a[198:199], v22 offset:25600\n\t" \
    "v_mfma_f32_16x16x32_bf16 a[28:31], a[252:255], a[132:135], a[28:31]\n\t" \
    "v_mfma_f32_16x16x32_bf16 a[48:51], a[224:227], a[136:139], a[48:51]\n\t" \
    "ds_read_b128 a[128:131], v20 offset:0\n\t" \
    "v_mfma_f32_16x16x32_bf16 a[48:51], a[228:231], a[140:143], a[48:51]\n\t" \
    "v_mfma_f32_16x16x32_bf16 a[52:55], a[232:235], a[136:139], a[52:55]\n\t" \
    "ds_read_b128 a[132:135], v21 offset:0\n\t" \
    "v_mfma_f32_16x16x32_bf16 a[52:55], a[236:239], a[140:143], a[52:55]\n\t" \
    "v_mfma_f32_16x16x32_bf16 a[56:59], a[240:243], a[136:139], a[56:59]\n\t" \
    "ds_read_b64_tr_b16 a[200:201], v23 offset:16384\n\t" \
    "v_mfma_f32_16x16x32_bf16 a[56:59], a[244:247], a[140:143], a[56:59]\n\t" \
    "v_mfma_f32_16x16x32_bf16 a[60:63], a[248:251], a[136:139], a[60:63]\n\t" \
    "ds_read_b64_tr_b16 a[202:203], v23 offset:17408\n\t" \
    "v_mfma_f32_16x16x32_bf16 a[60:63], a[252:255], a[140:143], a[60:63]\n\t" \
    "v_mfma_f32_16x16x32_bf16 v[64:67], a[224:227], a[144:147], v[64:67]\n\t" \
    "ds_read_b128 a[136:139], v20 offset:2048\n\t" \
    "v_mfma_f32_16x16x32_bf16 v[64:67], a[228:231], a[148:151], v[64:67]\n\t" \
    "v_mfma_f32_16x16x32_bf16 v[68:71], a[232:235], a[144:147], v[68:71]\n\t" \
    "ds_read_b128 a[140:143], v21 offset:2048\n\t" \
    "v_mfma_f32_16x16x32_bf16 v[68:71], a[236:239], a[148:151], v[68:71]\n\t" \
    "v_mfma_f32_16x16x32_bf16 v[72:75], a[240:243], a[144:147], v[72:75]\n\t" \
    "ds_read_b64_tr_b16 a[204:205], v23 offset:24576\n\t" \
    "v_mfma_f32_16x16x32_bf16 v[72:75], a[244:247], a[148:151], v[72:75]\n\t" \
    "v_mfma_f32_16x16x32_bf16 v[76:79], a[248:251], a[144:147], v[76:79]\n\t" \
    "ds_read_b64_tr_b16 a[206:207], v23 offset:25600\n\t" \
    "v_mfma_f32_16x16x32_bf16 v[76:79], a[252:255], a[148:151], v[76:79]\n\t" \
    "buffer_load_dword v16, v17, %[rs], %[so] offen \n\t" \
    "v_mfma_f32_16x16x32_bf16 v[96:99], a[224:227], a[152:155], v[96:99]\n\t" \
    "ds_read_b128 a[144:147], v20 offset:4096\n\t" \
    "v_mfma_f32_16x16x32_bf16 v[96:99], a[228:231], a[156:159], v[96:99]\n\t" \
    "s_add_u32 %[so], %[so], %[pfi]\n\ts_nop 0\n\t" \
    "v_mfma_f32_16x16x32_bf16 v[100:103], a[232:235], a[152:155], v[100:103]\n\t" \
    "ds_read_b128 a[148:151], v21 offset:4096\n\t" \
    "v_mfma_f32_16x16x32_bf16 v[100:103], a[236:239], a[156:159], v[100:103]\n\t" \
    "s_cmp_eq_u32 %[rem], %[psw]\n\ts_cselect_b32 %[so], %[nx], %[so]\n\t" \
    "v_mfma_f32_16x16x32_bf16 v[104:107], a[240:243], a[152:155], v[104:107]\n\t" \
    "ds_read_b64_tr_b16 a[208:209], v24 offset:16384\n\t" \
    "v_mfma_f32_16x16x32_bf16 v[104:107], a[244:247], a[156:159], v[104:107]\n\t" \
    "v_mfma_f32_16x16x32_bf16 v[108:111], a[248:251], a[152:155], v[108:111]\n\t" \
    "ds_read_b64_tr_b16 a[210:211], v24 offset:17408\n\t" \
    "v_mfma_f32_16x16x32_bf16 v[108:111], a[252:255], a[156:159], v[108:111]\n\t" \
    "v_mfma_f32_16x16x32_bf16 v[128:131], a[224:227], a[160:163], v[128:131]\n\t" \
    "ds_read_b128 a[152:155], v20 offset:6144\n\t" \
    "v_mfma_f32_16x16x32_bf16 v[128:131], a[228:231], a[164:167], v[128:131]\n\t" \
    "v_mfma_f32_16x16x32_bf16 v[132:135], a[232:235], a[160:163], v[132:135]\n\t" \
    "ds_read_b128 a[156:159], v21 offset:6144\n\t" \
    "v_mfma_f32_16x16x32_bf16 v[132:135], a[236:239], a[164:167], v[132:135]\n\t" \
    "v_mfma_f32_16x16x32_bf16 v[136:139], a[240:243], a[160:163], v[136:139]\n\t" \
    "ds_read_b64_tr_b16 a[212:213], v24 offset:24576\n\t" \
    "v_mfma_f32_16x16x32_bf16 v[136:139], a[244:247], a[164:167], v[136:139]\n\t" \
    "v_mfma_f32_16x16x32_bf16 v[140:143], a[248:251], a[160:163], v[140:143]\n\t" \
    "ds_read_b64_tr_b16 a[214:215], v24 offset:25600\n\t" \
    "v_mfma_f32_16x16x32_bf16 v[140:143], a[252:255], a[164:167], v[140:143]\n\t" \
    "v_mfma_f32_16x16x32_bf16 v[160:163], a[224:227], a[168:171], v[160:163]\n\t" \
    "ds_read_b128 a[160:163], v20 offset:8192\n\t" \
    "s_sub_u32 %[gap], %[gap], 1\n\t" \
    "v_mfma_f32_16x16x32_bf16 v[160:163], a[228:231], a[172:175], v[160:163]\n\t" \
    "v_mfma_f32_16x16x32_bf16 v[164:167], a[232:235], a[168:171], v[164:167]\n\t" \
    "ds_read_b128 a[164:167], v21 offset:8192\n\t" \
    "v_mfma_f32_16x16x32_bf16 v[164:167], a[236:239], a[172:175], v[164:167]\n\t" \
    "v_mfma_f32_16x16x32_bf16 v[168:171], a[240:243], a[168:171], v[168:171]\n\t" \
    "ds_read_b64_tr_b16 a[216:217], v25 offset:16384\n\t" \
    "v_mfma_f32_16x16x32_bf16 v[168:171], a[244:247], a[172:175], v[168:171]\n\t" \
    "v_mfma_f32_16x16x32_bf16 v[172:175], a[248:251], a[168:171], v[172:175]\n\t" \
    "ds_read_b64_tr_b16 a[218:219], v25 offset:17408\n\t" \
    "v_mfma_f32_16x16x32_bf16 v[172:175], a[252:255], a[172:175], v[172:175]\n\t" \
    "v_mfma_f32_16x16x32_bf16 v[192:195], a[224:227], a[176:179], v[192:195]\n\t" \
    "ds_read_b128 a[168:171], v20 offset:10240\n\t" \
    "v_mfma_f32_16x16x32_bf16 v[192:195], a[228:231], a[180:183], v[192:195]\n\t" \
    "v_mfma_f32_16x16x32_bf16 v[196:199], a[232:235], a[176:179], v[196:199]\n\t" \
    "ds_read_b128 a[172:175], v21 offset:10240\n\t" \
    "v_mfma_f32_16x16x32_bf16 v[196:199], a[236:239], a[180:183], v[196:199]\n\t" \
    "v_readfirstlane_b32 %[go], v18\n\t" \
    "v_readfirstlane_b32 %[t0], v44\n\t" \
    "v_mfma_f32_16x16x32_bf16 v[200:203], a[240:243], a[176:179], v[200:203]\n\t" \
    "ds_read_b64_tr_b16 a[220:221], v25 offset:24576\n\t" \
    "v_mfma_f32_16x16x32_bf16 v[200:203], a[244:247], a[180:183], v[200:203]\n\t" \
    "s_bitcmp1_b32 %[c], 0\n\ts_cselect_b32 %[t0], %[t0], %[go]\n\t" \
    "v_mfma_f32_16x16x32_bf16 v[204:207], a[248:251], a[176:179], v[204:207]\n\t" \
    "ds_read_b64_tr_b16 a[222:223], v25 offset:25600\n\t" \
    "v_readfirstlane_b32 %[go], v19\n\t" \
    "v_mfma_f32_16x16x32_bf16 v[204:207], a[252:255], a[180:183], v[204:207]\n\t" \
    "s_cmp_ge_u32 %[ph], 2\n\ts_cselect_b32 %[go], %[t0], %[go]\n\t" \
    "v_mfma_f32_16x16x32_bf16 v[224:227], a[224:227], a[184:187], v[224:227]\n\t" \
    "v_add_u32 v23, 0x8000, v22\n\t" \
    "v_add_u32 v22, 0xffff0000, v22\n\t" \
    "v_mfma_f32_16x16x32_bf16 v[224:227], a[228:231], a[188:191], v[224:227]\n\t" \
    "v_min_u32_e64 v22, v22, v23\n\t" \
    "v_mfma_f32_16x16x32_bf16 v[228:231], a[232:235], a[184:187], v[228:231]\n\t" \
    "v_xor_b32_e64 v23, 32, v22\n\t" \
    "v_xor_b32_e64 v24, 64, v22\n\t" \
    "v_mfma_f32_16x16x32_bf16 v[228:231], a[236:239], a[188:191], v[228:231]\n\t" \
    "v_xor_b32 v25, 0x60, v22\n\t" \
    "v_mfma_f32_16x16x32_bf16 v[232:235], a[240:243], a[184:187], v[232:235]\n\t" \
    "s_cmp_ge_u32 %[go], %[thr]\n\ts_cselect_b32 %[go], %[zls], 0\n\t" \
    "v_mfma_f32_16x16x32_bf16 v[232:235], a[244:247], a[188:191], v[232:235]\n\t" \
    "s_cmp_le_i32 %[gap], 0\n\ts_cselect_b32 %[go], %[go], 0\n\t" \
    "v_mfma_f32_16x16x32_bf16 v[236:239], a[248:251], a[184:187], v[236:239]\n\t" \
    "v_mfma_f32_16x16x32_bf16 v[236:239], a[252:255], a[188:191], v[236:239]\n\t" \
    "s_sub_u32 %[rem], %[rem], 1\n\t" \
    "s_cmp_le_i32 %[rem], %[stop]\n\t" \
    "s_waitcnt lgkmcnt(0)\n\ts_barrier\n\t" \
    "s_cbranch_scc1 90f\n\t" \
    "s_cmp_eq_u32 %[go], 0\n\t" \
    "s_cbranch_scc1 10b\n\t" \
    "20:\n\t" \
    "s_and_b32 %[t0], %[c], 1\n\t" \
    "s_lshl1_add_u32 %[go], %[go], %[t0]\n\t" \
    "s_cmp_ge_u32 %[ph], 2\n\t" \
    "s_cbranch_scc1 21f\n\t" \
    "s_cmp_lg_u32 %[t0], 0\n\t" \
    "s_cbranch_scc1 430f\n\t" \
    "buffer_load_dwordx4 a[64:67], v47, %[rss], %[ld] offen offset:0 " LAUXS "\n\t" \
    "buffer_load_dwordx4 a[68:71], v47, %[rss], %[ld] offen offset:256 " LAUXS "\n\t" \
    "s_add_u32 %[t0], %[ld], 0x40000\n\tbuffer_load_dwordx4 a[72:75], v47, %[rss], %[t0] offen offset:0 " LAUXS "\n\t" \
    "s_add_u32 %[t0], %[ld], 0x40000\n\tbuffer_load_dwordx4 a[76:79], v47, %[rss], %[t0] offen offset:256 " LAUXS "\n\t" \
    "s_add_u32 %[ld], %[ld], %[dl]\n\tbuffer_load_dwordx4 a[80:83], v47, %[rss], %[ld] offen offset:0 " LAUXS "\n\t" \
    "buffer_load_dwordx4 a[84:87], v47, %[rss], %[ld] offen offset:256 " LAUXS "\n\t" \
    "s_add_u32 %[t0], %[ld], 0x40000\n\tbuffer_load_dwordx4 a[88:91], v47, %[rss], %[t0] offen offset:0 " LAUXS "\n\t" \
    "s_add_u32 %[t0], %[ld], 0x40000\n\tbuffer_load_dwordx4 a[92:95], v47, %[rss], %[t0] offen offset:256 " LAUXS "\n\t" \
    "v_mov_b32 v18, 0\n\tbuffer_load_dword v18, off, %[rss], %[pc] sc1\n\t" \
    "s_add_u32 %[c], %[c], 1\n\t" \
    "s_add_u32 %[ph], %[ph], 1\n\t" \
    "s_mov_b32 %[pw], 0xf0000000\n\t" \
    "s_lshr_b32 %[thr], %[ph], 1\n\t" \
    "v_readlane_b32 %[ld], v14, %[c]\n\t" \
    "v_readlane_b32 %[oo], v15, %[c]\n\t" \
    "s_branch 10b\n\t" \
    "430:\n\t" \
    "buffer_load_dwordx4 a[96:99], v47, %[rss], %[ld] offen offset:0 " LAUXS "\n\t" \
    "buffer_load_dwordx4 a[100:103], v47, %[rss], %[ld] offen offset:256 " LAUXS "\n\t" \
    "s_add_u32 %[t0], %[ld], 0x40000\n\tbuffer_load_dwordx4 a[104:107], v47, %[rss], %[t0] offen offset:0 " LAUXS "\n\t" \
    "s_add_u32 %[t0], %[ld], 0x40000\n\tbuffer_load_dwordx4 a[108:111], v47, %[rss], %[t0] offen offset:256 " LAUXS "\n\t" \
    "s_add_u32 %[ld], %[ld], %[dl]\n\tbuffer_load_dwordx4 a[112:115], v47, %[rss], %[ld] offen offset:0 " LAUXS "\n\t" \
    "buffer_load_dwordx4 a[116:119], v47, %[rss], %[ld] offen offset:256 " LAUXS "\n\t" \
    "s_add_u32 %[t0], %[ld], 0x40000\n\tbuffer_load_dwordx4 a[120:123], v47, %[rss], %[t0] offen offset:0 " LAUXS "\n\t" \
    "s_add_u32 %[t0], %[ld], 0x40000\n\tbuffer_load_dwordx4 a[124:127], v47, %[rss], %[t0] offen offset:256 " LAUXS "\n\t" \
    "v_mov_b32 v44, 0\n\tbuffer_load_dword v44, off, %[rss], %[pc] sc1\n\t" \
    "s_add_u32 %[c], %[c], 1\n\t" \
    "s_add_u32 %[ph], %[ph], 1\n\t" \
    "s_mov_b32 %[pw], 0xf0000000\n\t" \
    "s_lshr_b32 %[thr], %[ph], 1\n\t" \
    "v_readlane_b32 %[ld], v14, %[c]\n\t" \
    "v_readlane_b32 %[oo], v15, %[c]\n\t" \
    "s_branch 10b\n\t" \
    "21:\n\t" \
    "s_cmp_eq_u32 %[go], 4\n\t" \
    "s_cbranch_scc1 320f\n\t" \
    "321:\n\t" \
    "v_mfma_f32_16x16x32_bf16 v[36:39], v[26:29], a[96:99], 0\n\t" \
    "v_mfma_f32_16x16x32_bf16 v[36:39], v[26:29], a[104:107], v[36:39]\n\t" \
    "v_mfma_f32_16x16x32_bf16 v[40:43], v[26:29], a[100:103], 0\n\t" \
    "v_mfma_f32_16x16x32_bf16 v[40:43], v[26:29], a[108:111], v[40:43]\n\t" \
    "v_mfma_f32_16x16x32_bf16 v[240:243], v[26:29], a[112:115], 0\n\t" \
    "v_mfma_f32_16x16x32_bf16 v[240:243], v[26:29], a[120:123], v[240:243]\n\t" \
    "v_mfma_f32_16x16x32_bf16 v[244:247], v[26:29], a[116:119], 0\n\t" \
    "v_mfma_f32_16x16x32_bf16 v[244:247], v[26:29], a[124:127], v[244:247]\n\t" \
    "v_mfma_f32_16x16x32_bf16 a[0:3], a[192:195], a[128:131], a[0:3]\n\t" \
    "ds_read_b128 a[176:179], v20 offset:12288\n\t" \
    "v_mfma_f32_16x16x32_bf16 a[0:3], a[196:199], a[132:135], a[0:3]\n\t" \
    "buffer_load_dwordx4 a[96:99], v47, %[rss], %[ld] offen offset:0 " LAUXS "\n\t" \
    "v_mfma_f32_16x16x32_bf16 a[4:7], a[200:203], a[128:131], a[4:7]\n\t" \
    "ds_read_b128 a[180:183], v21 offset:12288\n\t" \
    "v_mfma_f32_16x16x32_bf16 a[4:7], a[204:207], a[132:135], a[4:7]\n\t" \
    "v_mfma_f32_16x16x32_bf16 a[8:11], a[208:211], a[128:131], a[8:11]\n\t" \
    "ds_read_b128 a[184:187], v20 offset:14336\n\t" \
    "v_mfma_f32_16x16x32_bf16 a[8:11], a[212:215], a[132:135], a[8:11]\n\t" \
    "v_mfma_f32_16x16x32_bf16 a[12:15], a[216:219], a[128:131], a[12:15]\n\t" \
    "ds_read_b128 a[188:191], v21 offset:14336\n\t" \
    "v_mfma_f32_16x16x32_bf16 a[12:15], a[220:223], a[132:135], a[12:15]\n\t" \
    "v_mfma_f32_16x16x32_bf16 a[32:35], a[192:195], a[136:139], a[32:35]\n\t" \
    "ds_read_b64_tr_b16 a[224:225], v22 offset:0\n\t" \
    "v_mfma_f32_16x16x32_bf16 a[32:35], a[196:199], a[140:143], a[32:35]\n\t" \
    "v_xor_b32 v20, 0x38000, v20\n\t" \
    "v_xor_b32 v21, 0x38000, v21\n\t" \
    "v_mfma_f32_16x16x32_bf16 a[36:39], a[200:203], a[136:139], a[36:39]\n\t" \
    "ds_read_b64_tr_b16 a[226:227], v22 offset:1024\n\t" \
    "v_med3_f32 v32, v36, -%[c60], %[c60]\n\t" \
    "v_mfma_f32_16x16x32_bf16 a[36:39], a[204:207], a[140:143], a[36:39]\n\t" \
    "v_med3_f32 v33, v37, -%[c60], %[c60]\n\t" \
    "v_med3_f32 v34, v38, -%[c60], %[c60]\n\t" \
    "v_mfma_f32_16x16x32_bf16 a[40:43], a[208:211], a[136:139], a[40:43]\n\t" \
    "ds_read_b64_tr_b16 a[228:229], v22 offset:8192\n\t" \
    "v_med3_f32 v35, v39, -%[c60], %[c60]\n\t" \
    "v_mfma_f32_16x16x32_bf16 a[40:43], a[212:215], a[140:143], a[40:43]\n\t" \
    "v_mul_f32 v32, 0xbfb8aa3b, v32\n\t" \
    "v_mul_f32 v33, 0xbfb8aa3b, v33\n\t" \
    "v_mfma_f32_16x16x32_bf16 a[44:47], a[216:219], a[136:139], a[44:47]\n\t" \
    "ds_read_b64_tr_b16 a[230:231], v22 offset:9216\n\t" \
    "v_mul_f32 v34, 0xbfb8aa3b, v34\n\t" \
    "v_mfma_f32_16x16x32_bf16 a[44:47], a[220:223], a[140:143], a[44:47]\n\t" \
    "buffer_load_dwordx4 a[100:103], v47, %[rss], %[ld] offen offset:256 " LAUXS "\n\t" \
    "v_mfma_f32_16x16x32_bf16 v[48:51], a[192:195], a[144:147], v[48:51]\n\t" \
    "ds_read_b64_tr_b16 a[232:233], v23 offset:0\n\t" \
    "v_mul_f32 v35, 0xbfb8aa3b, v35\n\t" \
    "v_mfma_f32_16x16x32_bf16 v[48:51], a[196:199], a[148:151], v[48:51]\n\t" \
    "v_exp_f32_e64 v32, v32\n\t" \
    "v_mfma_f32_16x16x32_bf16 v[52:55], a[200:203], a[144:147], v[52:55]\n\t" \
    "ds_read_b64_tr_b16 a[234:235], v23 offset:1024\n\t" \
    "v_mfma_f32_16x16x32_bf16 v[52:55], a[204:207], a[148:151], v[52:55]\n\t" \
    "v_exp_f32_e64 v33, v33\n\t" \
    "v_mfma_f32_16x16x32_bf16 v[56:59], a[208:211], a[144:147], v[56:59]\n\t" \
    "ds_read_b64_tr_b16 a[236:237], v23 offset:8192\n\t" \
    "v_mfma_f32_16x16x32_bf16 v[56:59], a[212:215], a[148:151], v[56:59]\n\t" \
    "v_exp_f32_e64 v34, v34\n\t" \
    "v_mfma_f32_16x16x32_bf16 v[60:63], a[216:219], a[144:147], v[60:63]\n\t" \
    "ds_read_b64_tr_b16 a[238:239], v23 offset:9216\n\t" \
    "v_mfma_f32_16x16x32_bf16 v[60:63], a[220:223], a[148:151], v[60:63]\n\t" \
    "v_exp_f32_e64 v35, v35\n\t" \
    "v_mfma_f32_16x16x32_bf16 v[80:83], a[192:195], a[152:155], v[80:83]\n\t" \
    "ds_read_b64_tr_b16 a[240:241], v24 offset:0\n\t" \
    "v_add_f32_e64 v32, 1.0, v32\n\t" \
    "v_mfma_f32_16x16x32_bf16 v[80:83], a[196:199], a[156:159], v[80:83]\n\t" \
    "v_add_f32_e64 v33, 1.0, v33\n\t" \
    "v_add_f32_e64 v34, 1.0, v34\n\t" \
    "v_mfma_f32_16x16x32_bf16 v[84:87], a[200:203], a[152:155], v[84:87]\n\t" \
    "ds_read_b64_tr_b16 a[242:243], v24 offset:1024\n\t" \
    "v_add_f32_e64 v35, 1.0, v35\n\t" \
    "v_mfma_f32_16x16x32_bf16 v[84:87], a[204:207], a[156:159], v[84:87]\n\t" \
    "v_rcp_f32_e64 v32, v32\n\t" \
    "v_mfma_f32_16x16x32_bf16 v[88:91], a[208:211], a[152:155], v[88:91]\n\t" \
    "ds_read_b64_tr_b16 a[244:245], v24 offset:8192\n\t" \
    "v_mfma_f32_16x16x32_bf16 v[88:91], a[212:215], a[156:159], v[88:91]\n\t" \
    "s_add_u32 %[t0], %[ld], 0x40000\n\tbuffer_load_dwordx4 a[104:107], v47, %[rss], %[t0] offen offset:0 " LAUXS "\n\t" \
    "v_mfma_f32_16x16x32_bf16 v[92:95], a[216:219], a[152:155], v[92:95]\n\t" \
    "ds_read_b64_tr_b16 a[246:247], v24 offset:9216\n\t" \
    "v_mfma_f32_16x16x32_bf16 v[92:95], a[220:223], a[156:159], v[92:95]\n\t" \
    "v_rcp_f32_e64 v33, v33\n\t" \
    "v_mfma_f32_16x16x32_bf16 v[112:115], a[192:195], a[160:163], v[112:115]\n\t" \
    "ds_read_b64_tr_b16 a[248:249], v25 offset:0\n\t" \
    "v_mfma_f32_16x16x32_bf16 v[112:115], a[196:199], a[164:167], v[112:115]\n\t" \
    "v_rcp_f32_e64 v34, v34\n\t" \
    "v_mfma_f32_16x16x32_bf16 v[116:119], a[200:203], a[160:163], v[116:119]\n\t" \
    "ds_read_b64_tr_b16 a[250:251], v25 offset:1024\n\t" \
    "v_mfma_f32_16x16x32_bf16 v[116:119], a[204:207], a[164:167], v[116:119]\n\t" \
    "v_rcp_f32_e64 v35, v35\n\t" \
    "v_mfma_f32_16x16x32_bf16 v[120:123], a[208:211], a[160:163], v[120:123]\n\t" \
    "ds_read_b64_tr_b16 a[252:253], v25 offset:8192\n\t" \
    "v_mul_f32_e64 v36, v36, v32\n\t" \
    "v_mfma_f32_16x16x32_bf16 v[120:123], a[212:215], a[164:167], v[120:123]\n\t" \
    "v_mul_f32_e64 v37, v37, v33\n\t" \
    "v_mul_f32_e64 v38, v38, v34\n\t" \
    "v_mfma_f32_16x16x32_bf16 v[124:127], a[216:219], a[160:163], v[124:127]\n\t" \
    "ds_read_b64_tr_b16 a[254:255], v25 offset:9216\n\t" \
    "v_mul_f32_e64 v39, v39, v35\n\t" \
    "v_mfma_f32_16x16x32_bf16 v[124:127], a[220:223], a[164:167], v[124:127]\n\t" \
    "v_med3_f32 v32, v40, -%[c60], %[c60]\n\t" \
    "v_med3_f32 v33, v41, -%[c60], %[c60]\n\t" \
    "v_mfma_f32_16x16x32_bf16 v[144:147], a[192:195], a[168:171], v[144:147]\n\t" \
    "v_med3_f32 v34, v42, -%[c60], %[c60]\n\t" \
    "v_med3_f32 v35, v43, -%[c60], %[c60]\n\t" \
    "v_mfma_f32_16x16x32_bf16 v[144:147], a[196:199], a[172:175], v[144:147]\n\t" \
    "v_mul_f32 v32, 0xbfb8aa3b, v32\n\t" \
    "v_mul_f32 v33, 0xbfb8aa3b, v33\n\t" \
    "v_mfma_f32_16x16x32_bf16 v[148:151], a[200:203], a[168:171], v[148:151]\n\t" \
    "v_mul_f32 v34, 0xbfb8aa3b, v34\n\t" \
    "v_mul_f32 v35, 0xbfb8aa3b, v35\n\t" \
    "v_mfma_f32_16x16x32_bf16 v[148:151], a[204:207], a[172:175], v[148:151]\n\t" \
    "s_add_u32 %[t0], %[ld], 0x40000\n\tbuffer_load_dwordx4 a[108:111], v47, %[rss], %[t0] offen offset:256 " LAUXS "\n\t" \
    "v_mfma_f32_16x16x32_bf16 v[152:155], a[208:211], a[168:171], v[152:155]\n\t" \
    "v_exp_f32_e64 v32, v32\n\t" \
    "v_mfma_f32_16x16x32_bf16 v[152:155], a[212:215], a[172:175], v[152:155]\n\t" \
    "v_exp_f32_e64 v33, v33\n\t" \
    "v_mfma_f32_16x16x32_bf16 v[156:159], a[216:219], a[168:171], v[156:159]\n\t" \
    "v_exp_f32_e64 v34, v34\n\t" \
    "v_mfma_f32_16x16x32_bf16 v[156:159], a[220:223], a[172:175], v[156:159]\n\t" \
    "s_waitcnt lgkmcnt(8)\n\ts_nop 0\n\t" \
    "v_mfma_f32_16x16x32_bf16 v[176:179], a[192:195], a[176:179], v[176:179]\n\t" \
    "v_exp_f32_e64 v35, v35\n\t" \
    "v_mfma_f32_16x16x32_bf16 v[176:179], a[196:199], a[180:183], v[176:179]\n\t" \
    "v_add_f32_e64 v32, 1.0, v32\n\t" \
    "v_add_f32_e64 v33, 1.0, v33\n\t" \
    "v_mfma_f32_16x16x32_bf16 v[180:183], a[200:203], a[176:179], v[180:183]\n\t" \
    "v_add_f32_e64 v34, 1.0, v34\n\t" \
    "v_add_f32_e64 v35, 1.0, v35\n\t" \
    "v_mfma_f32_16x16x32_bf16 v[180:183], a[204:207], a[180:183], v[180:183]\n\t" \
    "v_rcp_f32_e64 v32, v32\n\t" \
    "v_mfma_f32_16x16x32_bf16 v[184:187], a[208:211], a[176:179], v[184:187]\n\t" \
    "v_rcp_f32_e64 v33, v33\n\t" \
    "v_mfma_f32_16x16x32_bf16 v[184:187], a[212:215], a[180:183], v[184:187]\n\t" \
    "v_rcp_f32_e64 v34, v34\n\t" \
    "v_mfma_f32_16x16x32_bf16 v[188:191], a[216:219], a[176:179], v[188:191]\n\t" \
    "v_rcp_f32_e64 v35, v35\n\t" \
    "v_mfma_f32_16x16x32_bf16 v[188:191], a[220:223], a[180:183], v[188:191]\n\t" \
    "v_mul_f32_e64 v40, v40, v32\n\t" \
    "v_mul_f32_e64 v41, v41, v33\n\t" \
    "v_mfma_f32_16x16x32_bf16 v[208:211], a[192:195], a[184:187], v[208:211]\n\t" \
    "v_mul_f32_e64 v42, v42, v34\n\t" \
    "v_mul_f32_e64 v43, v43, v35\n\t" \
    "v_mfma_f32_16x16x32_bf16 v[208:211], a[196:199], a[188:191], v[208:211]\n\t" \
    "s_add_u32 %[ld], %[ld], %[dl]\n\tbuffer_load_dwordx4 a[112:115], v47, %[rss], %[ld] offen offset:0 " LAUXS "\n\t" \
    "v_mfma_f32_16x16x32_bf16 v[212:215], a[200:203], a[184:187], v[212:215]\n\t" \
    "v_cvt_pk_bf16_f32 v36, v36, v37\n\t" \
    "v_cvt_pk_bf16_f32 v37, v38, v39\n\t" \
    "v_mfma_f32_16x16x32_bf16 v[212:215], a[204:207], a[188:191], v[212:215]\n\t" \
    "v_cvt_pk_bf16_f32 v38, v40, v41\n\t" \
    "v_cvt_pk_bf16_f32 v39, v42, v43\n\t" \
    "v_mfma_f32_16x16x32_bf16 v[216:219], a[208:211], a[184:187], v[216:219]\n\t" \
    "buffer_store_dwordx4 v[36:39], v31, %[rso], %[oo] offen sc0 sc1\n\t" \
    "v_mfma_f32_16x16x32_bf16 v[216:219], a[212:215], a[188:191], v[216:219]\n\t" \
    "v_mfma_f32_16x16x32_bf16 v[220:223], a[216:219], a[184:187], v[220:223]\n\t" \
    "v_mfma_f32_16x16x32_bf16 v[220:223], a[220:223], a[188:191], v[220:223]\n\t" \
    "s_waitcnt lgkmcnt(0)\n\ts_barrier\n\t" \
    "v_mfma_f32_16x16x32_bf16 a[16:19], a[224:227], a[128:131], a[16:19]\n\t" \
    "ds_read_b64_tr_b16 a[192:193], v22 offset:16384\n\t" \
    "v_med3_f32 v32, v240, -%[c60], %[c60]\n\t" \
    "v_med3_f32 v33, v241, -%[c60], %[c60]\n\t" \
    "v_mfma_f32_16x16x32_bf16 a[16:19], a[228:231], a[132:135], a[16:19]\n\t" \
    "v_med3_f32 v34, v242, -%[c60], %[c60]\n\t" \
    "v_med3_f32 v35, v243, -%[c60], %[c60]\n\t" \
    "v_mul_f32 v32, 0xbfb8aa3b, v32\n\t" \
    "v_mfma_f32_16x16x32_bf16 a[20:23], a[232:235], a[128:131], a[20:23]\n\t" \
    "ds_read_b64_tr_b16 a[194:195], v22 offset:17408\n\t" \
    "v_mul_f32 v33, 0xbfb8aa3b, v33\n\t" \
    "v_mul_f32 v34, 0xbfb8aa3b, v34\n\t" \
    "v_mfma_f32_16x16x32_bf16 a[20:23], a[236:239], a[132:135], a[20:23]\n\t" \
    "v_mul_f32 v35, 0xbfb8aa3b, v35\n\t" \
    "v_exp_f32_e64 v32, v32\n\t" \
    "v_mfma_f32_16x16x32_bf16 a[24:27], a[240:243], a[128:131], a[24:27]\n\t" \
    "ds_read_b64_tr_b16 a[196:197], v22 offset:24576\n\t" \
    "v_exp_f32_e64 v33, v33\n\t" \
    "v_mfma_f32_16x16x32_bf16 a[24:27], a[244:247], a[132:135], a[24:27]\n\t" \
    "buffer_load_dwordx4 a[116:119], v47, %[rss], %[ld] offen offset:256 " LAUXS "\n\t" \
    "v_mfma_f32_16x16x32_bf16 a[28:31], a[248:251], a[128:131], a[28:31]\n\t" \
    "ds_read_b64_tr_b16 a[198:199], v22 offset:25600\n\t" \
    "v_exp_f32_e64 v34, v34\n\t" \
    "v_mfma_f32_16x16x32_bf16 a[28:31], a[252:255], a[132:135], a[28:31]\n\t" \
    "v_exp_f32_e64 v35, v35\n\t" \
    "v_add_f32_e64 v32, 1.0, v32\n\t" \
    "v_mfma_f32_16x16x32_bf16 a[48:51], a[224:227], a[136:139], a[48:51]\n\t" \
    "ds_read_b128 a[128:131], v20 offset:0\n\t" \
    "v_add_f32_e64 v33, 1.0, v33\n\t" \
    "v_add_f32_e64 v34, 1.0, v34\n\t" \
    "v_mfma_f32_16x16x32_bf16 a[48:51], a[228:231], a[140:143], a[48:51]\n\t" \
    "v_add_f32_e64 v35, 1.0, v35\n\t" \
    "v_rcp_f32_e64 v32, v32\n\t" \
    "v_mfma_f32_16x16x32_bf16 a[52:55], a[232:235], a[136:139], a[52:55]\n\t" \
    "ds_read_b128 a[132:135], v21 offset:0\n\t" \
    "v_rcp_f32_e64 v33, v33\n\t" \
    "v_mfma_f32_16x16x32_bf16 a[52:55], a[236:239], a[140:143], a[52:55]\n\t" \
    "v_rcp_f32_e64 v34, v34\n\t" \
    "v_mfma_f32_16x16x32_bf16 a[56:59], a[240:243], a[136:139], a[56:59]\n\t" \
    "ds_read_b64_tr_b16 a[200:201], v23 offset:16384\n\t" \
    "v_rcp_f32_e64 v35, v35\n\t" \
    "v_mfma_f32_16x16x32_bf16 a[56:59], a[244:247], a[140:143], a[56:59]\n\t" \
    "v_mul_f32_e64 v240, v240, v32\n\t" \
    "v_mul_f32_e64 v241, v241, v33\n\t" \
    "v_mul_f32_e64 v242, v242, v34\n\t" \
    "v_mfma_f32_16x16x32_bf16 a[60:63], a[248:251], a[136:139], a[60:63]\n\t" \
    "ds_read_b64_tr_b16 a[202:203], v23 offset:17408\n\t" \
    "v_mul_f32_e64 v243, v243, v35\n\t" \
    "v_med3_f32 v32, v244, -%[c60], %[c60]\n\t" \
    "v_mfma_f32_16x16x32_bf16 a[60:63], a[252:255], a[140:143], a[60:63]\n\t" \
    "s_add_u32 %[t0], %[ld], 0x40000\n\tbuffer_load_dwordx4 a[120:123], v47, %[rss], %[t0] offen offset:0 " LAUXS "\n\t" \
    "v_med3_f32 v33, v245, -%[c60], %[c60]\n\t" \
    "v_mfma_f32_16x16x32_bf16 v[64:67], a[224:227], a[144:147], v[64:67]\n\t" \
    "ds_read_b128 a[136:139], v20 offset:2048\n\t" \
    "v_med3_f32 v34, v246, -%[c60], %[c60]\n\t" \
    "v_med3_f32 v35, v247, -%[c60], %[c60]\n\t" \
    "v_mfma_f32_16x16x32_bf16 v[64:67], a[228:231], a[148:151], v[64:67]\n\t" \
    "v_mul_f32 v32, 0xbfb8aa3b, v32\n\t" \
    "v_mul_f32 v33, 0xbfb8aa3b, v33\n\t" \
    "v_mul_f32 v34, 0xbfb8aa3b, v34\n\t" \
    "v_mfma_f32_16x16x32_bf16 v[68:71], a[232:235], a[144:147], v[68:71]\n\t" \
    "ds_read_b128 a[140:143], v21 offset:2048\n\t" \
    "v_mul_f32 v35, 0xbfb8aa3b, v35\n\t" \
    "v_mfma_f32_16x16x32_bf16 v[68:71], a[236:239], a[148:151], v[68:71]\n\t" \
    "v_exp_f32_e64 v32, v32\n\t" \
    "v_mfma_f32_16x16x32_bf16 v[72:75], a[240:243], a[144:147], v[72:75]\n\t" \
    "ds_read_b64_tr_b16 a[204:205], v23 offset:24576\n\t" \
    "v_exp_f32_e64 v33, v33\n\t" \
    "v_mfma_f32_16x16x32_bf16 v[72:75], a[244:247], a[148:151], v[72:75]\n\t" \
    "v_exp_f32_e64 v34, v34\n\t" \
    "v_mfma_f32_16x16x32_bf16 v[76:79], a[248:251], a[144:147], v[76:79]\n\t" \
    "ds_read_b64_tr_b16 a[206:207], v23 offset:25600\n\t" \
    "v_exp_f32_e64 v35, v35\n\t" \
    "v_mfma_f32_16x16x32_bf16 v[76:79], a[252:255], a[148:151], v[76:79]\n\t" \
    "buffer_load_dword v16, v17, %[rs], %[so] offen \n\t" \
    "v_add_f32_e64 v32, 1.0, v32\n\t" \
    "v_mfma_f32_16x16x32_bf16 v[96:99], a[224:227], a[152:155], v[96:99]\n\t" \
    "ds_read_b128 a[144:147], v20 offset:4096\n\t" \
    "v_add_f32_e64 v33, 1.0, v33\n\t" \
    "v_add_f32_e64 v34, 1.0, v34\n\t" \
    "v_mfma_f32_16x16x32_bf16 v[96:99], a[228:231], a[156:159], v[96:99]\n\t" \
    "s_add_u32 %[so], %[so], %[pfi]\n\ts_nop 0\n\t" \
    "s_add_u32 %[t0], %[ld], 0x40000\n\tbuffer_load_dwordx4 a[124:127], v47, %[rss], %[t0] offen offset:256 " LAUXS "\n\t" \
    "v_mfma_f32_16x16x32_bf16 v[100:103], a[232:235], a[152:155], v[100:103]\n\t" \
    "ds_read_b128 a[148:151], v21 offset:4096\n\t" \
    "v_add_f32_e64 v35, 1.0, v35\n\t" \
    "v_mfma_f32_16x16x32_bf16 v[100:103], a[236:239], a[156:159], v[100:103]\n\t" \
    "s_cmp_eq_u32 %[rem], %[psw]\n\ts_cselect_b32 %[so], %[nx], %[so]\n\t" \
    "v_mfma_f32_16x16x32_bf16 v[104:107], a[240:243], a[152:155], v[104:107]\n\t" \
    "ds_read_b64_tr_b16 a[208:209], v24 offset:16384\n\t" \
    "v_rcp_f32_e64 v32, v32\n\t" \
    "v_mfma_f32_16x16x32_bf16 v[104:107], a[244:247], a[156:159], v[104:107]\n\t" \
    "v_rcp_f32_e64 v33, v33\n\t" \
    "v_mfma_f32_16x16x32_bf16 v[108:111], a[248:251], a[152:155], v[108:111]\n\t" \
    "ds_read_b64_tr_b16 a[210:211], v24 offset:17408\n\t" \
    "v_rcp_f32_e64 v34, v34\n\t" \
    "v_mfma_f32_16x16x32_bf16 v[108:111], a[252:255], a[156:159], v[108:111]\n\t" \
    "v_rcp_f32_e64 v35, v35\n\t" \
    "v_mul_f32_e64 v244, v244, v32\n\t" \
    "v_mfma_f32_16x16x32_bf16 v[128:131], a[224:227], a[160:163], v[128:131]\n\t" \
    "ds_read_b128 a[152:155], v20 offset:6144\n\t" \
    "v_mul_f32_e64 v245, v245, v33\n\t" \
    "v_mul_f32_e64 v246, v246, v34\n\t" \
    "v_mfma_f32_16x16x32_bf16 v[128:131], a[228:231], a[164:167], v[128:131]\n\t" \
    "v_mul_f32_e64 v247, v247, v35\n\t" \
    "v_cvt_pk_bf16_f32 v240, v240, v241\n\t" \
    "v_cvt_pk_bf16_f32 v241, v242, v243\n\t" \
    "v_mfma_f32_16x16x32_bf16 v[132:135], a[232:235], a[160:163], v[132:135]\n\t" \
    "ds_read_b128 a[156:159], v21 offset:6144\n\t" \
    "v_cvt_pk_bf16_f32 v242, v244, v245\n\t" \
    "v_cvt_pk_bf16_f32 v243, v246, v247\n\t" \
    "v_mfma_f32_16x16x32_bf16 v[132:135], a[236:239], a[164:167], v[132:135]\n\t" \
    "v_mov_b32 v44, 0\n\tbuffer_load_dword v44, off, %[rss], %[pc] sc1\n\t" \
    "v_mfma_f32_16x16x32_bf16 v[136:139], a[240:243], a[160:163], v[136:139]\n\t" \
    "ds_read_b64_tr_b16 a[212:213], v24 offset:24576\n\t" \
    "s_add_u32 %[t0], %[oo], 0x4000\n\tbuffer_store_dwordx4 v[240:243], v31, %[rso], %[t0] offen sc0 sc1\n\t" \
    "v_mfma_f32_16x16x32_bf16 v[136:139], a[244:247], a[164:167], v[136:139]\n\t" \
    "s_add_u32 %[c], %[c], 1\n\t" \
    "s_mov_b32 %[gap], 1\n\t" \
    "v_mfma_f32_16x16x32_bf16 v[140:143], a[248:251], a[160:163], v[140:143]\n\t" \
    "ds_read_b64_tr_b16 a[214:215], v24 offset:25600\n\t" \
    "v_mov_b32 v19, 0\n\t" \
    "v_readlane_b32 %[ld], v14, %[c]\n\t" \
    "v_mfma_f32_16x16x32_bf16 v[140:143], a[252:255], a[164:167], v[140:143]\n\t" \
    "v_readlane_b32 %[oo], v15, %[c]\n\t" \
    "v_mfma_f32_16x16x32_bf16 v[160:163], a[224:227], a[168:171], v[160:163]\n\t" \
    "ds_read_b128 a[160:163], v20 offset:8192\n\t" \
    "v_mfma_f32_16x16x32_bf16 v[160:163], a[228:231], a[172:175], v[160:163]\n\t" \
    "v_mfma_f32_16x16x32_bf16 v[164:167], a[232:235], a[168:171], v[164:167]\n\t" \
    "ds_read_b128 a[164:167], v21 offset:8192\n\t" \
    "v_mfma_f32_16x16x32_bf16 v[164:167], a[236:239], a[172:175], v[164:167]\n\t" \
    "v_mfma_f32_16x16x32_bf16 v[168:171], a[240:243], a[168:171], v[168:171]\n\t" \
    "ds_read_b64_tr_b16 a[216:217], v25 offset:16384\n\t" \
    "v_mfma_f32_16x16x32_bf16 v[168:171], a[244:247], a[172:175], v[168:171]\n\t" \
    "v_mfma_f32_16x16x32_bf16 v[172:175], a[248:251], a[168:171], v[172:175]\n\t" \
    "ds_read_b64_tr_b16 a[218:219], v25 offset:17408\n\t" \
    "v_mfma_f32_16x16x32_bf16 v[172:175], a[252:255], a[172:175], v[172:175]\n\t" \
    "v_mfma_f32_16x16x32_bf16 v[192:195], a[224:227], a[176:179], v[192:195]\n\t" \
    "ds_read_b128 a[168:171], v20 offset:10240\n\t" \
    "v_mfma_f32_16x16x32_bf16 v[192:195], a[228:231], a[180:183], v[192:195]\n\t" \
    "v_mfma_f32_16x16x32_bf16 v[196:199], a[232:235], a[176:179], v[196:199]\n\t" \
    "ds_read_b128 a[172:175], v21 offset:10240\n\t" \
    "v_mfma_f32_16x16x32_bf16 v[196:199], a[236:239], a[180:183], v[196:199]\n\t" \
    "v_readfirstlane_b32 %[go], v18\n\t" \
    "v_readfirstlane_b32 %[t0], v44\n\t" \
    "v_mfma_f32_16x16x32_bf16 v[200:203], a[240:243], a[176:179], v[200:203]\n\t" \
    "ds_read_b64_tr_b16 a[220:221], v25 offset:24576\n\t" \
    "s_bitcmp1_b32 %[c], 0\n\ts_cselect_b32 %[t0], %[t0], %[go]\n\t" \
    "v_mfma_f32_16x16x32_bf16 v[200:203], a[244:247], a[180:183], v[200:203]\n\t" \
    "v_readfirstlane_b32 %[go], v19\n\t" \
    "s_cmp_ge_u32 %[ph], 2\n\ts_cselect_b32 %[go], %[t0], %[go]\n\t" \
    "v_mfma_f32_16x16x32_bf16 v[204:207], a[248:251], a[176:179], v[204:207]\n\t" \
    "ds_read_b64_tr_b16 a[222:223], v25 offset:25600\n\t" \
    "s_cmp_ge_u32 %[go], %[thr]\n\ts_cselect_b32 %[go], %[zls], 0\n\t" \
    "v_mfma_f32_16x16x32_bf16 v[204:207], a[252:255], a[180:183], v[204:207]\n\t" \
    "s_cmp_le_i32 %[gap], 0\n\ts_cselect_b32 %[go], %[go], 0\n\t" \
    "v_mfma_f32_16x16x32_bf16 v[224:227], a[224:227], a[184:187], v[224:227]\n\t" \
    "v_add_u32 v23, 0x8000, v22\n\t" \
    "v_add_u32 v22, 0xffff0000, v22\n\t" \
    "v_mfma_f32_16x16x32_bf16 v[224:227], a[228:231], a[188:191], v[224:227]\n\t" \
    "v_min_u32_e64 v22, v22, v23\n\t" \
    "v_mfma_f32_16x16x32_bf16 v[228:231], a[232:235], a[184:187], v[228:231]\n\t" \
    "v_xor_b32_e64 v23, 32, v22\n\t" \
    "v_xor_b32_e64 v24, 64, v22\n\t" \
    "v_mfma_f32_16x16x32_bf16 v[228:231], a[236:239], a[188:191], v[228:231]\n\t" \
    "v_xor_b32 v25, 0x60, v22\n\t" \
    "v_mfma_f32_16x16x32_bf16 v[232:235], a[240:243], a[184:187], v[232:235]\n\t" \
    "v_mfma_f32_16x16x32_bf16 v[232:235], a[244:247], a[188:191], v[232:235]\n\t" \
    "v_mfma_f32_16x16x32_bf16 v[236:239], a[248:251], a[184:187], v[236:239]\n\t" \
    "v_mfma_f32_16x16x32_bf16 v[236:239], a[252:255], a[188:191], v[236:239]\n\t" \
    "s_sub_u32 %[rem], %[rem], 1\n\t" \
    "s_cmp_eq_u32 %[c], %[cend]\n\t" \
    "s_waitcnt lgkmcnt(0)\n\ts_barrier\n\t" \
    "s_cbranch_scc1 90f\n\t" \
    "s_cmp_le_i32 %[rem], %[stop]\n\t" \
    "s_cbranch_scc1 90f\n\t" \
    "s_cmp_lg_u32 %[go], 0\n\t" \
    "s_cbranch_scc1 20b\n\t" \
    "s_branch 10b\n\t" \
    "320:\n\t" \
    "v_mfma_f32_16x16x32_bf16 v[36:39], v[26:29], a[64:67], 0\n\t" \
    "v_mfma_f32_16x16x32_bf16 v[36:39], v[26:29], a[72:75], v[36:39]\n\t" \
    "v_mfma_f32_16x16x32_bf16 v[40:43], v[26:29], a[68:71], 0\n\t" \
    "v_mfma_f32_16x16x32_bf16 v[40:43], v[26:29], a[76:79], v[40:43]\n\t" \
    "v_mfma_f32_16x16x32_bf16 v[240:243], v[26:29], a[80:83], 0\n\t" \
    "v_mfma_f32_16x16x32_bf16 v[240:243], v[26:29], a[88:91], v[240:243]\n\t" \
    "v_mfma_f32_16x16x32_bf16 v[244:247], v[26:29], a[84:87], 0\n\t" \
    "v_mfma_f32_16x16x32_bf16 v[244:247], v[26:29], a[92:95], v[244:247]\n\t" \
    "v_mfma_f32_16x16x32_bf16 a[0:3], a[192:195], a[128:131], a[0:3]\n\t" \
    "ds_read_b128 a[176:179], v20 offset:12288\n\t" \
    "v_mfma_f32_16x16x32_bf16 a[0:3], a[196:199], a[132:135], a[0:3]\n\t" \
    "buffer_load_dwordx4 a[64:67], v47, %[rss], %[ld] offen offset:0 " LAUXS "\n\t" \
    "v_mfma_f32_16x16x32_bf16 a[4:7], a[200:203], a[128:131], a[4:7]\n\t" \
    "ds_read_b128 a[180:183], v21 offset:12288\n\t" \
    "v_mfma_f32_16x16x32_bf16 a[4:7], a[204:207], a[132:135], a[4:7]\n\t" \
    "v_mfma_f32_16x16x32_bf16 a[8:11], a[208:211], a[128:131], a[8:11]\n\t" \
    "ds_read_b128 a[184:187], v20 offset:14336\n\t" \
    "v_mfma_f32_16x16x32_bf16 a[8:11], a[212:215], a[132:135], a[8:11]\n\t" \
    "v_mfma_f32_16x16x32_bf16 a[12:15], a[216:219], a[128:131], a[12:15]\n\t" \
    "ds_read_b128 a[188:191], v21 offset:14336\n\t" \
    "v_mfma_f32_16x16x32_bf16 a[12:15], a[220:223], a[132:135], a[12:15]\n\t" \
    "v_mfma_f32_16x16x32_bf16 a[32:35], a[192:195], a[136:139], a[32:35]\n\t" \
    "ds_read_b64_tr_b16 a[224:225], v22 offset:0\n\t" \
    "v_mfma_f32_16x16x32_bf16 a[32:35], a[196:199], a[140:143], a[32:35]\n\t" \
    "v_xor_b32 v20, 0x38000, v20\n\t" \
    "v_xor_b32 v21, 0x38000, v21\n\t" \
    "v_mfma_f32_16x16x32_bf16 a[36:39], a[200:203], a[136:139], a[36:39]\n\t" \
    "ds_read_b64_tr_b16 a[226:227], v22 offset:1024\n\t" \
    "v_med3_f32 v32, v36, -%[c60], %[c60]\n\t" \
    "v_mfma_f32_16x16x32_bf16 a[36:39], a[204:207], a[140:143], a[36:39]\n\t" \
    "v_med3_f32 v33, v37, -%[c60], %[c60]\n\t" \
    "v_med3_f32 v34, v38, -%[c60], %[c60]\n\t" \
    "v_mfma_f32_16x16x32_bf16 a[40:43], a[208:211], a[136:139], a[40:43]\n\t" \
    "ds_read_b64_tr_b16 a[228:229], v22 offset:8192\n\t" \
    "v_med3_f32 v35, v39, -%[c60], %[c60]\n\t" \
    "v_mfma_f32_16x16x32_bf16 a[40:43], a[212:215], a[140:143], a[40:43]\n\t" \
    "v_mul_f32 v32, 0xbfb8aa3b, v32\n\t" \
    "v_mul_f32 v33, 0xbfb8aa3b, v33\n\t" \
    "v_mfma_f32_16x16x32_bf16 a[44:47], a[216:219], a[136:139], a[44:47]\n\t" \
    "ds_read_b64_tr_b16 a[230:231], v22 offset:9216\n\t" \
    "v_mul_f32 v34, 0xbfb8aa3b, v34\n\t" \
    "v_mfma_f32_16x16x32_bf16 a[44:47], a[220:223], a[140:143], a[44:47]\n\t" \
    "buffer_load_dwordx4 a[68:71], v47, %[rss], %[ld] offen offset:256 " LAUXS "\n\t" \
    "v_mfma_f32_16x16x32_bf16 v[48:51], a[192:195], a[144:147], v[48:51]\n\t" \
    "ds_read_b64_tr_b16 a[232:233], v23 offset:0\n\t" \
    "v_mul_f32 v35, 0xbfb8aa3b, v35\n\t" \
    "v_mfma_f32_16x16x32_bf16 v[48:51], a[196:199], a[148:151], v[48:51]\n\t" \
    "v_exp_f32_e64 v32, v32\n\t" \
    "v_mfma_f32_16x16x32_bf16 v[52:55], a[200:203], a[144:147], v[52:55]\n\t" \
    "ds_read_b64_tr_b16 a[234:235], v23 offset:1024\n\t" \
    "v_mfma_f32_16x16x32_bf16 v[52:55], a[204:207], a[148:151], v[52:55]\n\t" \
    "v_exp_f32_e64 v33, v33\n\t" \
    "v_mfma_f32_16x16x32_bf16 v[56:59], a[208:211], a[144:147], v[56:59]\n\t" \
    "ds_read_b64_tr_b16 a[236:237], v23 offset:8192\n\t" \
    "v_mfma_f32_16x16x32_bf16 v[56:59], a[212:215], a[148:151], v[56:59]\n\t" \
    "v_exp_f32_e64 v34, v34\n\t" \
    "v_mfma_f32_16x16x32_bf16 v[60:63], a[216:219], a[144:147], v[60:63]\n\t" \
    "ds_read_b64_tr_b16 a[238:239], v23 offset:9216\n\t" \
    "v_mfma_f32_16x16x32_bf16 v[60:63], a[220:223], a[148:151], v[60:63]\n\t" \
    "v_exp_f32_e64 v35, v35\n\t" \
    "v_mfma_f32_16x16x32_bf16 v[80:83], a[192:195], a[152:155], v[80:83]\n\t" \
    "ds_read_b64_tr_b16 a[240:241], v24 offset:0\n\t" \
    "v_add_f32_e64 v32, 1.0, v32\n\t" \
    "v_mfma_f32_16x16x32_bf16 v[80:83], a[196:199], a[156:159], v[80:83]\n\t" \
    "v_add_f32_e64 v33, 1.0, v33\n\t" \
    "v_add_f32_e64 v34, 1.0, v34\n\t" \
    "v_mfma_f32_16x16x32_bf16 v[84:87], a[200:203], a[152:155], v[84:87]\n\t" \
    "ds_read_b64_tr_b16 a[242:243], v24 offset:1024\n\t" \
    "v_add_f32_e64 v35, 1.0, v35\n\t" \
    "v_mfma_f32_16x16x32_bf16 v[84:87], a[204:207], a[156:159], v[84:87]\n\t" \
    "v_rcp_f32_e64 v32, v32\n\t" \
    "v_mfma_f32_16x16x32_bf16 v[88:91], a[208:211], a[152:155], v[88:91]\n\t" \
    "ds_read_b64_tr_b16 a[244:245], v24 offset:8192\n\t" \
    "v_mfma_f32_16x16x32_bf16 v[88:91], a[212:215], a[156:159], v[88:91]\n\t" \
    "s_add_u32 %[t0], %[ld], 0x40000\n\tbuffer_load_dwordx4 a[72:75], v47, %[rss], %[t0] offen offset:0 " LAUXS "\n\t" \
    "v_mfma_f32_16x16x32_bf16 v[92:95], a[216:219], a[152:155], v[92:95]\n\t" \
    "ds_read_b64_tr_b16 a[246:247], v24 offset:9216\n\t" \
    "v_mfma_f32_16x16x32_bf16 v[92:95], a[220:223], a[156:159], v[92:95]\n\t" \
    "v_rcp_f32_e64 v33, v33\n\t" \
    "v_mfma_f32_16x16x32_bf16 v[112:115], a[192:195], a[160:163], v[112:115]\n\t" \
    "ds_read_b64_tr_b16 a[248:249], v25 offset:0\n\t" \
    "v_mfma_f32_16x16x32_bf16 v[112:115], a[196:199], a[164:167], v[112:115]\n\t" \
    "v_rcp_f32_e64 v34, v34\n\t" \
    "v_mfma_f32_16x16x32_bf16 v[116:119], a[200:203], a[160:163], v[116:119]\n\t" \
    "ds_read_b64_tr_b16 a[250:251], v25 offset:1024\n\t" \
    "v_mfma_f32_16x16x32_bf16 v[116:119], a[204:207], a[164:167], v[116:119]\n\t" \
    "v_rcp_f32_e64 v35, v35\n\t" \
    "v_mfma_f32_16x16x32_bf16 v[120:123], a[208:211], a[160:163], v[120:123]\n\t" \
    "ds_read_b64_tr_b16 a[252:253], v25 offset:8192\n\t" \
    "v_mul_f32_e64 v36, v36, v32\n\t" \
    "v_mfma_f32_16x16x32_bf16 v[120:123], a[212:215], a[164:167], v[120:123]\n\t" \
    "v_mul_f32_e64 v37, v37, v33\n\t" \
    "v_mul_f32_e64 v38, v38, v34\n\t" \
    "v_mfma_f32_16x16x32_bf16 v[124:127], a[216:219], a[160:163], v[124:127]\n\t" \
    "ds_read_b64_tr_b16 a[254:255], v25 offset:9216\n\t" \
    "v_mul_f32_e64 v39, v39, v35\n\t" \
    "v_mfma_f32_16x16x32_bf16 v[124:127], a[220:223], a[164:167], v[124:127]\n\t" \
    "v_med3_f32 v32, v40, -%[c60], %[c60]\n\t" \
    "v_med3_f32 v33, v41, -%[c60], %[c60]\n\t" \
    "v_mfma_f32_16x16x32_bf16 v[144:147], a[192:195], a[168:171], v[144:147]\n\t" \
    "v_med3_f32 v34, v42, -%[c60], %[c60]\n\t" \
    "v_med3_f32 v35, v43, -%[c60], %[c60]\n\t" \
    "v_mfma_f32_16x16x32_bf16 v[144:147], a[196:199], a[172:175], v[144:147]\n\t" \
    "v_mul_f32 v32, 0xbfb8aa3b, v32\n\t" \
    "v_mul_f32 v33, 0xbfb8aa3b, v33\n\t" \
    "v_mfma_f32_16x16x32_bf16 v[148:151], a[200:203], a[168:171], v[148:151]\n\t" \
    "v_mul_f32 v34, 0xbfb8aa3b, v34\n\t" \
    "v_mul_f32 v35, 0xbfb8aa3b, v35\n\t" \
    "v_mfma_f32_16x16x32_bf16 v[148:151], a[204:207], a[172:175], v[148:151]\n\t" \
    "s_add_u32 %[t0], %[ld], 0x40000\n\tbuffer_load_dwordx4 a[76:79], v47, %[rss], %[t0] offen offset:256 " LAUXS "\n\t" \
    "v_mfma_f32_16x16x32_bf16 v[152:155], a[208:211], a[168:171], v[152:155]\n\t" \
    "v_exp_f32_e64 v32, v32\n\t" \
    "v_mfma_f32_16x16x32_bf16 v[152:155], a[212:215], a[172:175], v[152:155]\n\t" \
    "v_exp_f32_e64 v33, v33\n\t" \
    "v_mfma_f32_16x16x32_bf16 v[156:159], a[216:219], a[168:171], v[156:159]\n\t" \
    "v_exp_f32_e64 v34, v34\n\t" \
    "v_mfma_f32_16x16x32_bf16 v[156:159], a[220:223], a[172:175], v[156:159]\n\t" \
    "s_waitcnt lgkmcnt(8)\n\ts_nop 0\n\t" \
    "v_mfma_f32_16x16x32_bf16 v[176:179], a[192:195], a[176:179], v[176:179]\n\t" \
    "v_exp_f32_e64 v35, v35\n\t" \
    "v_mfma_f32_16x16x32_bf16 v[176:179], a[196:199], a[180:183], v[176:179]\n\t" \
    "v_add_f32_e64 v32, 1.0, v32\n\t" \
    "v_add_f32_e64 v33, 1.0, v33\n\t" \
    "v_mfma_f32_16x16x32_bf16 v[180:183], a[200:203], a[176:179], v[180:183]\n\t" \
    "v_add_f32_e64 v34, 1.0, v34\n\t" \
    "v_add_f32_e64 v35, 1.0, v35\n\t" \
    "v_mfma_f32_16x16x32_bf16 v[180:183], a[204:207], a[180:183], v[180:183]\n\t" \
    "v_rcp_f32_e64 v32, v32\n\t" \
    "v_mfma_f32_16x16x32_bf16 v[184:187], a[208:211], a[176:179], v[184:187]\n\t" \
    "v_rcp_f32_e64 v33, v33\n\t" \
    "v_mfma_f32_16x16x32_bf16 v[184:187], a[212:215], a[180:183], v[184:187]\n\t" \
    "v_rcp_f32_e64 v34, v34\n\t" \
    "v_mfma_f32_16x16x32_bf16 v[188:191], a[216:219], a[176:179], v[188:191]\n\t" \
    "v_rcp_f32_e64 v35, v35\n\t" \
    "v_mfma_f32_16x16x32_bf16 v[188:191], a[220:223], a[180:183], v[188:191]\n\t" \
    "v_mul_f32_e64 v40, v40, v32\n\t" \
    "v_mul_f32_e64 v41, v41, v33\n\t" \
    "v_mfma_f32_16x16x32_bf16 v[208:211], a[192:195], a[184:187], v[208:211]\n\t" \
    "v_mul_f32_e64 v42, v42, v34\n\t" \
    "v_mul_f32_e64 v43, v43, v35\n\t" \
    "v_mfma_f32_16x16x32_bf16 v[208:211], a[196:199], a[188:191], v[208:211]\n\t" \
    "s_add_u32 %[ld], %[ld], %[dl]\n\tbuffer_load_dwordx4 a[80:83], v47, %[rss], %[ld] offen offset:0 " LAUXS "\n\t" \
    "v_mfma_f32_16x16x32_bf16 v[212:215], a[200:203], a[184:187], v[212:215]\n\t" \
    "v_cvt_pk_bf16_f32 v36, v36, v37\n\t" \
    "v_cvt_pk_bf16_f32 v37, v38, v39\n\t" \
    "v_mfma_f32_16x16x32_bf16 v[212:215], a[204:207], a[188:191], v[212:215]\n\t" \
    "v_cvt_pk_bf16_f32 v38, v40, v41\n\t" \
    "v_cvt_pk_bf16_f32 v39, v42, v43\n\t" \
    "v_mfma_f32_16x16x32_bf16 v[216:219], a[208:211], a[184:187], v[216:219]\n\t" \
    "buffer_store_dwordx4 v[36:39], v31, %[rso], %[oo] offen sc0 sc1\n\t" \
    "v_mfma_f32_16x16x32_bf16 v[216:219], a[212:215], a[188:191], v[216:219]\n\t" \
    "v_mfma_f32_16x16x32_bf16 v[220:223], a[216:219], a[184:187], v[220:223]\n\t" \
    "v_mfma_f32_16x16x32_bf16 v[220:223], a[220:223], a[188:191], v[220:223]\n\t" \
    "s_waitcnt lgkmcnt(0)\n\ts_barrier\n\t" \
    "v_mfma_f32_16x16x32_bf16 a[16:19], a[224:227], a[128:131], a[16:19]\n\t" \
    "ds_read_b64_tr_b16 a[192:193], v22 offset:16384\n\t" \
    "v_med3_f32 v32, v240, -%[c60], %[c60]\n\t" \
    "v_med3_f32 v33, v241, -%[c60], %[c60]\n\t" \
    "v_mfma_f32_16x16x32_bf16 a[16:19], a[228:231], a[132:135], a[16:19]\n\t" \
    "v_med3_f32 v34, v242, -%[c60], %[c60]\n\t" \
    "v_med3_f32 v35, v243, -%[c60], %[c60]\n\t" \
    "v_mul_f32 v32, 0xbfb8aa3b, v32\n\t" \
    "v_mfma_f32_16x16x32_bf16 a[20:23], a[232:235], a[128:131], a[20:23]\n\t" \
    "ds_read_b64_tr_b16 a[194:195], v22 offset:17408\n\t" \
    "v_mul_f32 v33, 0xbfb8aa3b, v33\n\t" \
    "v_mul_f32 v34, 0xbfb8aa3b, v34\n\t" \
    "v_mfma_f32_16x16x32_bf16 a[20:23], a[236:239], a[132:135], a[20:23]\n\t" \
    "v_mul_f32 v35, 0xbfb8aa3b, v35\n\t" \
    "v_exp_f32_e64 v32, v32\n\t" \
    "v_mfma_f32_16x16x32_bf16 a[24:27], a[240:243], a[128:131], a[24:27]\n\t" \
    "ds_read_b64_tr_b16 a[196:197], v22 offset:24576\n\t" \
    "v_exp_f32_e64 v33, v33\n\t" \
    "v_mfma_f32_16x16x32_bf16 a[24:27], a[244:247], a[132:135], a[24:27]\n\t" \
    "buffer_load_dwordx4 a[84:87], v47, %[rss], %[ld] offen offset:256 " LAUXS "\n\t" \
    "v_mfma_f32_16x16x32_bf16 a[28:31], a[248:251], a[128:131], a[28:31]\n\t" \
    "ds_read_b64_tr_b16 a[198:199], v22 offset:25600\n\t" \
    "v_exp_f32_e64 v34, v34\n\t" \
    "v_mfma_f32_16x16x32_bf16 a[28:31], a[252:255], a[132:135], a[28:31]\n\t" \
    "v_exp_f32_e64 v35, v35\n\t" \
    "v_add_f32_e64 v32, 1.0, v32\n\t" \
    "v_mfma_f32_16x16x32_bf16 a[48:51], a[224:227], a[136:139], a[48:51]\n\t" \
    "ds_read_b128 a[128:131], v20 offset:0\n\t" \
    "v_add_f32_e64 v33, 1.0, v33\n\t" \
    "v_add_f32_e64 v34, 1.0, v34\n\t" \
    "v_mfma_f32_16x16x32_bf16 a[48:51], a[228:231], a[140:143], a[48:51]\n\t" \
    "v_add_f32_e64 v35, 1.0, v35\n\t" \
    "v_rcp_f32_e64 v32, v32\n\t" \
    "v_mfma_f32_16x16x32_bf16 a[52:55], a[232:235], a[136:139], a[52:55]\n\t" \
    "ds_read_b128 a[132:135], v21 offset:0\n\t" \
    "v_rcp_f32_e64 v33, v33\n\t" \
    "v_mfma_f32_16x16x32_bf16 a[52:55], a[236:239], a[140:143], a[52:55]\n\t" \
    "v_rcp_f32_e64 v34, v34\n\t" \
    "v_mfma_f32_16x16x32_bf16 a[56:59], a[240:243], a[136:139], a[56:59]\n\t" \
    "ds_read_b64_tr_b16 a[200:201], v23 offset:16384\n\t" \
    "v_rcp_f32_e64 v35, v35\n\t" \
    "v_mfma_f32_16x16x32_bf16 a[56:59], a[244:247], a[140:143], a[56:59]\n\t" \
    "v_mul_f32_e64 v240, v240, v32\n\t" \
    "v_mul_f32_e64 v241, v241, v33\n\t" \
    "v_mul_f32_e64 v242, v242, v34\n\t" \
    "v_mfma_f32_16x16x32_bf16 a[60:63], a[248:251], a[136:139], a[60:63]\n\t" \
    "ds_read_b64_tr_b16 a[202:203], v23 offset:17408\n\t" \
    "v_mul_f32_e64 v243, v243, v35\n\t" \
    "v_med3_f32 v32, v244, -%[c60], %[c60]\n\t" \
    "v_mfma_f32_16x16x32_bf16 a[60:63], a[252:255], a[140:143], a[60:63]\n\t" \
    "s_add_u32 %[t0], %[ld], 0x40000\n\tbuffer_load_dwordx4 a[88:91], v47, %[rss], %[t0] offen offset:0 " LAUXS "\n\t" \
    "v_med3_f32 v33, v245, -%[c60], %[c60]\n\t" \
    "v_mfma_f32_16x16x32_bf16 v[64:67], a[224:227], a[144:147], v[64:67]\n\t" \
    "ds_read_b128 a[136:139], v20 offset:2048\n\t" \
    "v_med3_f32 v34, v246, -%[c60], %[c60]\n\t" \
    "v_med3_f32 v35, v247, -%[c60], %[c60]\n\t" \
    "v_mfma_f32_16x16x32_bf16 v[64:67], a[228:231], a[148:151], v[64:67]\n\t" \
    "v_mul_f32 v32, 0xbfb8aa3b, v32\n\t" \
    "v_mul_f32 v33, 0xbfb8aa3b, v33\n\t" \
    "v_mul_f32 v34, 0xbfb8aa3b, v34\n\t" \
    "v_mfma_f32_16x16x32_bf16 v[68:71], a[232:235], a[144:147], v[68:71]\n\t" \
    "ds_read_b128 a[140:143], v21 offset:2048\n\t" \
    "v_mul_f32 v35, 0xbfb8aa3b, v35\n\t" \
    "v_mfma_f32_16x16x32_bf16 v[68:71], a[236:239], a[148:151], v[68:71]\n\t" \
    "v_exp_f32_e64 v32, v32\n\t" \
    "v_mfma_f32_16x16x32_bf16 v[72:75], a[240:243], a[144:147], v[72:75]\n\t" \
    "ds_read_b64_tr_b16 a[204:205], v23 offset:24576\n\t" \
    "v_exp_f32_e64 v33, v33\n\t" \
    "v_mfma_f32_16x16x32_bf16 v[72:75], a[244:247], a[148:151], v[72:75]\n\t" \
    "v_exp_f32_e64 v34, v34\n\t" \
    "v_mfma_f32_16x16x32_bf16 v[76:79], a[248:251], a[144:147], v[76:79]\n\t" \
    "ds_read_b64_tr_b16 a[206:207], v23 offset:25600\n\t" \
    "v_exp_f32_e64 v35, v35\n\t" \
    "v_mfma_f32_16x16x32_bf16 v[76:79], a[252:255], a[148:151], v[76:79]\n\t" \
    "buffer_load_dword v16, v17, %[rs], %[so] offen \n\t" \
    "v_add_f32_e64 v32, 1.0, v32\n\t" \
    "v_mfma_f32_16x16x32_bf16 v[96:99], a[224:227], a[152:155], v[96:99]\n\t" \
    "ds_read_b128 a[144:147], v20 offset:4096\n\t" \
    "v_add_f32_e64 v33, 1.0, v33\n\t" \
    "v_add_f32_e64 v34, 1.0, v34\n\t" \
    "v_mfma_f32_16x16x32_bf16 v[96:99], a[228:231], a[156:159], v[96:99]\n\t" \
    "s_add_u32 %[so], %[so], %[pfi]\n\ts_nop 0\n\t" \
    "s_add_u32 %[t0], %[ld], 0x40000\n\tbuffer_load_dwordx4 a[92:95], v47, %[rss], %[t0] offen offset:256 " LAUXS "\n\t" \
    "v_mfma_f32_16x16x32_bf16 v[100:103], a[232:235], a[152:155], v[100:103]\n\t" \
    "ds_read_b128 a[148:151], v21 offset:4096\n\t" \
    "v_add_f32_e64 v35, 1.0, v35\n\t" \
    "v_mfma_f32_16x16x32_bf16 v[100:103], a[236:239], a[156:159], v[100:103]\n\t" \
    "s_cmp_eq_u32 %[rem], %[psw]\n\ts_cselect_b32 %[so], %[nx], %[so]\n\t" \
    "v_mfma_f32_16x16x32_bf16 v[104:107], a[240:243], a[152:155], v[104:107]\n\t" \
    "ds_read_b64_tr_b16 a[208:209], v24 offset:16384\n\t" \
    "v_rcp_f32_e64 v32, v32\n\t" \
    "v_mfma_f32_16x16x32_bf16 v[104:107], a[244:247], a[156:159], v[104:107]\n\t" \
    "v_rcp_f32_e64 v33, v33\n\t" \
    "v_mfma_f32_16x16x32_bf16 v[108:111], a[248:251], a[152:155], v[108:111]\n\t" \
    "ds_read_b64_tr_b16 a[210:211], v24 offset:17408\n\t" \
    "v_rcp_f32_e64 v34, v34\n\t" \
    "v_mfma_f32_16x16x32_bf16 v[108:111], a[252:255], a[156:159], v[108:111]\n\t" \
    "v_rcp_f32_e64 v35, v35\n\t" \
    "v_mul_f32_e64 v244, v244, v32\n\t" \
    "v_mfma_f32_16x16x32_bf16 v[128:131], a[224:227], a[160:163], v[128:131]\n\t" \
    "ds_read_b128 a[152:155], v20 offset:6144\n\t" \
    "v_mul_f32_e64 v245, v245, v33\n\t" \
    "v_mul_f32_e64 v246, v246, v34\n\t" \
    "v_mfma_f32_16x16x32_bf16 v[128:131], a[228:231], a[164:167], v[128:131]\n\t" \
    "v_mul_f32_e64 v247, v247, v35\n\t" \
    "v_cvt_pk_bf16_f32 v240, v240, v241\n\t" \
    "v_cvt_pk_bf16_f32 v241, v242, v243\n\t" \
    "v_mfma_f32_16x16x32_bf16 v[132:135], a[232:235], a[160:163], v[132:135]\n\t" \
    "ds_read_b128 a[156:159], v21 offset:6144\n\t" \
    "v_cvt_pk_bf16_f32 v242, v244, v245\n\t" \
    "v_cvt_pk_bf16_f32 v243, v246, v247\n\t" \
    "v_mfma_f32_16x16x32_bf16 v[132:135], a[236:239], a[164:167], v[132:135]\n\t" \
    "v_mov_b32 v18, 0\n\tbuffer_load_dword v18, off, %[rss], %[pc] sc1\n\t" \
    "v_mfma_f32_16x16x32_bf16 v[136:139], a[240:243], a[160:163], v[136:139]\n\t" \
    "ds_read_b64_tr_b16 a[212:213], v24 offset:24576\n\t" \
    "s_add_u32 %[t0], %[oo], 0x4000\n\tbuffer_store_dwordx4 v[240:243], v31, %[rso], %[t0] offen sc0 sc1\n\t" \
    "v_mfma_f32_16x16x32_bf16 v[136:139], a[244:247], a[164:167], v[136:139]\n\t" \
    "s_add_u32 %[c], %[c], 1\n\t" \
    "s_mov_b32 %[gap], 1\n\t" \
    "v_mfma_f32_16x16x32_bf16 v[140:143], a[248:251], a[160:163], v[140:143]\n\t" \
    "ds_read_b64_tr_b16 a[214:215], v24 offset:25600\n\t" \
    "v_mov_b32 v19, 0\n\t" \
    "v_readlane_b32 %[ld], v14, %[c]\n\t" \
    "v_mfma_f32_16x16x32_bf16 v[140:143], a[252:255], a[164:167], v[140:143]\n\t" \
    "v_readlane_b32 %[oo], v15, %[c]\n\t" \
    "v_mfma_f32_16x16x32_bf16 v[160:163], a[224:227], a[168:171], v[160:163]\n\t" \
    "ds_read_b128 a[160:163], v20 offset:8192\n\t" \
    "v_mfma_f32_16x16x32_bf16 v[160:163], a[228:231], a[172:175], v[160:163]\n\t" \
    "v_mfma_f32_16x16x32_bf16 v[164:167], a[232:235], a[168:171], v[164:167]\n\t" \
    "ds_read_b128 a[164:167], v21 offset:8192\n\t" \
    "v_mfma_f32_16x16x32_bf16 v[164:167], a[236:239], a[172:175], v[164:167]\n\t" \
    "v_mfma_f32_16x16x32_bf16 v[168:171], a[240:243], a[168:171], v[168:171]\n\t" \
    "ds_read_b64_tr_b16 a[216:217], v25 offset:16384\n\t" \
    "v_mfma_f32_16x16x32_bf16 v[168:171], a[244:247], a[172:175], v[168:171]\n\t" \
    "v_mfma_f32_16x16x32_bf16 v[172:175], a[248:251], a[168:171], v[172:175]\n\t" \
    "ds_read_b64_tr_b16 a[218:219], v25 offset:17408\n\t" \
    "v_mfma_f32_16x16x32_bf16 v[172:175], a[252:255], a[172:175], v[172:175]\n\t" \
    "v_mfma_f32_16x16x32_bf16 v[192:195], a[224:227], a[176:179], v[192:195]\n\t" \
    "ds_read_b128 a[168:171], v20 offset:10240\n\t" \
    "v_mfma_f32_16x16x32_bf16 v[192:195], a[228:231], a[180:183], v[192:195]\n\t" \
    "v_mfma_f32_16x16x32_bf16 v[196:199], a[232:235], a[176:179], v[196:199]\n\t" \
    "ds_read_b128 a[172:175], v21 offset:10240\n\t" \
    "v_mfma_f32_16x16x32_bf16 v[196:199], a[236:239], a[180:183], v[196:199]\n\t" \
    "v_readfirstlane_b32 %[go], v18\n\t" \
    "v_readfirstlane_b32 %[t0], v44\n\t" \
    "v_mfma_f32_16x16x32_bf16 v[200:203], a[240:243], a[176:179], v[200:203]\n\t" \
    "ds_read_b64_tr_b16 a[220:221], v25 offset:24576\n\t" \
    "s_bitcmp1_b32 %[c], 0\n\ts_cselect_b32 %[t0], %[t0], %[go]\n\t" \
    "v_mfma_f32_16x16x32_bf16 v[200:203], a[244:247], a[180:183], v[200:203]\n\t" \
    "v_readfirstlane_b32 %[go], v19\n\t" \
    "s_cmp_ge_u32 %[ph], 2\n\ts_cselect_b32 %[go], %[t0], %[go]\n\t" \
    "v_mfma_f32_16x16x32_bf16 v[204:207], a[248:251], a[176:179], v[204:207]\n\t" \
    "ds_read_b64_tr_b16 a[222:223], v25 offset:25600\n\t" \
    "s_cmp_ge_u32 %[go], %[thr]\n\ts_cselect_b32 %[go], %[zls], 0\n\t" \
    "v_mfma_f32_16x16x32_bf16 v[204:207], a[252:255], a[180:183], v[204:207]\n\t" \
    "s_cmp_le_i32 %[gap], 0\n\ts_cselect_b32 %[go], %[go], 0\n\t" \
    "v_mfma_f32_16x16x32_bf16 v[224:227], a[224:227], a[184:187], v[224:227]\n\t" \
    "v_add_u32 v23, 0x8000, v22\n\t" \
    "v_add_u32 v22, 0xffff0000, v22\n\t" \
    "v_mfma_f32_16x16x32_bf16 v[224:227], a[228:231], a[188:191], v[224:227]\n\t" \
    "v_min_u32_e64 v22, v22, v23\n\t" \
    "v_mfma_f32_16x16x32_bf16 v[228:231], a[232:235], a[184:187], v[228:231]\n\t" \
    "v_xor_b32_e64 v23, 32, v22\n\t" \
    "v_xor_b32_e64 v24, 64, v22\n\t" \
    "v_mfma_f32_16x16x32_bf16 v[228:231], a[236:239], a[188:191], v[228:231]\n\t" \
    "v_xor_b32 v25, 0x60, v22\n\t" \
    "v_mfma_f32_16x16x32_bf16 v[232:235], a[240:243], a[184:187], v[232:235]\n\t" \
    "v_mfma_f32_16x16x32_bf16 v[232:235], a[244:247], a[188:191], v[232:235]\n\t" \
    "v_mfma_f32_16x16x32_bf16 v[236:239], a[248:251], a[184:187], v[236:239]\n\t" \
    "v_mfma_f32_16x16x32_bf16 v[236:239], a[252:255], a[188:191], v[236:239]\n\t" \
    "s_sub_u32 %[rem], %[rem], 1\n\t" \
    "s_cmp_eq_u32 %[c], %[cend]\n\t" \
    "s_waitcnt lgkmcnt(0)\n\ts_barrier\n\t" \
    "s_cbranch_scc1 90f\n\t" \
    "s_cmp_le_i32 %[rem], %[stop]\n\t" \
    "s_cbranch_scc1 90f\n\t" \
    "s_cmp_lg_u32 %[go], 0\n\t" \
    "s_cbranch_scc1 20b\n\t" \
    "s_branch 10b\n\t" \
    "90:\n\t" \
    ""
#define ASM_C0 \
    "v_mfma_f32_16x16x32_bf16 a[0:3], a[192:195], a[128:131], a[0:3]\n\t" \
    "ds_read_b128 a[176:179], v20 offset:12288\n\t" \
    "v_mfma_f32_16x16x32_bf16 a[0:3], a[196:199], a[132:135], a[0:3]\n\t" \
    "v_mfma_f32_16x16x32_bf16 a[4:7], a[200:203], a[128:131], a[4:7]\n\t" \
    "ds_read_b128 a[180:183], v21 offset:12288\n\t" \
    "v_mfma_f32_16x16x32_bf16 a[4:7], a[204:207], a[132:135], a[4:7]\n\t" \
    "v_mfma_f32_16x16x32_bf16 a[8:11], a[208:211], a[128:131], a[8:11]\n\t" \
    "ds_read_b128 a[184:187], v20 offset:14336\n\t" \
    "v_mfma_f32_16x16x32_bf16 a[8:11], a[212:215], a[132:135], a[8:11]\n\t" \
    "v_mfma_f32_16x16x32_bf16 a[12:15], a[216:219], a[128:131], a[12:15]\n\t" \
    "ds_read_b128 a[188:191], v21 offset:14336\n\t" \
    "v_mfma_f32_16x16x32_bf16 a[12:15], a[220:223], a[132:135], a[12:15]\n\t" \
    "v_mfma_f32_16x16x32_bf16 a[32:35], a[192:195], a[136:139], a[32:35]\n\t" \
    "ds_read_b64_tr_b16 a[224:225], v22 offset:0\n\t" \
    "v_mfma_f32_16x16x32_bf16 a[32:35], a[196:199], a[140:143], a[32:35]\n\t" \
    "v_xor_b32 v20, 0x38000, v20\n\t" \
    "v_xor_b32 v21, 0x38000, v21\n\t" \
    "v_mfma_f32_16x16x32_bf16 a[36:39], a[200:203], a[136:139], a[36:39]\n\t" \
    "ds_read_b64_tr_b16 a[226:227], v22 offset:1024\n\t" \
    "v_mfma_f32_16x16x32_bf16 a[36:39], a[204:207], a[140:143], a[36:39]\n\t" \
    "v_mfma_f32_16x16x32_bf16 a[40:43], a[208:211], a[136:139], a[40:43]\n\t" \
    "ds_read_b64_tr_b16 a[228:229], v22 offset:8192\n\t" \
    "v_mfma_f32_16x16x32_bf16 a[40:43], a[212:215], a[140:143], a[40:43]\n\t" \
    "v_mfma_f32_16x16x32_bf16 a[44:47], a[216:219], a[136:139], a[44:47]\n\t" \
    "ds_read_b64_tr_b16 a[230:231], v22 offset:9216\n\t" \
    "v_mfma_f32_16x16x32_bf16 a[44:47], a[220:223], a[140:143], a[44:47]\n\t" \
    "v_mfma_f32_16x16x32_bf16 v[48:51], a[192:195], a[144:147], v[48:51]\n\t" \
    "ds_read_b64_tr_b16 a[232:233], v23 offset:0\n\t" \
    "v_mfma_f32_16x16x32_bf16 v[48:51], a[196:199], a[148:151], v[48:51]\n\t" \
    "v_mfma_f32_16x16x32_bf16 v[52:55], a[200:203], a[144:147], v[52:55]\n\t" \
    "ds_read_b64_tr_b16 a[234:235], v23 offset:1024\n\t" \
    "v_mfma_f32_16x16x32_bf16 v[52:55], a[204:207], a[148:151], v[52:55]\n\t" \
    "v_mfma_f32_16x16x32_bf16 v[56:59], a[208:211], a[144:147], v[56:59]\n\t" \
    "ds_read_b64_tr_b16 a[236:237], v23 offset:8192\n\t" \
    "v_mfma_f32_16x16x32_bf16 v[56:59], a[212:215], a[148:151], v[56:59]\n\t" \
    "v_mfma_f32_16x16x32_bf16 v[60:63], a[216:219], a[144:147], v[60:63]\n\t" \
    "ds_read_b64_tr_b16 a[238:239], v23 offset:9216\n\t" \
    "v_mfma_f32_16x16x32_bf16 v[60:63], a[220:223], a[148:151], v[60:63]\n\t" \
    "v_mfma_f32_16x16x32_bf16 v[80:83], a[192:195], a[152:155], v[80:83]\n\t" \
    "ds_read_b64_tr_b16 a[240:241], v24 offset:0\n\t" \
    "v_mfma_f32_16x16x32_bf16 v[80:83], a[196:199], a[156:159], v[80:83]\n\t" \
    "v_mfma_f32_16x16x32_bf16 v[84:87], a[200:203], a[152:155], v[84:87]\n\t" \
    "ds_read_b64_tr_b16 a[242:243], v24 offset:1024\n\t" \
    "v_mfma_f32_16x16x32_bf16 v[84:87], a[204:207], a[156:159], v[84:87]\n\t" \
    "v_mfma_f32_16x16x32_bf16 v[88:91], a[208:211], a[152:155], v[88:91]\n\t" \
    "ds_read_b64_tr_b16 a[244:245], v24 offset:8192\n\t" \
    "v_mfma_f32_16x16x32_bf16 v[88:91], a[212:215], a[156:159], v[88:91]\n\t" \
    "v_mfma_f32_16x16x32_bf16 v[92:95], a[216:219], a[152:155], v[92:95]\n\t" \
    "ds_read_b64_tr_b16 a[246:247], v24 offset:9216\n\t" \
    "v_mfma_f32_16x16x32_bf16 v[92:95], a[220:223], a[156:159], v[92:95]\n\t" \
    "v_mfma_f32_16x16x32_bf16 v[112:115], a[192:195], a[160:163], v[112:115]\n\t" \
    "ds_read_b64_tr_b16 a[248:249], v25 offset:0\n\t" \
    "v_mfma_f32_16x16x32_bf16 v[112:115], a[196:199], a[164:167], v[112:115]\n\t" \
    "v_mfma_f32_16x16x32_bf16 v[116:119], a[200:203], a[160:163], v[116:119]\n\t" \
    "ds_read_b64_tr_b16 a[250:251], v25 offset:1024\n\t" \
    "v_mfma_f32_16x16x32_bf16 v[116:119], a[204:207], a[164:167], v[116:119]\n\t" \
    "v_mfma_f32_16x16x32_bf16 v[120:123], a[208:211], a[160:163], v[120:123]\n\t" \
    "ds_read_b64_tr_b16 a[252:253], v25 offset:8192\n\t" \
    "v_mfma_f32_16x16x32_bf16 v[120:123], a[212:215], a[164:167], v[120:123]\n\t" \
    "v_mfma_f32_16x16x32_bf16 v[124:127], a[216:219], a[160:163], v[124:127]\n\t" \
    "ds_read_b64_tr_b16 a[254:255], v25 offset:9216\n\t" \
    "v_mfma_f32_16x16x32_bf16 v[124:127], a[220:223], a[164:167], v[124:127]\n\t" \
    "v_mfma_f32_16x16x32_bf16 v[144:147], a[192:195], a[168:171], v[144:147]\n\t" \
    "v_mfma_f32_16x16x32_bf16 v[144:147], a[196:199], a[172:175], v[144:147]\n\t" \
    "v_mfma_f32_16x16x32_bf16 v[148:151], a[200:203], a[168:171], v[148:151]\n\t" \
    "v_mfma_f32_16x16x32_bf16 v[148:151], a[204:207], a[172:175], v[148:151]\n\t" \
    "v_mfma_f32_16x16x32_bf16 v[152:155], a[208:211], a[168:171], v[152:155]\n\t" \
    "v_mfma_f32_16x16x32_bf16 v[152:155], a[212:215], a[172:175], v[152:155]\n\t" \
    "v_mfma_f32_16x16x32_bf16 v[156:159], a[216:219], a[168:171], v[156:159]\n\t" \
    "v_mfma_f32_16x16x32_bf16 v[156:159], a[220:223], a[172:175], v[156:159]\n\t" \
    "s_waitcnt lgkmcnt(8)\n\ts_nop 0\n\t" \
    "v_mfma_f32_16x16x32_bf16 v[176:179], a[192:195], a[176:179], v[176:179]\n\t" \
    "v_mfma_f32_16x16x32_bf16 v[176:179], a[196:199], a[180:183], v[176:179]\n\t" \
    "v_mfma_f32_16x16x32_bf16 v[180:183], a[200:203], a[176:179], v[180:183]\n\t" \
    "v_mfma_f32_16x16x32_bf16 v[180:183], a[204:207], a[180:183], v[180:183]\n\t" \
    "v_mfma_f32_16x16x32_bf16 v[184:187], a[208:211], a[176:179], v[184:187]\n\t" \
    "v_mfma_f32_16x16x32_bf16 v[184:187], a[212:215], a[180:183], v[184:187]\n\t" \
    "v_mfma_f32_16x16x32_bf16 v[188:191], a[216:219], a[176:179], v[188:191]\n\t" \
    "v_mfma_f32_16x16x32_bf16 v[188:191], a[220:223], a[180:183], v[188:191]\n\t" \
    "v_mfma_f32_16x16x32_bf16 v[208:211], a[192:195], a[184:187], v[208:211]\n\t" \
    "v_mfma_f32_16x16x32_bf16 v[208:211], a[196:199], a[188:191], v[208:211]\n\t" \
    "v_mfma_f32_16x16x32_bf16 v[212:215], a[200:203], a[184:187], v[212:215]\n\t" \
    "v_mfma_f32_16x16x32_bf16 v[212:215], a[204:207], a[188:191], v[212:215]\n\t" \
    "v_mfma_f32_16x16x32_bf16 v[216:219], a[208:211], a[184:187], v[216:219]\n\t" \
    "v_mfma_f32_16x16x32_bf16 v[216:219], a[212:215], a[188:191], v[216:219]\n\t" \
    "v_mfma_f32_16x16x32_bf16 v[220:223], a[216:219], a[184:187], v[220:223]\n\t" \
    "v_mfma_f32_16x16x32_bf16 v[220:223], a[220:223], a[188:191], v[220:223]\n\t" \
    "s_waitcnt vmcnt(17)\n\t" \
    "s_waitcnt lgkmcnt(0)\n\ts_barrier\n\t" \
    "v_mfma_f32_16x16x32_bf16 a[16:19], a[224:227], a[128:131], a[16:19]\n\t" \
    "ds_read_b64_tr_b16 a[192:193], v22 offset:16384\n\t" \
    "v_mfma_f32_16x16x32_bf16 a[16:19], a[228:231], a[132:135], a[16:19]\n\t" \
    "v_mfma_f32_16x16x32_bf16 a[20:23], a[232:235], a[128:131], a[20:23]\n\t" \
    "ds_read_b64_tr_b16 a[194:195], v22 offset:17408\n\t" \
    "v_mfma_f32_16x16x32_bf16 a[20:23], a[236:239], a[132:135], a[20:23]\n\t" \
    "v_mfma_f32_16x16x32_bf16 a[24:27], a[240:243], a[128:131], a[24:27]\n\t" \
    "ds_read_b64_tr_b16 a[196:197], v22 offset:24576\n\t" \
    "v_mfma_f32_16x16x32_bf16 a[24:27], a[244:247], a[132:135], a[24:27]\n\t" \
    "v_mfma_f32_16x16x32_bf16 a[28:31], a[248:251], a[128:131], a[28:31]\n\t" \
    "ds_read_b64_tr_b16 a[198:199], v22 offset:25600\n\t" \
    "v_mfma_f32_16x16x32_bf16 a[28:31], a[252:255], a[132:135], a[28:31]\n\t" \
    "v_mfma_f32_16x16x32_bf16 a[48:51], a[224:227], a[136:139], a[48:51]\n\t" \
    "ds_read_b128 a[128:131], v20 offset:0\n\t" \
    "v_mfma_f32_16x16x32_bf16 a[48:51], a[228:231], a[140:143], a[48:51]\n\t" \
    "v_mfma_f32_16x16x32_bf16 a[52:55], a[232:235], a[136:139], a[52:55]\n\t" \
    "ds_read_b128 a[132:135], v21 offset:0\n\t" \
    "v_mfma_f32_16x16x32_bf16 a[52:55], a[236:239], a[140:143], a[52:55]\n\t" \
    "v_mfma_f32_16x16x32_bf16 a[56:59], a[240:243], a[136:139], a[56:59]\n\t" \
    "ds_read_b64_tr_b16 a[200:201], v23 offset:16384\n\t" \
    "v_mfma_f32_16x16x32_bf16 a[56:59], a[244:247], a[140:143], a[56:59]\n\t" \
    "v_mfma_f32_16x16x32_bf16 a[60:63], a[248:251], a[136:139], a[60:63]\n\t" \
    "ds_read_b64_tr_b16 a[202:203], v23 offset:17408\n\t" \
    "v_mfma_f32_16x16x32_bf16 a[60:63], a[252:255], a[140:143], a[60:63]\n\t" \
    "v_mfma_f32_16x16x32_bf16 v[64:67], a[224:227], a[144:147], v[64:67]\n\t" \
    "ds_read_b128 a[136:139], v20 offset:2048\n\t" \
    "v_mfma_f32_16x16x32_bf16 v[64:67], a[228:231], a[148:151], v[64:67]\n\t" \
    "v_mfma_f32_16x16x32_bf16 v[68:71], a[232:235], a[144:147], v[68:71]\n\t" \
    "ds_read_b128 a[140:143], v21 offset:2048\n\t" \
    "v_mfma_f32_16x16x32_bf16 v[68:71], a[236:239], a[148:151], v[68:71]\n\t" \
    "v_mfma_f32_16x16x32_bf16 v[72:75], a[240:243], a[144:147], v[72:75]\n\t" \
    "ds_read_b64_tr_b16 a[204:205], v23 offset:24576\n\t" \
    "v_mfma_f32_16x16x32_bf16 v[72:75], a[244:247], a[148:151], v[72:75]\n\t" \
    "v_mfma_f32_16x16x32_bf16 v[76:79], a[248:251], a[144:147], v[76:79]\n\t" \
    "ds_read_b64_tr_b16 a[206:207], v23 offset:25600\n\t" \
    "v_mfma_f32_16x16x32_bf16 v[76:79], a[252:255], a[148:151], v[76:79]\n\t" \
    "buffer_load_dword v16, v17, %[rs], %[so] offen \n\t" \
    "v_mfma_f32_16x16x32_bf16 v[96:99], a[224:227], a[152:155], v[96:99]\n\t" \
    "ds_read_b128 a[144:147], v20 offset:4096\n\t" \
    "v_mfma_f32_16x16x32_bf16 v[96:99], a[228:231], a[156:159], v[96:99]\n\t" \
    "s_add_u32 %[so], %[so], %[pfi]\n\ts_nop 0\n\t" \
    "v_mfma_f32_16x16x32_bf16 v[100:103], a[232:235], a[152:155], v[100:103]\n\t" \
    "ds_read_b128 a[148:151], v21 offset:4096\n\t" \
    "v_mfma_f32_16x16x32_bf16 v[100:103], a[236:239], a[156:159], v[100:103]\n\t" \
    "s_cmp_eq_u32 %[rem], %[psw]\n\ts_cselect_b32 %[so], %[nx], %[so]\n\t" \
    "v_mfma_f32_16x16x32_bf16 v[104:107], a[240:243], a[152:155], v[104:107]\n\t" \
    "ds_read_b64_tr_b16 a[208:209], v24 offset:16384\n\t" \
    "v_mfma_f32_16x16x32_bf16 v[104:107], a[244:247], a[156:159], v[104:107]\n\t" \
    "v_mfma_f32_16x16x32_bf16 v[108:111], a[248:251], a[152:155], v[108:111]\n\t" \
    "ds_read_b64_tr_b16 a[210:211], v24 offset:17408\n\t" \
    "v_mfma_f32_16x16x32_bf16 v[108:111], a[252:255], a[156:159], v[108:111]\n\t" \
    "v_mfma_f32_16x16x32_bf16 v[128:131], a[224:227], a[160:163], v[128:131]\n\t" \
    "ds_read_b128 a[152:155], v20 offset:6144\n\t" \
    "v_mfma_f32_16x16x32_bf16 v[128:131], a[228:231], a[164:167], v[128:131]\n\t" \
    "v_mfma_f32_16x16x32_bf16 v[132:135], a[232:235], a[160:163], v[132:135]\n\t" \
    "ds_read_b128 a[156:159], v21 offset:6144\n\t" \
    "v_mfma_f32_16x16x32_bf16 v[132:135], a[236:239], a[164:167], v[132:135]\n\t" \
    "v_mfma_f32_16x16x32_bf16 v[136:139], a[240:243], a[160:163], v[136:139]\n\t" \
    "ds_read_b64_tr_b16 a[212:213], v24 offset:24576\n\t" \
    "v_mfma_f32_16x16x32_bf16 v[136:139], a[244:247], a[164:167], v[136:139]\n\t" \
    "v_mfma_f32_16x16x32_bf16 v[140:143], a[248:251], a[160:163], v[140:143]\n\t" \
    "ds_read_b64_tr_b16 a[214:215], v24 offset:25600\n\t" \
    "v_mfma_f32_16x16x32_bf16 v[140:143], a[252:255], a[164:167], v[140:143]\n\t" \
    "v_mfma_f32_16x16x32_bf16 v[160:163], a[224:227], a[168:171], v[160:163]\n\t" \
    "ds_read_b128 a[160:163], v20 offset:8192\n\t" \
    "v_mfma_f32_16x16x32_bf16 v[160:163], a[228:231], a[172:175], v[160:163]\n\t" \
    "v_mfma_f32_16x16x32_bf16 v[164:167], a[232:235], a[168:171], v[164:167]\n\t" \
    "ds_read_b128 a[164:167], v21 offset:8192\n\t" \
    "v_mfma_f32_16x16x32_bf16 v[164:167], a[236:239], a[172:175], v[164:167]\n\t" \
    "v_mfma_f32_16x16x32_bf16 v[168:171], a[240:243], a[168:171], v[168:171]\n\t" \
    "ds_read_b64_tr_b16 a[216:217], v25 offset:16384\n\t" \
    "v_mfma_f32_16x16x32_bf16 v[168:171], a[244:247], a[172:175], v[168:171]\n\t" \
    "v_mfma_f32_16x16x32_bf16 v[172:175], a[248:251], a[168:171], v[172:175]\n\t" \
    "ds_read_b64_tr_b16 a[218:219], v25 offset:17408\n\t" \
    "v_mfma_f32_16x16x32_bf16 v[172:175], a[252:255], a[172:175], v[172:175]\n\t" \
    "v_mfma_f32_16x16x32_bf16 v[192:195], a[224:227], a[176:179], v[192:195]\n\t" \
    "ds_read_b128 a[168:171], v20 offset:10240\n\t" \
    "v_mfma_f32_16x16x32_bf16 v[192:195], a[228:231], a[180:183], v[192:195]\n\t" \
    "v_mfma_f32_16x16x32_bf16 v[196:199], a[232:235], a[176:179], v[196:199]\n\t" \
    "ds_read_b128 a[172:175], v21 offset:10240\n\t" \
    "v_mfma_f32_16x16x32_bf16 v[196:199], a[236:239], a[180:183], v[196:199]\n\t" \
    "v_mfma_f32_16x16x32_bf16 v[200:203], a[240:243], a[176:179], v[200:203]\n\t" \
    "ds_read_b64_tr_b16 a[220:221], v25 offset:24576\n\t" \
    "v_mfma_f32_16x16x32_bf16 v[200:203], a[244:247], a[180:183], v[200:203]\n\t" \
    "v_mfma_f32_16x16x32_bf16 v[204:207], a[248:251], a[176:179], v[204:207]\n\t" \
    "ds_read_b64_tr_b16 a[222:223], v25 offset:25600\n\t" \
    "v_mfma_f32_16x16x32_bf16 v[204:207], a[252:255], a[180:183], v[204:207]\n\t" \
    "v_mfma_f32_16x16x32_bf16 v[224:227], a[224:227], a[184:187], v[224:227]\n\t" \
    "v_add_u32 v23, 0x8000, v22\n\t" \
    "v_add_u32 v22, 0xffff0000, v22\n\t" \
    "v_mfma_f32_16x16x32_bf16 v[224:227], a[228:231], a[188:191], v[224:227]\n\t" \
    "v_min_u32_e64 v22, v22, v23\n\t" \
    "v_mfma_f32_16x16x32_bf16 v[228:231], a[232:235], a[184:187], v[228:231]\n\t" \
    "v_xor_b32_e64 v23, 32, v22\n\t" \
    "v_xor_b32_e64 v24, 64, v22\n\t" \
    "v_mfma_f32_16x16x32_bf16 v[228:231], a[236:239], a[188:191], v[228:231]\n\t" \
    "v_xor_b32 v25, 0x60, v22\n\t" \
    "v_mfma_f32_16x16x32_bf16 v[232:235], a[240:243], a[184:187], v[232:235]\n\t" \
    "v_mfma_f32_16x16x32_bf16 v[232:235], a[244:247], a[188:191], v[232:235]\n\t" \
    "v_mfma_f32_16x16x32_bf16 v[236:239], a[248:251], a[184:187], v[236:239]\n\t" \
    "v_mfma_f32_16x16x32_bf16 v[236:239], a[252:255], a[188:191], v[236:239]\n\t" \
    "s_sub_u32 %[rem], %[rem], 1\n\t" \
    "s_waitcnt vmcnt(1)\n\t" \
    "s_waitcnt lgkmcnt(0)\n\ts_barrier\n\t" \
    ""
#define ASM_ZERO \
    "v_accvgpr_write_b32 a0, 0\n\t" \
    "v_accvgpr_write_b32 a1, 0\n\t" \
    "v_accvgpr_write_b32 a2, 0\n\t" \
    "v_accvgpr_write_b32 a3, 0\n\t" \
    "v_accvgpr_write_b32 a4, 0\n\t" \
    "v_accvgpr_write_b32 a5, 0\n\t" \
    "v_accvgpr_write_b32 a6, 0\n\t" \
    "v_accvgpr_write_b32 a7, 0\n\t" \
    "v_accvgpr_write_b32 a8, 0\n\t" \
    "v_accvgpr_write_b32 a9, 0\n\t" \
    "v_accvgpr_write_b32 a10, 0\n\t" \
    "v_accvgpr_write_b32 a11, 0\n\t" \
    "v_accvgpr_write_b32 a12, 0\n\t" \
    "v_accvgpr_write_b32 a13, 0\n\t" \
    "v_accvgpr_write_b32 a14, 0\n\t" \
    "v_accvgpr_write_b32 a15, 0\n\t" \
    "v_accvgpr_write_b32 a16, 0\n\t" \
    "v_accvgpr_write_b32 a17, 0\n\t" \
    "v_accvgpr_write_b32 a18, 0\n\t" \
    "v_accvgpr_write_b32 a19, 0\n\t" \
    "v_accvgpr_write_b32 a20, 0\n\t" \
    "v_accvgpr_write_b32 a21, 0\n\t" \
    "v_accvgpr_write_b32 a22, 0\n\t" \
    "v_accvgpr_write_b32 a23, 0\n\t" \
    "v_accvgpr_write_b32 a24, 0\n\t" \
    "v_accvgpr_write_b32 a25, 0\n\t" \
    "v_accvgpr_write_b32 a26, 0\n\t" \
    "v_accvgpr_write_b32 a27, 0\n\t" \
    "v_accvgpr_write_b32 a28, 0\n\t" \
    "v_accvgpr_write_b32 a29, 0\n\t" \
    "v_accvgpr_write_b32 a30, 0\n\t" \
    "v_accvgpr_write_b32 a31, 0\n\t" \
    "v_accvgpr_write_b32 a32, 0\n\t" \
    "v_accvgpr_write_b32 a33, 0\n\t" \
    "v_accvgpr_write_b32 a34, 0\n\t" \
    "v_accvgpr_write_b32 a35, 0\n\t" \
    "v_accvgpr_write_b32 a36, 0\n\t" \
    "v_accvgpr_write_b32 a37, 0\n\t" \
    "v_accvgpr_write_b32 a38, 0\n\t" \
    "v_accvgpr_write_b32 a39, 0\n\t" \
    "v_accvgpr_write_b32 a40, 0\n\t" \
    "v_accvgpr_write_b32 a41, 0\n\t" \
    "v_accvgpr_write_b32 a42, 0\n\t" \
    "v_accvgpr_write_b32 a43, 0\n\t" \
    "v_accvgpr_write_b32 a44, 0\n\t" \
    "v_accvgpr_write_b32 a45, 0\n\t" \
    "v_accvgpr_write_b32 a46, 0\n\t" \
    "v_accvgpr_write_b32 a47, 0\n\t" \
    "v_accvgpr_write_b32 a48, 0\n\t" \
    "v_accvgpr_write_b32 a49, 0\n\t" \
    "v_accvgpr_write_b32 a50, 0\n\t" \
    "v_accvgpr_write_b32 a51, 0\n\t" \
    "v_accvgpr_write_b32 a52, 0\n\t" \
    "v_accvgpr_write_b32 a53, 0\n\t" \
    "v_accvgpr_write_b32 a54, 0\n\t" \
    "v_accvgpr_write_b32 a55, 0\n\t" \
    "v_accvgpr_write_b32 a56, 0\n\t" \
    "v_accvgpr_write_b32 a57, 0\n\t" \
    "v_accvgpr_write_b32 a58, 0\n\t" \
    "v_accvgpr_write_b32 a59, 0\n\t" \
    "v_accvgpr_write_b32 a60, 0\n\t" \
    "v_accvgpr_write_b32 a61, 0\n\t" \
    "v_accvgpr_write_b32 a62, 0\n\t" \
    "v_accvgpr_write_b32 a63, 0\n\t" \
    "v_mov_b32 v48, 0\n\t" \
    "v_mov_b32 v49, 0\n\t" \
    "v_mov_b32 v50, 0\n\t" \
    "v_mov_b32 v51, 0\n\t" \
    "v_mov_b32 v52, 0\n\t" \
    "v_mov_b32 v53, 0\n\t" \
    "v_mov_b32 v54, 0\n\t" \
    "v_mov_b32 v55, 0\n\t" \
    "v_mov_b32 v56, 0\n\t" \
    "v_mov_b32 v57, 0\n\t" \
    "v_mov_b32 v58, 0\n\t" \
    "v_mov_b32 v59, 0\n\t" \
    "v_mov_b32 v60, 0\n\t" \
    "v_mov_b32 v61, 0\n\t" \
    "v_mov_b32 v62, 0\n\t" \
    "v_mov_b32 v63, 0\n\t" \
    "v_mov_b32 v64, 0\n\t" \
    "v_mov_b32 v65, 0\n\t" \
    "v_mov_b32 v66, 0\n\t" \
    "v_mov_b32 v67, 0\n\t" \
    "v_mov_b32 v68, 0\n\t" \
    "v_mov_b32 v69, 0\n\t" \
    "v_mov_b32 v70, 0\n\t" \
    "v_mov_b32 v71, 0\n\t" \
    "v_mov_b32 v72, 0\n\t" \
    "v_mov_b32 v73, 0\n\t" \
    "v_mov_b32 v74, 0\n\t" \
    "v_mov_b32 v75, 0\n\t" \
    "v_mov_b32 v76, 0\n\t" \
    "v_mov_b32 v77, 0\n\t" \
    "v_mov_b32 v78, 0\n\t" \
    "v_mov_b32 v79, 0\n\t" \
    "v_mov_b32 v80, 0\n\t" \
    "v_mov_b32 v81, 0\n\t" \
    "v_mov_b32 v82, 0\n\t" \
    "v_mov_b32 v83, 0\n\t" \
    "v_mov_b32 v84, 0\n\t" \
    "v_mov_b32 v85, 0\n\t" \
    "v_mov_b32 v86, 0\n\t" \
    "v_mov_b32 v87, 0\n\t" \
    "v_mov_b32 v88, 0\n\t" \
    "v_mov_b32 v89, 0\n\t" \
    "v_mov_b32 v90, 0\n\t" \
    "v_mov_b32 v91, 0\n\t" \
    "v_mov_b32 v92, 0\n\t" \
    "v_mov_b32 v93, 0\n\t" \
    "v_mov_b32 v94, 0\n\t" \
    "v_mov_b32 v95, 0\n\t" \
    "v_mov_b32 v96, 0\n\t" \
    "v_mov_b32 v97, 0\n\t" \
    "v_mov_b32 v98, 0\n\t" \
    "v_mov_b32 v99, 0\n\t" \
    "v_mov_b32 v100, 0\n\t" \
    "v_mov_b32 v101, 0\n\t" \
    "v_mov_b32 v102, 0\n\t" \
    "v_mov_b32 v103, 0\n\t" \
    "v_mov_b32 v104, 0\n\t" \
    "v_mov_b32 v105, 0\n\t" \
    "v_mov_b32 v106, 0\n\t" \
    "v_mov_b32 v107, 0\n\t" \
    "v_mov_b32 v108, 0\n\t" \
    "v_mov_b32 v109, 0\n\t" \
    "v_mov_b32 v110, 0\n\t" \
    "v_mov_b32 v111, 0\n\t" \
    "v_mov_b32 v112, 0\n\t" \
    "v_mov_b32 v113, 0\n\t" \
    "v_mov_b32 v114, 0\n\t" \
    "v_mov_b32 v115, 0\n\t" \
    "v_mov_b32 v116, 0\n\t" \
    "v_mov_b32 v117, 0\n\t" \
    "v_mov_b32 v118, 0\n\t" \
    "v_mov_b32 v119, 0\n\t" \
    "v_mov_b32 v120, 0\n\t" \
    "v_mov_b32 v121, 0\n\t" \
    "v_mov_b32 v122, 0\n\t" \
    "v_mov_b32 v123, 0\n\t" \
    "v_mov_b32 v124, 0\n\t" \
    "v_mov_b32 v125, 0\n\t" \
    "v_mov_b32 v126, 0\n\t" \
    "v_mov_b32 v127, 0\n\t" \
    "v_mov_b32 v128, 0\n\t" \
    "v_mov_b32 v129, 0\n\t" \
    "v_mov_b32 v130, 0\n\t" \
    "v_mov_b32 v131, 0\n\t" \
    "v_mov_b32 v132, 0\n\t" \
    "v_mov_b32 v133, 0\n\t" \
    "v_mov_b32 v134, 0\n\t" \
    "v_mov_b32 v135, 0\n\t" \
    "v_mov_b32 v136, 0\n\t" \
    "v_mov_b32 v137, 0\n\t" \
    "v_mov_b32 v138, 0\n\t" \
    "v_mov_b32 v139, 0\n\t" \
    "v_mov_b32 v140, 0\n\t" \
    "v_mov_b32 v141, 0\n\t" \
    "v_mov_b32 v142, 0\n\t" \
    "v_mov_b32 v143, 0\n\t" \
    "v_mov_b32 v144, 0\n\t" \
    "v_mov_b32 v145, 0\n\t" \
    "v_mov_b32 v146, 0\n\t" \
    "v_mov_b32 v147, 0\n\t" \
    "v_mov_b32 v148, 0\n\t" \
    "v_mov_b32 v149, 0\n\t" \
    "v_mov_b32 v150, 0\n\t" \
    "v_mov_b32 v151, 0\n\t" \
    "v_mov_b32 v152, 0\n\t" \
    "v_mov_b32 v153, 0\n\t" \
    "v_mov_b32 v154, 0\n\t" \
    "v_mov_b32 v155, 0\n\t" \
    "v_mov_b32 v156, 0\n\t" \
    "v_mov_b32 v157, 0\n\t" \
    "v_mov_b32 v158, 0\n\t" \
    "v_mov_b32 v159, 0\n\t" \
    "v_mov_b32 v160, 0\n\t" \
    "v_mov_b32 v161, 0\n\t" \
    "v_mov_b32 v162, 0\n\t" \
    "v_mov_b32 v163, 0\n\t" \
    "v_mov_b32 v164, 0\n\t" \
    "v_mov_b32 v165, 0\n\t" \
    "v_mov_b32 v166, 0\n\t" \
    "v_mov_b32 v167, 0\n\t" \
    "v_mov_b32 v168, 0\n\t" \
    "v_mov_b32 v169, 0\n\t" \
    "v_mov_b32 v170, 0\n\t" \
    "v_mov_b32 v171, 0\n\t" \
    "v_mov_b32 v172, 0\n\t" \
    "v_mov_b32 v173, 0\n\t" \
    "v_mov_b32 v174, 0\n\t" \
    "v_mov_b32 v175, 0\n\t" \
    "v_mov_b32 v176, 0\n\t" \
    "v_mov_b32 v177, 0\n\t" \
    "v_mov_b32 v178, 0\n\t" \
    "v_mov_b32 v179, 0\n\t" \
    "v_mov_b32 v180, 0\n\t" \
    "v_mov_b32 v181, 0\n\t" \
    "v_mov_b32 v182, 0\n\t" \
    "v_mov_b32 v183, 0\n\t" \
    "v_mov_b32 v184, 0\n\t" \
    "v_mov_b32 v185, 0\n\t" \
    "v_mov_b32 v186, 0\n\t" \
    "v_mov_b32 v187, 0\n\t" \
    "v_mov_b32 v188, 0\n\t" \
    "v_mov_b32 v189, 0\n\t" \
    "v_mov_b32 v190, 0\n\t" \
    "v_mov_b32 v191, 0\n\t" \
    "v_mov_b32 v192, 0\n\t" \
    "v_mov_b32 v193, 0\n\t" \
    "v_mov_b32 v194, 0\n\t" \
    "v_mov_b32 v195, 0\n\t" \
    "v_mov_b32 v196, 0\n\t" \
    "v_mov_b32 v197, 0\n\t" \
    "v_mov_b32 v198, 0\n\t" \
    "v_mov_b32 v199, 0\n\t" \
    "v_mov_b32 v200, 0\n\t" \
    "v_mov_b32 v201, 0\n\t" \
    "v_mov_b32 v202, 0\n\t" \
    "v_mov_b32 v203, 0\n\t" \
    "v_mov_b32 v204, 0\n\t" \
    "v_mov_b32 v205, 0\n\t" \
    "v_mov_b32 v206, 0\n\t" \
    "v_mov_b32 v207, 0\n\t" \
    "v_mov_b32 v208, 0\n\t" \
    "v_mov_b32 v209, 0\n\t" \
    "v_mov_b32 v210, 0\n\t" \
    "v_mov_b32 v211, 0\n\t" \
    "v_mov_b32 v212, 0\n\t" \
    "v_mov_b32 v213, 0\n\t" \
    "v_mov_b32 v214, 0\n\t" \
    "v_mov_b32 v215, 0\n\t" \
    "v_mov_b32 v216, 0\n\t" \
    "v_mov_b32 v217, 0\n\t" \
    "v_mov_b32 v218, 0\n\t" \
    "v_mov_b32 v219, 0\n\t" \
    "v_mov_b32 v220, 0\n\t" \
    "v_mov_b32 v221, 0\n\t" \
    "v_mov_b32 v222, 0\n\t" \
    "v_mov_b32 v223, 0\n\t" \
    "v_mov_b32 v224, 0\n\t" \
    "v_mov_b32 v225, 0\n\t" \
    "v_mov_b32 v226, 0\n\t" \
    "v_mov_b32 v227, 0\n\t" \
    "v_mov_b32 v228, 0\n\t" \
    "v_mov_b32 v229, 0\n\t" \
    "v_mov_b32 v230, 0\n\t" \
    "v_mov_b32 v231, 0\n\t" \
    "v_mov_b32 v232, 0\n\t" \
    "v_mov_b32 v233, 0\n\t" \
    "v_mov_b32 v234, 0\n\t" \
    "v_mov_b32 v235, 0\n\t" \
    "v_mov_b32 v236, 0\n\t" \
    "v_mov_b32 v237, 0\n\t" \
    "v_mov_b32 v238, 0\n\t" \
    "v_mov_b32 v239, 0\n\t" \
    ""
#define ASM_FIRST \
    "ds_read_b64_tr_b16 a[192:193], v22 offset:0\n\t" \
    "ds_read_b64_tr_b16 a[194:195], v22 offset:1024\n\t" \
    "ds_read_b64_tr_b16 a[196:197], v22 offset:8192\n\t" \
    "ds_read_b64_tr_b16 a[198:199], v22 offset:9216\n\t" \
    "ds_read_b64_tr_b16 a[200:201], v23 offset:0\n\t" \
    "ds_read_b64_tr_b16 a[202:203], v23 offset:1024\n\t" \
    "ds_read_b64_tr_b16 a[204:205], v23 offset:8192\n\t" \
    "ds_read_b64_tr_b16 a[206:207], v23 offset:9216\n\t" \
    "ds_read_b64_tr_b16 a[208:209], v24 offset:0\n\t" \
    "ds_read_b64_tr_b16 a[210:211], v24 offset:1024\n\t" \
    "ds_read_b64_tr_b16 a[212:213], v24 offset:8192\n\t" \
    "ds_read_b64_tr_b16 a[214:215], v24 offset:9216\n\t" \
    "ds_read_b64_tr_b16 a[216:217], v25 offset:0\n\t" \
    "ds_read_b64_tr_b16 a[218:219], v25 offset:1024\n\t" \
    "ds_read_b64_tr_b16 a[220:221], v25 offset:8192\n\t" \
    "ds_read_b64_tr_b16 a[222:223], v25 offset:9216\n\t" \
    "ds_read_b128 a[128:131], v20 offset:0\n\t" \
    "ds_read_b128 a[132:135], v21 offset:0\n\t" \
    "ds_read_b128 a[136:139], v20 offset:2048\n\t" \
    "ds_read_b128 a[140:143], v21 offset:2048\n\t" \
    "ds_read_b128 a[144:147], v20 offset:4096\n\t" \
    "ds_read_b128 a[148:151], v21 offset:4096\n\t" \
    "ds_read_b128 a[152:155], v20 offset:6144\n\t" \
    "ds_read_b128 a[156:159], v21 offset:6144\n\t" \
    "ds_read_b128 a[160:163], v20 offset:8192\n\t" \
    "ds_read_b128 a[164:167], v21 offset:8192\n\t" \
    "ds_read_b128 a[168:171], v20 offset:10240\n\t" \
    "ds_read_b128 a[172:175], v21 offset:10240\n\t" \
    ""
#define ASM_FINAL_HALF \
    "s_nop 7\n\t" \
    "v_accvgpr_read_b32 v32, a16\n\t" \
    "v_accvgpr_read_b32 v33, a17\n\t" \
    "v_accvgpr_read_b32 v34, a18\n\t" \
    "v_accvgpr_read_b32 v35, a19\n\t" \
    "v_cvt_pk_bf16_f32 v248, v32, v33\n\t" \
    "v_cvt_pk_bf16_f32 v249, v34, v35\n\t" \
    "v_accvgpr_read_b32 v32, a20\n\t" \
    "v_accvgpr_read_b32 v33, a21\n\t" \
    "v_accvgpr_read_b32 v34, a22\n\t" \
    "v_accvgpr_read_b32 v35, a23\n\t" \
    "v_cvt_pk_bf16_f32 v250, v32, v33\n\t" \
    "v_cvt_pk_bf16_f32 v251, v34, v35\n\t" \
    "s_xor_b32 %[t0], %[sd], 0x4000\n\tbuffer_store_dwordx4 v[248:251], v30, %[rss], %[t0] offen " DAUXS "\n\t" \
    "v_accvgpr_read_b32 v32, a24\n\t" \
    "v_accvgpr_read_b32 v33, a25\n\t" \
    "v_accvgpr_read_b32 v34, a26\n\t" \
    "v_accvgpr_read_b32 v35, a27\n\t" \
    "v_cvt_pk_bf16_f32 v252, v32, v33\n\t" \
    "v_cvt_pk_bf16_f32 v253, v34, v35\n\t" \
    "v_accvgpr_read_b32 v32, a28\n\t" \
    "v_accvgpr_read_b32 v33, a29\n\t" \
    "v_accvgpr_read_b32 v34, a30\n\t" \
    "v_accvgpr_read_b32 v35, a31\n\t" \
    "v_cvt_pk_bf16_f32 v254, v32, v33\n\t" \
    "v_cvt_pk_bf16_f32 v255, v34, v35\n\t" \
    "s_xor_b32 %[t0], %[sd], 0x4400\n\tbuffer_store_dwordx4 v[252:255], v30, %[rss], %[t0] offen " DAUXS "\n\t" \
    "v_accvgpr_read_b32 v32, a48\n\t" \
    "v_accvgpr_read_b32 v33, a49\n\t" \
    "v_accvgpr_read_b32 v34, a50\n\t" \
    "v_accvgpr_read_b32 v35, a51\n\t" \
    "v_cvt_pk_bf16_f32 v248, v32, v33\n\t" \
    "v_cvt_pk_bf16_f32 v249, v34, v35\n\t" \
    "v_accvgpr_read_b32 v32, a52\n\t" \
    "v_accvgpr_read_b32 v33, a53\n\t" \
    "v_accvgpr_read_b32 v34, a54\n\t" \
    "v_accvgpr_read_b32 v35, a55\n\t" \
    "v_cvt_pk_bf16_f32 v250, v32, v33\n\t" \
    "v_cvt_pk_bf16_f32 v251, v34, v35\n\t" \
    "s_xor_b32 %[t0], %[sd], 0x4800\n\tbuffer_store_dwordx4 v[248:251], v30, %[rss], %[t0] offen " DAUXS "\n\t" \
    "v_accvgpr_read_b32 v32, a56\n\t" \
    "v_accvgpr_read_b32 v33, a57\n\t" \
    "v_accvgpr_read_b32 v34, a58\n\t" \
    "v_accvgpr_read_b32 v35, a59\n\t" \
    "v_cvt_pk_bf16_f32 v252, v32, v33\n\t" \
    "v_cvt_pk_bf16_f32 v253, v34, v35\n\t" \
    "v_accvgpr_read_b32 v32, a60\n\t" \
    "v_accvgpr_read_b32 v33, a61\n\t" \
    "v_accvgpr_read_b32 v34, a62\n\t" \
    "v_accvgpr_read_b32 v35, a63\n\t" \
    "v_cvt_pk_bf16_f32 v254, v32, v33\n\t" \
    "v_cvt_pk_bf16_f32 v255, v34, v35\n\t" \
    "s_xor_b32 %[t0], %[sd], 0x4c00\n\tbuffer_store_dwordx4 v[252:255], v30, %[rss], %[t0] offen " DAUXS "\n\t" \
    "v_cvt_pk_bf16_f32 v248, v64, v65\n\t" \
    "v_cvt_pk_bf16_f32 v249, v66, v67\n\t" \
    "v_cvt_pk_bf16_f32 v250, v68, v69\n\t" \
    "v_cvt_pk_bf16_f32 v251, v70, v71\n\t" \
    "s_xor_b32 %[t0], %[sd], 0x5000\n\tbuffer_store_dwordx4 v[248:251], v30, %[rss], %[t0] offen " DAUXS "\n\t" \
    "v_cvt_pk_bf16_f32 v252, v72, v73\n\t" \
    "v_cvt_pk_bf16_f32 v253, v74, v75\n\t" \
    "v_cvt_pk_bf16_f32 v254, v76, v77\n\t" \
    "v_cvt_pk_bf16_f32 v255, v78, v79\n\t" \
    "s_xor_b32 %[t0], %[sd], 0x5400\n\tbuffer_store_dwordx4 v[252:255], v30, %[rss], %[t0] offen " DAUXS "\n\t" \
    "v_cvt_pk_bf16_f32 v248, v96, v97\n\t" \
    "v_cvt_pk_bf16_f32 v249, v98, v99\n\t" \
    "v_cvt_pk_bf16_f32 v250, v100, v101\n\t" \
    "v_cvt_pk_bf16_f32 v251, v102, v103\n\t" \
    "s_xor_b32 %[t0], %[sd], 0x5800\n\tbuffer_store_dwordx4 v[248:251], v30, %[rss], %[t0] offen " DAUXS "\n\t" \
    "v_cvt_pk_bf16_f32 v252, v104, v105\n\t" \
    "v_cvt_pk_bf16_f32 v253, v106, v107\n\t" \
    "v_cvt_pk_bf16_f32 v254, v108, v109\n\t" \
    "v_cvt_pk_bf16_f32 v255, v110, v111\n\t" \
    "s_xor_b32 %[t0], %[sd], 0x5c00\n\tbuffer_store_dwordx4 v[252:255], v30, %[rss], %[t0] offen " DAUXS "\n\t" \
    "v_cvt_pk_bf16_f32 v248, v128, v129\n\t" \
    "v_cvt_pk_bf16_f32 v249, v130, v131\n\t" \
    "v_cvt_pk_bf16_f32 v250, v132, v133\n\t" \
    "v_cvt_pk_bf16_f32 v251, v134, v135\n\t" \
    "s_xor_b32 %[t0], %[sd], 0x6000\n\tbuffer_store_dwordx4 v[248:251], v30, %[rss], %[t0] offen " DAUXS "\n\t" \
    "v_cvt_pk_bf16_f32 v252, v136, v137\n\t" \
    "v_cvt_pk_bf16_f32 v253, v138, v139\n\t" \
    "v_cvt_pk_bf16_f32 v254, v140, v141\n\t" \
    "v_cvt_pk_bf16_f32 v255, v142, v143\n\t" \
    "s_xor_b32 %[t0], %[sd], 0x6400\n\tbuffer_store_dwordx4 v[252:255], v30, %[rss], %[t0] offen " DAUXS "\n\t" \
    "v_cvt_pk_bf16_f32 v248, v160, v161\n\t" \
    "v_cvt_pk_bf16_f32 v249, v162, v163\n\t" \
    "v_cvt_pk_bf16_f32 v250, v164, v165\n\t" \
    "v_cvt_pk_bf16_f32 v251, v166, v167\n\t" \
    "s_xor_b32 %[t0], %[sd], 0x6800\n\tbuffer_store_dwordx4 v[248:251], v30, %[rss], %[t0] offen " DAUXS "\n\t" \
    "v_cvt_pk_bf16_f32 v252, v168, v169\n\t" \
    "v_cvt_pk_bf16_f32 v253, v170, v171\n\t" \
    "v_cvt_pk_bf16_f32 v254, v172, v173\n\t" \
    "v_cvt_pk_bf16_f32 v255, v174, v175\n\t" \
    "s_xor_b32 %[t0], %[sd], 0x6c00\n\tbuffer_store_dwordx4 v[252:255], v30, %[rss], %[t0] offen " DAUXS "\n\t" \
    "v_cvt_pk_bf16_f32 v248, v192, v193\n\t" \
    "v_cvt_pk_bf16_f32 v249, v194, v195\n\t" \
    "v_cvt_pk_bf16_f32 v250, v196, v197\n\t" \
    "v_cvt_pk_bf16_f32 v251, v198, v199\n\t" \
    "s_xor_b32 %[t0], %[sd], 0x7000\n\tbuffer_store_dwordx4 v[248:251], v30, %[rss], %[t0] offen " DAUXS "\n\t" \
    "v_cvt_pk_bf16_f32 v252, v200, v201\n\t" \
    "v_cvt_pk_bf16_f32 v253, v202, v203\n\t" \
    "v_cvt_pk_bf16_f32 v254, v204, v205\n\t" \
    "v_cvt_pk_bf16_f32 v255, v206, v207\n\t" \
    "s_xor_b32 %[t0], %[sd], 0x7400\n\tbuffer_store_dwordx4 v[252:255], v30, %[rss], %[t0] offen " DAUXS "\n\t" \
    "v_cvt_pk_bf16_f32 v248, v224, v225\n\t" \
    "v_cvt_pk_bf16_f32 v249, v226, v227\n\t" \
    "v_cvt_pk_bf16_f32 v250, v228, v229\n\t" \
    "v_cvt_pk_bf16_f32 v251, v230, v231\n\t" \
    "s_xor_b32 %[t0], %[sd], 0x7800\n\tbuffer_store_dwordx4 v[248:251], v30, %[rss], %[t0] offen " DAUXS "\n\t" \
    "v_cvt_pk_bf16_f32 v252, v232, v233\n\t" \
    "v_cvt_pk_bf16_f32 v253, v234, v235\n\t" \
    "v_cvt_pk_bf16_f32 v254, v236, v237\n\t" \
    "v_cvt_pk_bf16_f32 v255, v238, v239\n\t" \
    "s_xor_b32 %[t0], %[sd], 0x7c00\n\tbuffer_store_dwordx4 v[252:255], v30, %[rss], %[t0] offen " DAUXS "\n\t" \
    ""
#define ASM_DONE2_0 \
    "v_mfma_f32_16x16x32_bf16 v[36:39], v[26:29], a[64:67], 0\n\t" \
    "v_mfma_f32_16x16x32_bf16 v[36:39], v[26:29], a[72:75], v[36:39]\n\t" \
    "v_mfma_f32_16x16x32_bf16 v[40:43], v[26:29], a[68:71], 0\n\t" \
    "v_mfma_f32_16x16x32_bf16 v[40:43], v[26:29], a[76:79], v[40:43]\n\t" \
    "v_mfma_f32_16x16x32_bf16 v[240:243], v[26:29], a[80:83], 0\n\t" \
    "v_mfma_f32_16x16x32_bf16 v[240:243], v[26:29], a[88:91], v[240:243]\n\t" \
    "v_mfma_f32_16x16x32_bf16 v[244:247], v[26:29], a[84:87], 0\n\t" \
    "v_mfma_f32_16x16x32_bf16 v[244:247], v[26:29], a[92:95], v[244:247]\n\t" \
    "s_nop 7\n\t" \
    "v_med3_f32 v32, v36, -%[c60], %[c60]\n\t" \
    "v_med3_f32 v33, v37, -%[c60], %[c60]\n\t" \
    "v_med3_f32 v34, v38, -%[c60], %[c60]\n\t" \
    "v_med3_f32 v35, v39, -%[c60], %[c60]\n\t" \
    "v_mul_f32 v32, 0xbfb8aa3b, v32\n\t" \
    "v_mul_f32 v33, 0xbfb8aa3b, v33\n\t" \
    "v_mul_f32 v34, 0xbfb8aa3b, v34\n\t" \
    "v_mul_f32 v35, 0xbfb8aa3b, v35\n\t" \
    "v_exp_f32_e64 v32, v32\n\t" \
    "v_exp_f32_e64 v33, v33\n\t" \
    "v_exp_f32_e64 v34, v34\n\t" \
    "v_exp_f32_e64 v35, v35\n\t" \
    "v_add_f32_e64 v32, 1.0, v32\n\t" \
    "v_add_f32_e64 v33, 1.0, v33\n\t" \
    "v_add_f32_e64 v34, 1.0, v34\n\t" \
    "v_add_f32_e64 v35, 1.0, v35\n\t" \
    "v_rcp_f32_e64 v32, v32\n\t" \
    "v_rcp_f32_e64 v33, v33\n\t" \
    "v_rcp_f32_e64 v34, v34\n\t" \
    "v_rcp_f32_e64 v35, v35\n\t" \
    "v_mul_f32_e64 v36, v36, v32\n\t" \
    "v_mul_f32_e64 v37, v37, v33\n\t" \
    "v_mul_f32_e64 v38, v38, v34\n\t" \
    "v_mul_f32_e64 v39, v39, v35\n\t" \
    "v_med3_f32 v32, v40, -%[c60], %[c60]\n\t" \
    "v_med3_f32 v33, v41, -%[c60], %[c60]\n\t" \
    "v_med3_f32 v34, v42, -%[c60], %[c60]\n\t" \
    "v_med3_f32 v35, v43, -%[c60], %[c60]\n\t" \
    "v_mul_f32 v32, 0xbfb8aa3b, v32\n\t" \
    "v_mul_f32 v33, 0xbfb8aa3b, v33\n\t" \
    "v_mul_f32 v34, 0xbfb8aa3b, v34\n\t" \
    "v_mul_f32 v35, 0xbfb8aa3b, v35\n\t" \
    "v_exp_f32_e64 v32, v32\n\t" \
    "v_exp_f32_e64 v33, v33\n\t" \
    "v_exp_f32_e64 v34, v34\n\t" \
    "v_exp_f32_e64 v35, v35\n\t" \
    "v_add_f32_e64 v32, 1.0, v32\n\t" \
    "v_add_f32_e64 v33, 1.0, v33\n\t" \
    "v_add_f32_e64 v34, 1.0, v34\n\t" \
    "v_add_f32_e64 v35, 1.0, v35\n\t" \
    "v_rcp_f32_e64 v32, v32\n\t" \
    "v_rcp_f32_e64 v33, v33\n\t" \
    "v_rcp_f32_e64 v34, v34\n\t" \
    "v_rcp_f32_e64 v35, v35\n\t" \
    "v_mul_f32_e64 v40, v40, v32\n\t" \
    "v_mul_f32_e64 v41, v41, v33\n\t" \
    "v_mul_f32_e64 v42, v42, v34\n\t" \
    "v_mul_f32_e64 v43, v43, v35\n\t" \
    "v_cvt_pk_bf16_f32 v36, v36, v37\n\t" \
    "v_cvt_pk_bf16_f32 v37, v38, v39\n\t" \
    "v_cvt_pk_bf16_f32 v38, v40, v41\n\t" \
    "v_cvt_pk_bf16_f32 v39, v42, v43\n\t" \
    "s_waitcnt vmcnt(0)\n\t" \
    "buffer_store_dwordx4 v[36:39], v31, %[rso], %[oo] offen sc0 sc1\n\t" \
    "v_med3_f32 v32, v240, -%[c60], %[c60]\n\t" \
    "v_med3_f32 v33, v241, -%[c60], %[c60]\n\t" \
    "v_med3_f32 v34, v242, -%[c60], %[c60]\n\t" \
    "v_med3_f32 v35, v243, -%[c60], %[c60]\n\t" \
    "v_mul_f32 v32, 0xbfb8aa3b, v32\n\t" \
    "v_mul_f32 v33, 0xbfb8aa3b, v33\n\t" \
    "v_mul_f32 v34, 0xbfb8aa3b, v34\n\t" \
    "v_mul_f32 v35, 0xbfb8aa3b, v35\n\t" \
    "v_exp_f32_e64 v32, v32\n\t" \
    "v_exp_f32_e64 v33, v33\n\t" \
    "v_exp_f32_e64 v34, v34\n\t" \
    "v_exp_f32_e64 v35, v35\n\t" \
    "v_add_f32_e64 v32, 1.0, v32\n\t" \
    "v_add_f32_e64 v33, 1.0, v33\n\t" \
    "v_add_f32_e64 v34, 1.0, v34\n\t" \
    "v_add_f32_e64 v35, 1.0, v35\n\t" \
    "v_rcp_f32_e64 v32, v32\n\t" \
    "v_rcp_f32_e64 v33, v33\n\t" \
    "v_rcp_f32_e64 v34, v34\n\t" \
    "v_rcp_f32_e64 v35, v35\n\t" \
    "v_mul_f32_e64 v240, v240, v32\n\t" \
    "v_mul_f32_e64 v241, v241, v33\n\t" \
    "v_mul_f32_e64 v242, v242, v34\n\t" \
    "v_mul_f32_e64 v243, v243, v35\n\t" \
    "v_med3_f32 v32, v244, -%[c60], %[c60]\n\t" \
    "v_med3_f32 v33, v245, -%[c60], %[c60]\n\t" \
    "v_med3_f32 v34, v246, -%[c60], %[c60]\n\t" \
    "v_med3_f32 v35, v247, -%[c60], %[c60]\n\t" \
    "v_mul_f32 v32, 0xbfb8aa3b, v32\n\t" \
    "v_mul_f32 v33, 0xbfb8aa3b, v33\n\t" \
    "v_mul_f32 v34, 0xbfb8aa3b, v34\n\t" \
    "v_mul_f32 v35, 0xbfb8aa3b, v35\n\t" \
    "v_exp_f32_e64 v32, v32\n\t" \
    "v_exp_f32_e64 v33, v33\n\t" \
    "v_exp_f32_e64 v34, v34\n\t" \
    "v_exp_f32_e64 v35, v35\n\t" \
    "v_add_f32_e64 v32, 1.0, v32\n\t" \
    "v_add_f32_e64 v33, 1.0, v33\n\t" \
    "v_add_f32_e64 v34, 1.0, v34\n\t" \
    "v_add_f32_e64 v35, 1.0, v35\n\t" \
    "v_rcp_f32_e64 v32, v32\n\t" \
    "v_rcp_f32_e64 v33, v33\n\t" \
    "v_rcp_f32_e64 v34, v34\n\t" \
    "v_rcp_f32_e64 v35, v35\n\t" \
    "v_mul_f32_e64 v244, v244, v32\n\t" \
    "v_mul_f32_e64 v245, v245, v33\n\t" \
    "v_mul_f32_e64 v246, v246, v34\n\t" \
    "v_mul_f32_e64 v247, v247, v35\n\t" \
    "v_cvt_pk_bf16_f32 v240, v240, v241\n\t" \
    "v_cvt_pk_bf16_f32 v241, v242, v243\n\t" \
    "v_cvt_pk_bf16_f32 v242, v244, v245\n\t" \
    "v_cvt_pk_bf16_f32 v243, v246, v247\n\t" \
    "s_waitcnt vmcnt(0)\n\t" \
    "s_add_u32 %[t0], %[oo], 0x4000\n\tbuffer_store_dwordx4 v[240:243], v31, %[rso], %[t0] offen sc0 sc1\n\t" \
    "s_nop 1\n\t" \
    ""
#define ASM_DONE2_1 \
    "v_mfma_f32_16x16x32_bf16 v[36:39], v[26:29], a[96:99], 0\n\t" \
    "v_mfma_f32_16x16x32_bf16 v[36:39], v[26:29], a[104:107], v[36:39]\n\t" \
    "v_mfma_f32_16x16x32_bf16 v[40:43], v[26:29], a[100:103], 0\n\t" \
    "v_mfma_f32_16x16x32_bf16 v[40:43], v[26:29], a[108:111], v[40:43]\n\t" \
    "v_mfma_f32_16x16x32_bf16 v[240:243], v[26:29], a[112:115], 0\n\t" \
    "v_mfma_f32_16x16x32_bf16 v[240:243], v[26:29], a[120:123], v[240:243]\n\t" \
    "v_mfma_f32_16x16x32_bf16 v[244:247], v[26:29], a[116:119], 0\n\t" \
    "v_mfma_f32_16x16x32_bf16 v[244:247], v[26:29], a[124:127], v[244:247]\n\t" \
    "s_nop 7\n\t" \
    "v_med3_f32 v32, v36, -%[c60], %[c60]\n\t" \
    "v_med3_f32 v33, v37, -%[c60], %[c60]\n\t" \
    "v_med3_f32 v34, v38, -%[c60], %[c60]\n\t" \
    "v_med3_f32 v35, v39, -%[c60], %[c60]\n\t" \
    "v_mul_f32 v32, 0xbfb8aa3b, v32\n\t" \
    "v_mul_f32 v33, 0xbfb8aa3b, v33\n\t" \
    "v_mul_f32 v34, 0xbfb8aa3b, v34\n\t" \
    "v_mul_f32 v35, 0xbfb8aa3b, v35\n\t" \
    "v_exp_f32_e64 v32, v32\n\t" \
    "v_exp_f32_e64 v33, v33\n\t" \
    "v_exp_f32_e64 v34, v34\n\t" \
    "v_exp_f32_e64 v35, v35\n\t" \
    "v_add_f32_e64 v32, 1.0, v32\n\t" \
    "v_add_f32_e64 v33, 1.0, v33\n\t" \
    "v_add_f32_e64 v34, 1.0, v34\n\t" \
    "v_add_f32_e64 v35, 1.0, v35\n\t" \
    "v_rcp_f32_e64 v32, v32\n\t" \
    "v_rcp_f32_e64 v33, v33\n\t" \
    "v_rcp_f32_e64 v34, v34\n\t" \
    "v_rcp_f32_e64 v35, v35\n\t" \
    "v_mul_f32_e64 v36, v36, v32\n\t" \
    "v_mul_f32_e64 v37, v37, v33\n\t" \
    "v_mul_f32_e64 v38, v38, v34\n\t" \
    "v_mul_f32_e64 v39, v39, v35\n\t" \
    "v_med3_f32 v32, v40, -%[c60], %[c60]\n\t" \
    "v_med3_f32 v33, v41, -%[c60], %[c60]\n\t" \
    "v_med3_f32 v34, v42, -%[c60], %[c60]\n\t" \
    "v_med3_f32 v35, v43, -%[c60], %[c60]\n\t" \
    "v_mul_f32 v32, 0xbfb8aa3b, v32\n\t" \
    "v_mul_f32 v33, 0xbfb8aa3b, v33\n\t" \
    "v_mul_f32 v34, 0xbfb8aa3b, v34\n\t" \
    "v_mul_f32 v35, 0xbfb8aa3b, v35\n\t" \
    "v_exp_f32_e64 v32, v32\n\t" \
    "v_exp_f32_e64 v33, v33\n\t" \
    "v_exp_f32_e64 v34, v34\n\t" \
    "v_exp_f32_e64 v35, v35\n\t" \
    "v_add_f32_e64 v32, 1.0, v32\n\t" \
    "v_add_f32_e64 v33, 1.0, v33\n\t" \
    "v_add_f32_e64 v34, 1.0, v34\n\t" \
    "v_add_f32_e64 v35, 1.0, v35\n\t" \
    "v_rcp_f32_e64 v32, v32\n\t" \
    "v_rcp_f32_e64 v33, v33\n\t" \
    "v_rcp_f32_e64 v34, v34\n\t" \
    "v_rcp_f32_e64 v35, v35\n\t" \
    "v_mul_f32_e64 v40, v40, v32\n\t" \
    "v_mul_f32_e64 v41, v41, v33\n\t" \
    "v_mul_f32_e64 v42, v42, v34\n\t" \
    "v_mul_f32_e64 v43, v43, v35\n\t" \
    "v_cvt_pk_bf16_f32 v36, v36, v37\n\t" \
    "v_cvt_pk_bf16_f32 v37, v38, v39\n\t" \
    "v_cvt_pk_bf16_f32 v38, v40, v41\n\t" \
    "v_cvt_pk_bf16_f32 v39, v42, v43\n\t" \
    "s_waitcnt vmcnt(0)\n\t" \
    "buffer_store_dwordx4 v[36:39], v31, %[rso], %[oo] offen sc0 sc1\n\t" \
    "v_med3_f32 v32, v240, -%[c60], %[c60]\n\t" \
    "v_med3_f32 v33, v241, -%[c60], %[c60]\n\t" \
    "v_med3_f32 v34, v242, -%[c60], %[c60]\n\t" \
    "v_med3_f32 v35, v243, -%[c60], %[c60]\n\t" \
    "v_mul_f32 v32, 0xbfb8aa3b, v32\n\t" \
    "v_mul_f32 v33, 0xbfb8aa3b, v33\n\t" \
    "v_mul_f32 v34, 0xbfb8aa3b, v34\n\t" \
    "v_mul_f32 v35, 0xbfb8aa3b, v35\n\t" \
    "v_exp_f32_e64 v32, v32\n\t" \
    "v_exp_f32_e64 v33, v33\n\t" \
    "v_exp_f32_e64 v34, v34\n\t" \
    "v_exp_f32_e64 v35, v35\n\t" \
    "v_add_f32_e64 v32, 1.0, v32\n\t" \
    "v_add_f32_e64 v33, 1.0, v33\n\t" \
    "v_add_f32_e64 v34, 1.0, v34\n\t" \
    "v_add_f32_e64 v35, 1.0, v35\n\t" \
    "v_rcp_f32_e64 v32, v32\n\t" \
    "v_rcp_f32_e64 v33, v33\n\t" \
    "v_rcp_f32_e64 v34, v34\n\t" \
    "v_rcp_f32_e64 v35, v35\n\t" \
    "v_mul_f32_e64 v240, v240, v32\n\t" \
    "v_mul_f32_e64 v241, v241, v33\n\t" \
    "v_mul_f32_e64 v242, v242, v34\n\t" \
    "v_mul_f32_e64 v243, v243, v35\n\t" \
    "v_med3_f32 v32, v244, -%[c60], %[c60]\n\t" \
    "v_med3_f32 v33, v245, -%[c60], %[c60]\n\t" \
    "v_med3_f32 v34, v246, -%[c60], %[c60]\n\t" \
    "v_med3_f32 v35, v247, -%[c60], %[c60]\n\t" \
    "v_mul_f32 v32, 0xbfb8aa3b, v32\n\t" \
    "v_mul_f32 v33, 0xbfb8aa3b, v33\n\t" \
    "v_mul_f32 v34, 0xbfb8aa3b, v34\n\t" \
    "v_mul_f32 v35, 0xbfb8aa3b, v35\n\t" \
    "v_exp_f32_e64 v32, v32\n\t" \
    "v_exp_f32_e64 v33, v33\n\t" \
    "v_exp_f32_e64 v34, v34\n\t" \
    "v_exp_f32_e64 v35, v35\n\t" \
    "v_add_f32_e64 v32, 1.0, v32\n\t" \
    "v_add_f32_e64 v33, 1.0, v33\n\t" \
    "v_add_f32_e64 v34, 1.0, v34\n\t" \
    "v_add_f32_e64 v35, 1.0, v35\n\t" \
    "v_rcp_f32_e64 v32, v32\n\t" \
    "v_rcp_f32_e64 v33, v33\n\t" \
    "v_rcp_f32_e64 v34, v34\n\t" \
    "v_rcp_f32_e64 v35, v35\n\t" \
    "v_mul_f32_e64 v244, v244, v32\n\t" \
    "v_mul_f32_e64 v245, v245, v33\n\t" \
    "v_mul_f32_e64 v246, v246, v34\n\t" \
    "v_mul_f32_e64 v247, v247, v35\n\t" \
    "v_cvt_pk_bf16_f32 v240, v240, v241\n\t" \
    "v_cvt_pk_bf16_f32 v241, v242, v243\n\t" \
    "v_cvt_pk_bf16_f32 v242, v244, v245\n\t" \
    "v_cvt_pk_bf16_f32 v243, v246, v247\n\t" \
    "s_waitcnt vmcnt(0)\n\t" \
    "s_add_u32 %[t0], %[oo], 0x4000\n\tbuffer_store_dwordx4 v[240:243], v31, %[rso], %[t0] offen sc0 sc1\n\t" \
    "s_nop 1\n\t" \
    ""
#define ASM_DONE2_2 \
    "v_mfma_f32_16x16x32_bf16 v[36:39], v[26:29], v[48:51], 0\n\t" \
    "v_mfma_f32_16x16x32_bf16 v[36:39], v[26:29], v[56:59], v[36:39]\n\t" \
    "v_mfma_f32_16x16x32_bf16 v[40:43], v[26:29], v[52:55], 0\n\t" \
    "v_mfma_f32_16x16x32_bf16 v[40:43], v[26:29], v[60:63], v[40:43]\n\t" \
    "v_mfma_f32_16x16x32_bf16 v[240:243], v[26:29], v[64:67], 0\n\t" \
    "v_mfma_f32_16x16x32_bf16 v[240:243], v[26:29], v[72:75], v[240:243]\n\t" \
    "v_mfma_f32_16x16x32_bf16 v[244:247], v[26:29], v[68:71], 0\n\t" \
    "v_mfma_f32_16x16x32_bf16 v[244:247], v[26:29], v[76:79], v[244:247]\n\t" \
    "s_nop 7\n\t" \
    "v_med3_f32 v32, v36, -%[c60], %[c60]\n\t" \
    "v_med3_f32 v33, v37, -%[c60], %[c60]\n\t" \
    "v_med3_f32 v34, v38, -%[c60], %[c60]\n\t" \
    "v_med3_f32 v35, v39, -%[c60], %[c60]\n\t" \
    "v_mul_f32 v32, 0xbfb8aa3b, v32\n\t" \
    "v_mul_f32 v33, 0xbfb8aa3b, v33\n\t" \
    "v_mul_f32 v34, 0xbfb8aa3b, v34\n\t" \
    "v_mul_f32 v35, 0xbfb8aa3b, v35\n\t" \
    "v_exp_f32_e64 v32, v32\n\t" \
    "v_exp_f32_e64 v33, v33\n\t" \
    "v_exp_f32_e64 v34, v34\n\t" \
    "v_exp_f32_e64 v35, v35\n\t" \
    "v_add_f32_e64 v32, 1.0, v32\n\t" \
    "v_add_f32_e64 v33, 1.0, v33\n\t" \
    "v_add_f32_e64 v34, 1.0, v34\n\t" \
    "v_add_f32_e64 v35, 1.0, v35\n\t" \
    "v_rcp_f32_e64 v32, v32\n\t" \
    "v_rcp_f32_e64 v33, v33\n\t" \
    "v_rcp_f32_e64 v34, v34\n\t" \
    "v_rcp_f32_e64 v35, v35\n\t" \
    "v_mul_f32_e64 v36, v36, v32\n\t" \
    "v_mul_f32_e64 v37, v37, v33\n\t" \
    "v_mul_f32_e64 v38, v38, v34\n\t" \
    "v_mul_f32_e64 v39, v39, v35\n\t" \
    "v_med3_f32 v32, v40, -%[c60], %[c60]\n\t" \
    "v_med3_f32 v33, v41, -%[c60], %[c60]\n\t" \
    "v_med3_f32 v34, v42, -%[c60], %[c60]\n\t" \
    "v_med3_f32 v35, v43, -%[c60], %[c60]\n\t" \
    "v_mul_f32 v32, 0xbfb8aa3b, v32\n\t" \
    "v_mul_f32 v33, 0xbfb8aa3b, v33\n\t" \
    "v_mul_f32 v34, 0xbfb8aa3b, v34\n\t" \
    "v_mul_f32 v35, 0xbfb8aa3b, v35\n\t" \
    "v_exp_f32_e64 v32, v32\n\t" \
    "v_exp_f32_e64 v33, v33\n\t" \
    "v_exp_f32_e64 v34, v34\n\t" \
    "v_exp_f32_e64 v35, v35\n\t" \
    "v_add_f32_e64 v32, 1.0, v32\n\t" \
    "v_add_f32_e64 v33, 1.0, v33\n\t" \
    "v_add_f32_e64 v34, 1.0, v34\n\t" \
    "v_add_f32_e64 v35, 1.0, v35\n\t" \
    "v_rcp_f32_e64 v32, v32\n\t" \
    "v_rcp_f32_e64 v33, v33\n\t" \
    "v_rcp_f32_e64 v34, v34\n\t" \
    "v_rcp_f32_e64 v35, v35\n\t" \
    "v_mul_f32_e64 v40, v40, v32\n\t" \
    "v_mul_f32_e64 v41, v41, v33\n\t" \
    "v_mul_f32_e64 v42, v42, v34\n\t" \
    "v_mul_f32_e64 v43, v43, v35\n\t" \
    "v_cvt_pk_bf16_f32 v36, v36, v37\n\t" \
    "v_cvt_pk_bf16_f32 v37, v38, v39\n\t" \
    "v_cvt_pk_bf16_f32 v38, v40, v41\n\t" \
    "v_cvt_pk_bf16_f32 v39, v42, v43\n\t" \
    "s_waitcnt vmcnt(0)\n\t" \
    "buffer_store_dwordx4 v[36:39], v31, %[rso], %[oo] offen sc0 sc1\n\t" \
    "v_med3_f32 v32, v240, -%[c60], %[c60]\n\t" \
    "v_med3_f32 v33, v241, -%[c60], %[c60]\n\t" \
    "v_med3_f32 v34, v242, -%[c60], %[c60]\n\t" \
    "v_med3_f32 v35, v243, -%[c60], %[c60]\n\t" \
    "v_mul_f32 v32, 0xbfb8aa3b, v32\n\t" \
    "v_mul_f32 v33, 0xbfb8aa3b, v33\n\t" \
    "v_mul_f32 v34, 0xbfb8aa3b, v34\n\t" \
    "v_mul_f32 v35, 0xbfb8aa3b, v35\n\t" \
    "v_exp_f32_e64 v32, v32\n\t" \
    "v_exp_f32_e64 v33, v33\n\t" \
    "v_exp_f32_e64 v34, v34\n\t" \
    "v_exp_f32_e64 v35, v35\n\t" \
    "v_add_f32_e64 v32, 1.0, v32\n\t" \
    "v_add_f32_e64 v33, 1.0, v33\n\t" \
    "v_add_f32_e64 v34, 1.0, v34\n\t" \
    "v_add_f32_e64 v35, 1.0, v35\n\t" \
    "v_rcp_f32_e64 v32, v32\n\t" \
    "v_rcp_f32_e64 v33, v33\n\t" \
    "v_rcp_f32_e64 v34, v34\n\t" \
    "v_rcp_f32_e64 v35, v35\n\t" \
    "v_mul_f32_e64 v240, v240, v32\n\t" \
    "v_mul_f32_e64 v241, v241, v33\n\t" \
    "v_mul_f32_e64 v242, v242, v34\n\t" \
    "v_mul_f32_e64 v243, v243, v35\n\t" \
    "v_med3_f32 v32, v244, -%[c60], %[c60]\n\t" \
    "v_med3_f32 v33, v245, -%[c60], %[c60]\n\t" \
    "v_med3_f32 v34, v246, -%[c60], %[c60]\n\t" \
    "v_med3_f32 v35, v247, -%[c60], %[c60]\n\t" \
    "v_mul_f32 v32, 0xbfb8aa3b, v32\n\t" \
    "v_mul_f32 v33, 0xbfb8aa3b, v33\n\t" \
    "v_mul_f32 v34, 0xbfb8aa3b, v34\n\t" \
    "v_mul_f32 v35, 0xbfb8aa3b, v35\n\t" \
    "v_exp_f32_e64 v32, v32\n\t" \
    "v_exp_f32_e64 v33, v33\n\t" \
    "v_exp_f32_e64 v34, v34\n\t" \
    "v_exp_f32_e64 v35, v35\n\t" \
    "v_add_f32_e64 v32, 1.0, v32\n\t" \
    "v_add_f32_e64 v33, 1.0, v33\n\t" \
    "v_add_f32_e64 v34, 1.0, v34\n\t" \
    "v_add_f32_e64 v35, 1.0, v35\n\t" \
    "v_rcp_f32_e64 v32, v32\n\t" \
    "v_rcp_f32_e64 v33, v33\n\t" \
    "v_rcp_f32_e64 v34, v34\n\t" \
    "v_rcp_f32_e64 v35, v35\n\t" \
    "v_mul_f32_e64 v244, v244, v32\n\t" \
    "v_mul_f32_e64 v245, v245, v33\n\t" \
    "v_mul_f32_e64 v246, v246, v34\n\t" \
    "v_mul_f32_e64 v247, v247, v35\n\t" \
    "v_cvt_pk_bf16_f32 v240, v240, v241\n\t" \
    "v_cvt_pk_bf16_f32 v241, v242, v243\n\t" \
    "v_cvt_pk_bf16_f32 v242, v244, v245\n\t" \
    "v_cvt_pk_bf16_f32 v243, v246, v247\n\t" \
    "s_waitcnt vmcnt(0)\n\t" \
    "s_add_u32 %[t0], %[oo], 0x4000\n\tbuffer_store_dwordx4 v[240:243], v31, %[rso], %[t0] offen sc0 sc1\n\t" \
    "s_nop 1\n\t" \
    ""
#define ASM_DONE2_3 \
    "v_mfma_f32_16x16x32_bf16 v[36:39], v[26:29], v[80:83], 0\n\t" \
    "v_mfma_f32_16x16x32_bf16 v[36:39], v[26:29], v[88:91], v[36:39]\n\t" \
    "v_mfma_f32_16x16x32_bf16 v[40:43], v[26:29], v[84:87], 0\n\t" \
    "v_mfma_f32_16x16x32_bf16 v[40:43], v[26:29], v[92:95], v[40:43]\n\t" \
    "v_mfma_f32_16x16x32_bf16 v[240:243], v[26:29], v[96:99], 0\n\t" \
    "v_mfma_f32_16x16x32_bf16 v[240:243], v[26:29], v[104:107], v[240:243]\n\t" \
    "v_mfma_f32_16x16x32_bf16 v[244:247], v[26:29], v[100:103], 0\n\t" \
    "v_mfma_f32_16x16x32_bf16 v[244:247], v[26:29], v[108:111], v[244:247]\n\t" \
    "s_nop 7\n\t" \
    "v_med3_f32 v32, v36, -%[c60], %[c60]\n\t" \
    "v_med3_f32 v33, v37, -%[c60], %[c60]\n\t" \
    "v_med3_f32 v34, v38, -%[c60], %[c60]\n\t" \
    "v_med3_f32 v35, v39, -%[c60], %[c60]\n\t" \
    "v_mul_f32 v32, 0xbfb8aa3b, v32\n\t" \
    "v_mul_f32 v33, 0xbfb8aa3b, v33\n\t" \
    "v_mul_f32 v34, 0xbfb8aa3b, v34\n\t" \
    "v_mul_f32 v35, 0xbfb8aa3b, v35\n\t" \
    "v_exp_f32_e64 v32, v32\n\t" \
    "v_exp_f32_e64 v33, v33\n\t" \
    "v_exp_f32_e64 v34, v34\n\t" \
    "v_exp_f32_e64 v35, v35\n\t" \
    "v_add_f32_e64 v32, 1.0, v32\n\t" \
    "v_add_f32_e64 v33, 1.0, v33\n\t" \
    "v_add_f32_e64 v34, 1.0, v34\n\t" \
    "v_add_f32_e64 v35, 1.0, v35\n\t" \
    "v_rcp_f32_e64 v32, v32\n\t" \
    "v_rcp_f32_e64 v33, v33\n\t" \
    "v_rcp_f32_e64 v34, v34\n\t" \
    "v_rcp_f32_e64 v35, v35\n\t" \
    "v_mul_f32_e64 v36, v36, v32\n\t" \
    "v_mul_f32_e64 v37, v37, v33\n\t" \
    "v_mul_f32_e64 v38, v38, v34\n\t" \
    "v_mul_f32_e64 v39, v39, v35\n\t" \
    "v_med3_f32 v32, v40, -%[c60], %[c60]\n\t" \
    "v_med3_f32 v33, v41, -%[c60], %[c60]\n\t" \
    "v_med3_f32 v34, v42, -%[c60], %[c60]\n\t" \
    "v_med3_f32 v35, v43, -%[c60], %[c60]\n\t" \
    "v_mul_f32 v32, 0xbfb8aa3b, v32\n\t" \
    "v_mul_f32 v33, 0xbfb8aa3b, v33\n\t" \
    "v_mul_f32 v34, 0xbfb8aa3b, v34\n\t" \
    "v_mul_f32 v35, 0xbfb8aa3b, v35\n\t" \
    "v_exp_f32_e64 v32, v32\n\t" \
    "v_exp_f32_e64 v33, v33\n\t" \
    "v_exp_f32_e64 v34, v34\n\t" \
    "v_exp_f32_e64 v35, v35\n\t" \
    "v_add_f32_e64 v32, 1.0, v32\n\t" \
    "v_add_f32_e64 v33, 1.0, v33\n\t" \
    "v_add_f32_e64 v34, 1.0, v34\n\t" \
    "v_add_f32_e64 v35, 1.0, v35\n\t" \
    "v_rcp_f32_e64 v32, v32\n\t" \
    "v_rcp_f32_e64 v33, v33\n\t" \
    "v_rcp_f32_e64 v34, v34\n\t" \
    "v_rcp_f32_e64 v35, v35\n\t" \
    "v_mul_f32_e64 v40, v40, v32\n\t" \
    "v_mul_f32_e64 v41, v41, v33\n\t" \
    "v_mul_f32_e64 v42, v42, v34\n\t" \
    "v_mul_f32_e64 v43, v43, v35\n\t" \
    "v_cvt_pk_bf16_f32 v36, v36, v37\n\t" \
    "v_cvt_pk_bf16_f32 v37, v38, v39\n\t" \
    "v_cvt_pk_bf16_f32 v38, v40, v41\n\t" \
    "v_cvt_pk_bf16_f32 v39, v42, v43\n\t" \
    "s_waitcnt vmcnt(0)\n\t" \
    "buffer_store_dwordx4 v[36:39], v31, %[rso], %[oo] offen sc0 sc1\n\t" \
    "v_med3_f32 v32, v240, -%[c60], %[c60]\n\t" \
    "v_med3_f32 v33, v241, -%[c60], %[c60]\n\t" \
    "v_med3_f32 v34, v242, -%[c60], %[c60]\n\t" \
    "v_med3_f32 v35, v243, -%[c60], %[c60]\n\t" \
    "v_mul_f32 v32, 0xbfb8aa3b, v32\n\t" \
    "v_mul_f32 v33, 0xbfb8aa3b, v33\n\t" \
    "v_mul_f32 v34, 0xbfb8aa3b, v34\n\t" \
    "v_mul_f32 v35, 0xbfb8aa3b, v35\n\t" \
    "v_exp_f32_e64 v32, v32\n\t" \
    "v_exp_f32_e64 v33, v33\n\t" \
    "v_exp_f32_e64 v34, v34\n\t" \
    "v_exp_f32_e64 v35, v35\n\t" \
    "v_add_f32_e64 v32, 1.0, v32\n\t" \
    "v_add_f32_e64 v33, 1.0, v33\n\t" \
    "v_add_f32_e64 v34, 1.0, v34\n\t" \
    "v_add_f32_e64 v35, 1.0, v35\n\t" \
    "v_rcp_f32_e64 v32, v32\n\t" \
    "v_rcp_f32_e64 v33, v33\n\t" \
    "v_rcp_f32_e64 v34, v34\n\t" \
    "v_rcp_f32_e64 v35, v35\n\t" \
    "v_mul_f32_e64 v240, v240, v32\n\t" \
    "v_mul_f32_e64 v241, v241, v33\n\t" \
    "v_mul_f32_e64 v242, v242, v34\n\t" \
    "v_mul_f32_e64 v243, v243, v35\n\t" \
    "v_med3_f32 v32, v244, -%[c60], %[c60]\n\t" \
    "v_med3_f32 v33, v245, -%[c60], %[c60]\n\t" \
    "v_med3_f32 v34, v246, -%[c60], %[c60]\n\t" \
    "v_med3_f32 v35, v247, -%[c60], %[c60]\n\t" \
    "v_mul_f32 v32, 0xbfb8aa3b, v32\n\t" \
    "v_mul_f32 v33, 0xbfb8aa3b, v33\n\t" \
    "v_mul_f32 v34, 0xbfb8aa3b, v34\n\t" \
    "v_mul_f32 v35, 0xbfb8aa3b, v35\n\t" \
    "v_exp_f32_e64 v32, v32\n\t" \
    "v_exp_f32_e64 v33, v33\n\t" \
    "v_exp_f32_e64 v34, v34\n\t" \
    "v_exp_f32_e64 v35, v35\n\t" \
    "v_add_f32_e64 v32, 1.0, v32\n\t" \
    "v_add_f32_e64 v33, 1.0, v33\n\t" \
    "v_add_f32_e64 v34, 1.0, v34\n\t" \
    "v_add_f32_e64 v35, 1.0, v35\n\t" \
    "v_rcp_f32_e64 v32, v32\n\t" \
    "v_rcp_f32_e64 v33, v33\n\t" \
    "v_rcp_f32_e64 v34, v34\n\t" \
    "v_rcp_f32_e64 v35, v35\n\t" \
    "v_mul_f32_e64 v244, v244, v32\n\t" \
    "v_mul_f32_e64 v245, v245, v33\n\t" \
    "v_mul_f32_e64 v246, v246, v34\n\t" \
    "v_mul_f32_e64 v247, v247, v35\n\t" \
    "v_cvt_pk_bf16_f32 v240, v240, v241\n\t" \
    "v_cvt_pk_bf16_f32 v241, v242, v243\n\t" \
    "v_cvt_pk_bf16_f32 v242, v244, v245\n\t" \
    "v_cvt_pk_bf16_f32 v243, v246, v247\n\t" \
    "s_waitcnt vmcnt(0)\n\t" \
    "s_add_u32 %[t0], %[oo], 0x4000\n\tbuffer_store_dwordx4 v[240:243], v31, %[rso], %[t0] offen sc0 sc1\n\t" \
    "s_nop 1\n\t" \
    ""
#define ASM_LOADS_0 \
    "s_nop 4\n\t" \
    "buffer_load_dwordx4 a[64:67], v47, %[rss], %[ld] offen offset:0 " LAUXS "\n\t" \
    "buffer_load_dwordx4 a[68:71], v47, %[rss], %[ld] offen offset:256 " LAUXS "\n\t" \
    "s_add_u32 %[t0], %[ld], 0x40000\n\tbuffer_load_dwordx4 a[72:75], v47, %[rss], %[t0] offen offset:0 " LAUXS "\n\t" \
    "s_add_u32 %[t0], %[ld], 0x40000\n\tbuffer_load_dwordx4 a[76:79], v47, %[rss], %[t0] offen offset:256 " LAUXS "\n\t" \
    "s_add_u32 %[ld], %[ld], %[dl]\n\tbuffer_load_dwordx4 a[80:83], v47, %[rss], %[ld] offen offset:0 " LAUXS "\n\t" \
    "buffer_load_dwordx4 a[84:87], v47, %[rss], %[ld] offen offset:256 " LAUXS "\n\t" \
    "s_add_u32 %[t0], %[ld], 0x40000\n\tbuffer_load_dwordx4 a[88:91], v47, %[rss], %[t0] offen offset:0 " LAUXS "\n\t" \
    "s_add_u32 %[t0], %[ld], 0x40000\n\tbuffer_load_dwordx4 a[92:95], v47, %[rss], %[t0] offen offset:256 " LAUXS "\n\t" \
    "v_mov_b32 v18, 0\n\tbuffer_load_dword v18, off, %[rss], %[pc] sc1\n\t" \
    ""
#define ASM_LOADS_1 \
    "s_nop 4\n\t" \
    "buffer_load_dwordx4 a[96:99], v47, %[rss], %[ld] offen offset:0 " LAUXS "\n\t" \
    "buffer_load_dwordx4 a[100:103], v47, %[rss], %[ld] offen offset:256 " LAUXS "\n\t" \
    "s_add_u32 %[t0], %[ld], 0x40000\n\tbuffer_load_dwordx4 a[104:107], v47, %[rss], %[t0] offen offset:0 " LAUXS "\n\t" \
    "s_add_u32 %[t0], %[ld], 0x40000\n\tbuffer_load_dwordx4 a[108:111], v47, %[rss], %[t0] offen offset:256 " LAUXS "\n\t" \
    "s_add_u32 %[ld], %[ld], %[dl]\n\tbuffer_load_dwordx4 a[112:115], v47, %[rss], %[ld] offen offset:0 " LAUXS "\n\t" \
    "buffer_load_dwordx4 a[116:119], v47, %[rss], %[ld] offen offset:256 " LAUXS "\n\t" \
    "s_add_u32 %[t0], %[ld], 0x40000\n\tbuffer_load_dwordx4 a[120:123], v47, %[rss], %[t0] offen offset:0 " LAUXS "\n\t" \
    "s_add_u32 %[t0], %[ld], 0x40000\n\tbuffer_load_dwordx4 a[124:127], v47, %[rss], %[t0] offen offset:256 " LAUXS "\n\t" \
    "v_mov_b32 v44, 0\n\tbuffer_load_dword v44, off, %[rss], %[pc] sc1\n\t" \
    ""
#define ASM_LOADS_2 \
    "s_nop 4\n\t" \
    "buffer_load_dwordx4 v[48:51], v47, %[rss], %[ld] offen offset:0 " LAUXS "\n\t" \
    "buffer_load_dwordx4 v[52:55], v47, %[rss], %[ld] offen offset:256 " LAUXS "\n\t" \
    "s_add_u32 %[t0], %[ld], 0x40000\n\tbuffer_load_dwordx4 v[56:59], v47, %[rss], %[t0] offen offset:0 " LAUXS "\n\t" \
    "s_add_u32 %[t0], %[ld], 0x40000\n\tbuffer_load_dwordx4 v[60:63], v47, %[rss], %[t0] offen offset:256 " LAUXS "\n\t" \
    "s_add_u32 %[ld], %[ld], %[dl]\n\tbuffer_load_dwordx4 v[64:67], v47, %[rss], %[ld] offen offset:0 " LAUXS "\n\t" \
    "buffer_load_dwordx4 v[68:71], v47, %[rss], %[ld] offen offset:256 " LAUXS "\n\t" \
    "s_add_u32 %[t0], %[ld], 0x40000\n\tbuffer_load_dwordx4 v[72:75], v47, %[rss], %[t0] offen offset:0 " LAUXS "\n\t" \
    "s_add_u32 %[t0], %[ld], 0x40000\n\tbuffer_load_dwordx4 v[76:79], v47, %[rss], %[t0] offen offset:256 " LAUXS "\n\t" \
    ""
#define ASM_LOADS_3 \
    "s_nop 4\n\t" \
    "buffer_load_dwordx4 v[80:83], v47, %[rss], %[ld] offen offset:0 " LAUXS "\n\t" \
    "buffer_load_dwordx4 v[84:87], v47, %[rss], %[ld] offen offset:256 " LAUXS "\n\t" \
    "s_add_u32 %[t0], %[ld], 0x40000\n\tbuffer_load_dwordx4 v[88:91], v47, %[rss], %[t0] offen offset:0 " LAUXS "\n\t" \
    "s_add_u32 %[t0], %[ld], 0x40000\n\tbuffer_load_dwordx4 v[92:95], v47, %[rss], %[t0] offen offset:256 " LAUXS "\n\t" \
    "s_add_u32 %[ld], %[ld], %[dl]\n\tbuffer_load_dwordx4 v[96:99], v47, %[rss], %[ld] offen offset:0 " LAUXS "\n\t" \
    "buffer_load_dwordx4 v[100:103], v47, %[rss], %[ld] offen offset:256 " LAUXS "\n\t" \
    "s_add_u32 %[t0], %[ld], 0x40000\n\tbuffer_load_dwordx4 v[104:107], v47, %[rss], %[t0] offen offset:0 " LAUXS "\n\t" \
    "s_add_u32 %[t0], %[ld], 0x40000\n\tbuffer_load_dwordx4 v[108:111], v47, %[rss], %[t0] offen offset:256 " LAUXS "\n\t" \
    ""

constexpr int NR = 8;
constexpr int ML = 1024, NN = 8192, KK = 8192, NB = 1024;
constexpr int LDS_BYTES = 163840;
constexpr int A_BASE = 0x18000;
constexpr int NWG = 256;
constexpr int NU = 2;
constexpr unsigned long long DEADLINE = 50000000ull;
constexpr int PFB = 5;

struct Sched { unsigned w[NWG][2 * NU]; };
constexpr int skew(int c) { return 2 * (c & 3) - 3; }
constexpr int pskew(int par) { return par ? -5 : 5; }
constexpr Sched make_sched() {
    Sched s{};
    for (int b = 0; b < NWG; ++b) {
        const int x = b & 7, e = b >> 5, c = (b >> 3) & 3;
        auto put = [&](int r, int kt, int len, int kap, int tm, int blk, int pfd) {
            s.w[b][2 * r] = kt | (len << 8) | (kap << 18) | (tm << 22) | (e << 24) | (((blk + e) & 3) << 27);
            s.w[b][2 * r + 1] = (blk * 128 + (e * 4 + tm) * 4) | (c << 10) | (pfd << 12); };
        constexpr int g[4] = {0, 3, 1, 2};
        const int xq = x >> 1, par = x & 1;
        auto off = [&](int cc) { int o = 0; for (int k = 0; k < xq; ++k) o += skew(cc ^ g[xq] ^ g[k]); return o; };
        int omax = off(0);
        for (int cc = 1; cc < 4; ++cc) omax = off(cc) > omax ? off(cc) : omax;
        const int tm0 = c ^ g[xq], blk0 = par ^ (xq & 1);
        put(0, 32 * xq + off(c) + ((xq & 1) ? pskew(par ^ 1) : 0), 32 + skew(c) + pskew(par), xq, tm0, blk0, PFB + omax - off(c));
        put(1, 32 * xq, 32, xq, c, 2 + par, PFB + skew(c) - skew(0));
    }
    return s;
}
__device__ constexpr Sched SCHED = make_sched();

constexpr int WS_TMO = 512;
constexpr int WS_ENTER = 1024;
constexpr int WS_DONE = 2048;
constexpr int WS_CNT = 4096;
constexpr int WS_TICKET = 32768;
constexpr unsigned WS_SCR = 1u << 20;
constexpr unsigned WS_END = WS_SCR + NU * (32u << 20);
constexpr unsigned OOB = 0xf0000000u;

constexpr int CV = 14, VTL = 14, VTO = 15, VPF = 17, SEN0 = 18, RPOLL = 19, AAD0 = 20, AAD1 = 21, BAD = 22, VOFF = 26, SEL = 26,
              VLIN = 30, VOUT = 31, RT = 32, SEN1 = 44, VUT = 45, VLD = 47;

struct Params {
    const void* x; const void* w; void* wsme;
    int rank;
    int pad;
    void* out[NR];
    void* ws[NR];
};

struct Unit { unsigned w0, w1;
    __device__ int kt() const { return w0 & 255; }          __device__ int len() const { return (w0 >> 8) & 127; }
    __device__ int kap() const { return (w0 >> 18) & 15; }
    __device__ int tm() const { return (w0 >> 22) & 3; }    __device__ int e() const { return (w0 >> 24) & 7; }
    __device__ int sub() const { return (w0 >> 27) & 3; }   __device__ int J() const { return (w0 >> 22) & 127; }
    __device__ unsigned slot() const { return w1 & 1023; }  __device__ unsigned pq() const { return (w1 >> 10) & 3; }
    __device__ int pfd() const { return (w1 >> 12) & 31; }
};
template <int R> __device__ __forceinline__ void setv(unsigned v) { asm volatile("v_mov_b32 v[%c0], %1" :: "i"(R), "v"(v)); }
template <int R> __device__ __forceinline__ unsigned getv() { unsigned f; asm volatile("v_mov_b32 %0, v[%c1]" : "=v"(f) : "i"(R)); return f; }
#define WAITVM(n) asm volatile("s_waitcnt vmcnt(" #n ")" ::: "memory")

__global__ __launch_bounds__(256, 1) __attribute__((amdgpu_num_vgpr(CV))) void gemm_a2a(Params p) {
    extern __shared__ __attribute__((aligned(1024))) unsigned char lds_raw[];
    asm volatile("" ::: "v255", "a255");
    const int tid = threadIdx.x;
    const int wave = __builtin_amdgcn_readfirstlane(tid >> 6);
    const int lane = tid & 63;
    const int b = blockIdx.x;
    const int rank = p.rank;
    char* ws = (char*)p.wsme;
    const void* px = p.x; const void* pw_ = p.w;
    unsigned tw[2 * NU];
#pragma unroll
    for (int k = 0; k < 2 * NU; ++k) tw[k] = SCHED.w[b][k];
    asm volatile("" :: "s"(px), "s"(pw_), "s"(ws), "s"(rank), "s"(tw[0]), "s"(tw[1]), "s"(tw[2]), "s"(tw[3]));
    const rsrc_t rs = wave == 0 ? __builtin_amdgcn_make_buffer_rsrc((void*)px, 0, ML * KK * 2, 0x00020000) : __builtin_amdgcn_make_buffer_rsrc((void*)pw_, 0, KK * NN * 2, 0x00020000);
    const rsrc_t rss = __builtin_amdgcn_make_buffer_rsrc((void*)ws, 0, WS_END, 0x00020000);
    const int dest = (rank + 1 + (b >> 5)) & 7;
    const Unit u0{tw[0], tw[1]};
    auto unit = [&](int r) { const unsigned v = getv<VUT>(); return Unit{(unsigned)__builtin_amdgcn_readlane(v, 2 * r), (unsigned)__builtin_amdgcn_readlane(v, 2 * r + 1)}; };
    auto src = [&](const Unit& u) {
        if (wave == 0) return (unsigned)(u.tm() * 256 * (KK * 2) + u.kt() * 128);
        const unsigned o = (unsigned)(u.kt() * 64 * (NN * 2) + (dest * 4 + u.sub()) * 512);
        return wave == 1 ? o : (wave == 2 ? o + u.pq() * (16 * NN * 2) : OOB); };
    auto ld_of = [&](const Unit& u, int im, int k) { const int gam = 8 * u.kap() + 2 * k;
        return WS_SCR + u.slot() * 131072u + (unsigned)im * 32768u + (unsigned)((gam ^ ((u.J() ^ (im << 3)) & 30)) << 10); };
    auto out_of = [&](const Unit& u, int im, int k) { const int gam = 8 * u.kap() + 2 * k;
        return (unsigned)((rank * ML + u.tm() * 256 + (im >> 1) * 128 + ((gam >> 1) & 7) * 16) * (NB * 2) + u.sub() * 512 + (im & 1) * 256 + (gam >> 4) * 128); };
    auto set_sel = [&](int l) {
        const int i = l & 15, g = l >> 4;
        const int jj = 4 * (i >> 3) + (i & 3);
        const unsigned one = ((g & 1) == ((i >> 2) & 1)) ? ((jj & 1) ? 0x3f800000u : 0x00003f80u) : 0u;
        const int d1 = jj >> 1;
        setv<SEL>(d1 == 0 ? one : 0u); setv<SEL + 1>(d1 == 1 ? one : 0u); setv<SEL + 2>(d1 == 2 ? one : 0u); setv<SEL + 3>(d1 == 3 ? one : 0u);
    };
    unsigned so = src(u0);
    const unsigned pfi = 64u * NN * 2u;
    {
        auto vo = [&](int i) {
            const unsigned a = (unsigned)(((lane >> 3) + 8 * i) * (KK * 2) + (((lane & 7) ^ (((i & 1) << 2) | (lane >> 4))) << 4) - i * 1024);
            const int kq = lane >> 4, sw = (kq | ((i >> 1) << 2)) << 1, ch = (lane & 15) ^ sw;
            const unsigned bo = (unsigned)((4 * i + kq) * (NN * 2) + (ch >> 3) * 256 + (ch & 7) * 16 - i * 1024);
            return wave == 0 ? a : bo; };
        setv<VOFF>(vo(0)); setv<VOFF + 1>(vo(1)); setv<VOFF + 2>(vo(2)); setv<VOFF + 3>(vo(3));
    }
#define G4(dm, ds) asm volatile("buffer_load_dwordx4 v26, %1, %0 offen lds\n\tbuffer_load_dwordx4 v27, %1, %0 offen offset:1024 lds\n\tbuffer_load_dwordx4 v28, %1, %0 offen offset:2048 lds\n\tbuffer_load_dwordx4 v29, %1, %0 offen offset:3072 lds\n\t" \
        "s_add_u32 m0, m0, %2\n\ts_add_u32 %0, %0, %3" : "+s"(so) : "s"(rs), "i"(dm), "i"(ds) : "scc")
#define BU(dml, dsl) do { G4(0x1000, 16 * NN * 2); G4(0x1000, 16 * NN * 2); G4(0x1000, 16 * NN * 2); G4(dml, dsl); } while (0)
#define AT() do { G4(0x1000, 0x80000); G4(0x1000, 0x80000); G4(0x2000, 0x100000); G4(0x1000, 0x80000); G4(0x1000, 0x80000); G4(-0x3000, -0x180000); G4(0x4000, 0x200000); G4(0x1000, -7 * 0x80000 + 128); } while (0)
    if (wave == 0) { asm volatile("s_mov_b32 m0, 0x18000\n\ts_nop 4"); AT(); }
    else if (wave == 1) { asm volatile("s_mov_b32 m0, 0x14000\n\ts_nop 4"); BU(0x1000 - 0x18000, -48 * NN * 2 + 128); BU(0x11000, 144 * NN * 2 - 128); }
    else {
        setv<VPF>((unsigned)((lane >> 2) * (NN * 2) + (lane & 3) * 128));
        so += u0.pfd() * pfi;
        setv<VTL>(lane < 8 ? ld_of(u0, wave - 2 * (lane >> 2), lane & 3) : OOB);
        setv<VTO>(lane >= 2 ? out_of(u0, wave - 2 * (((lane - 2) >> 2) & 1), (lane - 2) & 3) : OOB);
    }
#pragma unroll
    for (int k = 0; k < 2 * NU; ++k) asm volatile("v_writelane_b32 v[%c0], %1, %c2" :: "i"(VUT), "s"(tw[k]), "i"(k));
    {
        const int g = lane >> 4, r16 = lane & 15, wm = wave >> 1, wn = wave & 1;
        const unsigned a0 = (unsigned)(A_BASE + (wm * 128 + r16) * 128 + ((g ^ ((r16 >> 1) & 7)) << 4));
        setv<AAD0>(a0); setv<AAD1>(a0 ^ 64u);
        const int q = r16 >> 2, pp = r16 & 3, k = 8 * g + q, sw = ((k & 3) | ((k >> 1) & 4)) << 1;
        const unsigned b0 = (unsigned)(k * 256 + (((wn * 8 + (pp >> 1)) ^ sw) << 4) + ((pp & 1) << 3));
        setv<BAD>(b0 + 0x14000); setv<BAD + 1>((b0 ^ 32u) + 0x14000); setv<BAD + 2>((b0 ^ 64u) + 0x14000); setv<BAD + 3>((b0 ^ 96u) + 0x14000);
        setv<VLIN>(lane * 16); setv<VOUT>((r16 & 7) * (NB * 2) + (r16 >> 3) * 64 + g * 16); setv<VLD>((g >> 1) * 131072 + (g & 1) * 512 + (r16 & 7) * 16 + (r16 >> 3) * 1024);
    }
    asm volatile(ASM_ZERO);
#define LANE ((int)(getv<VLIN>() >> 4))
    if (wave == 0) { AT(); asm volatile("s_mov_b32 m0, 0x18000"); }
    else if (wave >= 2) {
        const unsigned keep = so;
        so = (unsigned)((u0.kt() + 1) * 64 * (NN * 2) + (dest * 4 + u0.sub()) * 512 + (wave == 2 ? 128 : 0));
        if (wave == 2) { asm volatile("s_mov_b32 m0, 0x8000\n\ts_nop 4"); BU(0x1000, 16 * NN * 2 - 128); BU(0, 0); }
        else { asm volatile("s_mov_b32 m0, 0x4000\n\ts_nop 4"); BU(0x9000, 16 * NN * 2 + 128); BU(0, 0); }
        so = keep;
        set_sel(LANE);
    }
    asm volatile("v_mov_b32 v[%c0], %2\n\tv_mov_b32 v[%c1], 1\n\ts_mov_b64 exec, 1\n\tglobal_atomic_add v[%c0], v[%c0], v[%c1], %3 sc0\n\ts_mov_b64 exec, -1"
                 :: "i"(RPOLL), "i"(SEN0), "s"(WS_TICKET + 128 * (4 * (b & 7) + wave)), "s"(ws) : "memory");
    if (wave == 0) WAITVM(33); else if (wave == 1) WAITVM(17);
    asm volatile("s_barrier" ::: "memory");
    asm volatile(ASM_FIRST);
#pragma unroll
    for (int i = 0; i < 4; ++i) asm volatile("v_add_u32 v[%c0], 0xfffec000, v[%c0]" :: "i"(BAD + i));
    if (wave == 1) WAITVM(1);
    asm volatile("s_waitcnt lgkmcnt(0)\n\ts_barrier" ::: "memory");

    GAS unsigned* tmo = (GAS unsigned*)(ws + WS_TMO);
    auto signal = [&](int r) {
        asm volatile("v_mov_b32 v[%c0], %2\n\tv_mov_b32 v[%c1], 1\n\ts_mov_b64 exec, 1\n\tglobal_atomic_add v[%c0], v[%c1], %3\n\ts_mov_b64 exec, -1"
                     :: "i"(RT), "i"(RT + 1), "s"(WS_CNT + 128 * unit(r).J()), "s"(ws) : "memory"); };
    auto dump_base = [&](int r) { const Unit u = unit(r);
        return WS_SCR + ((u.slot() + u.kap()) * 4 + wave) * 32768u + (unsigned)(((u.J() ^ (wave << 3)) & 30) << 10); };
    unsigned call = 0;
    auto get_call = [&]() { call = (unsigned)__builtin_amdgcn_readfirstlane(getv<RPOLL>()) / 32u + 1u; };
    int rem = 0; unsigned t0, sd = 0;
    const float c60 = 60.0f;

    int c = 0, ph = 0;
    unsigned pc = WS_CNT + 128 * u0.J();
    const rsrc_t rso = __builtin_amdgcn_make_buffer_rsrc(p.out[dest], 0, NR * ML * NB * 2, 0x00020000);
#define LOADS(M, l1) do { unsigned l_ = (l1); asm volatile(M : [t0] "=&s"(t0), [ld] "+s"(l_) : [rss] "s"(rss), [pc] "s"(pc), [dl] "s"(128) : "scc", "memory"); } while (0)
#define DONE(M, oo_) asm volatile(M : [t0] "=&s"(t0) : [c60] "s"(c60), [rso] "s"(rso), [oo] "s"(oo_) : "scc", "memory")

    auto wait_ge = [&](unsigned off, unsigned want, int nap) -> bool {
        const unsigned long long t0_ = __builtin_amdgcn_s_memrealtime();
        const bool hi = LANE >= 32;
        GAS unsigned* a_ = (GAS unsigned*)(ws + (hi ? (unsigned)WS_TMO : off));
        for (;;) {
            const unsigned v = __hip_atomic_load(a_, __ATOMIC_RELAXED, __HIP_MEMORY_SCOPE_SYSTEM);
            const unsigned long long m = __builtin_amdgcn_ballot_w64(hi ? v == call : v < want);
            if (m >> 32) return true;
            if (m == 0) return false;
            if (__builtin_amdgcn_s_memrealtime() - t0_ > DEADLINE) { __hip_atomic_store(tmo, call, __ATOMIC_RELAXED, __HIP_MEMORY_SCOPE_AGENT); return true; }
            if (nap) __builtin_amdgcn_s_sleep(4);
        }
    };
    if (wave < 2) {
#pragma clang loop unroll(disable)
        for (int r = 0; r < NU; ++r) {
            const unsigned nx = src(unit(r + 1 < NU ? r + 1 : r));
            rem = unit(r).len();
#define EDGE(M) asm volatile(M : [so] "+s"(so), [rem] "+s"(rem), [t0] "=&s"(t0) : [rs] "s"(rs), [nx] "s"(nx), [sd] "s"(sd), [rss] "s"(rss) : "scc", "memory")
            if (r > 0) {
                if (wave == 0) { EDGE(ASM_START_A); WAITVM(8); } else { EDGE(ASM_START_B); WAITVM(16); }
                signal(r - 1);
            }
            sd = dump_base(r);
#define PLAIN(M) asm volatile(M : [so] "+s"(so), [rem] "+s"(rem) : [rs] "s"(rs), [nx] "s"(nx), [stop] "s"(1) : "scc", "memory")
            if (wave == 0) { PLAIN(ASM_PLAIN_A); EDGE(ASM_END_A); } else { PLAIN(ASM_PLAIN_B); EDGE(ASM_END_B); }
        }
        get_call();
        set_sel(LANE);
    } else {
        int gap = 0;
        unsigned thr = ~0u, pw = OOB, go, l1, oo;
#pragma clang loop unroll(disable)
        for (int r = 0; r < NU; ++r) {
            rem = unit(r).len();
            const Unit un = unit(r + 1 < NU ? r + 1 : r);
            const unsigned nx = r + 1 < NU ? src(un) : OOB;
            const int psw = un.pfd() + 1;
#define PFOPS [rs] "s"(rs), [nx] "s"(nx), [pfi] "s"(pfi), [psw] "s"(psw)
            if (r > 0) {
                if (r == 1) asm volatile("global_load_dword v[%c0], %1, %2 sc0 sc1" :: "i"(RT + 4), "v"(WS_ENTER + 128 * (LANE & 7)), "s"(ws) : "memory");
#define EDGEC(M) asm volatile(M : [rem] "+s"(rem), [t0] "=&s"(t0), [so] "+s"(so) : [sd] "s"(sd), [rss] "s"(rss), PFOPS : "scc", "memory")
                EDGEC(ASM_START_C);
                WAITVM(1);
                signal(r - 1);
                thr = call * 16u; pw = pc;
                if (r == 1) {
                    if (__builtin_amdgcn_ballot_w64(getv<RT + 4>() < call) != 0) wait_ge(WS_ENTER + 128 * (LANE & 7), call, 1);
                }
            } else {
                asm volatile(ASM_C0 : [rem] "+s"(rem), [so] "+s"(so) : PFOPS : "scc", "memory");
                get_call();
                setv<RPOLL>(0u);
                if (b == 0 && wave == 2)
                    __hip_atomic_store((GAS unsigned*)((char*)p.ws[LANE & 7] + WS_ENTER + 128 * rank), call, __ATOMIC_RELAXED, __HIP_MEMORY_SCOPE_SYSTEM);
            }
            sd = dump_base(r);
#pragma clang loop unroll(disable)
            for (;;) {
                if (rem == 1) break;
                asm volatile(ASM_BG : [rem] "+s"(rem), [so] "+s"(so), [c] "+s"(c), [thr] "+s"(thr), [pw] "+s"(pw), [ph] "+s"(ph), [gap] "+s"(gap),
                                      [t0] "=&s"(t0), [go] "=&s"(go), [ld] "=&s"(l1), [oo] "=&s"(oo)
                             : [stop] "s"(1), PFOPS, [rss] "s"(rss), [c60] "s"(c60), [rso] "s"(rso), [cend] "s"(10), [pc] "s"(pc), [zls] "s"(2), [dl] "s"(128) : "scc", "memory");
                if (c == 10) thr = ~0u;
            }
            EDGEC(ASM_END_C);
        }
    }
    auto nan_out = [&](unsigned oo) { const u32x4 nv = {0x7fc07fc0u, 0x7fc07fc0u, 0x7fc07fc0u, 0x7fc07fc0u}; __builtin_amdgcn_raw_buffer_store_b128(nv, rso, getv<VOUT>(), oo, 17); };
    asm volatile("s_waitcnt lgkmcnt(0)" ::: "memory");
    asm volatile(ASM_FINAL_HALF : [t0] "=&s"(t0) : [sd] "s"(sd), [rss] "s"(rss) : "scc", "memory");
    WAITVM(0);
    signal(NU - 1);
    const unsigned target = call * 16u;
    if (wave >= 2 && c < 10) {
        const bool bad = c == 0 && wait_ge(pc, target, 1);
#pragma clang loop unroll(disable)
        for (; c < 10; ++c) {
            const unsigned oo = __builtin_amdgcn_readlane(getv<VTO>(), c), l1 = __builtin_amdgcn_readlane(getv<VTL>(), c);
#define BK(P) do { if (ph < 2) ph += 1; else { WAITVM(9); if (bad) nan_out(oo); else DONE(ASM_DONE2_##P, oo); } LOADS(ASM_LOADS_##P, l1); } while (0)
            if (c & 1) BK(1); else BK(0);
        }
    }
    {
        Unit ul = unit(1);
        pc = WS_CNT + 128 * ul.J();
        if (b & 1) {
            if (wait_ge(pc, target, 1)) { for (int k = 0; k < 4; ++k) nan_out(out_of(ul, wave, k)); }
            else {
                LOADS(ASM_LOADS_0, ld_of(ul, wave, 0)); LOADS(ASM_LOADS_1, ld_of(ul, wave, 1)); LOADS(ASM_LOADS_2, ld_of(ul, wave, 2)); LOADS(ASM_LOADS_3, ld_of(ul, wave, 3));
                WAITVM(25);
                DONE(ASM_DONE2_0, out_of(ul, wave, 0)); DONE(ASM_DONE2_1, out_of(ul, wave, 1)); DONE(ASM_DONE2_2, out_of(ul, wave, 2)); DONE(ASM_DONE2_3, out_of(ul, wave, 3));
            }
            ul.w0 = (ul.w0 & ~(3u << 27)) | (unsigned)((2 + (b >> 5)) & 3) << 27; ul.w1 ^= 128;
            pc = WS_CNT + 128 * ul.J();
        }
        const int k0 = 2 * (b & 1);
        if (wait_ge(pc, target, 1)) { nan_out(out_of(ul, wave, k0)); nan_out(out_of(ul, wave, k0 + 1)); }
        else {
            LOADS(ASM_LOADS_2, ld_of(ul, wave, k0)); LOADS(ASM_LOADS_3, ld_of(ul, wave, k0 + 1));
            WAITVM(8);
            DONE(ASM_DONE2_2, out_of(ul, wave, k0)); DONE(ASM_DONE2_3, out_of(ul, wave, k0 + 1));
        }
    }
    WAITVM(0);
    if (dest != rank && LANE == 0)
        __hip_atomic_fetch_add((GAS unsigned*)((char*)p.ws[dest] + WS_DONE + 128 * rank), 1u, __ATOMIC_RELAXED, __HIP_MEMORY_SCOPE_SYSTEM);
    if (b == NWG - 25 && wave == 2) {
        if (wait_ge(WS_DONE + 128 * (LANE & 7), (LANE & 7) != rank ? call * 128u : 0u, 0))
            *(GAS unsigned short*)((char*)p.out[rank] + (size_t)(rank * ML) * NB * 2) = 0x7fc0;
    }
}

extern "C" void kernel_launch(int rank, int nranks, ncclComm_t comm, hipStream_t stream,
                              void* const* d_in, const int* in_sizes, int n_in,
                              void* d_out, int out_size, const int* in_dtypes, int out_dtype,
                              void* const* d_out_all, void* const* d_ws_all, size_t ws_bytes) {
    if (nranks != NR || (unsigned)rank >= (unsigned)NR || ws_bytes < WS_END || n_in < 2 || in_sizes[0] != ML * KK || in_sizes[1] != KK * NN || out_size != NR * ML * NB) {
        fprintf(stderr, "unexpected configuration\n"); return; }
    Params p;
    p.x = d_in[0]; p.w = d_in[1];
    for (int i = 0; i < NR; ++i) { p.out[i] = d_out_all[i]; p.ws[i] = d_ws_all[i]; }
    p.rank = rank; p.pad = 0; p.wsme = d_ws_all[rank];
    HIP_CHECK(hipFuncSetAttribute((const void*)gemm_a2a, hipFuncAttributeMaxDynamicSharedMemorySize, LDS_BYTES));
    hipLaunchKernelGGL(gemm_a2a, dim3(NWG), dim3(256), LDS_BYTES, stream, p);
    HIP_CHECK(hipGetLastError());
}
```
